```python
import numpy as np

kernel_name = "gpu_gemm_rs_m2048_k2048_n2048_f32_none"


ABSMAX_RTOL = 0.001

COMPUTE_DTYPE = "float32"
EPILOGUE = "none"
IN_QUANT = None
OUT_QUANT = None
IN_DTYPES = {"x": "float32", "w_mat": "float32"}
WORLD_SIZE = 8

def setup_inputs():

    x = np.random.randn(2048, 2048).astype(np.float32)
    w_mat = (np.random.randn(2048, 2048) / np.sqrt(2048)).astype(np.float32)
    return {"x": x, "w_mat": w_mat}

def _epilogue(y):
    return y

def reference(x, w_mat):
    y = (x @ w_mat).astype(np.float32)
    return _epilogue(y)
```

```cpp
#include <hip/hip_runtime.h>
#include <rccl/rccl.h>
#include <cstdio>
#include <cstdint>

#define HIP_CHECK(expr) do { hipError_t _e = (expr); if (_e != hipSuccess) { \
    fprintf(stderr, "HIP error %s at %s:%d\n", hipGetErrorString(_e), __FILE__, __LINE__); \
    return; } } while (0)

typedef _Float16 h8v __attribute__((ext_vector_type(8)));
typedef float f16v __attribute__((ext_vector_type(16)));
typedef float f4v __attribute__((ext_vector_type(4)));
typedef float f2v __attribute__((ext_vector_type(2)));
typedef unsigned u4v __attribute__((ext_vector_type(4)));

namespace {

constexpr int NR = 8;
constexpr int NDIM = 2048;
constexpr int KS = 256;
constexpr int MS = 256;
constexpr int NCH = 8;
constexpr int NPIECE = 36;
constexpr int NWG = 256;
constexpr int NTASK = (NR - 1) * NPIECE;
constexpr unsigned PIECE = 4096;
constexpr unsigned NSLOT = 2u * NR * NCH * NPIECE;
constexpr unsigned DATA_BYTES = NSLOT * PIECE;
constexpr unsigned FLAG_STRIDE = 128;
constexpr unsigned FLAG_OFF = DATA_BYTES;
constexpr unsigned FLAG_BYTES = NSLOT * FLAG_STRIDE;
constexpr unsigned CTR_OFF = FLAG_OFF + FLAG_BYTES;
constexpr unsigned CTR_STRIDE = 64;
constexpr unsigned TMO_OFF = CTR_OFF + NWG * CTR_STRIDE;
constexpr unsigned WS_NEED = TMO_OFF + 128;
constexpr unsigned long long DEADLINE_TICKS = 50000000ull;

#define PUSH_LAG 2
#define AUX_ST 17
#define AUX_LD 0
#define USE_ACQ 0

struct Args {
    const float* x;
    const float* w;
    float* out;
    char* ws[NR];
    int rank;
};

__device__ __forceinline__ unsigned slot_of(int par, int src, int c, int q) {
    return (unsigned)(((par * NR + src) * NCH + c) * NPIECE + q);
}

template <int NV>
__device__ __forceinline__ int block_exp(const float (&v)[NV]) {
    float m = 0.f;
#pragma unroll
    for (int t = 0; t < NV; ++t) m = fmaxf(m, fabsf(v[t]));
#pragma unroll
    for (int o = 32; o > 0; o >>= 1) m = fmaxf(m, __shfl_xor(m, o));
    unsigned E = (__float_as_uint(m) >> 23) & 0xffu;
    E = E < 1u ? 1u : E;
    return __builtin_amdgcn_readfirstlane(141 - (int)E);
}

__device__ __forceinline__ h8v pack8(const float* v, int e) {
    h8v r;
#pragma unroll
    for (int j = 0; j < 8; ++j) r[j] = (_Float16)ldexpf(v[j], e);
    return r;
}

__device__ __forceinline__ void load_w_piece(const float* __restrict__ w, int c, int cg, int i, int h,
                                             float (&v)[32]) {
    const float* base = w + (size_t)(32 * c + 8 * h) * NDIM + 64 * cg + 2 * i;
#pragma unroll
    for (int s = 0; s < 2; ++s)
#pragma unroll
        for (int j = 0; j < 8; ++j) {
            f2v t = *(const f2v*)(base + (size_t)(16 * s + j) * NDIM);
            v[(s * 2 + 0) * 8 + j] = t[0];
            v[(s * 2 + 1) * 8 + j] = t[1];
        }
}

__device__ __forceinline__ void load_x_half(const float* __restrict__ x, int row0, int c, int i, int h,
                                            float* v) {
    const float* base = x + (size_t)(row0 + i) * KS + 32 * c + 8 * h;
#pragma unroll
    for (int s = 0; s < 2; ++s) {
        f4v t0 = *(const f4v*)(base + 16 * s);
        f4v t1 = *(const f4v*)(base + 16 * s + 4);
#pragma unroll
        for (int j = 0; j < 4; ++j) { v[s * 8 + j] = t0[j]; v[s * 8 + 4 + j] = t1[j]; }
    }
}

__device__ __forceinline__ void mma_chunk(h8v a0, h8v a1, h8v b00, h8v b01, h8v b10, h8v b11, int e,
                                          f16v& acc0, f16v& acc1) {
    f16v z = {0.f, 0.f, 0.f, 0.f, 0.f, 0.f, 0.f, 0.f, 0.f, 0.f, 0.f, 0.f, 0.f, 0.f, 0.f, 0.f};
    f16v t0 = __builtin_amdgcn_mfma_f32_32x32x16_f16(a0, b00, z, 0, 0, 0);
    f16v t1 = __builtin_amdgcn_mfma_f32_32x32x16_f16(a0, b01, z, 0, 0, 0);
    t0 = __builtin_amdgcn_mfma_f32_32x32x16_f16(a1, b10, t0, 0, 0, 0);
    t1 = __builtin_amdgcn_mfma_f32_32x32x16_f16(a1, b11, t1, 0, 0, 0);
#pragma unroll
    for (int r = 0; r < 16; ++r) {
        acc0[r] += ldexpf(t0[r], e);
        acc1[r] += ldexpf(t1[r], e);
    }
}

template <int N>
__device__ __forceinline__ void wait_vm() {
    asm volatile("s_waitcnt vmcnt(%0)" ::"i"(N) : "memory");
}

}

__global__ void __launch_bounds__(576) gemm_rs_kernel(Args a) {
    __shared__ __attribute__((aligned(16))) u4v stage[NCH][256];
    __shared__ __attribute__((aligned(16))) float red[NR][8][64][4];
    __shared__ int s_exp[NCH];
    __shared__ unsigned s_call;
    __shared__ unsigned s_bad;

    const int b = blockIdx.x;
    const int wv = __builtin_amdgcn_readfirstlane(threadIdx.x >> 6);
    const int L = threadIdx.x & 63;
    const int i = L & 31, h = L >> 5;
    const int rank = a.rank & (NR - 1);
    const bool has_task = b < NTASK;
    const int q = b % NPIECE;
    const int dest = (rank + 1 + b / NPIECE) & (NR - 1);
    char* const myws = a.ws[rank];

    if (wv == NR) {
        if (L == 0) {
            unsigned* ctr = (unsigned*)(myws + CTR_OFF + b * CTR_STRIDE);
            unsigned v = __hip_atomic_load(ctr, __ATOMIC_RELAXED, __HIP_MEMORY_SCOPE_AGENT) + 1u;
            __hip_atomic_store(ctr, v, __ATOMIC_RELAXED, __HIP_MEMORY_SCOPE_AGENT);
            s_call = v;
            s_bad = 0u;
        }
    } else if (has_task) {
        float v[32];
        if (q < 32) {
            load_w_piece(a.w, wv, q, i, h, v);
        } else {
            const int row0 = MS * dest + 64 * (q - 32);
            load_x_half(a.x, row0, wv, i, h, v);
            load_x_half(a.x, row0 + 32, wv, i, h, v + 16);
        }
        const int e = block_exp<32>(v);
#pragma unroll
        for (int f = 0; f < 4; ++f) stage[wv][f * 64 + L] = __builtin_bit_cast(u4v, pack8(v + 8 * f, e));
        if (L == 0) s_exp[wv] = e;
    }
    __syncthreads();
    const unsigned call = __builtin_amdgcn_readfirstlane(s_call);
    const int par = (int)(call & 1u);

    if (wv == NR) {
        if (!has_task) return;
        __builtin_amdgcn_s_setprio(3);
        char* dws = a.ws[dest];
        const __amdgpu_buffer_rsrc_t rs = __builtin_amdgcn_make_buffer_rsrc(dws, 0, (int)DATA_BYTES, 0x00020000);
        const unsigned s0 = slot_of(par, rank, 0, q);
        auto publish = [&](int c) {
            if (L == 0) {
                unsigned long long* fl =
                    (unsigned long long*)(dws + FLAG_OFF + (s0 + (unsigned)c * NPIECE) * FLAG_STRIDE);
                const unsigned long long val = ((unsigned long long)call << 32) | (unsigned)s_exp[c];
                __hip_atomic_store(fl, val, __ATOMIC_RELAXED, __HIP_MEMORY_SCOPE_SYSTEM);
            }
        };
#pragma unroll
        for (int c = 0; c < NCH; ++c) {
            const unsigned off = (s0 + (unsigned)c * NPIECE) * PIECE + (unsigned)L * 16u;
            u4v t[4];
#pragma unroll
            for (int f = 0; f < 4; ++f) t[f] = stage[c][f * 64 + L];
#pragma unroll
            for (int f = 0; f < 4; ++f) __builtin_amdgcn_raw_buffer_store_b128(t[f], rs, (int)(off + f * 1024u), 0, AUX_ST);
            if (c >= PUSH_LAG - 1 && PUSH_LAG >= 1) {
                wait_vm<4 * (PUSH_LAG - 1)>();
                publish(c - (PUSH_LAG - 1));
            }
        }
#pragma unroll
        for (int c = NCH - (PUSH_LAG - 1); c < NCH; ++c) {
            if (c == NCH - 1) wait_vm<0>();
            else if (c == NCH - 2) wait_vm<4>();
            else if (c == NCH - 3) wait_vm<8>();
            else wait_vm<12>();
            publish(c);
        }
        return;
    }

    const int cg = (b & 7) * 4 + (b >> 6);
    const int rg = (b >> 3) & 7;
    f16v acc0, acc1;
#pragma unroll
    for (int r = 0; r < 16; ++r) { acc0[r] = 0.f; acc1[r] = 0.f; }

    if (wv == rank) {
#pragma unroll 2
        for (int c = 0; c < NCH; ++c) {
            float vw[32], vx[16];
            load_w_piece(a.w, c, cg, i, h, vw);
            load_x_half(a.x, MS * rank + 32 * rg, c, i, h, vx);
            const int ew = block_exp<32>(vw);
            const int ex = block_exp<16>(vx);
            mma_chunk(pack8(vx, ex), pack8(vx + 8, ex), pack8(vw, ew), pack8(vw + 8, ew), pack8(vw + 16, ew),
                      pack8(vw + 24, ew), -(ew + ex), acc0, acc1);
        }
    } else {
        const __amdgpu_buffer_rsrc_t rs = __builtin_amdgcn_make_buffer_rsrc(myws, 0, (int)DATA_BYTES, 0x00020000);
        const unsigned sW = slot_of(par, wv, 0, cg);
        const unsigned sX = slot_of(par, wv, 0, 32 + (rg >> 1));
        unsigned* tmo = (unsigned*)(myws + TMO_OFF);
        const unsigned long long t0 = __builtin_amdgcn_s_memrealtime();
        bool bad = false;
        for (int c = 0; c < NCH; ++c) {
            const unsigned cw = sW + (unsigned)c * NPIECE, cx = sX + (unsigned)c * NPIECE;
            const unsigned long long* pa = (const unsigned long long*)(
                myws + (L == 0 ? FLAG_OFF + cw * FLAG_STRIDE : L == 1 ? FLAG_OFF + cx * FLAG_STRIDE : TMO_OFF));
            int e;
            for (;;) {
                const unsigned long long v = __hip_atomic_load(pa, __ATOMIC_RELAXED, __HIP_MEMORY_SCOPE_SYSTEM);
                const unsigned hi = (unsigned)(v >> 32), lo = (unsigned)v;
                const unsigned hw = __builtin_amdgcn_readlane(hi, 0), hx = __builtin_amdgcn_readlane(hi, 1);
                const unsigned tm = __builtin_amdgcn_readlane(lo, 2);
                e = -((int)__builtin_amdgcn_readlane(lo, 0) + (int)__builtin_amdgcn_readlane(lo, 1));
                if (tm == call) { bad = true; break; }
                if (hw >= call && hx >= call) break;
                if (__builtin_amdgcn_s_memrealtime() - t0 > DEADLINE_TICKS) {
                    if (L == 0) __hip_atomic_store(tmo, call, __ATOMIC_RELAXED, __HIP_MEMORY_SCOPE_AGENT);
                    bad = true;
                    break;
                }
                __builtin_amdgcn_s_sleep(1);
            }
            if (bad) break;
            __builtin_amdgcn_fence(__ATOMIC_ACQUIRE, "wavefront");
            asm volatile("" ::: "memory");
            const int ow = (int)(cw * PIECE + (unsigned)L * 16u);
            const int ox = (int)(cx * PIECE + (unsigned)(rg & 1) * 2048u + (unsigned)L * 16u);
            const u4v a0 = __builtin_amdgcn_raw_buffer_load_b128(rs, ox, 0, AUX_LD);
            const u4v a1 = __builtin_amdgcn_raw_buffer_load_b128(rs, ox + 1024, 0, AUX_LD);
            const u4v b00 = __builtin_amdgcn_raw_buffer_load_b128(rs, ow, 0, AUX_LD);
            const u4v b01 = __builtin_amdgcn_raw_buffer_load_b128(rs, ow + 1024, 0, AUX_LD);
            const u4v b10 = __builtin_amdgcn_raw_buffer_load_b128(rs, ow + 2048, 0, AUX_LD);
            const u4v b11 = __builtin_amdgcn_raw_buffer_load_b128(rs, ow + 3072, 0, AUX_LD);
            mma_chunk(__builtin_bit_cast(h8v, a0), __builtin_bit_cast(h8v, a1), __builtin_bit_cast(h8v, b00),
                      __builtin_bit_cast(h8v, b01), __builtin_bit_cast(h8v, b10), __builtin_bit_cast(h8v, b11), e,
                      acc0, acc1);
        }
        if (bad && L == 0) s_bad = 1u;
    }

#pragma unroll
    for (int g = 0; g < 4; ++g) {
        f4v t0 = {acc0[4 * g], acc0[4 * g + 1], acc0[4 * g + 2], acc0[4 * g + 3]};
        f4v t1 = {acc1[4 * g], acc1[4 * g + 1], acc1[4 * g + 2], acc1[4 * g + 3]};
        *(f4v*)&red[wv][g][L][0] = t0;
        *(f4v*)&red[wv][4 + g][L][0] = t1;
    }
    __syncthreads();
    {
        const int g = wv >> 1, p2 = (wv & 1) * 2;
        f2v s0 = {0.f, 0.f}, s1 = {0.f, 0.f};
#pragma unroll
        for (int p = 0; p < NR; ++p) {
            s0 += *(const f2v*)&red[p][g][L][p2];
            s1 += *(const f2v*)&red[p][4 + g][L][p2];
        }
        if (s_bad) {
            const float qn = __builtin_nanf("");
            s0 = f2v{qn, qn};
            s1 = f2v{qn, qn};
        }
        float* o = a.out + (size_t)(32 * rg + 8 * g + 4 * h + p2) * NDIM + 64 * cg + 2 * i;
        *(f2v*)o = f2v{s0[0], s1[0]};
        *(f2v*)(o + NDIM) = f2v{s0[1], s1[1]};
    }
}

extern "C" void kernel_launch(int rank, int nranks, ncclComm_t comm,
                              hipStream_t stream,
                              void* const* d_in, const int* in_sizes, int n_in,
                              void* d_out, int out_size,
                              const int* in_dtypes, int out_dtype,
                              void* const* d_out_all,
                              void* const* d_ws_all, size_t ws_bytes) {
    if (nranks != NR || n_in < 2 || ws_bytes < (size_t)WS_NEED || out_size != MS * NDIM) {
        fprintf(stderr, "unexpected problem geometry\n");
        return;
    }
    Args a;
    a.x = (const float*)d_in[0];
    a.w = (const float*)d_in[1];
    a.out = (float*)d_out;
    for (int p = 0; p < NR; ++p) a.ws[p] = (char*)d_ws_all[p];
    a.rank = rank;
    gemm_rs_kernel<<<NWG, 576, 0, stream>>>(a);
    HIP_CHECK(hipGetLastError());
}
```

```cpp
#include <hip/hip_runtime.h>
#include <rccl/rccl.h>
#include <cstdio>
#include <cstdint>

#define HIP_CHECK(expr) do { hipError_t _e = (expr); if (_e != hipSuccess) { \
    fprintf(stderr, "HIP error %s at %s:%d\n", hipGetErrorString(_e), __FILE__, __LINE__); \
    return; } } while (0)

typedef _Float16 h8v __attribute__((ext_vector_type(8)));
typedef float f16v __attribute__((ext_vector_type(16)));
typedef float f4v __attribute__((ext_vector_type(4)));
typedef float f2v __attribute__((ext_vector_type(2)));
typedef unsigned u4v __attribute__((ext_vector_type(4)));

namespace {

constexpr int NR = 8;
constexpr int NDIM = 2048;
constexpr int KS = 256;
constexpr int MS = 256;
constexpr int NCH = 8;
constexpr int NPIECE = 36;
constexpr int NWG = 256;
constexpr int NTASK = (NR - 1) * NPIECE;
constexpr unsigned PIECE = 4096;
constexpr unsigned NSLOT = 2u * NR * NCH * NPIECE;
constexpr unsigned DATA_BYTES = NSLOT * PIECE;
constexpr unsigned FLAG_STRIDE = 128;
constexpr unsigned FLAG_OFF = DATA_BYTES;
constexpr unsigned FLAG_BYTES = NSLOT * FLAG_STRIDE;
constexpr unsigned CTR_OFF = FLAG_OFF + FLAG_BYTES;
constexpr unsigned CTR_STRIDE = 64;
constexpr unsigned TMO_OFF = CTR_OFF + NWG * CTR_STRIDE;
constexpr unsigned WS_NEED = TMO_OFF + 128;
constexpr unsigned long long DEADLINE_TICKS = 50000000ull;

#define INFL 2
#define SLEEP_N 1
#define AUX_ST 17
#define AUX_LD 17
#define USE_ACQ 0

#define STAMP(idx) do {} while (0)

struct Args {
    const float* x;
    const float* w;
    float* out;
    char* ws[NR];
    int rank;
};

__device__ __forceinline__ unsigned slot_of(int par, int src, int c, int q) {
    return (unsigned)(((par * NR + src) * NCH + c) * NPIECE + q);
}

template <int NV>
__device__ __forceinline__ int block_exp(const float (&v)[NV]) {
    float m = 0.f;
#pragma unroll
    for (int t = 0; t < NV; ++t) m = fmaxf(m, fabsf(v[t]));
#pragma unroll
    for (int o = 32; o > 0; o >>= 1) m = fmaxf(m, __shfl_xor(m, o));
    unsigned E = (__float_as_uint(m) >> 23) & 0xffu;
    E = E < 1u ? 1u : E;
    return __builtin_amdgcn_readfirstlane(141 - (int)E);
}

__device__ __forceinline__ h8v pack8(const float* v, int e) {
    h8v r;
#pragma unroll
    for (int j = 0; j < 8; ++j) r[j] = (_Float16)ldexpf(v[j], e);
    return r;
}

__device__ __forceinline__ void load_w_piece(const float* __restrict__ w, int c, int cg, int i, int h,
                                             float (&v)[32]) {
    const float* base = w + (size_t)(32 * c + 8 * h) * NDIM + 64 * cg + 2 * i;
#pragma unroll
    for (int s = 0; s < 2; ++s)
#pragma unroll
        for (int j = 0; j < 8; ++j) {
            f2v t = *(const f2v*)(base + (size_t)(16 * s + j) * NDIM);
            v[(s * 2 + 0) * 8 + j] = t[0];
            v[(s * 2 + 1) * 8 + j] = t[1];
        }
}

__device__ __forceinline__ void load_x_half(const float* __restrict__ x, int row0, int c, int i, int h,
                                            float* v) {
    const float* base = x + (size_t)(row0 + i) * KS + 32 * c + 8 * h;
#pragma unroll
    for (int s = 0; s < 2; ++s) {
        f4v t0 = *(const f4v*)(base + 16 * s);
        f4v t1 = *(const f4v*)(base + 16 * s + 4);
#pragma unroll
        for (int j = 0; j < 4; ++j) { v[s * 8 + j] = t0[j]; v[s * 8 + 4 + j] = t1[j]; }
    }
}

__device__ __forceinline__ void mma_chunk(h8v a0, h8v a1, h8v b00, h8v b01, h8v b10, h8v b11, int e,
                                          f16v& acc0, f16v& acc1) {
    f16v z = {0.f, 0.f, 0.f, 0.f, 0.f, 0.f, 0.f, 0.f, 0.f, 0.f, 0.f, 0.f, 0.f, 0.f, 0.f, 0.f};
    f16v t0 = __builtin_amdgcn_mfma_f32_32x32x16_f16(a0, b00, z, 0, 0, 0);
    f16v t1 = __builtin_amdgcn_mfma_f32_32x32x16_f16(a0, b01, z, 0, 0, 0);
    t0 = __builtin_amdgcn_mfma_f32_32x32x16_f16(a1, b10, t0, 0, 0, 0);
    t1 = __builtin_amdgcn_mfma_f32_32x32x16_f16(a1, b11, t1, 0, 0, 0);
#pragma unroll
    for (int r = 0; r < 16; ++r) {
        acc0[r] += ldexpf(t0[r], e);
        acc1[r] += ldexpf(t1[r], e);
    }
}

template <int N>
__device__ __forceinline__ void wait_vm() {
    asm volatile("s_waitcnt vmcnt(%0)" ::"i"(N) : "memory");
}

}

__global__ void __launch_bounds__(576) gemm_rs_kernel(Args a) {
    __shared__ __attribute__((aligned(16))) u4v stage[NCH][256];
    __shared__ __attribute__((aligned(16))) float red[NR][8][64][4];
    __shared__ int s_exp[NCH];
    __shared__ unsigned s_call;
    __shared__ unsigned s_bad;

    const int b = blockIdx.x;
    const int wv = __builtin_amdgcn_readfirstlane(threadIdx.x >> 6);
    const int L = threadIdx.x & 63;
    const int i = L & 31, h = L >> 5;
    const int rank = a.rank & (NR - 1);
    const bool has_task = b < NTASK;
    const int q = b % NPIECE;
    const int dest = (rank + 1 + b / NPIECE) & (NR - 1);
    char* const myws = a.ws[rank];

    if (wv == NR) {
        STAMP(0);
        if (L == 0) {
            unsigned* ctr = (unsigned*)(myws + CTR_OFF + b * CTR_STRIDE);
            unsigned v = __hip_atomic_load(ctr, __ATOMIC_RELAXED, __HIP_MEMORY_SCOPE_AGENT) + 1u;
            __hip_atomic_store(ctr, v, __ATOMIC_RELAXED, __HIP_MEMORY_SCOPE_AGENT);
            s_call = v;
            s_bad = 0u;
        }
    } else if (has_task) {
        float v[32];
        if (q < 32) {
            load_w_piece(a.w, wv, q, i, h, v);
        } else {
            const int row0 = MS * dest + 64 * (q - 32);
            load_x_half(a.x, row0, wv, i, h, v);
            load_x_half(a.x, row0 + 32, wv, i, h, v + 16);
        }
        const int e = block_exp<32>(v);
#pragma unroll
        for (int f = 0; f < 4; ++f) stage[wv][f * 64 + L] = __builtin_bit_cast(u4v, pack8(v + 8 * f, e));
        if (L == 0) s_exp[wv] = e;
    }
    __syncthreads();
    const unsigned call = __builtin_amdgcn_readfirstlane(s_call);
    const int par = (int)(call & 1u);

    if (wv == NR) {
        STAMP(1);
        if (!has_task) return;
        __builtin_amdgcn_s_setprio(3);
        char* dws = a.ws[dest];
        const __amdgpu_buffer_rsrc_t rs = __builtin_amdgcn_make_buffer_rsrc(dws, 0, (int)DATA_BYTES, 0x00020000);
        const unsigned s0 = slot_of(par, rank, 0, q);
        auto publish = [&](int c) {
            if (L == 0) {
                unsigned long long* fl =
                    (unsigned long long*)(dws + FLAG_OFF + (s0 + (unsigned)c * NPIECE) * FLAG_STRIDE);
                const unsigned long long val = ((unsigned long long)call << 32) | (unsigned)s_exp[c];
                __hip_atomic_store(fl, val, __ATOMIC_RELAXED, __HIP_MEMORY_SCOPE_SYSTEM);
            }
            STAMP(2 + c);
        };
#pragma unroll
        for (int c = 0; c < NCH; ++c) {
            const unsigned off = (s0 + (unsigned)c * NPIECE) * PIECE + (unsigned)L * 16u;
            u4v t[4];
#pragma unroll
            for (int f = 0; f < 4; ++f) t[f] = stage[c][f * 64 + L];
#pragma unroll
            for (int f = 0; f < 4; ++f) {
                __builtin_amdgcn_raw_buffer_store_b128(t[f], rs, (int)(off + f * 1024u), 0, AUX_ST);
                wait_vm<INFL - 1>();
                const int done = 4 * c + f - (INFL - 1);
                if (done >= 3 && (done & 3) == 3) publish(done >> 2);
            }
        }
#pragma unroll
        for (int n = 4 * NCH - (INFL - 1); n < 4 * NCH; ++n) {
            if ((n & 3) == 3) {
                if (n == 4 * NCH - 1) wait_vm<0>();
                else wait_vm<4>();
                publish(n >> 2);
            }
        }
        return;
    }

    const int cg = (b & 7) * 4 + (b >> 6);
    const int rg = (b >> 3) & 7;
    f16v acc0, acc1;
#pragma unroll
    for (int r = 0; r < 16; ++r) { acc0[r] = 0.f; acc1[r] = 0.f; }

    if (wv == rank) {
#pragma unroll 2
        for (int c = 0; c < NCH; ++c) {
            float vw[32], vx[16];
            load_w_piece(a.w, c, cg, i, h, vw);
            load_x_half(a.x, MS * rank + 32 * rg, c, i, h, vx);
            const int ew = block_exp<32>(vw);
            const int ex = block_exp<16>(vx);
            mma_chunk(pack8(vx, ex), pack8(vx + 8, ex), pack8(vw, ew), pack8(vw + 8, ew), pack8(vw + 16, ew),
                      pack8(vw + 24, ew), -(ew + ex), acc0, acc1);
        }
    } else {
        const __amdgpu_buffer_rsrc_t rs = __builtin_amdgcn_make_buffer_rsrc(myws, 0, (int)DATA_BYTES, 0x00020000);
        const unsigned sW = slot_of(par, wv, 0, cg);
        const unsigned sX = slot_of(par, wv, 0, 32 + (rg >> 1));
        unsigned* tmo = (unsigned*)(myws + TMO_OFF);
        const unsigned long long t0 = __builtin_amdgcn_s_memrealtime();
        bool bad = false;
        for (int c = 0; c < NCH; ++c) {
            const unsigned cw = sW + (unsigned)c * NPIECE, cx = sX + (unsigned)c * NPIECE;
            const unsigned long long* pa = (const unsigned long long*)(
                myws + (L == 0 ? FLAG_OFF + cw * FLAG_STRIDE : L == 1 ? FLAG_OFF + cx * FLAG_STRIDE : TMO_OFF));
            int e;
            for (;;) {
                const unsigned long long v = __hip_atomic_load(pa, __ATOMIC_RELAXED, __HIP_MEMORY_SCOPE_SYSTEM);
                const unsigned hi = (unsigned)(v >> 32), lo = (unsigned)v;
                const unsigned hw = __builtin_amdgcn_readlane(hi, 0), hx = __builtin_amdgcn_readlane(hi, 1);
                const unsigned tm = __builtin_amdgcn_readlane(lo, 2);
                e = -((int)__builtin_amdgcn_readlane(lo, 0) + (int)__builtin_amdgcn_readlane(lo, 1));
                if (tm == call) { bad = true; break; }
                if (hw >= call && hx >= call) break;
                if (__builtin_amdgcn_s_memrealtime() - t0 > DEADLINE_TICKS) {
                    if (L == 0) __hip_atomic_store(tmo, call, __ATOMIC_RELAXED, __HIP_MEMORY_SCOPE_AGENT);
                    bad = true;
                    break;
                }
                __builtin_amdgcn_s_sleep(SLEEP_N);
            }
            if (bad) break;
            STAMP(16 + wv * 8 + c);
            __builtin_amdgcn_fence(__ATOMIC_ACQUIRE, "wavefront");
            asm volatile("" ::: "memory");
            const int ow = (int)(cw * PIECE + (unsigned)L * 16u);
            const int ox = (int)(cx * PIECE + (unsigned)(rg & 1) * 2048u + (unsigned)L * 16u);
            const u4v a0 = __builtin_amdgcn_raw_buffer_load_b128(rs, ox, 0, AUX_LD);
            const u4v a1 = __builtin_amdgcn_raw_buffer_load_b128(rs, ox + 1024, 0, AUX_LD);
            const u4v b00 = __builtin_amdgcn_raw_buffer_load_b128(rs, ow, 0, AUX_LD);
            const u4v b01 = __builtin_amdgcn_raw_buffer_load_b128(rs, ow + 1024, 0, AUX_LD);
            const u4v b10 = __builtin_amdgcn_raw_buffer_load_b128(rs, ow + 2048, 0, AUX_LD);
            const u4v b11 = __builtin_amdgcn_raw_buffer_load_b128(rs, ow + 3072, 0, AUX_LD);
            mma_chunk(__builtin_bit_cast(h8v, a0), __builtin_bit_cast(h8v, a1), __builtin_bit_cast(h8v, b00),
                      __builtin_bit_cast(h8v, b01), __builtin_bit_cast(h8v, b10), __builtin_bit_cast(h8v, b11), e,
                      acc0, acc1);
        }
        if (bad && L == 0) s_bad = 1u;
    }

    STAMP(80 + wv);
#pragma unroll
    for (int g = 0; g < 4; ++g) {
        f4v t0 = {acc0[4 * g], acc0[4 * g + 1], acc0[4 * g + 2], acc0[4 * g + 3]};
        f4v t1 = {acc1[4 * g], acc1[4 * g + 1], acc1[4 * g + 2], acc1[4 * g + 3]};
        *(f4v*)&red[wv][g][L][0] = t0;
        *(f4v*)&red[wv][4 + g][L][0] = t1;
    }
    __syncthreads();
    {
        const int g = wv >> 1, p2 = (wv & 1) * 2;
        f2v s0 = {0.f, 0.f}, s1 = {0.f, 0.f};
#pragma unroll
        for (int p = 0; p < NR; ++p) {
            s0 += *(const f2v*)&red[p][g][L][p2];
            s1 += *(const f2v*)&red[p][4 + g][L][p2];
        }
        if (s_bad) {
            const float qn = __builtin_nanf("");
            s0 = f2v{qn, qn};
            s1 = f2v{qn, qn};
        }
        float* o = a.out + (size_t)(32 * rg + 8 * g + 4 * h + p2) * NDIM + 64 * cg + 2 * i;
        *(f2v*)o = f2v{s0[0], s1[0]};
        *(f2v*)(o + NDIM) = f2v{s0[1], s1[1]};
    }
    STAMP(88 + wv);
}

extern "C" void kernel_launch(int rank, int nranks, ncclComm_t comm,
                              hipStream_t stream,
                              void* const* d_in, const int* in_sizes, int n_in,
                              void* d_out, int out_size,
                              const int* in_dtypes, int out_dtype,
                              void* const* d_out_all,
                              void* const* d_ws_all, size_t ws_bytes) {
    if (nranks != NR || n_in < 2 || ws_bytes < (size_t)WS_NEED || out_size != MS * NDIM) {
        fprintf(stderr, "unexpected problem geometry\n");
        return;
    }
    Args a;
    a.x = (const float*)d_in[0];
    a.w = (const float*)d_in[1];
    a.out = (float*)d_out;
    for (int p = 0; p < NR; ++p) a.ws[p] = (char*)d_ws_all[p];
    a.rank = rank;
    gemm_rs_kernel<<<NWG, 576, 0, stream>>>(a);
    HIP_CHECK(hipGetLastError());
}
```

```cpp
#include <hip/hip_runtime.h>
#include <rccl/rccl.h>
#include <cstdio>
#include <cstdint>

#define HIP_CHECK(expr) do { hipError_t _e = (expr); if (_e != hipSuccess) { \
    fprintf(stderr, "HIP error %s at %s:%d\n", hipGetErrorString(_e), __FILE__, __LINE__); \
    return; } } while (0)

typedef _Float16 h8v __attribute__((ext_vector_type(8)));
typedef float f16v __attribute__((ext_vector_type(16)));
typedef float f4v __attribute__((ext_vector_type(4)));
typedef float f2v __attribute__((ext_vector_type(2)));
typedef unsigned u4v __attribute__((ext_vector_type(4)));
typedef unsigned u2v __attribute__((ext_vector_type(2)));

namespace {

constexpr int NR = 8;
constexpr int NDIM = 2048;
constexpr int KS = 256;
constexpr int MS = 256;
constexpr int NCH = 8;
constexpr int NPIECE = 36;
constexpr int NWG = 256;
constexpr int NST = 4 * NCH;
constexpr unsigned Q_STR = 1024, F_STR = 37 * 1024, C_STR = 4 * 37 * 1024;
constexpr unsigned S_STR = NCH * C_STR;
constexpr unsigned DATA_BYTES = 2u * NR * S_STR;
constexpr unsigned FLAG_STRIDE = 128;
constexpr unsigned FLAG_OFF = DATA_BYTES;
constexpr unsigned FLAG_BYTES = 2u * NR * NPIECE * FLAG_STRIDE;
constexpr unsigned CTR_OFF = FLAG_OFF + FLAG_BYTES;
constexpr unsigned CTR_STRIDE = 64;
constexpr unsigned TMO_OFF = CTR_OFF + NWG * CTR_STRIDE;
constexpr unsigned PRG_OFF = TMO_OFF + 128;
constexpr unsigned WS_NEED = PRG_OFF + NR * 128;
#define DLT 50000000ull
constexpr unsigned long long DEADLINE_TICKS = DLT;

#define INFL 2
#define INFLX 4
#define SLEEP_N 1
#define FB_T1 50
#define FB_T2 110
#define FB_T3 200
#define FB_TL 60
#define FB_END 99
#define FB_WE 6
#define FASTLAST 1
#define SLEEP_E 16
#define LATE_C 6
#define TMO_EVERY 16
#define FB_SLEEP 4
#define AUX_ST 17
#define AUX_LD 17
#define AUX_POLL 17

#define LDS_LD(v) __builtin_amdgcn_readfirstlane(__hip_atomic_load(&(v), __ATOMIC_RELAXED, __HIP_MEMORY_SCOPE_WORKGROUP))
#define LDS_ST(v, x) __hip_atomic_store(&(v), (x), __ATOMIC_RELAXED, __HIP_MEMORY_SCOPE_WORKGROUP)

struct Args {
    const float* x;
    const float* w;
    float* out;
    char* ws[NR];
    int rank;
};

__device__ __forceinline__ unsigned data_of(int par, int src, int q) {
    return (unsigned)(par * NR + src) * S_STR + (unsigned)q * Q_STR;
}
__device__ __forceinline__ unsigned flag_of(int par, int src, int q) {
    return FLAG_OFF + (unsigned)((par * NR + src) * NPIECE + q) * FLAG_STRIDE;
}

template <int NV>
__device__ __forceinline__ int block_exp(const float (&v)[NV]) {
    float m = 0.f;
#pragma unroll
    for (int t = 0; t < NV; ++t) m = fmaxf(m, fabsf(v[t]));
#pragma unroll
    for (int o = 32; o > 0; o >>= 1) m = fmaxf(m, __shfl_xor(m, o));
    unsigned E = (__float_as_uint(m) >> 23) & 0xffu;
    E = E < 1u ? 1u : E;
    return __builtin_amdgcn_readfirstlane(141 - (int)E);
}

__device__ __forceinline__ h8v pack8(const float* v, int e) {
    h8v r;
#pragma unroll
    for (int j = 0; j < 8; ++j) r[j] = (_Float16)ldexpf(v[j], e);
    return r;
}

__device__ __forceinline__ void load_w_piece(const float* __restrict__ w, int c, int cg, int i, int h,
                                             float (&v)[32]) {
    const float* base = w + (size_t)(32 * c + 8 * h) * NDIM + 64 * cg + 2 * i;
#pragma unroll
    for (int s = 0; s < 2; ++s)
#pragma unroll
        for (int j = 0; j < 8; ++j) {
            f2v t = *(const f2v*)(base + (size_t)(16 * s + j) * NDIM);
            v[(s * 2 + 0) * 8 + j] = t[0];
            v[(s * 2 + 1) * 8 + j] = t[1];
        }
}

__device__ __forceinline__ void load_x_half(const float* __restrict__ x, int row0, int c, int i, int h,
                                            float* v) {
    const float* base = x + (size_t)(row0 + i) * KS + 32 * c + 8 * h;
#pragma unroll
    for (int s = 0; s < 2; ++s) {
        f4v t0 = *(const f4v*)(base + 16 * s);
        f4v t1 = *(const f4v*)(base + 16 * s + 4);
#pragma unroll
        for (int j = 0; j < 4; ++j) { v[s * 8 + j] = t0[j]; v[s * 8 + 4 + j] = t1[j]; }
    }
}

__device__ __forceinline__ void mma_chunk(h8v a0, h8v a1, h8v b00, h8v b01, h8v b10, h8v b11, int e,
                                          f16v& acc0, f16v& acc1) {
    f16v z = {0.f, 0.f, 0.f, 0.f, 0.f, 0.f, 0.f, 0.f, 0.f, 0.f, 0.f, 0.f, 0.f, 0.f, 0.f, 0.f};
    f16v t0 = __builtin_amdgcn_mfma_f32_32x32x16_f16(a0, b00, z, 0, 0, 0);
    f16v t1 = __builtin_amdgcn_mfma_f32_32x32x16_f16(a0, b01, z, 0, 0, 0);
    t0 = __builtin_amdgcn_mfma_f32_32x32x16_f16(a1, b10, t0, 0, 0, 0);
    t1 = __builtin_amdgcn_mfma_f32_32x32x16_f16(a1, b11, t1, 0, 0, 0);
    if (e >= -126 && e <= 127) {
        const float sc = __uint_as_float((unsigned)(e + 127) << 23);
        const f2v s2 = {sc, sc};
#pragma unroll
        for (int r = 0; r < 16; r += 2) {
            f2v u0 = __builtin_elementwise_fma(f2v{t0[r], t0[r + 1]}, s2, f2v{acc0[r], acc0[r + 1]});
            f2v u1 = __builtin_elementwise_fma(f2v{t1[r], t1[r + 1]}, s2, f2v{acc1[r], acc1[r + 1]});
            acc0[r] = u0[0]; acc0[r + 1] = u0[1];
            acc1[r] = u1[0]; acc1[r + 1] = u1[1];
        }
        return;
    }
#pragma unroll
    for (int r = 0; r < 16; ++r) {
        acc0[r] += ldexpf(t0[r], e);
        acc1[r] += ldexpf(t1[r], e);
    }
}

struct Frag { u4v a0, a1, b00, b01, b10, b11; };

template <int AUX = AUX_LD>
__device__ __forceinline__ Frag load_frag(__amdgpu_buffer_rsrc_t rs, int ow, int ox) {
    Frag f;
    f.a0 = __builtin_amdgcn_raw_buffer_load_b128(rs, ox, 0, AUX);
    f.a1 = __builtin_amdgcn_raw_buffer_load_b128(rs, ox + (int)F_STR, 0, AUX);
    f.b00 = __builtin_amdgcn_raw_buffer_load_b128(rs, ow, 0, AUX);
    f.b01 = __builtin_amdgcn_raw_buffer_load_b128(rs, ow + (int)F_STR, 0, AUX);
    f.b10 = __builtin_amdgcn_raw_buffer_load_b128(rs, ow + 2 * (int)F_STR, 0, AUX);
    f.b11 = __builtin_amdgcn_raw_buffer_load_b128(rs, ow + 3 * (int)F_STR, 0, AUX);
    return f;
}

__device__ __forceinline__ void mma_frag(const Frag& f, int e, f16v& acc0, f16v& acc1) {
    mma_chunk(__builtin_bit_cast(h8v, f.a0), __builtin_bit_cast(h8v, f.a1), __builtin_bit_cast(h8v, f.b00),
              __builtin_bit_cast(h8v, f.b01), __builtin_bit_cast(h8v, f.b10), __builtin_bit_cast(h8v, f.b11), e,
              acc0, acc1);
}

__device__ __forceinline__ u4v no_zero(u4v u) {
#pragma unroll
    for (int j = 0; j < 4; ++j)
        u[j] |= ((u[j] & 0xffffu) == 0u ? 0x8000u : 0u) | ((u[j] >> 16) == 0u ? 0x80000000u : 0u);
    return u;
}

__device__ __forceinline__ bool frag_valid(const Frag& f) {
    typedef unsigned short us8 __attribute__((ext_vector_type(8)));
    us8 m = __builtin_elementwise_min(__builtin_bit_cast(us8, f.a0), __builtin_bit_cast(us8, f.a1));
    m = __builtin_elementwise_min(m, __builtin_bit_cast(us8, f.b00));
    m = __builtin_elementwise_min(m, __builtin_bit_cast(us8, f.b01));
    m = __builtin_elementwise_min(m, __builtin_bit_cast(us8, f.b10));
    m = __builtin_elementwise_min(m, __builtin_bit_cast(us8, f.b11));
    typedef unsigned short us2 __attribute__((ext_vector_type(2)));
    us2 r = __builtin_elementwise_min(__builtin_elementwise_min(us2{m[0], m[1]}, us2{m[2], m[3]}),
                                      __builtin_elementwise_min(us2{m[4], m[5]}, us2{m[6], m[7]}));
    return __builtin_amdgcn_ballot_w64(r[0] == 0 || r[1] == 0) == 0ull;
}

template <int N>
__device__ __forceinline__ void wait_vm() {
    asm volatile("s_waitcnt vmcnt(%0)" ::"i"(N) : "memory");
}

__device__ __forceinline__ void wait_vm_n(int n) {
    switch (n) {
        case 0: wait_vm<0>(); break;
        case 1: wait_vm<1>(); break;
        case 2: wait_vm<2>(); break;
        case 3: wait_vm<3>(); break;
        case 4: wait_vm<4>(); break;
        case 5: wait_vm<5>(); break;
        case 6: wait_vm<6>(); break;
        case 7: wait_vm<7>(); break;
        default: wait_vm<8>(); break;
    }
}

__device__ __forceinline__ void raw_barrier() {
    asm volatile("s_waitcnt lgkmcnt(0)\n\ts_barrier" ::: "memory");
}

template <int SLP, class T>
__device__ __forceinline__ unsigned lds_wait(T& v) {
    unsigned r, it = 0u;
    unsigned long long t0 = 0ull;
    while ((r = (unsigned)LDS_LD(v)) == 0u) {
        __builtin_amdgcn_s_sleep(SLP);
        if ((++it & 1023u) == 0u) {
            const unsigned long long t = __builtin_amdgcn_s_memrealtime();
            if (t0 == 0ull) t0 = t;
            else if (t - t0 > DEADLINE_TICKS) break;
        }
    }
    asm volatile("" ::: "memory");
    return r;
}

}

__global__ void __launch_bounds__(640) gemm_rs_kernel(Args a) {
    __shared__ __attribute__((aligned(16))) u4v stage[NCH][256];
    __shared__ __attribute__((aligned(16))) float red[NR][8][64][4];
    __shared__ int s_exp[NCH];
    __shared__ int s_rdy[NCH];
    __shared__ unsigned s_call;
    __shared__ unsigned s_bad;
    __shared__ int s_prog;
    __shared__ int s_win;
    __shared__ int s_zok;

    const int b = blockIdx.x;
    const int wv = __builtin_amdgcn_readfirstlane(threadIdx.x >> 6);
    const int L = threadIdx.x & 63;
    const int i = L & 31, h = L >> 5;
    const int rank = a.rank & (NR - 1);
    const int xcd = b & 7, idx = b >> 3;
    const int cg = 4 * xcd + (idx & 3);
    const int rg = idx >> 2;
    const bool is_w = idx < 28;
    const int tx = 4 * xcd + (idx - 28);
    const bool has_task = is_w || tx < 28;
    const int q = is_w ? cg : 32 + (tx & 3);
    const int dest = (rank + 1 + (is_w ? rg : (tx >> 2))) & (NR - 1);
    char* const myws = a.ws[rank];

    if (wv == NR) {
        unsigned cv = 0u;
        unsigned* ctr = (unsigned*)(myws + CTR_OFF + b * CTR_STRIDE);
        if (L == 0) cv = __hip_atomic_load(ctr, __ATOMIC_RELAXED, __HIP_MEMORY_SCOPE_AGENT);
        if (L < NCH) s_rdy[L] = 0;
        if (L == 0) {
            s_call = 0u;
            s_bad = 0u;
            s_prog = 0;
            s_zok = 0;
            s_win = is_w ? INFL : INFLX;
        }
        raw_barrier();
        if (L == 0) {
            cv += 1u;
            __hip_atomic_store(ctr, cv, __ATOMIC_RELAXED, __HIP_MEMORY_SCOPE_AGENT);
            LDS_ST(s_call, cv);
        }
        if (!has_task) return;
        const unsigned call = __builtin_amdgcn_readfirstlane(cv);
        const int par = (int)(call & 1u);
        __builtin_amdgcn_s_setprio(3);
        char* dws = a.ws[dest];
        const __amdgpu_buffer_rsrc_t rs = __builtin_amdgcn_make_buffer_rsrc(dws, 0, (int)DATA_BYTES, 0x00020000);
        const unsigned s0 = data_of(par, rank, q) + (unsigned)L * 16u;
        unsigned long long* const fl = (unsigned long long*)(dws + flag_of(par, rank, q));
        auto publish = [&](int c) {
            if (L == 0) {
                const unsigned long long val = ((unsigned long long)call << 32) | (unsigned)s_exp[c];
                __hip_atomic_store(fl + c, val, __ATOMIC_RELAXED, __HIP_MEMORY_SCOPE_SYSTEM);
            }
        };
        int issued = 0, pub = 0;
        int P[NCH];
#pragma unroll
        for (int c = 0; c < NCH; ++c) {
            if (!lds_wait<0>(s_rdy[c])) {
                if (L == 0) { LDS_ST(s_bad, 1u); LDS_ST(s_prog, NST); }
                return;
            }
            const unsigned off = s0 + (unsigned)c * C_STR;
            u4v t[4];
#pragma unroll
            for (int f = 0; f < 4; ++f) t[f] = stage[c][f * 64 + L];
#pragma unroll
            for (int f = 0; f < 4; ++f) {
                const int win = LDS_LD(s_win);
                __builtin_amdgcn_raw_buffer_store_b128(t[f], rs, (int)(off + f * F_STR), 0, AUX_ST);
                ++issued;
                if (f == 3) P[c] = issued;
                if (4 * c + f + 1 == NST && L == 0) LDS_ST(s_prog, NST);
                if (FASTLAST && 4 * c + f + 1 == NST) break;
                if (win <= 2) { if (win == 2) wait_vm<1>(); else wait_vm<0>(); }
                else if (win == 3) wait_vm<2>();
                else if (win == 4) wait_vm<3>();
                else if (win == 6) wait_vm<5>();
                else wait_vm<7>();
                const int done = issued - (win - 1);
                if (L == 0 && 4 * c + f + 1 < NST) LDS_ST(s_prog, 4 * c + f + 2 - win);
#pragma unroll
                for (int cc = (c >= 2 ? c - 2 : 0); cc <= c; ++cc)
                    if ((cc < c || f == 3) && pub == cc && P[cc] <= done) { publish(cc); ++issued; ++pub; }
            }
        }
#pragma unroll
        for (int cc = NCH - 3; cc < NCH; ++cc)
            if (pub == cc) {
                if (!(FASTLAST && cc == NCH - 1)) wait_vm_n(issued - P[cc]);
                else if (!lds_wait<1>(s_zok)) {
                    if (L == 0) LDS_ST(s_bad, 1u);
                    return;
                }
                publish(cc); ++issued; ++pub;
            }
        return;
    }

    if (wv == NR + 1) {
        raw_barrier();
        if (!has_task) return;
        const unsigned call = lds_wait<1>(s_call);
        if (call == 0u) return;
        {
            const __amdgpu_buffer_rsrc_t rz = __builtin_amdgcn_make_buffer_rsrc(myws, 0, (int)DATA_BYTES, 0x00020000);
            const unsigned zo = data_of((int)(call & 1u) ^ 1, dest, q) + (unsigned)(NCH - 1) * C_STR + (unsigned)L * 16u;
#pragma unroll
            for (int f = 0; f < 4; ++f)
                __builtin_amdgcn_raw_buffer_store_b128(u4v{0u, 0u, 0u, 0u}, rz, (int)(zo + f * F_STR), 0, AUX_ST);
            wait_vm<0>();
            if (L == 0) LDS_ST(s_zok, 1);
        }
        unsigned* pc = (unsigned*)(myws + PRG_OFF + (unsigned)dest * 128u);
        const unsigned base = (call - 1u) * (unsigned)(NPIECE * NST);
        int last = 0;
        unsigned it = 0u;
        unsigned long long t0 = 0ull;
        for (;;) {
            int p = LDS_LD(s_prog);
            if ((++it & 255u) == 0u) {
                const unsigned long long t = __builtin_amdgcn_s_memrealtime();
                if (t0 == 0ull) t0 = t;
                else if (t - t0 > DEADLINE_TICKS) p = NST;
            }
            const int dl = p > last ? p - last : 0;
            last += dl;
            if (last >= NST) {
                if (L == 0) __hip_atomic_fetch_add(pc, (unsigned)dl, __ATOMIC_RELAXED, __HIP_MEMORY_SCOPE_AGENT);
                break;
            }
            unsigned g = 0u;
            if (L == 0) g = __hip_atomic_fetch_add(pc, (unsigned)dl, __ATOMIC_RELAXED, __HIP_MEMORY_SCOPE_AGENT) + (unsigned)dl;
            const int lag = (int)(__builtin_amdgcn_readfirstlane(g) - base) - NPIECE * last;
            int win = lag > FB_T3 ? 6 : lag > FB_T2 ? 4 : lag > FB_T1 ? 3 : lag < -FB_TL ? 1 : 2;
            if (lag + NPIECE * last >= FB_END * NPIECE && win < FB_WE) win = FB_WE;
            if (L == 0 && last > 0) LDS_ST(s_win, win);
            __builtin_amdgcn_s_sleep(FB_SLEEP);
        }
        return;
    }

    f16v acc0, acc1;
#pragma unroll
    for (int r = 0; r < 16; ++r) { acc0[r] = 0.f; acc1[r] = 0.f; }
    {
        float vw[32], vx[16];
        h8v bf[4];
        int ew;
        if (!is_w) {
            if (has_task) {
                float vt[32];
                const int row0 = MS * dest + 64 * (q - 32);
                load_x_half(a.x, row0, wv, i, h, vt);
                load_x_half(a.x, row0 + 32, wv, i, h, vt + 16);
                raw_barrier();
                load_w_piece(a.w, wv, cg, i, h, vw);
                load_x_half(a.x, MS * rank + 32 * rg, wv, i, h, vx);
                const int e = block_exp<32>(vt);
#pragma unroll
                for (int f = 0; f < 4; ++f) {
                    u4v u = __builtin_bit_cast(u4v, pack8(vt + 8 * f, e));
                    if (FASTLAST && wv == NCH - 1) u = no_zero(u);
                    stage[wv][f * 64 + L] = u;
                }
                if (L == 0) s_exp[wv] = e;
                asm volatile("s_waitcnt lgkmcnt(0)" ::: "memory");
                if (L == 0) LDS_ST(s_rdy[wv], 1);
            } else {
                raw_barrier();
                load_w_piece(a.w, wv, cg, i, h, vw);
                load_x_half(a.x, MS * rank + 32 * rg, wv, i, h, vx);
            }
            ew = block_exp<32>(vw);
#pragma unroll
            for (int f = 0; f < 4; ++f) bf[f] = pack8(vw + 8 * f, ew);
        } else {
            load_w_piece(a.w, wv, cg, i, h, vw);
            raw_barrier();
            load_x_half(a.x, MS * rank + 32 * rg, wv, i, h, vx);
            ew = block_exp<32>(vw);
#pragma unroll
            for (int f = 0; f < 4; ++f) bf[f] = pack8(vw + 8 * f, ew);
#pragma unroll
            for (int f = 0; f < 4; ++f) {
                u4v u = __builtin_bit_cast(u4v, bf[f]);
                if (FASTLAST && wv == NCH - 1) u = no_zero(u);
                stage[wv][f * 64 + L] = u;
            }
            if (L == 0) s_exp[wv] = ew;
            asm volatile("s_waitcnt lgkmcnt(0)" ::: "memory");
            if (L == 0) LDS_ST(s_rdy[wv], 1);
        }
        const int ex = block_exp<16>(vx);
        mma_chunk(pack8(vx, ex), pack8(vx + 8, ex), bf[0], bf[1], bf[2], bf[3], -(ew + ex), acc0, acc1);
    }

    if (wv < NR - 1) {
        const unsigned call = lds_wait<1>(s_call);
        const int par = (int)(call & 1u);
        const int src = (rank + 1 + wv) & (NR - 1);
        const __amdgpu_buffer_rsrc_t rs = __builtin_amdgcn_make_buffer_rsrc(myws, 0, (int)WS_NEED, 0x00020000);
        const int owb = (int)(data_of(par, src, cg) + (unsigned)L * 16u);
        const int oxb = (int)(data_of(par, src, 32 + (rg >> 1)) + (unsigned)(rg & 1) * 2u * F_STR + (unsigned)L * 16u);
        const int poff1 = (int)(L < 8 ? flag_of(par, src, cg) + 8u * L
                               : L < 16 ? flag_of(par, src, 32 + (rg >> 1)) + 8u * (L - 8) : TMO_OFF);
        const int poff0 = TMO_EVERY == 1 ? poff1 : (int)(L < 16 ? (unsigned)poff1 : flag_of(par, src, cg) + 64u);
        int poff = poff0;
        unsigned npoll = 0u;
        const unsigned long long t0 = __builtin_amdgcn_s_memrealtime();
        bool bad = call == 0u, retry = false;
        int next = 0;
        u2v pv = __builtin_amdgcn_raw_buffer_load_b64(rs, poff, 0, AUX_POLL);
        while (!bad) {
            const unsigned lo = pv[0], hi = pv[1];
            const unsigned m = (unsigned)__builtin_amdgcn_ballot_w64(hi >= call);
            const unsigned rdy = (m & (m >> 8) & 0xffu) >> next;
            if (__builtin_amdgcn_readlane(lo, 16) == call) { bad = true; break; }
            if (!(rdy & 1u)) {
                if (__builtin_amdgcn_s_memrealtime() - t0 > DEADLINE_TICKS) {
                    if (L == 0)
                        __hip_atomic_store((unsigned*)(myws + TMO_OFF), call, __ATOMIC_RELAXED, __HIP_MEMORY_SCOPE_AGENT);
                    bad = true;
                    break;
                }
                if (next < LATE_C) __builtin_amdgcn_s_sleep(SLEEP_E);
                else __builtin_amdgcn_s_sleep(SLEEP_N);
                asm volatile("" ::: "memory");
                ++npoll;
                pv = __builtin_amdgcn_raw_buffer_load_b64(rs, (npoll % TMO_EVERY) == 0u ? poff1 : poff0, 0, AUX_POLL);
                continue;
            }
            asm volatile("" ::: "memory");
            const int co = next * (int)C_STR;
            const int e0 = -((int)__builtin_amdgcn_readlane(lo, next) + (int)__builtin_amdgcn_readlane(lo, next + 8));
            if ((rdy & 3u) == 3u) {
                const int e1 = -((int)__builtin_amdgcn_readlane(lo, next + 1) + (int)__builtin_amdgcn_readlane(lo, next + 9));
                const Frag f0 = load_frag(rs, owb + co, oxb + co);
                const Frag f1 = load_frag(rs, owb + co + (int)C_STR, oxb + co + (int)C_STR);
                if (next + 2 < NCH) pv = __builtin_amdgcn_raw_buffer_load_b64(rs, poff, 0, AUX_POLL);
                __builtin_amdgcn_sched_barrier(0);
                mma_frag(f0, e0, acc0, acc1);
                if (!FASTLAST || next + 2 < NCH || frag_valid(f1)) {
                    mma_frag(f1, e1, acc0, acc1);
                    next += 2;
                } else {
                    next += 1;
                    retry = true;
                }
            } else {
                const Frag f0 = retry ? load_frag<17>(rs, owb + co, oxb + co) : load_frag(rs, owb + co, oxb + co);
                if (next + 1 < NCH) pv = __builtin_amdgcn_raw_buffer_load_b64(rs, poff, 0, AUX_POLL);
                __builtin_amdgcn_sched_barrier(0);
                if (!FASTLAST || next + 1 < NCH || frag_valid(f0)) {
                    mma_frag(f0, e0, acc0, acc1);
                    next += 1;
                } else {
                    retry = true;
                    if (__builtin_amdgcn_s_memrealtime() - t0 > DEADLINE_TICKS) {
                        if (L == 0)
                            __hip_atomic_store((unsigned*)(myws + TMO_OFF), call, __ATOMIC_RELAXED, __HIP_MEMORY_SCOPE_AGENT);
                        bad = true;
                        break;
                    }
                    asm volatile("" ::: "memory");
                }
            }
            if (next >= NCH) break;
        }
        if (bad && L == 0) LDS_ST(s_bad, 1u);
    }

#pragma unroll
    for (int g = 0; g < 4; ++g) {
        f4v t0 = {acc0[4 * g], acc0[4 * g + 1], acc0[4 * g + 2], acc0[4 * g + 3]};
        f4v t1 = {acc1[4 * g], acc1[4 * g + 1], acc1[4 * g + 2], acc1[4 * g + 3]};
        *(f4v*)&red[wv][g][L][0] = t0;
        *(f4v*)&red[wv][4 + g][L][0] = t1;
    }
    __syncthreads();
    {
        const int g = wv >> 1, p2 = (wv & 1) * 2;
        f2v s0 = {0.f, 0.f}, s1 = {0.f, 0.f};
#pragma unroll
        for (int p = 0; p < NR; ++p) {
            s0 += *(const f2v*)&red[p][g][L][p2];
            s1 += *(const f2v*)&red[p][4 + g][L][p2];
        }
        if (s_bad) {
            const float qn = __builtin_nanf("");
            s0 = f2v{qn, qn};
            s1 = f2v{qn, qn};
        }
        float* o = a.out + (size_t)(32 * rg + 8 * g + 4 * h + p2) * NDIM + 64 * cg + 2 * i;
        const __amdgpu_buffer_rsrc_t ro = __builtin_amdgcn_make_buffer_rsrc(a.out, 0, MS * NDIM * 4, 0x00020000);
        const int oo = (int)((char*)o - (char*)a.out);
        __builtin_amdgcn_raw_buffer_store_b64(__builtin_bit_cast(u2v, f2v{s0[0], s1[0]}), ro, oo, 0, 17);
        __builtin_amdgcn_raw_buffer_store_b64(__builtin_bit_cast(u2v, f2v{s0[1], s1[1]}), ro, oo + NDIM * 4, 0, 17);
    }
}

extern "C" void kernel_launch(int rank, int nranks, ncclComm_t comm,
                              hipStream_t stream,
                              void* const* d_in, const int* in_sizes, int n_in,
                              void* d_out, int out_size,
                              const int* in_dtypes, int out_dtype,
                              void* const* d_out_all,
                              void* const* d_ws_all, size_t ws_bytes) {
    if (nranks != NR || n_in < 2 || ws_bytes < (size_t)WS_NEED || out_size != MS * NDIM) {
        fprintf(stderr, "unexpected problem geometry\n");
        return;
    }
    Args a;
    a.x = (const float*)d_in[0];
    a.w = (const float*)d_in[1];
    a.out = (float*)d_out;
    for (int p = 0; p < NR; ++p) a.ws[p] = (char*)d_ws_all[p];
    a.rank = rank;
    gemm_rs_kernel<<<NWG, 640, 0, stream>>>(a);
    HIP_CHECK(hipGetLastError());
}
```

```cpp
#include <hip/hip_runtime.h>
#include <rccl/rccl.h>
#include <cstdio>
#include <cstdint>

#define HIP_CHECK(expr) do { hipError_t _e = (expr); if (_e != hipSuccess) { \
    fprintf(stderr, "HIP error %s at %s:%d\n", hipGetErrorString(_e), __FILE__, __LINE__); \
    return; } } while (0)

typedef _Float16 h8v __attribute__((ext_vector_type(8)));
typedef float f16v __attribute__((ext_vector_type(16)));
typedef float f4v __attribute__((ext_vector_type(4)));
typedef float f2v __attribute__((ext_vector_type(2)));
typedef unsigned u4v __attribute__((ext_vector_type(4)));
typedef unsigned u2v __attribute__((ext_vector_type(2)));

namespace {

constexpr int NR = 8;
constexpr int NDIM = 2048;
constexpr int KS = 256;
constexpr int MS = 256;
constexpr int NCH = 8;
constexpr int NPIECE = 36;
constexpr int NWG = 256;
constexpr int NST = 4 * NCH;
constexpr unsigned Q_STR = 1024, F_STR = 37 * 1024, C_STR = 4 * 37 * 1024;
constexpr unsigned S_STR = NCH * C_STR;
constexpr unsigned DATA_BYTES = 2u * NR * S_STR;
constexpr unsigned FLAG_STRIDE = 128;
constexpr unsigned FLAG_OFF = DATA_BYTES;
constexpr unsigned FLAG_BYTES = 2u * NR * NPIECE * FLAG_STRIDE;
constexpr unsigned CTR_OFF = FLAG_OFF + FLAG_BYTES;
constexpr unsigned CTR_STRIDE = 64;
constexpr unsigned TMO_OFF = CTR_OFF + NWG * CTR_STRIDE;
constexpr unsigned PRG_OFF = TMO_OFF + 128;
constexpr unsigned WS_NEED = PRG_OFF + NR * 128;
#define DLT 50000000ull
constexpr unsigned long long DEADLINE_TICKS = DLT;

#define INFL 2
#define INFLX 4
#define SLEEP_N 1
#define FB_T1 50
#define FB_T2 110
#define FB_T3 200
#define FB_TL 60
#define FB_END 26
#define FB_WE 4
#define FASTLAST 1
#define SLEEP_E 16
#define LATE_C 6
#define TMO_EVERY 16
#define FB_SLEEP 4
#define STAG_S 1
#define AUX_ST 17
#define AUX_LD 17
#define AUX_POLL 17

#define LDS_LD(v) __builtin_amdgcn_readfirstlane(__hip_atomic_load(&(v), __ATOMIC_RELAXED, __HIP_MEMORY_SCOPE_WORKGROUP))
#define LDS_ST(v, x) __hip_atomic_store(&(v), (x), __ATOMIC_RELAXED, __HIP_MEMORY_SCOPE_WORKGROUP)

struct Args {
    const float* x;
    const float* w;
    float* out;
    char* ws[NR];
    int rank;
};

__device__ __forceinline__ unsigned data_of(int par, int src, int q) {
    return (unsigned)(par * NR + src) * S_STR + (unsigned)q * Q_STR;
}
__device__ __forceinline__ unsigned flag_of(int par, int src, int q) {
    return FLAG_OFF + (unsigned)((par * NR + src) * NPIECE + q) * FLAG_STRIDE;
}

template <int NV>
__device__ __forceinline__ int block_exp(const float (&v)[NV]) {
    float m = 0.f;
#pragma unroll
    for (int t = 0; t < NV; ++t) m = fmaxf(m, fabsf(v[t]));
    unsigned u = __float_as_uint(m);
    u = max(u, (unsigned)__builtin_amdgcn_update_dpp(0, (int)u, 0xB1, 0xf, 0xf, true));
    u = max(u, (unsigned)__builtin_amdgcn_update_dpp(0, (int)u, 0x4E, 0xf, 0xf, true));
    u = max(u, (unsigned)__builtin_amdgcn_update_dpp(0, (int)u, 0x141, 0xf, 0xf, true));
    u = max(u, (unsigned)__builtin_amdgcn_update_dpp(0, (int)u, 0x140, 0xf, 0xf, true));
    const unsigned r = max(max((unsigned)__builtin_amdgcn_readlane((int)u, 0), (unsigned)__builtin_amdgcn_readlane((int)u, 16)),
                           max((unsigned)__builtin_amdgcn_readlane((int)u, 32), (unsigned)__builtin_amdgcn_readlane((int)u, 48)));
    unsigned E = r >> 23;
    E = E < 14u ? 14u : E;
    return 141 - (int)E;
}

__device__ __forceinline__ h8v pack8(const float* v, int e) {
    h8v r;
    typedef _Float16 h2v __attribute__((ext_vector_type(2)));
    const float sc = __uint_as_float((unsigned)(e + 127) << 23);
#pragma unroll
    for (int j = 0; j < 8; j += 2) {
        const h2v t = __builtin_convertvector(f2v{v[j] * sc, v[j + 1] * sc}, h2v);
        r[j] = t[0];
        r[j + 1] = t[1];
    }
    return r;
}

__device__ __forceinline__ void pack_w(float (&v)[32], int e, h8v (&bf)[4]) {
    typedef _Float16 h2v __attribute__((ext_vector_type(2)));
    const float sc = __uint_as_float((unsigned)(e + 127) << 23);
    const f2v s2 = {sc, sc};
#pragma unroll
    for (int s = 0; s < 2; ++s)
#pragma unroll
        for (int j = 0; j < 8; j += 2) {
            const f2v p = f2v{v[16 * s + j], v[16 * s + 8 + j]} * s2;
            const f2v q = f2v{v[16 * s + j + 1], v[16 * s + 8 + j + 1]} * s2;
            const h2v lo = __builtin_convertvector(f2v{p[0], q[0]}, h2v);
            const h2v hi = __builtin_convertvector(f2v{p[1], q[1]}, h2v);
            bf[2 * s][j] = lo[0]; bf[2 * s][j + 1] = lo[1];
            bf[2 * s + 1][j] = hi[0]; bf[2 * s + 1][j + 1] = hi[1];
        }
}

__device__ __forceinline__ void load_w_piece(const float* __restrict__ w, int c, int cg, int i, int h,
                                             float (&v)[32]) {
    const float* base = w + (size_t)(32 * c + 8 * h) * NDIM + 64 * cg + 2 * i;
#pragma unroll
    for (int s = 0; s < 2; ++s)
#pragma unroll
        for (int j = 0; j < 8; ++j) {
            f2v t = *(const f2v*)(base + (size_t)(16 * s + j) * NDIM);
            v[(s * 2 + 0) * 8 + j] = t[0];
            v[(s * 2 + 1) * 8 + j] = t[1];
        }
}

__device__ __forceinline__ void load_x_half(const float* __restrict__ x, int row0, int c, int i, int h,
                                            float* v) {
    const float* base = x + (size_t)(row0 + i) * KS + 32 * c + 8 * h;
#pragma unroll
    for (int s = 0; s < 2; ++s) {
        f4v t0 = *(const f4v*)(base + 16 * s);
        f4v t1 = *(const f4v*)(base + 16 * s + 4);
#pragma unroll
        for (int j = 0; j < 4; ++j) { v[s * 8 + j] = t0[j]; v[s * 8 + 4 + j] = t1[j]; }
    }
}

__device__ __forceinline__ void mma_chunk(h8v a0, h8v a1, h8v b00, h8v b01, h8v b10, h8v b11, int e,
                                          f16v& acc0, f16v& acc1) {
    f16v z = {0.f, 0.f, 0.f, 0.f, 0.f, 0.f, 0.f, 0.f, 0.f, 0.f, 0.f, 0.f, 0.f, 0.f, 0.f, 0.f};
    f16v t0 = __builtin_amdgcn_mfma_f32_32x32x16_f16(a0, b00, z, 0, 0, 0);
    f16v t1 = __builtin_amdgcn_mfma_f32_32x32x16_f16(a0, b01, z, 0, 0, 0);
    t0 = __builtin_amdgcn_mfma_f32_32x32x16_f16(a1, b10, t0, 0, 0, 0);
    t1 = __builtin_amdgcn_mfma_f32_32x32x16_f16(a1, b11, t1, 0, 0, 0);
    if (e >= -126 && e <= 127) {
        const float sc = __uint_as_float((unsigned)(e + 127) << 23);
        const f2v s2 = {sc, sc};
#pragma unroll
        for (int r = 0; r < 16; r += 2) {
            f2v u0 = __builtin_elementwise_fma(f2v{t0[r], t0[r + 1]}, s2, f2v{acc0[r], acc0[r + 1]});
            f2v u1 = __builtin_elementwise_fma(f2v{t1[r], t1[r + 1]}, s2, f2v{acc1[r], acc1[r + 1]});
            acc0[r] = u0[0]; acc0[r + 1] = u0[1];
            acc1[r] = u1[0]; acc1[r + 1] = u1[1];
        }
        return;
    }
#pragma unroll
    for (int r = 0; r < 16; ++r) {
        acc0[r] += ldexpf(t0[r], e);
        acc1[r] += ldexpf(t1[r], e);
    }
}

struct Frag { u4v a0, a1, b00, b01, b10, b11; };

template <int AUX = AUX_LD>
__device__ __forceinline__ Frag load_frag(__amdgpu_buffer_rsrc_t rs, int ow, int ox) {
    Frag f;
    f.a0 = __builtin_amdgcn_raw_buffer_load_b128(rs, ox, 0, AUX);
    f.a1 = __builtin_amdgcn_raw_buffer_load_b128(rs, ox + (int)F_STR, 0, AUX);
    f.b00 = __builtin_amdgcn_raw_buffer_load_b128(rs, ow, 0, AUX);
    f.b01 = __builtin_amdgcn_raw_buffer_load_b128(rs, ow + (int)F_STR, 0, AUX);
    f.b10 = __builtin_amdgcn_raw_buffer_load_b128(rs, ow + 2 * (int)F_STR, 0, AUX);
    f.b11 = __builtin_amdgcn_raw_buffer_load_b128(rs, ow + 3 * (int)F_STR, 0, AUX);
    return f;
}

__device__ __forceinline__ void mma_frag(const Frag& f, int e, f16v& acc0, f16v& acc1) {
    mma_chunk(__builtin_bit_cast(h8v, f.a0), __builtin_bit_cast(h8v, f.a1), __builtin_bit_cast(h8v, f.b00),
              __builtin_bit_cast(h8v, f.b01), __builtin_bit_cast(h8v, f.b10), __builtin_bit_cast(h8v, f.b11), e,
              acc0, acc1);
}

__device__ __forceinline__ u4v no_zero(u4v u) {
#pragma unroll
    for (int j = 0; j < 4; ++j)
        u[j] |= ((u[j] & 0xffffu) == 0u ? 0x8000u : 0u) | ((u[j] >> 16) == 0u ? 0x80000000u : 0u);
    return u;
}

__device__ __forceinline__ bool frag_valid(const Frag& f) {
    typedef unsigned short us8 __attribute__((ext_vector_type(8)));
    us8 m = __builtin_elementwise_min(__builtin_bit_cast(us8, f.a0), __builtin_bit_cast(us8, f.a1));
    m = __builtin_elementwise_min(m, __builtin_bit_cast(us8, f.b00));
    m = __builtin_elementwise_min(m, __builtin_bit_cast(us8, f.b01));
    m = __builtin_elementwise_min(m, __builtin_bit_cast(us8, f.b10));
    m = __builtin_elementwise_min(m, __builtin_bit_cast(us8, f.b11));
    typedef unsigned short us2 __attribute__((ext_vector_type(2)));
    us2 r = __builtin_elementwise_min(__builtin_elementwise_min(us2{m[0], m[1]}, us2{m[2], m[3]}),
                                      __builtin_elementwise_min(us2{m[4], m[5]}, us2{m[6], m[7]}));
    return __builtin_amdgcn_ballot_w64(r[0] == 0 || r[1] == 0) == 0ull;
}

template <int N>
__device__ __forceinline__ void wait_vm() {
    asm volatile("s_waitcnt vmcnt(%0)" ::"i"(N) : "memory");
}

__device__ __forceinline__ void wait_vm_n(int n) {
    switch (n) {
        case 0: wait_vm<0>(); break;
        case 1: wait_vm<1>(); break;
        case 2: wait_vm<2>(); break;
        case 3: wait_vm<3>(); break;
        case 4: wait_vm<4>(); break;
        case 5: wait_vm<5>(); break;
        case 6: wait_vm<6>(); break;
        case 7: wait_vm<7>(); break;
        default: wait_vm<8>(); break;
    }
}

__device__ __forceinline__ void raw_barrier() {
    asm volatile("s_waitcnt lgkmcnt(0)\n\ts_barrier" ::: "memory");
}

template <int SLP, class T>
__device__ __forceinline__ unsigned lds_wait(T& v) {
    unsigned r, it = 0u;
    unsigned long long t0 = 0ull;
    while ((r = (unsigned)LDS_LD(v)) == 0u) {
        __builtin_amdgcn_s_sleep(SLP);
        if ((++it & 1023u) == 0u) {
            const unsigned long long t = __builtin_amdgcn_s_memrealtime();
            if (t0 == 0ull) t0 = t;
            else if (t - t0 > DEADLINE_TICKS) break;
        }
    }
    asm volatile("" ::: "memory");
    return r;
}

}

__global__ void __launch_bounds__(640) gemm_rs_kernel(Args a) {
    __shared__ __attribute__((aligned(16))) u4v stage[NCH][256];
    __shared__ __attribute__((aligned(16))) float red[NR][8][64][4];
    __shared__ int s_exp[NCH];
    __shared__ int s_rdy[NCH];
    __shared__ unsigned s_call;
    __shared__ unsigned s_bad;
    __shared__ int s_prog;
    __shared__ int s_win;
    __shared__ int s_zok;

    const int b = blockIdx.x;
    const int wv = __builtin_amdgcn_readfirstlane(threadIdx.x >> 6);
    const int L = threadIdx.x & 63;
    const int i = L & 31, h = L >> 5;
    const int rank = a.rank & (NR - 1);
    const int xcd = b & 7, idx = b >> 3;
    const int cg = 4 * xcd + (idx & 3);
    const int rg = idx >> 2;
    const bool is_w = idx < 28;
    const int tx = 4 * xcd + (idx - 28);
    const bool has_task = is_w || tx < 28;
    const int q = is_w ? cg : 32 + (tx & 3);
    const int dest = (rank + 1 + (is_w ? rg : (tx >> 2))) & (NR - 1);
    char* const myws = a.ws[rank];

    if (wv == NR) {
        const unsigned cv = __hip_atomic_load((unsigned*)(myws + CTR_OFF + b * CTR_STRIDE), __ATOMIC_RELAXED,
                                              __HIP_MEMORY_SCOPE_AGENT);
        char* dws = a.ws[dest];
        unsigned call;
        auto init = [&]() {
            asm volatile("" ::"s"(dws));
            if (L < NCH) s_rdy[L] = 0;
            if (L == 0) {
                s_call = 0u;
                s_bad = 0u;
                s_prog = 0;
                s_zok = 0;
                s_win = is_w ? INFL : INFLX;
            }
            raw_barrier();
            call = __builtin_amdgcn_readfirstlane(cv) + 1u;
            if (L == 0) LDS_ST(s_call, call);
        };
        u4v t0[4];
        if (is_w) {
            float v0[32];
            load_w_piece(a.w, 0, cg, i, h, v0);
            init();
            const int e = block_exp<32>(v0);
            if (L == 0) { LDS_ST(s_rdy[0], 1); s_exp[0] = e; }
            h8v bf[4];
            pack_w(v0, e, bf);
#pragma unroll
            for (int f = 0; f < 4; ++f) t0[f] = __builtin_bit_cast(u4v, bf[f]);
        } else if (has_task) {
            float v0[32];
            load_x_half(a.x, MS * dest + 64 * (q - 32), 0, i, h, v0);
            load_x_half(a.x, MS * dest + 64 * (q - 32) + 32, 0, i, h, v0 + 16);
            init();
            const int e = block_exp<32>(v0);
            if (L == 0) { LDS_ST(s_rdy[0], 1); s_exp[0] = e; }
#pragma unroll
            for (int f = 0; f < 4; ++f) t0[f] = __builtin_bit_cast(u4v, pack8(v0 + 8 * f, e));
        } else {
            init();
            return;
        }
        const int par = (int)(call & 1u);
        __builtin_amdgcn_s_setprio(3);
        const __amdgpu_buffer_rsrc_t rs = __builtin_amdgcn_make_buffer_rsrc(dws, 0, (int)DATA_BYTES, 0x00020000);
        const unsigned s0 = data_of(par, rank, q) + (unsigned)L * 16u;
        unsigned long long* const fl = (unsigned long long*)(dws + flag_of(par, rank, q));
        auto publish = [&](int c) {
            if (L == 0) {
                const unsigned long long val = ((unsigned long long)call << 32) | (unsigned)s_exp[c];
                __hip_atomic_store(fl + c, val, __ATOMIC_RELAXED, __HIP_MEMORY_SCOPE_SYSTEM);
            }
        };
        int issued = 0, pub = 0;
        int P[NCH];
#pragma unroll
        for (int c = 0; c < NCH; ++c) {
            const unsigned off = s0 + (unsigned)c * C_STR;
            u4v t[4];
            if (c == 0) {
#pragma unroll
                for (int f = 0; f < 4; ++f) t[f] = t0[f];
            } else {
                if (!lds_wait<0>(s_rdy[c])) {
                    if (L == 0) { LDS_ST(s_bad, 1u); LDS_ST(s_prog, NST); }
                    return;
                }
#pragma unroll
                for (int f = 0; f < 4; ++f) t[f] = stage[c][f * 64 + L];
            }
#pragma unroll
            for (int f = 0; f < 4; ++f) {
                const int win = LDS_LD(s_win);
                __builtin_amdgcn_raw_buffer_store_b128(t[f], rs, (int)(off + f * F_STR), 0, AUX_ST);
                ++issued;
                if (f == 3) P[c] = issued;
                if (4 * c + f + 1 == NST && L == 0) LDS_ST(s_prog, NST);
                if (FASTLAST && 4 * c + f + 1 == NST) break;
                if (win <= 2) { if (win == 2) wait_vm<1>(); else wait_vm<0>(); }
                else if (win == 3) wait_vm<2>();
                else if (win == 4) wait_vm<3>();
                else if (win == 6) wait_vm<5>();
                else wait_vm<7>();
                const int done = issued - (win - 1);
                if (L == 0 && 4 * c + f + 1 < NST) LDS_ST(s_prog, 4 * c + f + 2 - win);
#pragma unroll
                for (int cc = (c >= 2 ? c - 2 : 0); cc <= c; ++cc)
                    if ((cc < c || f == 3) && pub == cc && P[cc] <= done) { publish(cc); ++issued; ++pub; }
            }
        }
#pragma unroll
        for (int cc = NCH - 3; cc < NCH; ++cc)
            if (pub == cc) {
                if (!(FASTLAST && cc == NCH - 1)) wait_vm_n(issued - P[cc]);
                else if (!lds_wait<1>(s_zok)) {
                    if (L == 0) LDS_ST(s_bad, 1u);
                    return;
                }
                publish(cc); ++issued; ++pub;
            }
        return;
    }

    if (wv == NR + 1) {
        raw_barrier();
        const unsigned call = lds_wait<1>(s_call);
        if (call == 0u) return;
        if (L == 0)
            __hip_atomic_store((unsigned*)(myws + CTR_OFF + b * CTR_STRIDE), call, __ATOMIC_RELAXED, __HIP_MEMORY_SCOPE_AGENT);
        if (!has_task) return;
        {
            const __amdgpu_buffer_rsrc_t rz = __builtin_amdgcn_make_buffer_rsrc(myws, 0, (int)DATA_BYTES, 0x00020000);
            const unsigned zo = data_of((int)(call & 1u) ^ 1, dest, q) + (unsigned)(NCH - 1) * C_STR + (unsigned)L * 16u;
#pragma unroll
            for (int f = 0; f < 4; ++f)
                __builtin_amdgcn_raw_buffer_store_b128(u4v{0u, 0u, 0u, 0u}, rz, (int)(zo + f * F_STR), 0, AUX_ST);
            wait_vm<0>();
            if (L == 0) LDS_ST(s_zok, 1);
        }
        unsigned* pc = (unsigned*)(myws + PRG_OFF + (unsigned)dest * 128u);
        const unsigned base = (call - 1u) * (unsigned)(NPIECE * NST);
        int last = 0;
        unsigned it = 0u;
        unsigned long long t0 = 0ull;
        for (;;) {
            int p = LDS_LD(s_prog);
            if ((++it & 255u) == 0u) {
                const unsigned long long t = __builtin_amdgcn_s_memrealtime();
                if (t0 == 0ull) t0 = t;
                else if (t - t0 > DEADLINE_TICKS) p = NST;
            }
            const int dl = p > last ? p - last : 0;
            last += dl;
            if (last >= NST) {
                if (L == 0) __hip_atomic_fetch_add(pc, (unsigned)dl, __ATOMIC_RELAXED, __HIP_MEMORY_SCOPE_AGENT);
                break;
            }
            unsigned g = 0u;
            if (L == 0) g = __hip_atomic_fetch_add(pc, (unsigned)dl, __ATOMIC_RELAXED, __HIP_MEMORY_SCOPE_AGENT) + (unsigned)dl;
            const int lag = (int)(__builtin_amdgcn_readfirstlane(g) - base) - NPIECE * last;
            int win = lag > FB_T3 ? 6 : lag > FB_T2 ? 4 : lag > FB_T1 ? 3 : lag < -FB_TL ? 1 : 2;
            if (lag + NPIECE * last >= FB_END * NPIECE && win < FB_WE) win = FB_WE;
            if (L == 0 && last > 0) LDS_ST(s_win, win);
            __builtin_amdgcn_s_sleep(FB_SLEEP);
        }
        return;
    }

    f16v acc0, acc1;
#pragma unroll
    for (int r = 0; r < 16; ++r) { acc0[r] = 0.f; acc1[r] = 0.f; }
    {
        float vw[32], vx[16];
        h8v bf[4];
        int ew;
        const bool feeds = wv != 0;
        if (!is_w) {
            if (has_task && feeds) {
                float vt[32];
                const int row0 = MS * dest + 64 * (q - 32);
                raw_barrier();
                lds_wait<STAG_S>(s_rdy[0]);
                load_x_half(a.x, row0, wv, i, h, vt);
                load_x_half(a.x, row0 + 32, wv, i, h, vt + 16);
                const int e = block_exp<32>(vt);
#pragma unroll
                for (int f = 0; f < 4; ++f) {
                    u4v u = __builtin_bit_cast(u4v, pack8(vt + 8 * f, e));
                    if (FASTLAST && wv == NCH - 1) u = no_zero(u);
                    stage[wv][f * 64 + L] = u;
                }
                if (L == 0) s_exp[wv] = e;
                asm volatile("s_waitcnt lgkmcnt(0)" ::: "memory");
                if (L == 0) LDS_ST(s_rdy[wv], 1);
                asm volatile("" ::: "memory");
                load_w_piece(a.w, wv, cg, i, h, vw);
                load_x_half(a.x, MS * rank + 32 * rg, wv, i, h, vx);
            } else {
                raw_barrier();
                if (has_task) lds_wait<STAG_S>(s_rdy[0]);
                load_w_piece(a.w, wv, cg, i, h, vw);
                load_x_half(a.x, MS * rank + 32 * rg, wv, i, h, vx);
            }
            ew = block_exp<32>(vw);
            pack_w(vw, ew, bf);
        } else {
            raw_barrier();
            lds_wait<STAG_S>(s_rdy[0]);
            load_w_piece(a.w, wv, cg, i, h, vw);
            ew = block_exp<32>(vw);
            pack_w(vw, ew, bf);
            if (feeds) {
#pragma unroll
                for (int f = 0; f < 4; ++f) {
                    u4v u = __builtin_bit_cast(u4v, bf[f]);
                    if (FASTLAST && wv == NCH - 1) u = no_zero(u);
                    stage[wv][f * 64 + L] = u;
                }
                if (L == 0) s_exp[wv] = ew;
                asm volatile("s_waitcnt lgkmcnt(0)" ::: "memory");
                if (L == 0) LDS_ST(s_rdy[wv], 1);
            }
            asm volatile("" ::: "memory");
            load_x_half(a.x, MS * rank + 32 * rg, wv, i, h, vx);
        }
        const int ex = block_exp<16>(vx);
        mma_chunk(pack8(vx, ex), pack8(vx + 8, ex), bf[0], bf[1], bf[2], bf[3], -(ew + ex), acc0, acc1);
    }

    if (wv < NR - 1) {
        const unsigned call = lds_wait<1>(s_call);
        const int par = (int)(call & 1u);
        const int src = (rank + 1 + wv) & (NR - 1);
        const __amdgpu_buffer_rsrc_t rs = __builtin_amdgcn_make_buffer_rsrc(myws, 0, (int)WS_NEED, 0x00020000);
        const int owb = (int)(data_of(par, src, cg) + (unsigned)L * 16u);
        const int oxb = (int)(data_of(par, src, 32 + (rg >> 1)) + (unsigned)(rg & 1) * 2u * F_STR + (unsigned)L * 16u);
        const int poff1 = (int)(L < 8 ? flag_of(par, src, cg) + 8u * L
                               : L < 16 ? flag_of(par, src, 32 + (rg >> 1)) + 8u * (L - 8) : TMO_OFF);
        const int poff0 = TMO_EVERY == 1 ? poff1 : (int)(L < 16 ? (unsigned)poff1 : flag_of(par, src, cg) + 64u);
        int poff = poff0;
        unsigned npoll = 0u;
        const unsigned long long t0 = __builtin_amdgcn_s_memrealtime();
        bool bad = call == 0u, retry = false;
        int next = 0;
        u2v pv = __builtin_amdgcn_raw_buffer_load_b64(rs, poff, 0, AUX_POLL);
        while (!bad) {
            const unsigned lo = pv[0], hi = pv[1];
            const unsigned m = (unsigned)__builtin_amdgcn_ballot_w64(hi >= call);
            const unsigned rdy = (m & (m >> 8) & 0xffu) >> next;
            if (__builtin_amdgcn_readlane(lo, 16) == call) { bad = true; break; }
            if (!(rdy & 1u)) {
                if (__builtin_amdgcn_s_memrealtime() - t0 > DEADLINE_TICKS) {
                    if (L == 0)
                        __hip_atomic_store((unsigned*)(myws + TMO_OFF), call, __ATOMIC_RELAXED, __HIP_MEMORY_SCOPE_AGENT);
                    bad = true;
                    break;
                }
                if (next < LATE_C) __builtin_amdgcn_s_sleep(SLEEP_E);
                else __builtin_amdgcn_s_sleep(SLEEP_N);
                asm volatile("" ::: "memory");
                ++npoll;
                pv = __builtin_amdgcn_raw_buffer_load_b64(rs, (npoll % TMO_EVERY) == 0u ? poff1 : poff0, 0, AUX_POLL);
                continue;
            }
            asm volatile("" ::: "memory");
            const int co = next * (int)C_STR;
            const int e0 = -((int)__builtin_amdgcn_readlane(lo, next) + (int)__builtin_amdgcn_readlane(lo, next + 8));
            if ((rdy & 3u) == 3u) {
                const int e1 = -((int)__builtin_amdgcn_readlane(lo, next + 1) + (int)__builtin_amdgcn_readlane(lo, next + 9));
                const Frag f0 = load_frag(rs, owb + co, oxb + co);
                const Frag f1 = load_frag(rs, owb + co + (int)C_STR, oxb + co + (int)C_STR);
                if (next + 2 < NCH) pv = __builtin_amdgcn_raw_buffer_load_b64(rs, poff, 0, AUX_POLL);
                __builtin_amdgcn_sched_barrier(0);
                mma_frag(f0, e0, acc0, acc1);
                if (!FASTLAST || next + 2 < NCH || frag_valid(f1)) {
                    mma_frag(f1, e1, acc0, acc1);
                    next += 2;
                } else {
                    next += 1;
                    retry = true;
                }
            } else {
                const Frag f0 = retry ? load_frag<17>(rs, owb + co, oxb + co) : load_frag(rs, owb + co, oxb + co);
                if (next + 1 < NCH) pv = __builtin_amdgcn_raw_buffer_load_b64(rs, poff, 0, AUX_POLL);
                __builtin_amdgcn_sched_barrier(0);
                if (!FASTLAST || next + 1 < NCH || frag_valid(f0)) {
                    mma_frag(f0, e0, acc0, acc1);
                    next += 1;
                } else {
                    retry = true;
                    if (__builtin_amdgcn_s_memrealtime() - t0 > DEADLINE_TICKS) {
                        if (L == 0)
                            __hip_atomic_store((unsigned*)(myws + TMO_OFF), call, __ATOMIC_RELAXED, __HIP_MEMORY_SCOPE_AGENT);
                        bad = true;
                        break;
                    }
                    asm volatile("" ::: "memory");
                }
            }
            if (next >= NCH) break;
        }
        if (bad && L == 0) LDS_ST(s_bad, 1u);
    }

#pragma unroll
    for (int g = 0; g < 4; ++g) {
        f4v t0 = {acc0[4 * g], acc0[4 * g + 1], acc0[4 * g + 2], acc0[4 * g + 3]};
        f4v t1 = {acc1[4 * g], acc1[4 * g + 1], acc1[4 * g + 2], acc1[4 * g + 3]};
        *(f4v*)&red[wv][g][L][0] = t0;
        *(f4v*)&red[wv][4 + g][L][0] = t1;
    }
    __syncthreads();
    {
        const int g = wv >> 1, p2 = (wv & 1) * 2;
        f2v s0 = {0.f, 0.f}, s1 = {0.f, 0.f};
#pragma unroll
        for (int p = 0; p < NR; ++p) {
            s0 += *(const f2v*)&red[p][g][L][p2];
            s1 += *(const f2v*)&red[p][4 + g][L][p2];
        }
        if (s_bad) {
            const float qn = __builtin_nanf("");
            s0 = f2v{qn, qn};
            s1 = f2v{qn, qn};
        }
        float* o = a.out + (size_t)(32 * rg + 8 * g + 4 * h + p2) * NDIM + 64 * cg + 2 * i;
        const __amdgpu_buffer_rsrc_t ro = __builtin_amdgcn_make_buffer_rsrc(a.out, 0, MS * NDIM * 4, 0x00020000);
        const int oo = (int)((char*)o - (char*)a.out);
        __builtin_amdgcn_raw_buffer_store_b64(__builtin_bit_cast(u2v, f2v{s0[0], s1[0]}), ro, oo, 0, 17);
        __builtin_amdgcn_raw_buffer_store_b64(__builtin_bit_cast(u2v, f2v{s0[1], s1[1]}), ro, oo + NDIM * 4, 0, 17);
    }
}

extern "C" void kernel_launch(int rank, int nranks, ncclComm_t comm,
                              hipStream_t stream,
                              void* const* d_in, const int* in_sizes, int n_in,
                              void* d_out, int out_size,
                              const int* in_dtypes, int out_dtype,
                              void* const* d_out_all,
                              void* const* d_ws_all, size_t ws_bytes) {
    if (nranks != NR || n_in < 2 || ws_bytes < (size_t)WS_NEED || out_size != MS * NDIM) {
        fprintf(stderr, "unexpected problem geometry\n");
        return;
    }
    Args a;
    a.x = (const float*)d_in[0];
    a.w = (const float*)d_in[1];
    a.out = (float*)d_out;
    for (int p = 0; p < NR; ++p) a.ws[p] = (char*)d_ws_all[p];
    a.rank = rank;
    gemm_rs_kernel<<<NWG, 640, 0, stream>>>(a);
    HIP_CHECK(hipGetLastError());
}
```

```cpp
#include <hip/hip_runtime.h>
#include <rccl/rccl.h>
#include <cstdio>
#include <cstdint>

#define HIP_CHECK(expr) do { hipError_t _e = (expr); if (_e != hipSuccess) { \
    fprintf(stderr, "HIP error %s at %s:%d\n", hipGetErrorString(_e), __FILE__, __LINE__); \
    return; } } while (0)

typedef _Float16 h8v __attribute__((ext_vector_type(8)));
typedef float f16v __attribute__((ext_vector_type(16)));
typedef float f4v __attribute__((ext_vector_type(4)));
typedef float f2v __attribute__((ext_vector_type(2)));
typedef unsigned u4v __attribute__((ext_vector_type(4)));
typedef unsigned u2v __attribute__((ext_vector_type(2)));

namespace {

constexpr int NR = 8;
constexpr int NDIM = 2048;
constexpr int KS = 256;
constexpr int MS = 256;
constexpr int NWG = 256;
constexpr int NSTR = 32;
constexpr int NPC = 48;
constexpr int RMAX = 7;
constexpr unsigned F_STR = 33 * 1024, R_STR = 4 * F_STR, S_STR = 6 * R_STR;
constexpr unsigned DATA_BYTES = 2u * NR * S_STR;
constexpr unsigned FLAG_STRIDE = 128;
constexpr unsigned FLAG_OFF = DATA_BYTES;
constexpr unsigned FLAG_BYTES = 2u * NR * NPC * FLAG_STRIDE;
constexpr unsigned OFL_OFF = FLAG_OFF + FLAG_BYTES;
constexpr unsigned OFL_STRIDE = 64;
constexpr unsigned CTR_OFF = OFL_OFF + NWG * OFL_STRIDE;
constexpr unsigned CTR_STRIDE = 64;
constexpr unsigned TMO_OFF = CTR_OFF + NWG * CTR_STRIDE;
constexpr unsigned PRG_OFF = TMO_OFF + 128;
constexpr unsigned WS_NEED = PRG_OFF + NR * 128;
#define DLT 50000000ull
constexpr unsigned long long DEADLINE_TICKS = DLT;

#define INFL 2
#define SLEEP_N 1
#define FB_T1 50
#define FB_T2 110
#define FB_T3 200
#define FB_TL 60
#define FB_W0 2
#define FB_END 6
#define FB_WE 4
#define SLEEP_E 16
#define SLEEP_W 8
#define SLEEP_O 2
#define SLEEP_T 2
#define SLEEP_R 2
#define TMO_EVERY 16
#define FB_SLEEP 4
#define STAG_S 1
#define TILE_WIN 1
#define TILE_WJ 24
#define AUX_ST 17
#define AUX_LD 17

#define STAMP(idx) do {} while (0)
#define STAMPV(idx, v) do {} while (0)
#define TNOW() 0ull

#define LDS_LD(v) __builtin_amdgcn_readfirstlane(__hip_atomic_load(&(v), __ATOMIC_RELAXED, __HIP_MEMORY_SCOPE_WORKGROUP))
#define LDS_ST(v, x) __hip_atomic_store(&(v), (x), __ATOMIC_RELAXED, __HIP_MEMORY_SCOPE_WORKGROUP)

struct Args {
    const float* x;
    const float* w;
    char* ws[NR];
    float* out[NR];
    int rank;
};

__device__ __forceinline__ unsigned data_of(int par, int src, int u) {
    return (unsigned)(par * NR + src) * S_STR + (unsigned)(u >> 5) * R_STR + (unsigned)(u & 31) * 1024u;
}
__device__ __forceinline__ unsigned flag_of(int par, int src, int u) {
    return FLAG_OFF + (unsigned)((par * NR + src) * NPC + (u >> 2)) * FLAG_STRIDE + (unsigned)(u & 3) * 8u;
}
__device__ __forceinline__ int frow(int r, int i) {
    return MS * (2 * (i >> 1) + 1 - (r & 1)) + 64 * (r >> 1) + 32 * (i & 1);
}

struct Ent { int kind, src, u, a0, kq; };
template <int NRND>
__device__ __forceinline__ Ent entry(int rank, int dest, int rho, int s) {
    Ent e;
    e.src = rank;
    if (NRND == 1) {
        const int j = 4 * (s & 7) + 3;
        e.kind = 1; e.kq = dest >> 1; e.a0 = 1024 * (dest & 1) + 32 * j; e.u = 32 + 4 * j + e.kq;
        return e;
    }
    if (rho == 0) { e.kind = 0; e.kq = s & 3; e.a0 = frow(dest, s >> 2); e.u = s; return e; }
    if (rho == NRND - 1) { e.kind = 0; e.kq = s & 3; e.a0 = MS * dest + 32 * (s >> 2); e.u = 160 + s; return e; }
    constexpr int G = NRND == 5 ? 12 : 20;
    const int n = 32 * (rho - 1) + s, g = n / G, t = n - g * G;
    if (t < 16) {
        const int j = 4 * g + (t >> 2);
        e.kind = 1; e.kq = t & 3; e.a0 = 1024 * (dest & 1) + 32 * j; e.u = 32 + 4 * j + e.kq;
    } else {
        e.kind = 2; e.kq = rank >> 1; e.a0 = 0; e.src = 2 * (t - 16) + 1 - (rank & 1); e.u = 32 + 4 * (4 * g + 3) + e.kq;
    }
    return e;
}

template <int NV>
__device__ __forceinline__ int block_exp(const float (&v)[NV]) {
    float m = 0.f;
#pragma unroll
    for (int t = 0; t < NV; ++t) m = fmaxf(m, fabsf(v[t]));
    unsigned u = __float_as_uint(m);
    u = max(u, (unsigned)__builtin_amdgcn_update_dpp(0, (int)u, 0xB1, 0xf, 0xf, true));
    u = max(u, (unsigned)__builtin_amdgcn_update_dpp(0, (int)u, 0x4E, 0xf, 0xf, true));
    u = max(u, (unsigned)__builtin_amdgcn_update_dpp(0, (int)u, 0x141, 0xf, 0xf, true));
    u = max(u, (unsigned)__builtin_amdgcn_update_dpp(0, (int)u, 0x140, 0xf, 0xf, true));
    const unsigned r = max(max((unsigned)__builtin_amdgcn_readlane((int)u, 0), (unsigned)__builtin_amdgcn_readlane((int)u, 16)),
                           max((unsigned)__builtin_amdgcn_readlane((int)u, 32), (unsigned)__builtin_amdgcn_readlane((int)u, 48)));
    unsigned E = r >> 23;
    E = E < 15u ? 15u : E;
    return 141 - (int)E;
}

__device__ __forceinline__ u4v pack8(const float* v, int e) {
    h8v r;
    typedef _Float16 h2v __attribute__((ext_vector_type(2)));
    const float sc = __uint_as_float((unsigned)(e + 127) << 23);
    const f2v s2 = {sc, sc};
#pragma unroll
    for (int j = 0; j < 8; j += 2) {
        const h2v t = __builtin_convertvector(f2v{v[j], v[j + 1]} * s2, h2v);
        r[j] = t[0];
        r[j + 1] = t[1];
    }
    return __builtin_bit_cast(u4v, r);
}

template <int NF>
__device__ __forceinline__ void load_w(const float* __restrict__ w, int k0, int col0, int i, int hh, float (&v)[8 * NF]) {
    const float* base = w + (size_t)(k0 + 8 * hh) * NDIM + col0 + i;
#pragma unroll
    for (int s = 0; s < NF; ++s)
#pragma unroll
        for (int j = 0; j < 8; ++j) v[8 * s + j] = base[(size_t)(16 * s + j) * NDIM];
}

template <int NF>
__device__ __forceinline__ void load_x(const float* __restrict__ x, int row0, int k0, int i, int hh, float (&v)[8 * NF]) {
    const float* base = x + (size_t)(row0 + i) * KS + k0 + 8 * hh;
#pragma unroll
    for (int s = 0; s < NF; ++s) {
        f4v t0 = *(const f4v*)(base + 16 * s);
        f4v t1 = *(const f4v*)(base + 16 * s + 4);
#pragma unroll
        for (int j = 0; j < 4; ++j) { v[s * 8 + j] = t0[j]; v[s * 8 + 4 + j] = t1[j]; }
    }
}

__device__ __forceinline__ f16v mm(u4v a, u4v b, f16v c) {
    return __builtin_amdgcn_mfma_f32_32x32x16_f16(__builtin_bit_cast(h8v, a), __builtin_bit_cast(h8v, b), c, 0, 0, 0);
}
#define ZERO16 f16v{0.f, 0.f, 0.f, 0.f, 0.f, 0.f, 0.f, 0.f, 0.f, 0.f, 0.f, 0.f, 0.f, 0.f, 0.f, 0.f}

__device__ __forceinline__ f16v scale16(f16v t, int e) {
    const int e1 = e >> 1, e2 = e - e1;
    const float c1 = __uint_as_float((unsigned)(e1 + 127) << 23), c2 = __uint_as_float((unsigned)(e2 + 127) << 23);
    const f2v s1 = {c1, c1}, s2 = {c2, c2};
#pragma unroll
    for (int r = 0; r < 16; r += 2) {
        const f2v u = (f2v{t[r], t[r + 1]} * s1) * s2;
        t[r] = u[0]; t[r + 1] = u[1];
    }
    return t;
}

__device__ __forceinline__ void fma16(f16v& acc, f16v t, int e) {
    const int e1 = e >> 1, e2 = e - e1;
    const float c1 = __uint_as_float((unsigned)(e1 + 127) << 23), c2 = __uint_as_float((unsigned)(e2 + 127) << 23);
    const f2v s1 = {c1, c1}, s2 = {c2, c2};
#pragma unroll
    for (int r = 0; r < 16; r += 2) {
        const f2v u = __builtin_elementwise_fma(f2v{t[r], t[r + 1]} * s1, s2, f2v{acc[r], acc[r + 1]});
        acc[r] = u[0]; acc[r + 1] = u[1];
    }
}

typedef unsigned u8v __attribute__((ext_vector_type(8)));
typedef unsigned u16v __attribute__((ext_vector_type(16)));
typedef u16v Quad;
#define QF(q, f) __builtin_shufflevector(q, q, 4 * (f), 4 * (f) + 1, 4 * (f) + 2, 4 * (f) + 3)

#define AUX_XF AUX_LD
#define AUX_W AUX_LD
#define AUX_XO AUX_LD
template <int AUX = AUX_LD>
__device__ __forceinline__ Quad load_quad(__amdgpu_buffer_rsrc_t rs, int off) {
    const u4v f0 = __builtin_amdgcn_raw_buffer_load_b128(rs, off, 0, AUX);
    const u4v f1 = __builtin_amdgcn_raw_buffer_load_b128(rs, off, (int)F_STR, AUX);
    const u4v f2 = __builtin_amdgcn_raw_buffer_load_b128(rs, off, 2 * (int)F_STR, AUX);
    const u4v f3 = __builtin_amdgcn_raw_buffer_load_b128(rs, off, 3 * (int)F_STR, AUX);
    const u8v l = __builtin_shufflevector(f0, f1, 0, 1, 2, 3, 4, 5, 6, 7);
    const u8v h = __builtin_shufflevector(f2, f3, 0, 1, 2, 3, 4, 5, 6, 7);
    return __builtin_shufflevector(l, h, 0, 1, 2, 3, 4, 5, 6, 7, 8, 9, 10, 11, 12, 13, 14, 15);
}

__device__ __forceinline__ f16v mm_quad(const Quad& w, const Quad& x) {
    f16v t = mm(QF(w, 0), QF(x, 0), ZERO16);
    t = mm(QF(w, 1), QF(x, 1), t);
    t = mm(QF(w, 2), QF(x, 2), t);
    return mm(QF(w, 3), QF(x, 3), t);
}

__device__ __forceinline__ void pin(f16v& v) {
    asm volatile("" : "+v"(v));
    __builtin_amdgcn_sched_barrier(0);
}

__device__ __forceinline__ u4v no_zero(u4v u) {
#pragma unroll
    for (int j = 0; j < 4; ++j)
        u[j] |= ((u[j] & 0xffffu) == 0u ? 0x8000u : 0u) | ((u[j] >> 16) == 0u ? 0x80000000u : 0u);
    return u;
}

__device__ __forceinline__ bool quad_valid(const Quad& q) {
    typedef unsigned short us8 __attribute__((ext_vector_type(8)));
    us8 m = __builtin_elementwise_min(__builtin_bit_cast(us8, QF(q, 0)), __builtin_bit_cast(us8, QF(q, 1)));
    m = __builtin_elementwise_min(m, __builtin_bit_cast(us8, QF(q, 2)));
    m = __builtin_elementwise_min(m, __builtin_bit_cast(us8, QF(q, 3)));
    typedef unsigned short us2 __attribute__((ext_vector_type(2)));
    us2 r = __builtin_elementwise_min(__builtin_elementwise_min(us2{m[0], m[1]}, us2{m[2], m[3]}),
                                      __builtin_elementwise_min(us2{m[4], m[5]}, us2{m[6], m[7]}));
    return __builtin_amdgcn_ballot_w64(r[0] == 0 || r[1] == 0) == 0ull;
}

template <int N>
__device__ __forceinline__ void wait_vm() {
    asm volatile("s_waitcnt vmcnt(%0)" ::"i"(N) : "memory");
}

__device__ __forceinline__ void wait_vm_n(int n) {
    switch (n) {
        case 0: wait_vm<0>(); break;
        case 1: wait_vm<1>(); break;
        case 2: wait_vm<2>(); break;
        case 3: wait_vm<3>(); break;
        case 4: wait_vm<4>(); break;
        case 5: wait_vm<5>(); break;
        case 6: wait_vm<6>(); break;
        case 7: wait_vm<7>(); break;
        default: wait_vm<8>(); break;
    }
}

__device__ __forceinline__ void raw_barrier() {
    asm volatile("s_waitcnt lgkmcnt(0)\n\ts_barrier" ::: "memory");
}

template <int SLP, class T>
__device__ __forceinline__ unsigned lds_wait(T& v, unsigned n = 1u) {
    unsigned r, it = 0u;
    unsigned long long t0 = 0ull;
    while ((r = (unsigned)LDS_LD(v)) < n) {
        __builtin_amdgcn_s_sleep(SLP);
        if ((++it & 1023u) == 0u) {
            const unsigned long long t = __builtin_amdgcn_s_memrealtime();
            if (t0 == 0ull) t0 = t;
            else if (t - t0 > 2ull * DEADLINE_TICKS) { r = 0u; break; }
        }
    }
    asm volatile("" ::: "memory");
    return r;
}

struct Lds {
    u4v stage[RMAX][256];
    f4v redF[NR][8 * 33];
    f4v redO[NR][8 * 33];
    int exp_[RMAX + 1];
    int rdy[RMAX + 1];
    unsigned call, bad, nloc, locdone, nfor, nown;
    int prog, win, zok;
};

__device__ __forceinline__ void deposit(f4v* slot, const f16v& t, int i, int hh) {
#pragma unroll
    for (int g = 0; g < 4; ++g)
        slot[(2 * g + hh) * 33 + i] = f4v{t[4 * g], t[4 * g + 1], t[4 * g + 2], t[4 * g + 3]};
}
__device__ __forceinline__ void lds_signal(unsigned& c, int L) {
    asm volatile("s_waitcnt lgkmcnt(0)" ::: "memory");
    if (L == 0) __hip_atomic_fetch_add(&c, 1u, __ATOMIC_RELAXED, __HIP_MEMORY_SCOPE_WORKGROUP);
}

template <int NRND>
__device__ __forceinline__ void pusher(const Args& a, Lds& S, int b, int rank, int dest, int s, int L) {
    constexpr int NST = 4 * NRND;
    char* const myws = a.ws[rank];
    const unsigned long long tq0 = TNOW();
    const unsigned cv = __hip_atomic_load((unsigned*)(myws + CTR_OFF + b * CTR_STRIDE), __ATOMIC_RELAXED,
                                          __HIP_MEMORY_SCOPE_AGENT);
    char* dws = a.ws[dest];
    const int i = L & 31, hh = L >> 5;
    u4v t0[4];
    {
        const Ent en = entry<NRND>(rank, dest, 0, s);
        float v0[32];
        if (NRND == 1) load_w<4>(a.w, 64 * en.kq, en.a0, i, hh, v0);
        else load_x<4>(a.x, en.a0, 64 * en.kq, i, hh, v0);
        asm volatile("" ::"s"(dws));
        if (L <= RMAX) S.rdy[L] = 0;
        if (L == 0) {
            S.call = 0u; S.bad = 0u; S.nloc = 0u; S.locdone = 0u; S.nfor = 0u; S.nown = 0u;
            S.prog = 0; S.zok = 0; S.win = INFL;
        }
        raw_barrier();
        const int e = block_exp<32>(v0);
        if (L == 0) { LDS_ST(S.rdy[0], 1); S.exp_[0] = e; }
#pragma unroll
        for (int f = 0; f < 4; ++f) t0[f] = pack8(v0 + 8 * f, e);
    }
    const unsigned call = __builtin_amdgcn_readfirstlane(cv) + 1u;
    if (L == 0) LDS_ST(S.call, call);
    const int par = (int)(call & 1u);
    __builtin_amdgcn_s_setprio(3);
    const __amdgpu_buffer_rsrc_t rs = __builtin_amdgcn_make_buffer_rsrc(dws, 0, (int)DATA_BYTES, 0x00020000);
    unsigned long long tp[RMAX + 3];
    auto publish = [&](int c) {
        const Ent en = entry<NRND>(rank, dest, c, s);
        if (L == 0) {
            const unsigned long long val = ((unsigned long long)call << 32) | (unsigned)S.exp_[c];
            __hip_atomic_store((unsigned long long*)(dws + flag_of(par, en.src, en.u)), val, __ATOMIC_RELAXED,
                               __HIP_MEMORY_SCOPE_SYSTEM);
        }
        tp[c] = TNOW();
    };
    if (NRND == 1) {
        const Ent en = entry<NRND>(rank, dest, 0, s);
        const unsigned off = data_of(par, en.src, en.u) + (unsigned)L * 16u;
        tp[RMAX] = TNOW();
#pragma unroll
        for (int f = 0; f < 4; ++f)
            __builtin_amdgcn_raw_buffer_store_b128(t0[f], rs, (int)(off + f * F_STR), 0, AUX_ST);
        wait_vm<0>();
        publish(0);
        STAMPV(0, tq0); STAMPV(1, tp[RMAX]); STAMPV(2, tp[0]);
        return;
    }
    int issued = 0, pub = 0;
    int P[NRND];
#pragma unroll
    for (int c = 0; c < NRND; ++c) {
        const Ent en = entry<NRND>(rank, dest, c, s);
        const unsigned off = data_of(par, en.src, en.u) + (unsigned)L * 16u;
        u4v t[4];
        if (c == 0) {
#pragma unroll
            for (int f = 0; f < 4; ++f) t[f] = t0[f];
            tp[RMAX] = TNOW();
        } else {
            if (!lds_wait<0>(S.rdy[c])) {
                if (L == 0) { LDS_ST(S.bad, 1u); LDS_ST(S.prog, NST); }
                return;
            }
#pragma unroll
            for (int f = 0; f < 4; ++f) t[f] = S.stage[c][f * 64 + L];
        }
#pragma unroll
        for (int f = 0; f < 4; ++f) {
            const int win = LDS_LD(S.win);
            __builtin_amdgcn_raw_buffer_store_b128(t[f], rs, (int)(off + f * F_STR), 0, AUX_ST);
            ++issued;
            if (f == 3) P[c] = issued;
            if (4 * c + f + 1 == NST) {
                if (L == 0) LDS_ST(S.prog, NST);
                break;
            }
            if (win <= 2) { if (win == 2) wait_vm<1>(); else wait_vm<0>(); }
            else if (win == 3) wait_vm<2>();
            else if (win == 4) wait_vm<3>();
            else if (win == 6) wait_vm<5>();
            else wait_vm<7>();
            const int done = issued - (win - 1);
            if (L == 0) LDS_ST(S.prog, 4 * c + f + 2 - win);
#pragma unroll
            for (int cc = (c >= 2 ? c - 2 : 0); cc <= c; ++cc)
                if ((cc < c || f == 3) && pub == cc && P[cc] <= done) { publish(cc); ++issued; ++pub; }
        }
    }
    tp[RMAX + 1] = TNOW();
#pragma unroll
    for (int cc = NRND - 3; cc < NRND; ++cc)
        if (pub == cc) {
            if (cc < NRND - 1) wait_vm_n(issued - P[cc]);
            else if (!lds_wait<1>(S.zok)) {
                if (L == 0) LDS_ST(S.bad, 1u);
                return;
            }
            publish(cc); ++issued; ++pub;
        }
}

template <int NRND>
__device__ __forceinline__ bool feed(const Args& a, Lds& S, int rank, int dest, int s, int wv, int L) {
    if (wv < 1 || wv >= NRND) return false;
    const Ent en = entry<NRND>(rank, dest, wv, s);
    if (en.kind == 2) return true;
    const int i = L & 31, hh = L >> 5;
    float v[32];
    if (en.kind == 1) load_w<4>(a.w, 64 * en.kq, en.a0, i, hh, v);
    else load_x<4>(a.x, en.a0, 64 * en.kq, i, hh, v);
    const int e = block_exp<32>(v);
#pragma unroll
    for (int f = 0; f < 4; ++f) {
        u4v u = pack8(v + 8 * f, e);
        if (wv == NRND - 1) u = no_zero(u);
        S.stage[wv][f * 64 + L] = u;
    }
    if (L == 0) S.exp_[wv] = e;
    asm volatile("s_waitcnt lgkmcnt(0)" ::: "memory");
    if (L == 0) LDS_ST(S.rdy[wv], 1);
    asm volatile("" ::: "memory");
    return false;
}

}

__global__ void __launch_bounds__(640) gemm_rs_kernel(Args a) {
    __shared__ __attribute__((aligned(16))) Lds S;

    const int b = blockIdx.x;
    const int wv = __builtin_amdgcn_readfirstlane(threadIdx.x >> 6);
    const int L = threadIdx.x & 63;
    const int i = L & 31, hh = L >> 5;
    const int rank = a.rank & (NR - 1);
    const int cls = rank & 1, mem = rank >> 1;
    const int ti = b & 7, tj = b >> 3;
    const int lk = b >> 5, s = b & 31;
    const int dest = lk < 7 ? (rank + 1 + lk) & (NR - 1) : 2 * (s >> 3) + 1 - cls;
    char* const myws = a.ws[rank];

    if (wv == NR) {
        if (lk == 7) pusher<1>(a, S, b, rank, dest, s, L);
        else if (lk & 1) pusher<7>(a, S, b, rank, dest, s, L);
        else pusher<5>(a, S, b, rank, dest, s, L);
        return;
    }

    if (wv == NR + 1) {
        raw_barrier();
        const unsigned call = lds_wait<1>(S.call);
        if (call == 0u) return;
        if (L == 0)
            __hip_atomic_store((unsigned*)(myws + CTR_OFF + b * CTR_STRIDE), call, __ATOMIC_RELAXED, __HIP_MEMORY_SCOPE_AGENT);
        if (lk < 7) {
            const int NST = (lk & 1) ? 28 : 20;
            {
                const __amdgpu_buffer_rsrc_t rz = __builtin_amdgcn_make_buffer_rsrc(myws, 0, (int)DATA_BYTES, 0x00020000);
                const unsigned zo = data_of((int)(call & 1u) ^ 1, dest, 160 + s) + (unsigned)L * 16u;
#pragma unroll
                for (int f = 0; f < 4; ++f)
                    __builtin_amdgcn_raw_buffer_store_b128(u4v{0u, 0u, 0u, 0u}, rz, (int)(zo + f * F_STR), 0, AUX_ST);
                wait_vm<0>();
                if (L == 0) LDS_ST(S.zok, 1);
            }
            unsigned* pc = (unsigned*)(myws + PRG_OFF + (unsigned)dest * 128u);
            const unsigned base = (call - 1u) * (unsigned)(NSTR * NST);
            int last = 0;
            unsigned it = 0u;
            unsigned long long t0 = 0ull;
            for (;;) {
                int p = LDS_LD(S.prog);
                if ((++it & 255u) == 0u) {
                    const unsigned long long t = __builtin_amdgcn_s_memrealtime();
                    if (t0 == 0ull) t0 = t;
                    else if (t - t0 > 2ull * DEADLINE_TICKS) p = NST;
                }
                const int dl = p > last ? p - last : 0;
                last += dl;
                if (last >= NST) {
                    if (L == 0) __hip_atomic_fetch_add(pc, (unsigned)dl, __ATOMIC_RELAXED, __HIP_MEMORY_SCOPE_AGENT);
                    break;
                }
                unsigned g = 0u;
                if (L == 0) g = __hip_atomic_fetch_add(pc, (unsigned)dl, __ATOMIC_RELAXED, __HIP_MEMORY_SCOPE_AGENT) + (unsigned)dl;
                const int tot = (int)(__builtin_amdgcn_readfirstlane(g) - base);
                const int lag = tot - NSTR * last;
                int win = lag > FB_T3 ? 6 : lag > FB_T2 ? 4 : lag > FB_T1 ? 3 : lag < -FB_TL ? 1 : 2;
                if (tot >= (NST - FB_END) * NSTR && win < FB_WE) win = FB_WE;
                if (L == 0 && last > 0) LDS_ST(S.win, win);
                __builtin_amdgcn_s_sleep(FB_SLEEP);
            }
        }
        {
            const unsigned* tf = (const unsigned*)(myws + (L == 0 ? OFL_OFF + (unsigned)b * OFL_STRIDE : TMO_OFF));
            const unsigned long long t0 = __builtin_amdgcn_s_memrealtime();
            bool bad = false;
            for (;;) {
                const unsigned v = __hip_atomic_load(tf, __ATOMIC_RELAXED, __HIP_MEMORY_SCOPE_SYSTEM);
                if ((unsigned)__builtin_amdgcn_readlane((int)v, 0) >= call) break;
                if ((unsigned)__builtin_amdgcn_readlane((int)v, 1) == call) { bad = true; break; }
                if (__builtin_amdgcn_s_memrealtime() - t0 > DEADLINE_TICKS) {
                    if (L == 0)
                        __hip_atomic_store((unsigned*)(myws + TMO_OFF), call, __ATOMIC_RELAXED, __HIP_MEMORY_SCOPE_AGENT);
                    bad = true;
                    break;
                }
                __builtin_amdgcn_s_sleep(SLEEP_T);
            }
            if (bad) {
                const float qn = __builtin_nanf("");
                float* o = a.out[rank] + (size_t)(32 * (b >> 5) + (L >> 3)) * NDIM + 1024 * (1 - cls) + 32 * (b & 31) + 4 * (L & 7);
#pragma unroll
                for (int u = 0; u < 4; ++u) *(f4v*)(o + (size_t)8 * u * NDIM) = f4v{qn, qn, qn, qn};
            }
            STAMP(59);
        }
        return;
    }

    raw_barrier();
    lds_wait<STAG_S>(S.rdy[0]);
    bool fwd = false;
    if (lk < 7) fwd = (lk & 1) ? feed<7>(a, S, rank, dest, s, wv, L) : feed<5>(a, S, rank, dest, s, wv, L);
    const int col0 = 1024 * cls + 32 * tj;
    {
        float vw[16], vf[16], vo[16];
        load_w<2>(a.w, 32 * wv, col0, i, hh, vw);
        load_x<2>(a.x, frow(rank, ti), 32 * wv, i, hh, vf);
        load_x<2>(a.x, MS * rank + 32 * ti, 32 * wv, i, hh, vo);
        const int ew = block_exp<16>(vw);
        const u4v w0 = pack8(vw, ew), w1 = pack8(vw + 8, ew);
        const int ef = block_exp<16>(vf);
        f16v t = mm(w0, pack8(vf, ef), ZERO16);
        t = mm(w1, pack8(vf + 8, ef), t);
        deposit(S.redF[wv], scale16(t, -(ew + ef)), i, hh);
        const int eo = block_exp<16>(vo);
        t = mm(w0, pack8(vo, eo), ZERO16);
        t = mm(w1, pack8(vo + 8, eo), t);
        deposit(S.redO[wv], scale16(t, -(ew + eo)), i, hh);
        lds_signal(S.nloc, L);
    }
    STAMP(60 + wv);
    const unsigned call = lds_wait<1>(S.call);
    const int par = (int)(call & 1u);

    if (wv == NR - 1) {
        const int rp = (L & 7) * 33 + (L >> 3);
        f4v aF[4], aO[4];
        bool bad = !lds_wait<1>(S.nloc, NR);
#pragma unroll
        for (int u = 0; u < 4; ++u) {
            aF[u] = S.redF[0][rp + 8 * u];
            aO[u] = S.redO[0][rp + 8 * u];
#pragma unroll
            for (int p = 1; p < NR; ++p) {
                aF[u] += S.redF[p][rp + 8 * u];
                aO[u] += S.redO[p][rp + 8 * u];
            }
            asm volatile("" : "+v"(aF[u]), "+v"(aO[u]));
        }
        asm volatile("s_waitcnt lgkmcnt(0)" ::: "memory");
        if (L == 0) LDS_ST(S.locdone, 1u);
        const float qn = __builtin_nanf("");
        const int oo = ((L >> 3) * NDIM + col0 + 4 * (L & 7)) * 4;
        bad |= !lds_wait<1>(S.nfor, NR - 1);
#pragma unroll
        for (int u = 0; u < 4; ++u)
#pragma unroll
            for (int p = 0; p < NR - 1; ++p) aF[u] += S.redF[p][rp + 8 * u];
        if (bad || LDS_LD(S.bad)) {
#pragma unroll
            for (int u = 0; u < 4; ++u) aF[u] = f4v{qn, qn, qn, qn};
        }
        {
            const int own = 2 * (ti >> 1) + 1 - cls;
            const __amdgpu_buffer_rsrc_t ro = __builtin_amdgcn_make_buffer_rsrc(a.out[own], 0, MS * NDIM * 4, 0x00020000);
            const int r0 = 64 * mem + 32 * (ti & 1);
            const unsigned long long tq = TNOW();
#pragma unroll
            for (int u = 0; u < 4; ++u) {
                __builtin_amdgcn_raw_buffer_store_b128(__builtin_bit_cast(u4v, aF[u]), ro, oo + (r0 + 8 * u) * NDIM * 4, 0, AUX_ST);
                if (tj < TILE_WJ) wait_vm<TILE_WIN - 1>();
            }
            wait_vm<0>();
            if (L == 0)
                __hip_atomic_store((unsigned*)(a.ws[own] + OFL_OFF + (unsigned)((2 * mem + (ti & 1)) * 32 + tj) * OFL_STRIDE), call,
                                   __ATOMIC_RELAXED, __HIP_MEMORY_SCOPE_SYSTEM);
            STAMPV(56, tq);
            STAMP(57);
        }
        bad |= !lds_wait<0>(S.nown, NR - 1);
#pragma unroll
        for (int u = 0; u < 4; ++u)
#pragma unroll
            for (int p = 0; p < NR - 1; ++p) aO[u] += S.redO[p][rp + 8 * u];
        if (bad || LDS_LD(S.bad)) {
#pragma unroll
            for (int u = 0; u < 4; ++u) aO[u] = f4v{qn, qn, qn, qn};
        }
        const __amdgpu_buffer_rsrc_t ro = __builtin_amdgcn_make_buffer_rsrc(a.out[rank], 0, MS * NDIM * 4, 0x00020000);
#pragma unroll
        for (int u = 0; u < 4; ++u)
            __builtin_amdgcn_raw_buffer_store_b128(__builtin_bit_cast(u4v, aO[u]), ro, oo + (32 * ti + 8 * u) * NDIM * 4, 0, AUX_ST);
        STAMP(58);
        return;
    }

    const __amdgpu_buffer_rsrc_t rs = __builtin_amdgcn_make_buffer_rsrc(myws, 0, (int)WS_NEED, 0x00020000);
    const unsigned long long t0 = __builtin_amdgcn_s_memrealtime();
    bool bad = call == 0u;
    unsigned npoll = 0u;
    auto late = [&]() -> bool {
        if (__builtin_amdgcn_s_memrealtime() - t0 > DEADLINE_TICKS) {
            if (L == 0)
                __hip_atomic_store((unsigned*)(myws + TMO_OFF), call, __ATOMIC_RELAXED, __HIP_MEMORY_SCOPE_AGENT);
            return true;
        }
        return false;
    };
    if (fwd && !bad) {
        const Ent en = entry<7>(rank, dest, wv, s);
        const int pf = (int)(L == 0 ? flag_of(par, en.src, en.u) : TMO_OFF);
        u2v pv;
        for (;;) {
            pv = __builtin_amdgcn_raw_buffer_load_b64(rs, pf, 0, AUX_LD);
            if ((unsigned)__builtin_amdgcn_readlane((int)pv[1], 0) >= call) break;
            if ((unsigned)__builtin_amdgcn_readlane((int)pv[0], 1) == call || late()) { bad = true; break; }
            __builtin_amdgcn_s_sleep(SLEEP_R);
            asm volatile("" ::: "memory");
        }
        asm volatile("" ::: "memory");
        const Quad qd = load_quad(rs, (int)(data_of(par, en.src, en.u) + (unsigned)L * 16u));
        S.stage[wv][L] = QF(qd, 0);
        S.stage[wv][64 + L] = QF(qd, 1);
        S.stage[wv][128 + L] = QF(qd, 2);
        S.stage[wv][192 + L] = QF(qd, 3);
        if (L == 0) S.exp_[wv] = (int)pv[0];
        asm volatile("s_waitcnt lgkmcnt(0)" ::: "memory");
        if (L == 0) LDS_ST(S.rdy[wv], 1);
        asm volatile("" ::: "memory");
        STAMP(70 + wv);
    }
    const int src = (rank + 1 + wv) & (NR - 1);
    const int oxf = (int)(data_of(par, src, 4 * ti) + (unsigned)L * 16u);
    const int ow = (int)(data_of(par, src, 32 + 4 * tj) + (unsigned)L * 16u);
    const int oxo = (int)(data_of(par, src, 160 + 4 * ti) + (unsigned)L * 16u);
    const int poff1 = (int)(L < 4 ? flag_of(par, src, 4 * ti + L)
                            : L < 8 ? flag_of(par, src, 32 + 4 * tj + L - 4)
                            : L < 12 ? flag_of(par, src, 160 + 4 * ti + L - 8) : TMO_OFF);
    const int poff0 = (int)(L < 12 ? (unsigned)poff1 : flag_of(par, src, 4 * ti) + 64u);
    unsigned lo = 0u, m = 0u;
    auto poll = [&]() {
        ++npoll;
        const u2v pv = __builtin_amdgcn_raw_buffer_load_b64(rs, (npoll % TMO_EVERY) == 0u ? poff1 : poff0, 0, AUX_LD);
        lo = pv[0];
        m = (unsigned)__builtin_amdgcn_ballot_w64(pv[1] >= call) & 0xfffu;
        if ((unsigned)__builtin_amdgcn_readlane((int)lo, 16) == call) bad = true;
    };
    Quad X[4], W[4];
    int ex[4], ew[4];
    for (poll(); !bad && (m & 0xfu) != 0xfu; poll()) {
        if (late()) { bad = true; break; }
        __builtin_amdgcn_s_sleep(SLEEP_E);
        asm volatile("" ::: "memory");
    }
    asm volatile("" ::: "memory");
    STAMP(16 + wv);
#pragma unroll
    for (int k = 0; k < 4; ++k) {
        X[k] = load_quad<AUX_XF>(rs, oxf + 1024 * k);
        ex[k] = __builtin_amdgcn_readlane((int)lo, k);
    }
    for (; !bad && (m & 0xf0u) != 0xf0u; poll()) {
        if (late()) { bad = true; break; }
        __builtin_amdgcn_s_sleep(SLEEP_W);
        asm volatile("" ::: "memory");
    }
    asm volatile("" ::: "memory");
    STAMP(24 + wv);
#pragma unroll
    for (int k = 0; k < 4; ++k) {
        W[k] = load_quad<AUX_W>(rs, ow + 1024 * k);
        ew[k] = __builtin_amdgcn_readlane((int)lo, 4 + k);
    }
    __builtin_amdgcn_sched_barrier(0);
    f16v acc = scale16(mm_quad(W[0], X[0]), -(ew[0] + ex[0]));
    pin(acc);
#pragma unroll
    for (int k = 1; k < 4; ++k) { fma16(acc, mm_quad(W[k], X[k]), -(ew[k] + ex[k])); pin(acc); }
    if (!lds_wait<1>(S.locdone)) bad = true;
    if (bad && L == 0) LDS_ST(S.bad, 1u);
    deposit(S.redF[wv], acc, i, hh);
    lds_signal(S.nfor, L);
    STAMP(32 + wv);
    for (poll(); !bad && (m & 0xf00u) != 0xf00u; poll()) {
        if (late()) { bad = true; break; }
        __builtin_amdgcn_s_sleep(SLEEP_O);
        asm volatile("" ::: "memory");
    }
    STAMP(40 + wv);
#pragma unroll
    for (int k = 0; k < 4; ++k) ex[k] = __builtin_amdgcn_readlane((int)lo, 8 + k);
    for (bool first = true;; first = false) {
        asm volatile("" ::: "memory");
        if (AUX_XO != AUX_LD && first) {
#pragma unroll
            for (int k = 0; k < 4; ++k) X[k] = load_quad<AUX_XO>(rs, oxo + 1024 * k);
        } else {
#pragma unroll
            for (int k = 0; k < 4; ++k) X[k] = load_quad(rs, oxo + 1024 * k);
        }
        __builtin_amdgcn_sched_barrier(0);
        const bool ok = quad_valid(X[0]) & quad_valid(X[1]) & quad_valid(X[2]) & quad_valid(X[3]);
        if (ok || bad) break;
        if (late()) { bad = true; break; }
    }
    __builtin_amdgcn_sched_barrier(0);
    acc = scale16(mm_quad(W[0], X[0]), -(ew[0] + ex[0]));
    pin(acc);
#pragma unroll
    for (int k = 1; k < 4; ++k) { fma16(acc, mm_quad(W[k], X[k]), -(ew[k] + ex[k])); pin(acc); }
    if (bad && L == 0) LDS_ST(S.bad, 1u);
    deposit(S.redO[wv], acc, i, hh);
    lds_signal(S.nown, L);
    STAMP(48 + wv);
}

extern "C" void kernel_launch(int rank, int nranks, ncclComm_t comm,
                              hipStream_t stream,
                              void* const* d_in, const int* in_sizes, int n_in,
                              void* d_out, int out_size,
                              const int* in_dtypes, int out_dtype,
                              void* const* d_out_all,
                              void* const* d_ws_all, size_t ws_bytes) {
    if (nranks != NR || n_in < 2 || ws_bytes < (size_t)WS_NEED || out_size != MS * NDIM) {
        fprintf(stderr, "unexpected problem geometry\n");
        return;
    }
    Args a;
    a.x = (const float*)d_in[0];
    a.w = (const float*)d_in[1];
    for (int p = 0; p < NR; ++p) { a.ws[p] = (char*)d_ws_all[p]; a.out[p] = (float*)d_out_all[p]; }
    a.out[rank] = (float*)d_out;
    a.rank = rank;
    gemm_rs_kernel<<<NWG, 640, 0, stream>>>(a);
    HIP_CHECK(hipGetLastError());
}
```

```cpp
#include <hip/hip_runtime.h>
#include <rccl/rccl.h>
#include <cstdio>
#include <cstdint>

#define HIP_CHECK(expr) do { hipError_t _e = (expr); if (_e != hipSuccess) { \
    fprintf(stderr, "HIP error %s at %s:%d\n", hipGetErrorString(_e), __FILE__, __LINE__); \
    return; } } while (0)

typedef _Float16 h8v __attribute__((ext_vector_type(8)));
typedef float f16v __attribute__((ext_vector_type(16)));
typedef float f4v __attribute__((ext_vector_type(4)));
typedef float f2v __attribute__((ext_vector_type(2)));
typedef unsigned u4v __attribute__((ext_vector_type(4)));
typedef unsigned u2v __attribute__((ext_vector_type(2)));

namespace {

constexpr int NR = 8;
constexpr int NDIM = 2048;
constexpr int KS = 256;
constexpr int MS = 256;
constexpr int NWG = 256;
constexpr int NSTR = 32;
constexpr int NPC = 48;
constexpr int RMAX = 7;
constexpr unsigned F_STR = 33 * 1024, R_STR = 4 * F_STR, S_STR = 6 * R_STR;
constexpr unsigned DATA_BYTES = 2u * NR * S_STR;
constexpr unsigned FLAG_STRIDE = 128;
constexpr unsigned FLAG_OFF = DATA_BYTES;
constexpr unsigned FLAG_BYTES = 2u * NR * NPC * FLAG_STRIDE;
constexpr unsigned OFL_OFF = FLAG_OFF + FLAG_BYTES;
constexpr unsigned OFL_STRIDE = 64;
constexpr unsigned CTR_OFF = OFL_OFF + NWG * OFL_STRIDE;
constexpr unsigned CTR_STRIDE = 64;
constexpr unsigned TMO_OFF = CTR_OFF + NWG * CTR_STRIDE;
constexpr unsigned PRG_OFF = TMO_OFF + 128;
constexpr unsigned WS_NEED = PRG_OFF + NR * 128;
#define DLT 50000000ull
constexpr unsigned long long DEADLINE_TICKS = DLT;

#define INFL 2
#define SLEEP_N 1
#define FB_T1 50
#define FB_T2 110
#define FB_T3 200
#define FB_TL 60
#define FB_W0 2
#define FB_END 6
#define FB_WE 4
#define SLEEP_E 16
#define SLEEP_W 8
#define SLEEP_O 2
#define SLEEP_T 2
#define SLEEP_R 2
#define TMO_EVERY 16
#define FB_SLEEP 4
#define STAG_S 1
#define EVS 64
#define EVC -32
#define GPD 16
#define RGD 3
#define LEAD_W 2
#define LEAD_O 2
#define ZAT 4
#define PREFETCH 1
#define TILE_WIN 1
#define TILE_WJ 12
constexpr int GP = GPD, RG = RGD, QREL = (32 / GP) * RG * 4, FH = QREL / 4;
constexpr int LS = 192 + QREL, LC = 192 - QREL;
static_assert(LS <= 32 * RMAX && QREL <= 32 && QREL > 0, "stream geometry");
#define AUX_ST 17
#define AUX_LD 17

#define STAMP(idx) do {} while (0)
#define STAMPV(idx, v) do {} while (0)
#define TNOW() 0ull

#define LDS_LD(v) __builtin_amdgcn_readfirstlane(__hip_atomic_load(&(v), __ATOMIC_RELAXED, __HIP_MEMORY_SCOPE_WORKGROUP))
#define LDS_ST(v, x) __hip_atomic_store(&(v), (x), __ATOMIC_RELAXED, __HIP_MEMORY_SCOPE_WORKGROUP)

struct Args {
    const float* x;
    const float* w;
    char* ws[NR];
    float* out[NR];
    int rank;
};

__device__ __forceinline__ unsigned data_of(int par, int src, int u) {
    return (unsigned)(par * NR + src) * S_STR + (unsigned)(u >> 5) * R_STR + (unsigned)(u & 31) * 1024u;
}
__device__ __forceinline__ unsigned flag_of(int par, int src, int u) {
    return FLAG_OFF + (unsigned)((par * NR + src) * NPC + (u >> 2)) * FLAG_STRIDE + (unsigned)(u & 3) * 8u;
}
__device__ __forceinline__ int frow(int r, int i) {
    return MS * (2 * (i >> 1) + 1 - (r & 1)) + 64 * (r >> 1) + 32 * (i & 1);
}

struct Ent { int kind, src, u, a0, kq; };
__device__ __forceinline__ int rounds(int typ, int s) {
    return typ == 2 ? (s < QREL ? 1 : 0) : ((typ ? LS : LC) - s + 31) >> 5;
}
__device__ __forceinline__ Ent entry(int typ, int rank, int dest, int rho, int s) {
    Ent e;
    e.src = rank;
    if (typ == 2) {
        const int x = s % FH, j = GP * (x / RG) + GP - RG + x % RG;
        e.kind = 1; e.kq = dest >> 1; e.a0 = 1024 * (dest & 1) + 32 * j; e.u = 32 + 4 * j + e.kq;
        return e;
    }
    if (rho == 0) { e.kind = 0; e.kq = s & 3; e.a0 = frow(dest, s >> 2); e.u = s; return e; }
    const int n = 32 * (rho - 1) + s, o = n - (typ ? LS - 64 : LC - 64);
    if (o >= 0) { e.kind = 0; e.kq = o & 3; e.a0 = MS * dest + 32 * (o >> 2); e.u = 160 + o; return e; }
    constexpr int GS = 4 * (GP + RG), GC = 4 * (GP - RG);
    const int g = typ ? n / GS : n / GC, t = n - g * (typ ? GS : GC);
    if (t < 4 * GP) {
        const int j = GP * g + (t >> 2);
        e.kind = 1; e.kq = t & 3; e.a0 = 1024 * (dest & 1) + 32 * j; e.u = 32 + 4 * j + e.kq;
    } else {
        const int r = t - 4 * GP;
        e.kind = 2; e.kq = rank >> 1; e.a0 = 0; e.src = 2 * (r & 3) + 1 - (rank & 1);
        e.u = 32 + 4 * (GP * g + GP - RG + (r >> 2)) + e.kq;
    }
    return e;
}
__device__ __forceinline__ int first_early(int typ, int s) {
    return typ == 1 ? (LS - 32 - EVS - s + 31) >> 5 : typ == 0 ? (LC - 32 - (EVC) - s + 31) >> 5 : 1;
}

template <int NV>
__device__ __forceinline__ int block_exp(const float (&v)[NV]) {
    float m = 0.f;
#pragma unroll
    for (int t = 0; t < NV; ++t) m = fmaxf(m, fabsf(v[t]));
    unsigned u = __float_as_uint(m);
    u = max(u, (unsigned)__builtin_amdgcn_update_dpp(0, (int)u, 0xB1, 0xf, 0xf, true));
    u = max(u, (unsigned)__builtin_amdgcn_update_dpp(0, (int)u, 0x4E, 0xf, 0xf, true));
    u = max(u, (unsigned)__builtin_amdgcn_update_dpp(0, (int)u, 0x141, 0xf, 0xf, true));
    u = max(u, (unsigned)__builtin_amdgcn_update_dpp(0, (int)u, 0x140, 0xf, 0xf, true));
    const unsigned r = max(max((unsigned)__builtin_amdgcn_readlane((int)u, 0), (unsigned)__builtin_amdgcn_readlane((int)u, 16)),
                           max((unsigned)__builtin_amdgcn_readlane((int)u, 32), (unsigned)__builtin_amdgcn_readlane((int)u, 48)));
    unsigned E = r >> 23;
    E = E < 15u ? 15u : E;
    return 141 - (int)E;
}

__device__ __forceinline__ u4v pack8(const float* v, int e) {
    h8v r;
    typedef _Float16 h2v __attribute__((ext_vector_type(2)));
    const float sc = __uint_as_float((unsigned)(e + 127) << 23);
    const f2v s2 = {sc, sc};
#pragma unroll
    for (int j = 0; j < 8; j += 2) {
        const h2v t = __builtin_convertvector(f2v{v[j], v[j + 1]} * s2, h2v);
        r[j] = t[0];
        r[j + 1] = t[1];
    }
    return __builtin_bit_cast(u4v, r);
}

template <int NF>
__device__ __forceinline__ void load_w(const float* __restrict__ w, int k0, int col0, int i, int hh, float (&v)[8 * NF]) {
    const float* base = w + (size_t)(k0 + 8 * hh) * NDIM + col0 + i;
#pragma unroll
    for (int s = 0; s < NF; ++s)
#pragma unroll
        for (int j = 0; j < 8; ++j) v[8 * s + j] = base[(size_t)(16 * s + j) * NDIM];
}

template <int NF>
__device__ __forceinline__ void load_x(const float* __restrict__ x, int row0, int k0, int i, int hh, float (&v)[8 * NF]) {
    const float* base = x + (size_t)(row0 + i) * KS + k0 + 8 * hh;
#pragma unroll
    for (int s = 0; s < NF; ++s) {
        f4v t0 = *(const f4v*)(base + 16 * s);
        f4v t1 = *(const f4v*)(base + 16 * s + 4);
#pragma unroll
        for (int j = 0; j < 4; ++j) { v[s * 8 + j] = t0[j]; v[s * 8 + 4 + j] = t1[j]; }
    }
}

__device__ __forceinline__ f16v mm(u4v a, u4v b, f16v c) {
    return __builtin_amdgcn_mfma_f32_32x32x16_f16(__builtin_bit_cast(h8v, a), __builtin_bit_cast(h8v, b), c, 0, 0, 0);
}
#define ZERO16 f16v{0.f, 0.f, 0.f, 0.f, 0.f, 0.f, 0.f, 0.f, 0.f, 0.f, 0.f, 0.f, 0.f, 0.f, 0.f, 0.f}

__device__ __forceinline__ f16v scale16(f16v t, int e) {
    const int e1 = e >> 1, e2 = e - e1;
    const float c1 = __uint_as_float((unsigned)(e1 + 127) << 23), c2 = __uint_as_float((unsigned)(e2 + 127) << 23);
    const f2v s1 = {c1, c1}, s2 = {c2, c2};
#pragma unroll
    for (int r = 0; r < 16; r += 2) {
        const f2v u = (f2v{t[r], t[r + 1]} * s1) * s2;
        t[r] = u[0]; t[r + 1] = u[1];
    }
    return t;
}

__device__ __forceinline__ void fma16(f16v& acc, f16v t, int e) {
    const int e1 = e >> 1, e2 = e - e1;
    const float c1 = __uint_as_float((unsigned)(e1 + 127) << 23), c2 = __uint_as_float((unsigned)(e2 + 127) << 23);
    const f2v s1 = {c1, c1}, s2 = {c2, c2};
#pragma unroll
    for (int r = 0; r < 16; r += 2) {
        const f2v u = __builtin_elementwise_fma(f2v{t[r], t[r + 1]} * s1, s2, f2v{acc[r], acc[r + 1]});
        acc[r] = u[0]; acc[r + 1] = u[1];
    }
}

typedef unsigned u8v __attribute__((ext_vector_type(8)));
typedef unsigned u16v __attribute__((ext_vector_type(16)));
typedef u16v Quad;
#define QF(q, f) __builtin_shufflevector(q, q, 4 * (f), 4 * (f) + 1, 4 * (f) + 2, 4 * (f) + 3)

#define AUX_XF AUX_LD
#define AUX_W AUX_LD
#define AUX_XO AUX_LD
template <int AUX = AUX_LD>
__device__ __forceinline__ Quad load_quad(__amdgpu_buffer_rsrc_t rs, int off) {
    const u4v f0 = __builtin_amdgcn_raw_buffer_load_b128(rs, off, 0, AUX);
    const u4v f1 = __builtin_amdgcn_raw_buffer_load_b128(rs, off, (int)F_STR, AUX);
    const u4v f2 = __builtin_amdgcn_raw_buffer_load_b128(rs, off, 2 * (int)F_STR, AUX);
    const u4v f3 = __builtin_amdgcn_raw_buffer_load_b128(rs, off, 3 * (int)F_STR, AUX);
    const u8v l = __builtin_shufflevector(f0, f1, 0, 1, 2, 3, 4, 5, 6, 7);
    const u8v h = __builtin_shufflevector(f2, f3, 0, 1, 2, 3, 4, 5, 6, 7);
    return __builtin_shufflevector(l, h, 0, 1, 2, 3, 4, 5, 6, 7, 8, 9, 10, 11, 12, 13, 14, 15);
}

__device__ __forceinline__ f16v mm_quad(const Quad& w, const Quad& x) {
    f16v t = mm(QF(w, 0), QF(x, 0), ZERO16);
    t = mm(QF(w, 1), QF(x, 1), t);
    t = mm(QF(w, 2), QF(x, 2), t);
    return mm(QF(w, 3), QF(x, 3), t);
}

__device__ __forceinline__ void pin(f16v& v) {
    asm volatile("" : "+v"(v));
    __builtin_amdgcn_sched_barrier(0);
}

__device__ __forceinline__ u4v no_zero(u4v u) {
    typedef unsigned short us2 __attribute__((ext_vector_type(2)));
#pragma unroll
    for (int j = 0; j < 4; ++j) {
        const unsigned v = u[j];
        const us2 d2 = __builtin_bit_cast(us2, v) - us2{1, 1};
        const unsigned d = __builtin_bit_cast(unsigned, d2);
        u[j] = v | ((d & ~v) & 0x80008000u);
    }
    return u;
}

__device__ __forceinline__ bool quad_valid(const Quad& q) {
    typedef unsigned short us8 __attribute__((ext_vector_type(8)));
    us8 m = __builtin_elementwise_min(__builtin_bit_cast(us8, QF(q, 0)), __builtin_bit_cast(us8, QF(q, 1)));
    m = __builtin_elementwise_min(m, __builtin_bit_cast(us8, QF(q, 2)));
    m = __builtin_elementwise_min(m, __builtin_bit_cast(us8, QF(q, 3)));
    typedef unsigned short us2 __attribute__((ext_vector_type(2)));
    us2 r = __builtin_elementwise_min(__builtin_elementwise_min(us2{m[0], m[1]}, us2{m[2], m[3]}),
                                      __builtin_elementwise_min(us2{m[4], m[5]}, us2{m[6], m[7]}));
    return __builtin_amdgcn_ballot_w64(r[0] == 0 || r[1] == 0) == 0ull;
}

template <int N>
__device__ __forceinline__ void wait_vm() {
    asm volatile("s_waitcnt vmcnt(%0)" ::"i"(N) : "memory");
}

__device__ __forceinline__ void wait_vm_n(int n) {
    switch (n) {
        case 0: wait_vm<0>(); break;
        case 1: wait_vm<1>(); break;
        case 2: wait_vm<2>(); break;
        case 3: wait_vm<3>(); break;
        case 4: wait_vm<4>(); break;
        case 5: wait_vm<5>(); break;
        case 6: wait_vm<6>(); break;
        case 7: wait_vm<7>(); break;
        case 8: wait_vm<8>(); break;
        case 9: wait_vm<9>(); break;
        default: wait_vm<10>(); break;
    }
}

__device__ __forceinline__ void raw_barrier() {
    asm volatile("s_waitcnt lgkmcnt(0)\n\ts_barrier" ::: "memory");
}

template <int SLP, class T>
__device__ __forceinline__ unsigned lds_wait(T& v, unsigned n = 1u) {
    unsigned r, it = 0u;
    unsigned long long t0 = 0ull;
    while ((r = (unsigned)LDS_LD(v)) < n) {
        __builtin_amdgcn_s_sleep(SLP);
        if ((++it & 1023u) == 0u) {
            const unsigned long long t = __builtin_amdgcn_s_memrealtime();
            if (t0 == 0ull) t0 = t;
            else if (t - t0 > 2ull * DEADLINE_TICKS) { r = 0u; break; }
        }
    }
    asm volatile("" ::: "memory");
    return r;
}

struct Lds {
    u4v stage[RMAX][256];
    f4v redF[NR][8 * 33];
    f4v redO[NR][8 * 33];
    int exp_[RMAX + 1];
    int rdy[RMAX + 1];
    unsigned call, bad, nloc, locdone, nfor, nown;
    int prog, win, zok;
};

__device__ __forceinline__ void deposit(f4v* slot, const f16v& t, int i, int hh) {
#pragma unroll
    for (int g = 0; g < 4; ++g)
        slot[(2 * g + hh) * 33 + i] = f4v{t[4 * g], t[4 * g + 1], t[4 * g + 2], t[4 * g + 3]};
}
__device__ __forceinline__ void lds_signal(unsigned& c, int L) {
    asm volatile("s_waitcnt lgkmcnt(0)" ::: "memory");
    if (L == 0) __hip_atomic_fetch_add(&c, 1u, __ATOMIC_RELAXED, __HIP_MEMORY_SCOPE_WORKGROUP);
}

__device__ __forceinline__ void pusher(const Args& a, Lds& S, int b, int rank, int dest, int s, int L, int typ) {
    const int nrnd = rounds(typ, s), NST = 4 * nrnd;
    char* const myws = a.ws[rank];
    const unsigned long long tq0 = TNOW();
    const unsigned cv = __hip_atomic_load((unsigned*)(myws + CTR_OFF + b * CTR_STRIDE), __ATOMIC_RELAXED,
                                          __HIP_MEMORY_SCOPE_AGENT);
    char* dws = a.ws[dest];
    const int i = L & 31, hh = L >> 5;
    u4v t[4];
    unsigned vdat, vflg;
    {
        const Ent en = entry(typ, rank, dest, 0, s);
        float v0[32];
        if (typ == 2) load_w<4>(a.w, 64 * en.kq, en.a0, i, hh, v0);
        else load_x<4>(a.x, en.a0, 64 * en.kq, i, hh, v0);
        asm volatile("" ::"s"(dws));
        const Ent le = entry(typ, rank, dest, L < nrnd ? L : 0, s);
        vdat = data_of(0, le.src, le.u);
        vflg = flag_of(0, le.src, le.u);
        asm volatile("" : "+v"(vdat), "+v"(vflg));
        if (L <= RMAX) S.rdy[L] = 0;
        if (L == 0) {
            S.call = 0u; S.bad = 0u; S.nloc = 0u; S.locdone = 0u; S.nfor = 0u; S.nown = 0u;
            S.prog = 0; S.zok = 0; S.win = typ == 2 ? 6 : INFL;
        }
        raw_barrier();
        const int e = block_exp<32>(v0);
        if (L == 0) { LDS_ST(S.rdy[0], 1); S.exp_[0] = e; }
#pragma unroll
        for (int f = 0; f < 4; ++f) t[f] = pack8(v0 + 8 * f, e);
        if (typ == 2) {
#pragma unroll
            for (int f = 0; f < 4; ++f) t[f] = no_zero(t[f]);
        }
    }
    const unsigned call = __builtin_amdgcn_readfirstlane(cv) + 1u;
    if (L == 0) LDS_ST(S.call, call);
    if (nrnd == 0) return;
    const int par = (int)(call & 1u);
    __builtin_amdgcn_s_setprio(3);
    const __amdgpu_buffer_rsrc_t rs = __builtin_amdgcn_make_buffer_rsrc(dws, 0, (int)DATA_BYTES, 0x00020000);
    vdat += (unsigned)par * (NR * S_STR);
    vflg += (unsigned)par * (NR * NPC * FLAG_STRIDE);
    auto publish = [&](int c, unsigned early) {
        if (L == c) {
            const unsigned long long val = ((unsigned long long)call << 32) | (((unsigned)S.exp_[c] & 0xffffu) | early);
            __hip_atomic_store((unsigned long long*)(dws + vflg), val, __ATOMIC_RELAXED, __HIP_MEMORY_SCOPE_SYSTEM);
        }
    };
    int issued = 0, pub = 0, dn = 0, h1 = 0, h2 = 0, h3 = 0, h4 = 0, h5 = 0, h6 = 0, h7 = 0;
    const int ev = first_early(typ, s);
    for (int c = 0;; ++c) {
        const int so = __builtin_amdgcn_readlane((int)vdat, c);
        bool have = false;
#pragma unroll
        for (int f = 0; f < 4; ++f) {
            const int n = 4 * c + f;
            __builtin_amdgcn_raw_buffer_store_b128(t[f], rs, L * 16, so + f * (int)F_STR, AUX_ST);
            const int h0 = ++issued;
            __builtin_amdgcn_sched_barrier(0);
            if (c >= ev && f == (c + 1 == nrnd ? LEAD_O : LEAD_W)) {
                if (c + 1 == nrnd && !lds_wait<1>(S.zok)) {
                    if (L == 0) { LDS_ST(S.bad, 1u); LDS_ST(S.prog, NST); }
                    return;
                }
                publish(c, 0x10000u); ++issued;
            }
            if (L == 0) LDS_ST(S.prog, n + 1 == NST ? NST : dn);
            while (pub < ev && 4 * (pub + 1) <= dn) { publish(pub, 0u); ++issued; ++pub; }
            if (f == 3) {
                if (n + 1 == NST) break;
                have = LDS_LD(S.rdy[c + 1]) != 0;
                asm volatile("" ::: "memory");
                if (have) {
#pragma unroll
                    for (int g = 0; g < 4; ++g) t[g] = S.stage[c + 1][g * 64 + L];
                }
            }
            const int win = LDS_LD(S.win);
            const int ik = win <= 1 ? h0 : win == 2 ? h1 : win == 3 ? h2 : win == 4 ? h3 : win <= 6 ? h5 : h7;
            const int k1 = n + 2 - (win > 6 ? 8 : win > 4 ? 6 : win);
            if (k1 > dn) {
                wait_vm_n(issued - ik);
                dn = k1;
            }
            h7 = h6; h6 = h5; h5 = h4; h4 = h3; h3 = h2; h2 = h1; h1 = h0;
        }
        if (c + 1 == nrnd) break;
        if (!have) {
            while (pub < ev && 4 * (pub + 1) <= dn) { publish(pub, 0u); ++issued; ++pub; }
            if (!lds_wait<0>(S.rdy[c + 1])) {
                if (L == 0) { LDS_ST(S.bad, 1u); LDS_ST(S.prog, NST); }
                return;
            }
#pragma unroll
            for (int g = 0; g < 4; ++g) t[g] = S.stage[c + 1][g * 64 + L];
        }
    }
    const unsigned long long tq1 = TNOW();
    if (pub < ev && pub == nrnd - 3) { wait_vm_n(issued - h7); publish(pub, 0u); ++issued; ++pub; }
    if (pub < ev && pub == nrnd - 2) { wait_vm_n(issued - h4); publish(pub, 0u); ++pub; }
    if (pub < ev) { wait_vm<0>(); publish(pub, 0u); }
}

__device__ __forceinline__ bool feed(const Args& a, Lds& S, int rank, int dest, int s, int wv, int L, int typ) {
    if (wv < 1 || wv >= rounds(typ, s)) return false;
    const Ent en = entry(typ, rank, dest, wv, s);
    if (en.kind == 2) return true;
    const int i = L & 31, hh = L >> 5;
    float v[32];
    if (en.kind == 1) load_w<4>(a.w, 64 * en.kq, en.a0, i, hh, v);
    else load_x<4>(a.x, en.a0, 64 * en.kq, i, hh, v);
    const int e = block_exp<32>(v);
#pragma unroll
    for (int f = 0; f < 4; ++f) {
        S.stage[wv][f * 64 + L] = no_zero(pack8(v + 8 * f, e));
    }
    if (L == 0) S.exp_[wv] = e;
    asm volatile("s_waitcnt lgkmcnt(0)" ::: "memory");
    if (L == 0) LDS_ST(S.rdy[wv], 1);
    asm volatile("" ::: "memory");
    return false;
}

}

__global__ void __launch_bounds__(640) gemm_rs_kernel(Args a) {
    __shared__ __attribute__((aligned(16))) Lds S;

    const int b = blockIdx.x;
    const int wv = __builtin_amdgcn_readfirstlane(threadIdx.x >> 6);
    const int L = threadIdx.x & 63;
    const int i = L & 31, hh = L >> 5;
    const int rank = a.rank & (NR - 1);
    const int cls = rank & 1, mem = rank >> 1;
    const int ti = b & 7, tj = b >> 3;
    const int lk = b >> 5, s = b & 31;
    const int dest = lk < 7 ? (rank + 1 + lk) & (NR - 1) : 2 * ((s / FH) & 3) + 1 - cls;
    const int typ = lk == 7 ? 2 : lk & 1;
    char* const myws = a.ws[rank];

    if (wv == NR) {
        pusher(a, S, b, rank, dest, s, L, typ);
        return;
    }

    if (wv == NR + 1) {
        raw_barrier();
        const unsigned call = lds_wait<1>(S.call);
        if (call == 0u) return;
        if (L == 0)
            __hip_atomic_store((unsigned*)(myws + CTR_OFF + b * CTR_STRIDE), call, __ATOMIC_RELAXED, __HIP_MEMORY_SCOPE_AGENT);
        if (lk < 7) {
            const int nrnd = rounds(typ, s), NST = 4 * nrnd, LT = typ ? 4 * LS : 4 * LC;
            int z = 0;
            auto clear = [&]() {
                const __amdgpu_buffer_rsrc_t rz = __builtin_amdgcn_make_buffer_rsrc(myws, 0, (int)DATA_BYTES, 0x00020000);
                for (int c = first_early(typ, s); c < nrnd; ++c) {
                    const Ent en = entry(typ, dest, rank, c, s);
                    const unsigned zo = data_of((int)(call & 1u) ^ 1, en.src, en.u);
#pragma unroll
                    for (int f = 0; f < 4; ++f)
                        __builtin_amdgcn_raw_buffer_store_b128(u4v{0u, 0u, 0u, 0u}, rz, L * 16, (int)(zo + f * F_STR), AUX_ST);
                }
                z = 1;
            };
            unsigned* pc = (unsigned*)(myws + PRG_OFF + (unsigned)dest * 128u);
            const unsigned base = (call - 1u) * (unsigned)LT;
            int last = 0;
            unsigned it = 0u;
            unsigned long long t0 = 0ull;
            for (;;) {
                int p = LDS_LD(S.prog);
                if ((++it & 255u) == 0u) {
                    const unsigned long long t = __builtin_amdgcn_s_memrealtime();
                    if (t0 == 0ull) t0 = t;
                    else if (t - t0 > 2ull * DEADLINE_TICKS) p = NST;
                }
                const int dl = p > last ? p - last : 0;
                last += dl;
                if (last >= NST) {
                    if (L == 0) __hip_atomic_fetch_add(pc, (unsigned)dl, __ATOMIC_RELAXED, __HIP_MEMORY_SCOPE_AGENT);
                    break;
                }
                if (z == 0 && last >= ZAT) clear();
                unsigned g = 0u;
                if (L == 0) g = __hip_atomic_fetch_add(pc, (unsigned)dl, __ATOMIC_RELAXED, __HIP_MEMORY_SCOPE_AGENT) + (unsigned)dl;
                const int tot = (int)(__builtin_amdgcn_readfirstlane(g) - base);
                if (z == 1) {
                    wait_vm<0>();
                    if (L == 0) LDS_ST(S.zok, 1);
                    z = 2;
                }
                const int lag = tot - NSTR * last;
                int win = lag > FB_T3 ? 6 : lag > FB_T2 ? 4 : lag > FB_T1 ? 3 : lag < -FB_TL ? 1 : 2;
                if (tot >= LT - FB_END * NSTR && win < FB_WE) win = FB_WE;
                if (L == 0 && last > 0) LDS_ST(S.win, win);
                __builtin_amdgcn_s_sleep(FB_SLEEP);
            }
            if (z == 0) clear();
            if (z == 1) {
                wait_vm<0>();
                if (L == 0) LDS_ST(S.zok, 1);
            }
        }
        {
            const unsigned* tf = (const unsigned*)(myws + (L == 0 ? OFL_OFF + (unsigned)b * OFL_STRIDE : TMO_OFF));
            const unsigned long long t0 = __builtin_amdgcn_s_memrealtime();
            bool bad = false;
            for (;;) {
                const unsigned v = __hip_atomic_load(tf, __ATOMIC_RELAXED, __HIP_MEMORY_SCOPE_SYSTEM);
                if ((unsigned)__builtin_amdgcn_readlane((int)v, 0) >= call) break;
                if ((unsigned)__builtin_amdgcn_readlane((int)v, 1) == call) { bad = true; break; }
                if (__builtin_amdgcn_s_memrealtime() - t0 > DEADLINE_TICKS) {
                    if (L == 0)
                        __hip_atomic_store((unsigned*)(myws + TMO_OFF), call, __ATOMIC_RELAXED, __HIP_MEMORY_SCOPE_AGENT);
                    bad = true;
                    break;
                }
                __builtin_amdgcn_s_sleep(SLEEP_T);
            }
            if (bad) {
                const float qn = __builtin_nanf("");
                float* o = a.out[rank] + (size_t)(32 * (b >> 5) + (L >> 3)) * NDIM + 1024 * (1 - cls) + 32 * (b & 31) + 4 * (L & 7);
#pragma unroll
                for (int u = 0; u < 4; ++u) *(f4v*)(o + (size_t)8 * u * NDIM) = f4v{qn, qn, qn, qn};
            }
            STAMP(59);
        }
        return;
    }

    raw_barrier();
    lds_wait<STAG_S>(S.rdy[0]);
    const bool fwd = feed(a, S, rank, dest, s, wv, L, typ);
    const int col0 = 1024 * cls + 32 * tj;
    {
        float vw[16], vf[16], vo[16];
        load_w<2>(a.w, 32 * wv, col0, i, hh, vw);
        load_x<2>(a.x, frow(rank, ti), 32 * wv, i, hh, vf);
        load_x<2>(a.x, MS * rank + 32 * ti, 32 * wv, i, hh, vo);
        const int ew = block_exp<16>(vw);
        const u4v w0 = pack8(vw, ew), w1 = pack8(vw + 8, ew);
        const int ef = block_exp<16>(vf);
        f16v t = mm(w0, pack8(vf, ef), ZERO16);
        t = mm(w1, pack8(vf + 8, ef), t);
        deposit(S.redF[wv], scale16(t, -(ew + ef)), i, hh);
        const int eo = block_exp<16>(vo);
        t = mm(w0, pack8(vo, eo), ZERO16);
        t = mm(w1, pack8(vo + 8, eo), t);
        deposit(S.redO[wv], scale16(t, -(ew + eo)), i, hh);
        lds_signal(S.nloc, L);
    }
    STAMP(60 + wv);
    const unsigned call = lds_wait<1>(S.call);
    const int par = (int)(call & 1u);

    if (wv == NR - 1) {
        const int rp = (L & 7) * 33 + (L >> 3);
        f4v aF[4], aO[4];
        bool bad = !lds_wait<1>(S.nloc, NR);
#pragma unroll
        for (int u = 0; u < 4; ++u) {
            aF[u] = S.redF[0][rp + 8 * u];
            aO[u] = S.redO[0][rp + 8 * u];
#pragma unroll
            for (int p = 1; p < NR; ++p) {
                aF[u] += S.redF[p][rp + 8 * u];
                aO[u] += S.redO[p][rp + 8 * u];
            }
            asm volatile("" : "+v"(aF[u]), "+v"(aO[u]));
        }
        asm volatile("s_waitcnt lgkmcnt(0)" ::: "memory");
        if (L == 0) LDS_ST(S.locdone, 1u);
        const float qn = __builtin_nanf("");
        const int oo = ((L >> 3) * NDIM + col0 + 4 * (L & 7)) * 4;
        bad |= !lds_wait<1>(S.nfor, NR - 1);
#pragma unroll
        for (int u = 0; u < 4; ++u)
#pragma unroll
            for (int p = 0; p < NR - 1; ++p) aF[u] += S.redF[p][rp + 8 * u];
        if (bad || LDS_LD(S.bad)) {
#pragma unroll
            for (int u = 0; u < 4; ++u) aF[u] = f4v{qn, qn, qn, qn};
        }
        {
            const int own = 2 * (ti >> 1) + 1 - cls;
            const __amdgpu_buffer_rsrc_t ro = __builtin_amdgcn_make_buffer_rsrc(a.out[own], 0, MS * NDIM * 4, 0x00020000);
            const int r0 = 64 * mem + 32 * (ti & 1);
            const unsigned long long tq = TNOW();
#pragma unroll
            for (int u = 0; u < 4; ++u) {
                __builtin_amdgcn_raw_buffer_store_b128(__builtin_bit_cast(u4v, aF[u]), ro, oo + (r0 + 8 * u) * NDIM * 4, 0, AUX_ST);
                if (tj < TILE_WJ) wait_vm<TILE_WIN - 1>();
            }
            wait_vm<0>();
            if (L == 0)
                __hip_atomic_store((unsigned*)(a.ws[own] + OFL_OFF + (unsigned)((2 * mem + (ti & 1)) * 32 + tj) * OFL_STRIDE), call,
                                   __ATOMIC_RELAXED, __HIP_MEMORY_SCOPE_SYSTEM);
            STAMPV(56, tq);
            STAMP(57);
        }
        bad |= !lds_wait<0>(S.nown, NR - 1);
#pragma unroll
        for (int u = 0; u < 4; ++u)
#pragma unroll
            for (int p = 0; p < NR - 1; ++p) aO[u] += S.redO[p][rp + 8 * u];
        if (bad || LDS_LD(S.bad)) {
#pragma unroll
            for (int u = 0; u < 4; ++u) aO[u] = f4v{qn, qn, qn, qn};
        }
        const __amdgpu_buffer_rsrc_t ro = __builtin_amdgcn_make_buffer_rsrc(a.out[rank], 0, MS * NDIM * 4, 0x00020000);
#pragma unroll
        for (int u = 0; u < 4; ++u)
            __builtin_amdgcn_raw_buffer_store_b128(__builtin_bit_cast(u4v, aO[u]), ro, oo + (32 * ti + 8 * u) * NDIM * 4, 0, AUX_ST);
        STAMP(58);
        return;
    }

    const __amdgpu_buffer_rsrc_t rs = __builtin_amdgcn_make_buffer_rsrc(myws, 0, (int)WS_NEED, 0x00020000);
    const unsigned long long t0 = __builtin_amdgcn_s_memrealtime();
    bool bad = call == 0u;
    unsigned npoll = 0u;
    auto late = [&]() -> bool {
        if (__builtin_amdgcn_s_memrealtime() - t0 > DEADLINE_TICKS) {
            if (L == 0)
                __hip_atomic_store((unsigned*)(myws + TMO_OFF), call, __ATOMIC_RELAXED, __HIP_MEMORY_SCOPE_AGENT);
            return true;
        }
        return false;
    };
    if (fwd && !bad) {
        const Ent en = entry(1, rank, dest, wv, s);
        const int pf = (int)(L == 0 ? flag_of(par, en.src, en.u) : TMO_OFF);
        u2v pv;
        for (;;) {
            pv = __builtin_amdgcn_raw_buffer_load_b64(rs, pf, 0, AUX_LD);
            if ((unsigned)__builtin_amdgcn_readlane((int)pv[1], 0) >= call) break;
            if ((unsigned)__builtin_amdgcn_readlane((int)pv[0], 1) == call || late()) { bad = true; break; }
            __builtin_amdgcn_s_sleep(SLEEP_R);
            asm volatile("" ::: "memory");
        }
        asm volatile("" ::: "memory");
        const Quad qd = load_quad(rs, (int)(data_of(par, en.src, en.u) + (unsigned)L * 16u));
        S.stage[wv][L] = QF(qd, 0);
        S.stage[wv][64 + L] = QF(qd, 1);
        S.stage[wv][128 + L] = QF(qd, 2);
        S.stage[wv][192 + L] = QF(qd, 3);
        if (L == 0) S.exp_[wv] = (int)(short)pv[0];
        asm volatile("s_waitcnt lgkmcnt(0)" ::: "memory");
        if (L == 0) LDS_ST(S.rdy[wv], 1);
        asm volatile("" ::: "memory");
        STAMP(70 + wv);
    }
    const int src = (rank + 1 + wv) & (NR - 1);
    const int oxf = (int)(data_of(par, src, 4 * ti) + (unsigned)L * 16u);
    const int ow = (int)(data_of(par, src, 32 + 4 * tj) + (unsigned)L * 16u);
    const int oxo = (int)(data_of(par, src, 160 + 4 * ti) + (unsigned)L * 16u);
    const int poff1 = (int)(L < 4 ? flag_of(par, src, 4 * ti + L)
                            : L < 8 ? flag_of(par, src, 32 + 4 * tj + L - 4)
                            : L < 12 ? flag_of(par, src, 160 + 4 * ti + L - 8) : TMO_OFF);
    const int poff0 = (int)(L < 12 ? (unsigned)poff1 : flag_of(par, src, 4 * ti) + 64u);
    unsigned lo = 0u, m = 0u;
    auto poll = [&]() {
        ++npoll;
        const u2v pv = __builtin_amdgcn_raw_buffer_load_b64(rs, (npoll % TMO_EVERY) == 0u ? poff1 : poff0, 0, AUX_LD);
        lo = pv[0];
        m = (unsigned)__builtin_amdgcn_ballot_w64(pv[1] >= call) & 0xfffu;
        if ((unsigned)__builtin_amdgcn_readlane((int)lo, 16) == call) bad = true;
    };
    Quad X[4], W[4];
    int ex[4], ew[4];
    for (poll(); !bad && (m & 0xfu) != 0xfu; poll()) {
        if (late()) { bad = true; break; }
        __builtin_amdgcn_s_sleep(SLEEP_E);
        asm volatile("" ::: "memory");
    }
    asm volatile("" ::: "memory");
    STAMP(16 + wv);
#pragma unroll
    for (int k = 0; k < 4; ++k) {
        X[k] = load_quad<AUX_XF>(rs, oxf + 1024 * k);
        ex[k] = (int)(short)__builtin_amdgcn_readlane((int)lo, k);
    }
    for (; !bad && (m & 0xf0u) != 0xf0u; poll()) {
        if (late()) { bad = true; break; }
        __builtin_amdgcn_s_sleep(SLEEP_W);
        asm volatile("" ::: "memory");
    }
    asm volatile("" ::: "memory");
    STAMP(24 + wv);
    const unsigned need = (unsigned)(__builtin_amdgcn_ballot_w64((lo & 0x10000u) != 0u) >> 4) & 0xfu;
#pragma unroll
    for (int k = 0; k < 4; ++k) ew[k] = (int)(short)__builtin_amdgcn_readlane((int)lo, 4 + k);
    for (;;) {
        asm volatile("" ::: "memory");
#pragma unroll
        for (int k = 0; k < 4; ++k) W[k] = load_quad(rs, ow + 1024 * k);
        __builtin_amdgcn_sched_barrier(0);
        if ((m & 0xf00u) != 0xf00u) poll();
        if (need == 0u || bad) break;
        const unsigned ok = (quad_valid(W[0]) ? 1u : 0u) | (quad_valid(W[1]) ? 2u : 0u) | (quad_valid(W[2]) ? 4u : 0u) |
                            (quad_valid(W[3]) ? 8u : 0u);
        if ((need & ~ok) == 0u) break;
        if (late()) { bad = true; break; }
    }
    __builtin_amdgcn_sched_barrier(0);
    f16v acc = scale16(mm_quad(W[0], X[0]), -(ew[0] + ex[0]));
    pin(acc);
#pragma unroll
    for (int k = 1; k < 4; ++k) { fma16(acc, mm_quad(W[k], X[k]), -(ew[k] + ex[k])); pin(acc); }
    if (!lds_wait<1>(S.locdone)) bad = true;
    if (bad && L == 0) LDS_ST(S.bad, 1u);
    deposit(S.redF[wv], acc, i, hh);
    lds_signal(S.nfor, L);
    STAMP(32 + wv);
    if ((m & 0xf00u) != 0xf00u) {
        for (poll(); !bad && (m & 0xf00u) != 0xf00u; poll()) {
            if (late()) { bad = true; break; }
            __builtin_amdgcn_s_sleep(SLEEP_O);
            asm volatile("" ::: "memory");
        }
    }
    STAMP(40 + wv);
#pragma unroll
    for (int k = 0; k < 4; ++k) ex[k] = (int)(short)__builtin_amdgcn_readlane((int)lo, 8 + k);
    const bool needo = (__builtin_amdgcn_readlane((int)lo, 8) & 0x10000) != 0;
    for (;;) {
        asm volatile("" ::: "memory");
#pragma unroll
        for (int k = 0; k < 4; ++k) X[k] = load_quad(rs, oxo + 1024 * k);
        __builtin_amdgcn_sched_barrier(0);
        if (!needo || bad) break;
        const bool ok = quad_valid(X[0]) & quad_valid(X[1]) & quad_valid(X[2]) & quad_valid(X[3]);
        if (ok) break;
        if (late()) { bad = true; break; }
    }
    __builtin_amdgcn_sched_barrier(0);
    acc = scale16(mm_quad(W[0], X[0]), -(ew[0] + ex[0]));
    pin(acc);
#pragma unroll
    for (int k = 1; k < 4; ++k) { fma16(acc, mm_quad(W[k], X[k]), -(ew[k] + ex[k])); pin(acc); }
    if (bad && L == 0) LDS_ST(S.bad, 1u);
    deposit(S.redO[wv], acc, i, hh);
    lds_signal(S.nown, L);
    STAMP(48 + wv);
}

extern "C" void kernel_launch(int rank, int nranks, ncclComm_t comm,
                              hipStream_t stream,
                              void* const* d_in, const int* in_sizes, int n_in,
                              void* d_out, int out_size,
                              const int* in_dtypes, int out_dtype,
                              void* const* d_out_all,
                              void* const* d_ws_all, size_t ws_bytes) {
    if (nranks != NR || n_in < 2 || ws_bytes < (size_t)WS_NEED || out_size != MS * NDIM) {
        fprintf(stderr, "unexpected problem geometry\n");
        return;
    }
    Args a;
    a.x = (const float*)d_in[0];
    a.w = (const float*)d_in[1];
    for (int p = 0; p < NR; ++p) { a.ws[p] = (char*)d_ws_all[p]; a.out[p] = (float*)d_out_all[p]; }
    a.out[rank] = (float*)d_out;
    a.rank = rank;
    gemm_rs_kernel<<<NWG, 640, 0, stream>>>(a);
    HIP_CHECK(hipGetLastError());
}
```

```cpp
#include <hip/hip_runtime.h>
#include <rccl/rccl.h>
#include <cstdio>
#include <cstdint>

#define HIP_CHECK(expr) do { hipError_t _e = (expr); if (_e != hipSuccess) { \
    fprintf(stderr, "HIP error %s at %s:%d\n", hipGetErrorString(_e), __FILE__, __LINE__); \
    return; } } while (0)

typedef _Float16 h8v __attribute__((ext_vector_type(8)));
typedef float f16v __attribute__((ext_vector_type(16)));
typedef float f4v __attribute__((ext_vector_type(4)));
typedef float f2v __attribute__((ext_vector_type(2)));
typedef unsigned u4v __attribute__((ext_vector_type(4)));
typedef unsigned u2v __attribute__((ext_vector_type(2)));

namespace {

constexpr int NR = 8;
constexpr int NDIM = 2048;
constexpr int KS = 256;
constexpr int MS = 256;
constexpr int NWG = 256;
constexpr int NSTR = 32;
constexpr int NPC = 48;
constexpr int RMAX = 7;
constexpr unsigned F_STR = 33 * 1024, R_STR = 4 * F_STR, S_STR = 6 * R_STR;
constexpr unsigned DATA_BYTES = 2u * NR * S_STR;
constexpr unsigned FLAG_STRIDE = 128;
constexpr unsigned FLAG_OFF = DATA_BYTES;
constexpr unsigned FLAG_BYTES = 2u * NR * NPC * FLAG_STRIDE;
constexpr unsigned OFL_OFF = FLAG_OFF + FLAG_BYTES;
constexpr unsigned OFL_STRIDE = 64;
constexpr unsigned CTR_OFF = OFL_OFF + NWG * OFL_STRIDE;
constexpr unsigned CTR_STRIDE = 64;
constexpr unsigned TMO_OFF = CTR_OFF + NWG * CTR_STRIDE;
constexpr unsigned PRG_OFF = TMO_OFF + 128;
constexpr unsigned WS_NEED = PRG_OFF + NR * 128;
#define DLT 50000000ull
constexpr unsigned long long DEADLINE_TICKS = DLT;

#define INFL 2
#define SLEEP_N 1
#define FB_T1 50
#define FB_T2 110
#define FB_T3 200
#define FB_TL 60
#define FB_W0 2
#define FB_END 6
#define FB_WE 4
#define SLEEP_E 16
#define SLEEP_W 8
#define SLEEP_O 2
#define SLEEP_T 2
#define SLEEP_T2 0
#define SLEEP_R 2
#define TMO_EVERY 16
#define FB_SLEEP 4
#define STAG_S 1
#define EVS 64
#define EVC -32
#define GPD 16
#define RGD 3
#define LEAD_W 2
#define LEAD_O 2
#define ZAT 4
#define PREFETCH 1
#define WL_S 1
#define WL_C 1
#define NL_S 1
#define NL_C 1
#define STAG_MODE 1
#define TILE_WIN 1
#define TILE_WJ 12
constexpr int GP = GPD, RG = RGD, QREL = (32 / GP) * RG * 4, FH = QREL / 4;
constexpr int LS = 192 + QREL, LC = 192 - QREL;
static_assert(LS <= 32 * RMAX && QREL <= 32 && QREL > 0, "stream geometry");
#define AUX_ST 17
#define AUX_LD 17

#define STAMP(idx) do {} while (0)
#define STAMPV(idx, v) do {} while (0)
#define TNOW() 0ull

#define LDS_LD(v) __builtin_amdgcn_readfirstlane(__hip_atomic_load(&(v), __ATOMIC_RELAXED, __HIP_MEMORY_SCOPE_WORKGROUP))
#define LDS_ST(v, x) __hip_atomic_store(&(v), (x), __ATOMIC_RELAXED, __HIP_MEMORY_SCOPE_WORKGROUP)

struct Args {
    const float* x;
    const float* w;
    char* ws[NR];
    float* out[NR];
    int rank;
};

__device__ __forceinline__ unsigned data_of(int par, int src, int u) {
    return (unsigned)(par * NR + src) * S_STR + (unsigned)(u >> 5) * R_STR + (unsigned)(u & 31) * 1024u;
}
__device__ __forceinline__ unsigned flag_of(int par, int src, int u) {
    return FLAG_OFF + (unsigned)((par * NR + src) * NPC + (u >> 2)) * FLAG_STRIDE + (unsigned)(u & 3) * 8u;
}
__device__ __forceinline__ int frow(int r, int i) {
    return MS * (2 * (i >> 1) + 1 - (r & 1)) + 64 * (r >> 1) + 32 * (i & 1);
}

struct Ent { int kind, src, u, a0, kq; };
__device__ __forceinline__ int rounds(int typ, int s) {
    return typ == 2 ? (s < QREL ? 1 : 0) : ((typ ? LS : LC) - s + 31) >> 5;
}
__device__ __forceinline__ Ent entry(int typ, int rank, int dest, int rho, int s) {
    Ent e;
    e.src = rank;
    if (typ == 2) {
        const int x = s % FH, j = GP * (x / RG) + GP - RG + x % RG;
        e.kind = 1; e.kq = dest >> 1; e.a0 = 1024 * (dest & 1) + 32 * j; e.u = 32 + 4 * j + e.kq;
        return e;
    }
    if (rho == 0) { e.kind = 0; e.kq = s & 3; e.a0 = frow(dest, s >> 2); e.u = s; return e; }
    const int n = 32 * (rho - 1) + s, o = n - (typ ? LS - 64 : LC - 64);
    if (o >= 0) { e.kind = 0; e.kq = o & 3; e.a0 = MS * dest + 32 * (o >> 2); e.u = 160 + o; return e; }
    constexpr int GS = 4 * (GP + RG), GC = 4 * (GP - RG);
    const int g = typ ? n / GS : n / GC, t = n - g * (typ ? GS : GC);
    if (t < 4 * GP) {
        const int j = GP * g + (t >> 2);
        e.kind = 1; e.kq = t & 3; e.a0 = 1024 * (dest & 1) + 32 * j; e.u = 32 + 4 * j + e.kq;
    } else {
        const int r = t - 4 * GP;
        e.kind = 2; e.kq = rank >> 1; e.a0 = 0; e.src = 2 * (r & 3) + 1 - (rank & 1);
        e.u = 32 + 4 * (GP * g + GP - RG + (r >> 2)) + e.kq;
    }
    return e;
}
__device__ __forceinline__ int first_early(int typ, int s) {
    return typ == 1 ? (LS - 32 - EVS - s + 31) >> 5 : typ == 0 ? (LC - 32 - (EVC) - s + 31) >> 5 : 1;
}

__device__ __forceinline__ bool clears(int typ, int rank, int s) { return typ == 0 && (s >> 3) == (rank >> 1); }
constexpr unsigned EMPTY = 0xffffffffu;

template <int NV>
__device__ __forceinline__ int block_exp(const float (&v)[NV]) {
    float m = 0.f;
#pragma unroll
    for (int t = 0; t < NV; ++t) m = fmaxf(m, fabsf(v[t]));
    unsigned u = __float_as_uint(m);
    u = max(u, (unsigned)__builtin_amdgcn_update_dpp(0, (int)u, 0xB1, 0xf, 0xf, true));
    u = max(u, (unsigned)__builtin_amdgcn_update_dpp(0, (int)u, 0x4E, 0xf, 0xf, true));
    u = max(u, (unsigned)__builtin_amdgcn_update_dpp(0, (int)u, 0x141, 0xf, 0xf, true));
    u = max(u, (unsigned)__builtin_amdgcn_update_dpp(0, (int)u, 0x140, 0xf, 0xf, true));
    const unsigned r = max(max((unsigned)__builtin_amdgcn_readlane((int)u, 0), (unsigned)__builtin_amdgcn_readlane((int)u, 16)),
                           max((unsigned)__builtin_amdgcn_readlane((int)u, 32), (unsigned)__builtin_amdgcn_readlane((int)u, 48)));
    unsigned E = r >> 23;
    E = E < 15u ? 15u : E;
    return 141 - (int)E;
}

__device__ __forceinline__ u4v pack8(const float* v, int e) {
    h8v r;
    typedef _Float16 h2v __attribute__((ext_vector_type(2)));
    const float sc = __uint_as_float((unsigned)(e + 127) << 23);
    const f2v s2 = {sc, sc};
#pragma unroll
    for (int j = 0; j < 8; j += 2) {
        const h2v t = __builtin_convertvector(f2v{v[j], v[j + 1]} * s2, h2v);
        r[j] = t[0];
        r[j + 1] = t[1];
    }
    return __builtin_bit_cast(u4v, r);
}

template <int NF>
__device__ __forceinline__ void load_w(const float* __restrict__ w, int k0, int col0, int i, int hh, float (&v)[8 * NF]) {
    const float* base = w + (size_t)(k0 + 8 * hh) * NDIM + col0 + i;
#pragma unroll
    for (int s = 0; s < NF; ++s)
#pragma unroll
        for (int j = 0; j < 8; ++j) v[8 * s + j] = base[(size_t)(16 * s + j) * NDIM];
}

template <int NF>
__device__ __forceinline__ void load_x(const float* __restrict__ x, int row0, int k0, int i, int hh, float (&v)[8 * NF]) {
    const float* base = x + (size_t)(row0 + i) * KS + k0 + 8 * hh;
#pragma unroll
    for (int s = 0; s < NF; ++s) {
        f4v t0 = *(const f4v*)(base + 16 * s);
        f4v t1 = *(const f4v*)(base + 16 * s + 4);
#pragma unroll
        for (int j = 0; j < 4; ++j) { v[s * 8 + j] = t0[j]; v[s * 8 + 4 + j] = t1[j]; }
    }
}

__device__ __forceinline__ f16v mm(u4v a, u4v b, f16v c) {
    return __builtin_amdgcn_mfma_f32_32x32x16_f16(__builtin_bit_cast(h8v, a), __builtin_bit_cast(h8v, b), c, 0, 0, 0);
}
#define ZERO16 f16v{0.f, 0.f, 0.f, 0.f, 0.f, 0.f, 0.f, 0.f, 0.f, 0.f, 0.f, 0.f, 0.f, 0.f, 0.f, 0.f}

__device__ __forceinline__ f16v scale16(f16v t, int e) {
    const int e1 = e >> 1, e2 = e - e1;
    const float c1 = __uint_as_float((unsigned)(e1 + 127) << 23), c2 = __uint_as_float((unsigned)(e2 + 127) << 23);
    const f2v s1 = {c1, c1}, s2 = {c2, c2};
#pragma unroll
    for (int r = 0; r < 16; r += 2) {
        const f2v u = (f2v{t[r], t[r + 1]} * s1) * s2;
        t[r] = u[0]; t[r + 1] = u[1];
    }
    return t;
}

__device__ __forceinline__ void fma16(f16v& acc, f16v t, int e) {
    const int e1 = e >> 1, e2 = e - e1;
    const float c1 = __uint_as_float((unsigned)(e1 + 127) << 23), c2 = __uint_as_float((unsigned)(e2 + 127) << 23);
    const f2v s1 = {c1, c1}, s2 = {c2, c2};
#pragma unroll
    for (int r = 0; r < 16; r += 2) {
        const f2v u = __builtin_elementwise_fma(f2v{t[r], t[r + 1]} * s1, s2, f2v{acc[r], acc[r + 1]});
        acc[r] = u[0]; acc[r + 1] = u[1];
    }
}

struct Quad { u4v f[4]; };
#define QF(q, n) ((q).f[n])

#define AUX_XF AUX_LD
#define AUX_W AUX_LD
#define AUX_XO AUX_LD
template <int AUX = AUX_LD>
__device__ __forceinline__ Quad load_quad(__amdgpu_buffer_rsrc_t rs, int off) {
    Quad q;
#pragma unroll
    for (int f = 0; f < 4; ++f) q.f[f] = __builtin_amdgcn_raw_buffer_load_b128(rs, off, f * (int)F_STR, AUX);
    return q;
}

__device__ __forceinline__ f16v mm_quad(const Quad& w, const Quad& x, f16v t) {
    t = mm(QF(w, 0), QF(x, 0), t);
    t = mm(QF(w, 1), QF(x, 1), t);
    t = mm(QF(w, 2), QF(x, 2), t);
    return mm(QF(w, 3), QF(x, 3), t);
}

__device__ __forceinline__ unsigned down(int d) {
    return d <= 14 ? (unsigned)(15 - d) << 10 : (1u << 24 >> min(d, 31));
}
__device__ __forceinline__ void scale_quad(Quad& q, unsigned c) {
    const _Float16 h = __builtin_bit_cast(_Float16, (unsigned short)c);
#pragma unroll
    for (int f = 0; f < 4; ++f) {
        q.f[f] = __builtin_bit_cast(u4v, __builtin_bit_cast(h8v, q.f[f]) * h);
        asm volatile("" : "+v"(q.f[f]));
    }
}

__device__ __forceinline__ u4v no_zero(u4v u) {
    u[0] = u[0] ? u[0] : 0x8000u;
    return u;
}

__device__ __forceinline__ bool quad_valid(const Quad& q) {
    return __builtin_amdgcn_ballot_w64(min(min(q.f[0][0], q.f[1][0]), min(q.f[2][0], q.f[3][0])) == 0u) == 0ull;
}

template <int N>
__device__ __forceinline__ void wait_vm() {
    asm volatile("s_waitcnt vmcnt(%0)" ::"i"(N) : "memory");
}

__device__ __forceinline__ void wait_vm_n(int n) {
    switch (n) {
        case 0: wait_vm<0>(); break;
        case 1: wait_vm<1>(); break;
        case 2: wait_vm<2>(); break;
        case 3: wait_vm<3>(); break;
        case 4: wait_vm<4>(); break;
        case 5: wait_vm<5>(); break;
        case 6: wait_vm<6>(); break;
        case 7: wait_vm<7>(); break;
        case 8: wait_vm<8>(); break;
        case 9: wait_vm<9>(); break;
        default: wait_vm<10>(); break;
    }
}

__device__ __forceinline__ void raw_barrier() {
    asm volatile("s_waitcnt lgkmcnt(0)\n\ts_barrier" ::: "memory");
}

template <int SLP, class T>
__device__ __forceinline__ unsigned lds_wait(T& v, unsigned n = 1u) {
    unsigned r, it = 0u;
    unsigned long long t0 = 0ull;
    while ((r = (unsigned)LDS_LD(v)) < n) {
        __builtin_amdgcn_s_sleep(SLP);
        if ((++it & 1023u) == 0u) {
            const unsigned long long t = __builtin_amdgcn_s_memrealtime();
            if (t0 == 0ull) t0 = t;
            else if (t - t0 > 2ull * DEADLINE_TICKS) { r = 0u; break; }
        }
    }
    asm volatile("" ::: "memory");
    return r;
}

struct Lds {
    u4v stage[RMAX][256];
    f4v redF[NR][8 * 33];
    f4v redO[NR][8 * 33];
    int exp_[RMAX + 1];
    int rdy[RMAX + 1];
    unsigned call, bad, nloc, locdone, nfor, nown, fdone, clr;
    int prog, win, zok;
};

__device__ __forceinline__ void deposit(f4v* slot, const f16v& t, int i, int hh) {
#pragma unroll
    for (int g = 0; g < 4; ++g)
        slot[(2 * g + hh) * 33 + i] = f4v{t[4 * g], t[4 * g + 1], t[4 * g + 2], t[4 * g + 3]};
}
__device__ __forceinline__ void arrive(unsigned& c, unsigned v) {
    unsigned long long ex;
    asm volatile("s_mov_b64 %0, exec\n\ts_mov_b64 exec, 1\n\tds_add_u32 %1, %2\n\ts_mov_b64 exec, %0"
                 : "=&s"(ex)
                 : "v"((unsigned)(uintptr_t)(__attribute__((address_space(3))) unsigned*)&c), "v"(v)
                 : "memory");
}
__device__ __forceinline__ void gather(f4v (&acc)[4], bool& bad, unsigned& flags, unsigned& badw, const f4v* red) {
    unsigned got = 0u, it = 0u;
    unsigned long long t0 = 0ull;
    auto late = [&]() -> bool {
        if ((++it & 1023u) == 0u) {
            const unsigned long long t = __builtin_amdgcn_s_memrealtime();
            if (t0 == 0ull) t0 = t;
            else if (t - t0 > 2ull * DEADLINE_TICKS) return true;
        }
        return false;
    };
    for (int n = 0; n < NR - 2 && !bad;) {
        const unsigned m = (unsigned)LDS_LD(flags) & ~got;
        asm volatile("" ::: "memory");
        if (m == 0u) {
            __builtin_amdgcn_s_sleep(0);
            bad = late();
            continue;
        }
        const int p = __builtin_ctz(m);
#pragma unroll
        for (int u = 0; u < 4; ++u) acc[u] += red[p * (8 * 33) + 8 * u];
        got |= 1u << p;
        ++n;
    }
    const int p = __builtin_ctz(~got);
    f4v v[4];
    for (;;) {
        asm volatile("" ::: "memory");
        const unsigned c = __hip_atomic_load(&flags, __ATOMIC_RELAXED, __HIP_MEMORY_SCOPE_WORKGROUP);
        __builtin_amdgcn_sched_barrier(0);
        asm volatile("" ::: "memory");
        const unsigned w = __hip_atomic_load(&badw, __ATOMIC_RELAXED, __HIP_MEMORY_SCOPE_WORKGROUP);
#pragma unroll
        for (int u = 0; u < 4; ++u) v[u] = red[p * (8 * 33) + 8 * u];
        __builtin_amdgcn_sched_barrier(0);
        if (__builtin_amdgcn_readfirstlane(c) == 0x7fu) { bad |= __builtin_amdgcn_readfirstlane(w) != 0u; break; }
        asm volatile("s_waitcnt lgkmcnt(0)" ::: "memory");
        if (bad || late()) { bad = true; break; }
    }
#pragma unroll
    for (int u = 0; u < 4; ++u) acc[u] += v[u];
    if (bad) {
        const float qn = __builtin_nanf("");
#pragma unroll
        for (int u = 0; u < 4; ++u) acc[u] = f4v{qn, qn, qn, qn};
    }
}

__device__ __forceinline__ void pusher(const Args& a, Lds& S, int b, int rank, int dest, int s, int L, int typ) {
    const int nrnd = rounds(typ, s), NST = 4 * nrnd;
    char* const myws = a.ws[rank];
    const unsigned long long tq0 = TNOW();
    const unsigned cv = __hip_atomic_load((unsigned*)(myws + CTR_OFF + b * CTR_STRIDE), __ATOMIC_RELAXED,
                                          __HIP_MEMORY_SCOPE_AGENT);
    char* dws = a.ws[dest];
    const int i = L & 31, hh = L >> 5;
    u4v t[4];
    unsigned vdat, vflg;
    {
        const Ent en = entry(typ, rank, dest, 0, s);
        float v0[32];
        if (typ == 2) load_w<4>(a.w, 64 * en.kq, en.a0, i, hh, v0);
        else load_x<4>(a.x, en.a0, 64 * en.kq, i, hh, v0);
        asm volatile("" ::"s"(dws));
        const Ent le = entry(typ, rank, dest, L < nrnd ? L : 0, s);
        vdat = data_of(0, le.src, le.u);
        vflg = flag_of(0, le.src, le.u);
        asm volatile("" : "+v"(vdat), "+v"(vflg));
        if (L <= RMAX) S.rdy[L] = 0;
        if (L == 0) {
            S.call = 0u; S.bad = 0u; S.nloc = 0u; S.locdone = 0u; S.nfor = 0u; S.nown = 0u; S.fdone = 0u;
            S.clr = clears(typ, rank, s) ? 0u : 1u;
            S.prog = 0; S.zok = 0; S.win = typ == 2 ? 6 : INFL;
        }
        raw_barrier();
        const int e = block_exp<32>(v0);
        if (L == 0) { LDS_ST(S.rdy[0], 1); S.exp_[0] = e; }
#pragma unroll
        for (int f = 0; f < 4; ++f) t[f] = pack8(v0 + 8 * f, e);
        if (typ == 2) {
#pragma unroll
            for (int f = 0; f < 4; ++f) t[f] = no_zero(t[f]);
        }
    }
    const unsigned call = __builtin_amdgcn_readfirstlane(cv) + 1u;
    if (L == 0) LDS_ST(S.call, call);
    if (nrnd == 0) return;
    const int par = (int)(call & 1u);
    __builtin_amdgcn_s_setprio(3);
    const __amdgpu_buffer_rsrc_t rs = __builtin_amdgcn_make_buffer_rsrc(dws, 0, (int)DATA_BYTES, 0x00020000);
    vdat += (unsigned)par * (NR * S_STR);
    vflg += (unsigned)par * (NR * NPC * FLAG_STRIDE);
    auto publish = [&](int c, unsigned early) {
        if (L == c) {
            const unsigned long long val = ((unsigned long long)call << 32) | (((unsigned)S.exp_[c] & 0xffffu) | early);
            __hip_atomic_store((unsigned long long*)(dws + vflg), val, __ATOMIC_RELAXED, __HIP_MEMORY_SCOPE_SYSTEM);
        }
    };
    int issued = 0, pub = 0, dn = 0, h1 = 0, h2 = 0, h3 = 0, h4 = 0, h5 = 0, h6 = 0, h7 = 0;
    const int ev = first_early(typ, s);
    const int nl = 4 * ((typ == 1 ? LS : LC) >> 5) - (typ == 1 ? NL_S : NL_C);
    for (int c = 0;; ++c) {
        const int so = __builtin_amdgcn_readlane((int)vdat, c);
        bool have = false;
#pragma unroll
        for (int f = 0; f < 4; ++f) {
            const int n = 4 * c + f;
            __builtin_amdgcn_raw_buffer_store_b128(t[f], rs, L * 16, so + f * (int)F_STR, AUX_ST);
            const int h0 = ++issued;
            __builtin_amdgcn_sched_barrier(0);
            if (c >= ev && f == (c + 1 == nrnd ? LEAD_O : LEAD_W)) {
                if (c + 1 == nrnd && !lds_wait<1>(S.zok)) {
                    if (L == 0) { LDS_ST(S.bad, 1u); LDS_ST(S.prog, NST); }
                    return;
                }
                publish(c, 0x10000u); ++issued;
            }
            if (L == 0) LDS_ST(S.prog, n + 1 == NST ? NST : dn);
            while (pub < ev && 4 * (pub + 1) <= dn && (pub > 0 || LDS_LD(S.clr))) { publish(pub, 0u); ++issued; ++pub; }
            if (f == 3) {
                if (n + 1 == NST) break;
                have = LDS_LD(S.rdy[c + 1]) != 0;
                asm volatile("" ::: "memory");
                if (have) {
#pragma unroll
                    for (int g = 0; g < 4; ++g) t[g] = S.stage[c + 1][g * 64 + L];
                }
            }
            int win = LDS_LD(S.win);
            if (n >= nl) win = max(win, typ == 1 ? WL_S : WL_C);
            const int ik = win <= 1 ? h0 : win == 2 ? h1 : win == 3 ? h2 : win == 4 ? h3 : win <= 6 ? h5 : h7;
            const int k1 = n + 2 - (win > 6 ? 8 : win > 4 ? 6 : win);
            if (k1 > dn) {
                wait_vm_n(issued - ik);
                dn = k1;
            }
            h7 = h6; h6 = h5; h5 = h4; h4 = h3; h3 = h2; h2 = h1; h1 = h0;
        }
        if (c + 1 == nrnd) break;
        if (!have) {
            while (pub < ev && 4 * (pub + 1) <= dn && (pub > 0 || LDS_LD(S.clr))) { publish(pub, 0u); ++issued; ++pub; }
            if (!lds_wait<0>(S.rdy[c + 1])) {
                if (L == 0) { LDS_ST(S.bad, 1u); LDS_ST(S.prog, NST); }
                return;
            }
#pragma unroll
            for (int g = 0; g < 4; ++g) t[g] = S.stage[c + 1][g * 64 + L];
        }
    }
    const unsigned long long tq1 = TNOW();
    if (pub == 0 && ev > 0) {
        if (!lds_wait<1>(S.clr)) {
            if (L == 0) LDS_ST(S.bad, 1u);
            return;
        }
        while (pub < ev && 4 * (pub + 1) <= dn) { publish(pub, 0u); ++issued; ++pub; }
    }
    if (pub < ev && pub == nrnd - 3) { wait_vm_n(issued - h7); publish(pub, 0u); ++issued; ++pub; }
    if (pub < ev && pub == nrnd - 2) { wait_vm_n(issued - h4); publish(pub, 0u); ++pub; }
    if (pub < ev) { wait_vm<0>(); publish(pub, 0u); }
}

__device__ __forceinline__ bool feed(const Args& a, Lds& S, int rank, int dest, int s, int wv, int L, int typ) {
    if (wv < 1 || wv >= rounds(typ, s)) return false;
    const Ent en = entry(typ, rank, dest, wv, s);
    if (en.kind == 2) return true;
    const int i = L & 31, hh = L >> 5;
    float v[32];
    if (en.kind == 1) load_w<4>(a.w, 64 * en.kq, en.a0, i, hh, v);
    else load_x<4>(a.x, en.a0, 64 * en.kq, i, hh, v);
    const int e = block_exp<32>(v);
#pragma unroll
    for (int f = 0; f < 4; ++f) {
        S.stage[wv][f * 64 + L] = no_zero(pack8(v + 8 * f, e));
    }
    if (L == 0) S.exp_[wv] = e;
    asm volatile("s_waitcnt lgkmcnt(0)" ::: "memory");
    if (L == 0) LDS_ST(S.rdy[wv], 1);
    asm volatile("" ::: "memory");
    return false;
}

}

__global__ void __launch_bounds__(640) gemm_rs_kernel(Args a) {
    __shared__ __attribute__((aligned(16))) Lds S;

    const int b = blockIdx.x;
    const int wv = __builtin_amdgcn_readfirstlane(threadIdx.x >> 6);
    const int L = threadIdx.x & 63;
    const int i = L & 31, hh = L >> 5;
    const int rank = a.rank & (NR - 1);
    const int cls = rank & 1, mem = rank >> 1;
    const int ti = b & 7, tj = b >> 3;
    const int lk = b >> 5, s = b & 31;
    const int dest = lk < 7 ? (rank + 1 + lk) & (NR - 1) : 2 * ((s / FH) & 3) + 1 - cls;
    const int typ = lk == 7 ? 2 : lk & 1;
    char* const myws = a.ws[rank];

    if (wv == NR) {
        pusher(a, S, b, rank, dest, s, L, typ);
        return;
    }

    if (wv == NR + 1) {
        raw_barrier();
        const unsigned call = lds_wait<1>(S.call);
        if (call == 0u) return;
        if (L == 0)
            __hip_atomic_store((unsigned*)(myws + CTR_OFF + b * CTR_STRIDE), call, __ATOMIC_RELAXED, __HIP_MEMORY_SCOPE_AGENT);
        if (lk < 7) {
            const int nrnd = rounds(typ, s), NST = 4 * nrnd, LT = typ ? 4 * LS : 4 * LC;
            int z = 0;
            auto clear = [&]() {
                const __amdgpu_buffer_rsrc_t rz = __builtin_amdgcn_make_buffer_rsrc(myws, 0, (int)DATA_BYTES, 0x00020000);
                for (int c = first_early(typ, s); c < nrnd; ++c) {
                    const Ent en = entry(typ, dest, rank, c, s);
                    const unsigned zo = data_of((int)(call & 1u) ^ 1, en.src, en.u);
#pragma unroll
                    for (int f = 0; f < 4; ++f)
                        __builtin_amdgcn_raw_buffer_store_b128(u4v{0u, 0u, 0u, 0u}, rz, L * 16, (int)(zo + f * F_STR), AUX_ST);
                }
                z = 1;
            };
            unsigned* pc = (unsigned*)(myws + PRG_OFF + (unsigned)dest * 128u);
            const unsigned base = (call - 1u) * (unsigned)LT;
            int last = 0;
            unsigned it = 0u;
            unsigned long long t0 = 0ull;
            for (;;) {
                int p = LDS_LD(S.prog);
                if ((++it & 255u) == 0u) {
                    const unsigned long long t = __builtin_amdgcn_s_memrealtime();
                    if (t0 == 0ull) t0 = t;
                    else if (t - t0 > 2ull * DEADLINE_TICKS) p = NST;
                }
                const int dl = p > last ? p - last : 0;
                last += dl;
                if (last >= NST) {
                    if (L == 0) __hip_atomic_fetch_add(pc, (unsigned)dl, __ATOMIC_RELAXED, __HIP_MEMORY_SCOPE_AGENT);
                    break;
                }
                if (z == 0 && last >= ZAT) clear();
                unsigned g = 0u;
                if (L == 0) g = __hip_atomic_fetch_add(pc, (unsigned)dl, __ATOMIC_RELAXED, __HIP_MEMORY_SCOPE_AGENT) + (unsigned)dl;
                const int tot = (int)(__builtin_amdgcn_readfirstlane(g) - base);
                if (z == 1) {
                    wait_vm<0>();
                    if (L == 0) LDS_ST(S.zok, 1);
                    z = 2;
                }
                const int lag = tot - NSTR * last;
                int win = lag > FB_T3 ? 6 : lag > FB_T2 ? 4 : lag > FB_T1 ? 3 : lag < -FB_TL ? 1 : 2;
                if (tot >= LT - FB_END * NSTR && win < FB_WE) win = FB_WE;
                if (L == 0 && last > 0) LDS_ST(S.win, win);
                __builtin_amdgcn_s_sleep(FB_SLEEP);
            }
            if (z == 0) clear();
            if (z == 1) {
                wait_vm<0>();
                if (L == 0) LDS_ST(S.zok, 1);
            }
        }
        {
            const __amdgpu_buffer_rsrc_t ro = __builtin_amdgcn_make_buffer_rsrc(a.out[rank], 0, MS * NDIM * 4, 0x00020000);
            const int oo = ((32 * ti + (L >> 3)) * NDIM + 1024 * (1 - cls) + 32 * tj + 4 * (L & 7)) * 4;
            bool bad = !lds_wait<SLEEP_T>(S.fdone);
            const unsigned long long t0 = __builtin_amdgcn_s_memrealtime();
            unsigned ok = 0u, it = 0u;
            struct T4 { u4v v[4]; };
            auto look = [&]() -> T4 {
                T4 r;
#pragma unroll
                for (int u = 0; u < 4; ++u)
                    r.v[u] = __builtin_amdgcn_raw_buffer_load_b128(ro, oo + ((ok >> u & 1u) ? 0x40000000 : 0), 8 * u * NDIM * 4, AUX_LD);
                __builtin_amdgcn_sched_barrier(0);
                return r;
            };
            auto check = [&](const T4& r) {
#pragma unroll
                for (int u = 0; u < 4; ++u) {
                    const u4v v = r.v[u];
                    if (__builtin_amdgcn_ballot_w64(v[0] == EMPTY) == 0ull) ok |= 1u << u;
                }
                __builtin_amdgcn_sched_barrier(0);
            };
            while (!bad) {
                asm volatile("" ::: "memory");
                check(look());
                if (ok != 15u && (++it & 15u) != 0u) {
                    __builtin_amdgcn_s_sleep(SLEEP_T2);
                    continue;
                }
                if (ok == 15u) break;
                const unsigned v = __hip_atomic_load((const unsigned*)(myws + TMO_OFF), __ATOMIC_RELAXED, __HIP_MEMORY_SCOPE_SYSTEM);
                if (__builtin_amdgcn_readfirstlane(v) == call) bad = true;
                if (__builtin_amdgcn_s_memrealtime() - t0 > DEADLINE_TICKS) {
                    if (L == 0)
                        __hip_atomic_store((unsigned*)(myws + TMO_OFF), call, __ATOMIC_RELAXED, __HIP_MEMORY_SCOPE_AGENT);
                    bad = true;
                }
            }
            if (bad) {
                const unsigned qn = 0x7fc00000u;
#pragma unroll
                for (int u = 0; u < 4; ++u)
                    __builtin_amdgcn_raw_buffer_store_b128(u4v{qn, qn, qn, qn}, ro, oo, 8 * u * NDIM * 4, AUX_ST);
            }
            STAMP(59);
        }
        return;
    }

    raw_barrier();
    const bool clr = wv == NR - 1 && clears(typ, rank, s);
    if (clr) {
        const __amdgpu_buffer_rsrc_t ro = __builtin_amdgcn_make_buffer_rsrc(a.out[rank], 0, MS * NDIM * 4, 0x00020000);
        const int oo = ((64 * (dest >> 1) + 32 * ((s >> 2) & 1)) * NDIM + 1024 * (dest & 1) + 256 * (s & 3) + 4 * L) * 4;
#pragma unroll
        for (int u = 0; u < 32; ++u)
            __builtin_amdgcn_raw_buffer_store_b128(u4v{EMPTY, EMPTY, EMPTY, EMPTY}, ro, oo, u * NDIM * 4, AUX_ST);
        wait_vm<0>();
        if (L == 0) LDS_ST(S.clr, 1u);
    }
    if (wv != 1) lds_wait<STAG_S>(S.rdy[0]);
    const bool fwd = feed(a, S, rank, dest, s, wv, L, typ);
    const int col0 = 1024 * cls + 32 * tj;
    {
        float vw[16], vf[16], vo[16];
        load_w<2>(a.w, 32 * wv, col0, i, hh, vw);
        load_x<2>(a.x, frow(rank, ti), 32 * wv, i, hh, vf);
        load_x<2>(a.x, MS * rank + 32 * ti, 32 * wv, i, hh, vo);
        const int ew = block_exp<16>(vw);
        const u4v w0 = pack8(vw, ew), w1 = pack8(vw + 8, ew);
        const int ef = block_exp<16>(vf);
        f16v t = mm(w0, pack8(vf, ef), ZERO16);
        t = mm(w1, pack8(vf + 8, ef), t);
        deposit(S.redF[wv], scale16(t, -(ew + ef)), i, hh);
        const int eo = block_exp<16>(vo);
        t = mm(w0, pack8(vo, eo), ZERO16);
        t = mm(w1, pack8(vo + 8, eo), t);
        deposit(S.redO[wv], scale16(t, -(ew + eo)), i, hh);
        arrive(S.nloc, 1u);
    }
    STAMP(60 + wv);
    if (wv == NR - 1) {
        float* pf = a.out[2 * (ti >> 1) + 1 - cls];
        float* po = a.out[rank];
        asm volatile("" : "+s"(pf), "+s"(po));
        const __amdgpu_buffer_rsrc_t rf = __builtin_amdgcn_make_buffer_rsrc(pf, 0, MS * NDIM * 4, 0x00020000);
        const __amdgpu_buffer_rsrc_t ro = __builtin_amdgcn_make_buffer_rsrc(po, 0, MS * NDIM * 4, 0x00020000);
        int of = ((64 * mem + 32 * (ti & 1) + (L >> 3)) * NDIM + col0 + 4 * (L & 7)) * 4;
        int oo = ((32 * ti + (L >> 3)) * NDIM + col0 + 4 * (L & 7)) * 4;
        asm volatile("" : "+v"(of), "+v"(oo));
        const int rp = (L & 7) * 33 + (L >> 3);
        f4v aF[4], aO[4];
        bool bad = !lds_wait<1>(S.nloc, NR);
#pragma unroll
        for (int u = 0; u < 4; ++u) {
            aF[u] = S.redF[0][rp + 8 * u];
            aO[u] = S.redO[0][rp + 8 * u];
#pragma unroll
            for (int p = 1; p < NR; ++p) {
                aF[u] += S.redF[p][rp + 8 * u];
                aO[u] += S.redO[p][rp + 8 * u];
            }
            asm volatile("" : "+v"(aF[u]), "+v"(aO[u]));
        }
        asm volatile("s_waitcnt lgkmcnt(0)" ::: "memory");
        if (L == 0) LDS_ST(S.locdone, 1u);
        gather(aF, bad, S.nfor, S.bad, &S.redF[0][rp]);
        u4v v[4];
#pragma unroll
        for (int u = 0; u < 4; ++u) v[u] = __builtin_bit_cast(u4v, aF[u]);
        const unsigned long long tq = TNOW();
#pragma unroll
        for (int u = 0; u < 4; ++u) {
            v[u][0] = min(v[u][0], EMPTY - 1u);
            asm volatile("" : "+v"(v[u]));
        }
        if (tj < TILE_WJ) {
#pragma unroll
            for (int u = 0; u < 4; ++u) {
                __builtin_amdgcn_raw_buffer_store_b128(v[u], rf, of, 8 * u * NDIM * 4, AUX_ST);
                if (u < 3) wait_vm<TILE_WIN - 1>();
            }
        } else {
#pragma unroll
            for (int u = 0; u < 4; ++u)
                __builtin_amdgcn_raw_buffer_store_b128(v[u], rf, of, 8 * u * NDIM * 4, AUX_ST);
        }
        __builtin_amdgcn_sched_barrier(0);
        asm volatile("s_nop 1" ::: "memory");
        __builtin_amdgcn_sched_barrier(0);
        if (L == 0) LDS_ST(S.fdone, 1u);
        STAMPV(56, tq);
        STAMP(57);
        gather(aO, bad, S.nown, S.bad, &S.redO[0][rp]);
#pragma unroll
        for (int u = 0; u < 4; ++u)
            __builtin_amdgcn_raw_buffer_store_b128(__builtin_bit_cast(u4v, aO[u]), ro, oo, 8 * u * NDIM * 4, AUX_ST);
        STAMP(58);
        return;
    }
    const unsigned call = lds_wait<1>(S.call);
    const int par = (int)(call & 1u);

    const __amdgpu_buffer_rsrc_t rs = __builtin_amdgcn_make_buffer_rsrc(myws, 0, (int)WS_NEED, 0x00020000);
    const unsigned long long t0 = __builtin_amdgcn_s_memrealtime();
    bool bad = call == 0u;
    unsigned npoll = 0u, nre = 0u;
    auto late = [&]() -> bool {
        if (__builtin_amdgcn_s_memrealtime() - t0 > DEADLINE_TICKS) {
            if (L == 0)
                __hip_atomic_store((unsigned*)(myws + TMO_OFF), call, __ATOMIC_RELAXED, __HIP_MEMORY_SCOPE_AGENT);
            return true;
        }
        return false;
    };
    if (fwd && !bad) {
        const Ent en = entry(1, rank, dest, wv, s);
        const int pf = (int)(L == 0 ? flag_of(par, en.src, en.u) : TMO_OFF);
        u2v pv;
        for (;;) {
            pv = __builtin_amdgcn_raw_buffer_load_b64(rs, pf, 0, AUX_LD);
            if ((unsigned)__builtin_amdgcn_readlane((int)pv[1], 0) >= call) break;
            if ((unsigned)__builtin_amdgcn_readlane((int)pv[0], 1) == call || late()) { bad = true; break; }
            __builtin_amdgcn_s_sleep(SLEEP_R);
            asm volatile("" ::: "memory");
        }
        asm volatile("" ::: "memory");
        const Quad qd = load_quad(rs, (int)(data_of(par, en.src, en.u) + (unsigned)L * 16u));
        S.stage[wv][L] = QF(qd, 0);
        S.stage[wv][64 + L] = QF(qd, 1);
        S.stage[wv][128 + L] = QF(qd, 2);
        S.stage[wv][192 + L] = QF(qd, 3);
        if (L == 0) S.exp_[wv] = (int)(short)pv[0];
        asm volatile("s_waitcnt lgkmcnt(0)" ::: "memory");
        if (L == 0) LDS_ST(S.rdy[wv], 1);
        asm volatile("" ::: "memory");
        STAMP(70 + wv);
    }
    const int src = (rank + 1 + wv) & (NR - 1);
    const int oxf = (int)(data_of(par, src, 4 * ti) + (unsigned)L * 16u);
    const int ow = (int)(data_of(par, src, 32 + 4 * tj) + (unsigned)L * 16u);
    const int oxo = (int)(data_of(par, src, 160 + 4 * ti) + (unsigned)L * 16u);
    const int poff1 = (int)(L < 4 ? flag_of(par, src, 4 * ti + L)
                            : L < 8 ? flag_of(par, src, 32 + 4 * tj + L - 4)
                            : L < 12 ? flag_of(par, src, 160 + 4 * ti + L - 8) : TMO_OFF);
    const int poff0 = (int)(L < 12 ? (unsigned)poff1 : flag_of(par, src, 4 * ti) + 64u);
    unsigned lo = 0u, m = 0u;
    auto poll = [&]() {
        ++npoll;
        const u2v pv = __builtin_amdgcn_raw_buffer_load_b64(rs, (npoll % TMO_EVERY) == 0u ? poff1 : poff0, 0, AUX_LD);
        lo = pv[0];
        m = (unsigned)__builtin_amdgcn_ballot_w64(pv[1] >= call) & 0xfffu;
        if ((unsigned)__builtin_amdgcn_readlane((int)lo, 16) == call) bad = true;
    };
    unsigned cf[4];
    auto level = [&](int xb) -> int {
        int e[4];
#pragma unroll
        for (int k = 0; k < 4; ++k)
            e[k] = (int)(short)__builtin_amdgcn_readlane((int)lo, 4 + k) + (int)(short)__builtin_amdgcn_readlane((int)lo, xb + k);
        const int E = min(min(e[0], e[1]), min(e[2], e[3]));
#pragma unroll
        for (int k = 0; k < 4; ++k) cf[k] = down(e[k] - E);
        return E;
    };
    auto apply = [&](Quad (&Q)[4]) {
#pragma unroll
        for (int k = 0; k < 4; ++k) {
            scale_quad(Q[k], cf[k]);
            cf[k] = 0x3c00u;
        }
    };
    Quad X[4], W[4];
    for (poll(); !bad && (m & 0xfu) != 0xfu; poll()) {
        if (late()) { bad = true; break; }
        __builtin_amdgcn_s_sleep(SLEEP_E);
        asm volatile("" ::: "memory");
    }
    asm volatile("" ::: "memory");
    STAMP(16 + wv);
#pragma unroll
    for (int k = 0; k < 4; ++k) X[k] = load_quad<AUX_XF>(rs, oxf + 1024 * k);
    if (!lds_wait<1>(S.locdone)) bad = true;
    for (; !bad && (m & 0xf0u) != 0xf0u; poll()) {
        if (late()) { bad = true; break; }
        __builtin_amdgcn_s_sleep(SLEEP_W);
        asm volatile("" ::: "memory");
    }
    asm volatile("" ::: "memory");
    STAMP(24 + wv);
    const unsigned need = (unsigned)(__builtin_amdgcn_ballot_w64((lo & 0x10000u) != 0u) >> 4) & 0xfu;
    int E = level(0);
    for (;;) {
        asm volatile("" ::: "memory");
#pragma unroll
        for (int k = 0; k < 4; ++k) W[k] = load_quad(rs, ow + 1024 * k);
        __builtin_amdgcn_sched_barrier(0);
        apply(X);
        __builtin_amdgcn_sched_barrier(0);
        if ((m & 0xf00u) != 0xf00u) poll();
        if (need == 0u || bad) break;
        const unsigned ok = (quad_valid(W[0]) ? 1u : 0u) | (quad_valid(W[1]) ? 2u : 0u) | (quad_valid(W[2]) ? 4u : 0u) |
                            (quad_valid(W[3]) ? 8u : 0u);
        if ((need & ~ok) == 0u) break;
        if ((++nre & 15u) == 0u && late()) { bad = true; break; }
    }
    __builtin_amdgcn_sched_barrier(0);
    f16v acc = ZERO16;
#pragma unroll
    for (int k = 0; k < 4; ++k) acc = mm_quad(W[k], X[k], acc);
    __builtin_amdgcn_sched_barrier(0);
    auto foreign = [&]() {
        if (bad && L == 0) LDS_ST(S.bad, 1u);
        deposit(S.redF[wv], scale16(acc, -E), i, hh);
        arrive(S.nfor, 1u << wv);
        STAMP(32 + wv);
    };
    bool pend = (m & 0xf00u) == 0xf00u;
    if (!pend) {
        foreign();
        for (poll(); !bad && (m & 0xf00u) != 0xf00u; poll()) {
            if (late()) { bad = true; break; }
            __builtin_amdgcn_s_sleep(SLEEP_O);
            asm volatile("" ::: "memory");
        }
    }
    const bool needo = (__builtin_amdgcn_readlane((int)lo, 8) & 0x10000) != 0;
    const int EO = level(8);
    for (;;) {
        asm volatile("" ::: "memory");
#pragma unroll
        for (int k = 0; k < 4; ++k) X[k] = load_quad(rs, oxo + 1024 * k);
        __builtin_amdgcn_sched_barrier(0);
        if (pend) {
            foreign();
            pend = false;
            asm volatile("" : "=v"(acc));
        }
        __builtin_amdgcn_sched_barrier(0);
        apply(W);
        __builtin_amdgcn_sched_barrier(0);
        if (!needo || bad) break;
        const bool ok = quad_valid(X[0]) & quad_valid(X[1]) & quad_valid(X[2]) & quad_valid(X[3]);
        if (ok) break;
        if ((++nre & 15u) == 0u && late()) { bad = true; break; }
    }
    STAMP(40 + wv);
    __builtin_amdgcn_sched_barrier(0);
    acc = ZERO16;
#pragma unroll
    for (int k = 0; k < 4; ++k) acc = mm_quad(W[k], X[k], acc);
    if (bad && L == 0) LDS_ST(S.bad, 1u);
    deposit(S.redO[wv], scale16(acc, -EO), i, hh);
    arrive(S.nown, 1u << wv);
    STAMP(48 + wv);
}

extern "C" void kernel_launch(int rank, int nranks, ncclComm_t comm,
                              hipStream_t stream,
                              void* const* d_in, const int* in_sizes, int n_in,
                              void* d_out, int out_size,
                              const int* in_dtypes, int out_dtype,
                              void* const* d_out_all,
                              void* const* d_ws_all, size_t ws_bytes) {
    if (nranks != NR || n_in < 2 || ws_bytes < (size_t)WS_NEED || out_size != MS * NDIM) {
        fprintf(stderr, "unexpected problem geometry\n");
        return;
    }
    Args a;
    a.x = (const float*)d_in[0];
    a.w = (const float*)d_in[1];
    for (int p = 0; p < NR; ++p) { a.ws[p] = (char*)d_ws_all[p]; a.out[p] = (float*)d_out_all[p]; }
    a.out[rank] = (float*)d_out;
    a.rank = rank;
    gemm_rs_kernel<<<NWG, 640, 0, stream>>>(a);
    HIP_CHECK(hipGetLastError());
}
```

```cpp
#include <hip/hip_runtime.h>
#include <rccl/rccl.h>
#include <cstdio>
#include <cstdint>

#define HIP_CHECK(expr) do { hipError_t _e = (expr); if (_e != hipSuccess) { \
    fprintf(stderr, "HIP error %s at %s:%d\n", hipGetErrorString(_e), __FILE__, __LINE__); \
    return; } } while (0)

typedef _Float16 h8v __attribute__((ext_vector_type(8)));
typedef float f16v __attribute__((ext_vector_type(16)));
typedef float f4v __attribute__((ext_vector_type(4)));
typedef float f2v __attribute__((ext_vector_type(2)));
typedef unsigned u4v __attribute__((ext_vector_type(4)));
typedef unsigned u2v __attribute__((ext_vector_type(2)));

namespace {

constexpr int NR = 8;
constexpr int NDIM = 2048;
constexpr int KS = 256;
constexpr int MS = 256;
constexpr int NWG = 256;
constexpr int NSTR = 32;
constexpr int NPC = 48;
constexpr int RMAX = 7;
constexpr unsigned F_STR = 33 * 1024, R_STR = 4 * F_STR, S_STR = 6 * R_STR;
constexpr unsigned DATA_BYTES = 2u * NR * S_STR;
constexpr unsigned FLAG_STRIDE = 128;
constexpr unsigned FLAG_OFF = DATA_BYTES;
constexpr unsigned FLAG_BYTES = 2u * NR * NPC * FLAG_STRIDE;
constexpr unsigned OFL_OFF = FLAG_OFF + FLAG_BYTES;
constexpr unsigned OFL_STRIDE = 64;
constexpr unsigned CTR_OFF = OFL_OFF + NWG * OFL_STRIDE;
constexpr unsigned CTR_STRIDE = 64;
constexpr unsigned TMO_OFF = CTR_OFF + NWG * CTR_STRIDE;
constexpr unsigned PRG_OFF = TMO_OFF + 128;
constexpr unsigned WS_NEED = PRG_OFF + NR * 128;
#define DLT 50000000ull
constexpr unsigned long long DEADLINE_TICKS = DLT;

#define INFL 2
#define SLEEP_N 1
#define FB_T1 50
#define FB_T2 110
#define FB_T3 200
#define FB_TL 60
#define FB_W0 2
#define FB_END 6
#define FB_WE 4
#define SLEEP_E 16
#define SLEEP_W 8
#define SLEEP_O 2
#define SLEEP_T 2
#define SLEEP_T2 0
#define SLEEP_R 2
#define TMO_EVERY 16
#define FB_SLEEP 4
#define STAG_S 1
#define EVS 64
#define EVC -32
#define GPD 16
#define RGD 3
#define LEAD_W 2
#define LEAD_O 2
#define ZAT 4
#define PREFETCH 1
#define WL_S 1
#define WL_C 1
#define NL_S 1
#define NL_C 1
#define STAG_MODE 1
#define TILE_WIN 1
#define TILE_WJ 12
#define PROG_F 1
#define PROG_O 1
#define AUX_OWN 17
#define PUSH_POL 0
#define PUSH_FIRST 1
#define PUSH_LIKELY 0
#define PUSH_EXPECT(c) (c)
constexpr int GP = GPD, RG = RGD, QREL = (32 / GP) * RG * 4, FH = QREL / 4;
constexpr int LS = 192 + QREL, LC = 192 - QREL;
static_assert(LS <= 32 * RMAX && QREL <= 32 && QREL > 0, "stream geometry");
#define AUX_ST 17
#define AUX_LD 17

#define STAMP(idx) do {} while (0)
#define STAMPV(idx, v) do {} while (0)
#define TNOW() 0ull

#define LDS_LD(v) __builtin_amdgcn_readfirstlane(__hip_atomic_load(&(v), __ATOMIC_RELAXED, __HIP_MEMORY_SCOPE_WORKGROUP))
#define LDS_ST(v, x) __hip_atomic_store(&(v), (x), __ATOMIC_RELAXED, __HIP_MEMORY_SCOPE_WORKGROUP)

struct Args {
    const float* x;
    const float* w;
    char* ws;
    float* out;
    int rank, pad;
    char* dws[NR + 3];
    float* fout[NR / 2];
};

__device__ __forceinline__ unsigned data_of(int par, int src, int u) {
    return (unsigned)(par * NR + src) * S_STR + (unsigned)(u >> 5) * R_STR + (unsigned)(u & 31) * 1024u;
}
__device__ __forceinline__ unsigned flag_of(int par, int src, int u) {
    return FLAG_OFF + (unsigned)((par * NR + src) * NPC + (u >> 2)) * FLAG_STRIDE + (unsigned)(u & 3) * 8u;
}
__device__ __forceinline__ int frow(int r, int i) {
    return MS * (2 * (i >> 1) + 1 - (r & 1)) + 64 * (r >> 1) + 32 * (i & 1);
}

struct Ent { int kind, src, u, a0, kq; };
__device__ __forceinline__ int rounds(int typ, int s) {
    return typ == 2 ? (s < QREL ? 1 : 0) : ((typ ? LS : LC) - s + 31) >> 5;
}
__device__ __forceinline__ Ent entry(int typ, int rank, int dest, int rho, int s) {
    Ent e;
    e.src = rank;
    if (typ == 2) {
        const int x = s % FH, j = GP * (x / RG) + GP - RG + x % RG;
        e.kind = 1; e.kq = dest >> 1; e.a0 = 1024 * (dest & 1) + 32 * j; e.u = 32 + 4 * j + e.kq;
        return e;
    }
    if (rho == 0) { e.kind = 0; e.kq = s & 3; e.a0 = frow(dest, s >> 2); e.u = s; return e; }
    const int n = 32 * (rho - 1) + s, o = n - (typ ? LS - 64 : LC - 64);
    if (o >= 0) { e.kind = 0; e.kq = o & 3; e.a0 = MS * dest + 32 * (o >> 2); e.u = 160 + o; return e; }
    constexpr int GS = 4 * (GP + RG), GC = 4 * (GP - RG);
    const int g = typ ? n / GS : n / GC, t = n - g * (typ ? GS : GC);
    if (t < 4 * GP) {
        const int j = GP * g + (t >> 2);
        e.kind = 1; e.kq = t & 3; e.a0 = 1024 * (dest & 1) + 32 * j; e.u = 32 + 4 * j + e.kq;
    } else {
        const int r = t - 4 * GP;
        e.kind = 2; e.kq = rank >> 1; e.a0 = 0; e.src = 2 * (r & 3) + 1 - (rank & 1);
        e.u = 32 + 4 * (GP * g + GP - RG + (r >> 2)) + e.kq;
    }
    return e;
}
__device__ __forceinline__ int first_early(int typ, int s) {
    return typ == 1 ? (LS - 32 - EVS - s + 31) >> 5 : typ == 0 ? (LC - 32 - (EVC) - s + 31) >> 5 : 1;
}

__device__ __forceinline__ bool clears(int typ, int rank, int s) { return typ == 0 && (s >> 3) == (rank >> 1); }
constexpr unsigned EMPTY = 0xffffffffu;

template <int NV>
__device__ __forceinline__ int block_exp(const float (&v)[NV]) {
    float m = 0.f;
#pragma unroll
    for (int t = 0; t < NV; ++t) m = fmaxf(m, fabsf(v[t]));
    unsigned u = __float_as_uint(m);
    u = max(u, (unsigned)__builtin_amdgcn_update_dpp(0, (int)u, 0xB1, 0xf, 0xf, true));
    u = max(u, (unsigned)__builtin_amdgcn_update_dpp(0, (int)u, 0x4E, 0xf, 0xf, true));
    u = max(u, (unsigned)__builtin_amdgcn_update_dpp(0, (int)u, 0x141, 0xf, 0xf, true));
    u = max(u, (unsigned)__builtin_amdgcn_update_dpp(0, (int)u, 0x140, 0xf, 0xf, true));
    const unsigned r = max(max((unsigned)__builtin_amdgcn_readlane((int)u, 0), (unsigned)__builtin_amdgcn_readlane((int)u, 16)),
                           max((unsigned)__builtin_amdgcn_readlane((int)u, 32), (unsigned)__builtin_amdgcn_readlane((int)u, 48)));
    unsigned E = r >> 23;
    E = E < 15u ? 15u : E;
    return 141 - (int)E;
}

__device__ __forceinline__ u4v pack8(const float* v, int e) {
    h8v r;
    typedef _Float16 h2v __attribute__((ext_vector_type(2)));
    const float sc = __uint_as_float((unsigned)(e + 127) << 23);
    const f2v s2 = {sc, sc};
#pragma unroll
    for (int j = 0; j < 8; j += 2) {
        const h2v t = __builtin_convertvector(f2v{v[j], v[j + 1]} * s2, h2v);
        r[j] = t[0];
        r[j + 1] = t[1];
    }
    return __builtin_bit_cast(u4v, r);
}

template <int NF>
__device__ __forceinline__ void load_w(const float* __restrict__ w, int k0, int col0, int i, int hh, float (&v)[8 * NF]) {
    const float* base = w + (size_t)(k0 + 8 * hh) * NDIM + col0 + i;
#pragma unroll
    for (int s = 0; s < NF; ++s)
#pragma unroll
        for (int j = 0; j < 8; ++j) v[8 * s + j] = base[(size_t)(16 * s + j) * NDIM];
}

template <int NF>
__device__ __forceinline__ void load_x(const float* __restrict__ x, int row0, int k0, int i, int hh, float (&v)[8 * NF]) {
    const float* base = x + (size_t)(row0 + i) * KS + k0 + 8 * hh;
#pragma unroll
    for (int s = 0; s < NF; ++s) {
        f4v t0 = *(const f4v*)(base + 16 * s);
        f4v t1 = *(const f4v*)(base + 16 * s + 4);
#pragma unroll
        for (int j = 0; j < 4; ++j) { v[s * 8 + j] = t0[j]; v[s * 8 + 4 + j] = t1[j]; }
    }
}

__device__ __forceinline__ f16v mm(u4v a, u4v b, f16v c) {
    return __builtin_amdgcn_mfma_f32_32x32x16_f16(__builtin_bit_cast(h8v, a), __builtin_bit_cast(h8v, b), c, 0, 0, 0);
}
#define ZERO16 f16v{0.f, 0.f, 0.f, 0.f, 0.f, 0.f, 0.f, 0.f, 0.f, 0.f, 0.f, 0.f, 0.f, 0.f, 0.f, 0.f}

__device__ __forceinline__ f16v scale16(f16v t, int e) {
    const int e1 = e >> 1, e2 = e - e1;
    const float c1 = __uint_as_float((unsigned)(e1 + 127) << 23), c2 = __uint_as_float((unsigned)(e2 + 127) << 23);
    const f2v s1 = {c1, c1}, s2 = {c2, c2};
#pragma unroll
    for (int r = 0; r < 16; r += 2) {
        const f2v u = (f2v{t[r], t[r + 1]} * s1) * s2;
        t[r] = u[0]; t[r + 1] = u[1];
    }
    return t;
}

__device__ __forceinline__ void fma16(f16v& acc, f16v t, int e) {
    const int e1 = e >> 1, e2 = e - e1;
    const float c1 = __uint_as_float((unsigned)(e1 + 127) << 23), c2 = __uint_as_float((unsigned)(e2 + 127) << 23);
    const f2v s1 = {c1, c1}, s2 = {c2, c2};
#pragma unroll
    for (int r = 0; r < 16; r += 2) {
        const f2v u = __builtin_elementwise_fma(f2v{t[r], t[r + 1]} * s1, s2, f2v{acc[r], acc[r + 1]});
        acc[r] = u[0]; acc[r + 1] = u[1];
    }
}

struct Quad { u4v f[4]; };
#define QF(q, n) ((q).f[n])

#define AUX_XF AUX_LD
#define AUX_W AUX_LD
#define AUX_XO AUX_LD
template <int AUX = AUX_LD>
__device__ __forceinline__ Quad load_quad(__amdgpu_buffer_rsrc_t rs, int off) {
    Quad q;
#pragma unroll
    for (int f = 0; f < 4; ++f) q.f[f] = __builtin_amdgcn_raw_buffer_load_b128(rs, off, f * (int)F_STR, AUX);
    return q;
}

__device__ __forceinline__ f16v mm_quad(const Quad& w, const Quad& x, f16v t) {
    t = mm(QF(w, 0), QF(x, 0), t);
    t = mm(QF(w, 1), QF(x, 1), t);
    t = mm(QF(w, 2), QF(x, 2), t);
    return mm(QF(w, 3), QF(x, 3), t);
}

__device__ __forceinline__ unsigned down(int d) {
    return d <= 14 ? (unsigned)(15 - d) << 10 : (1u << 24 >> min(d, 31));
}
__device__ __forceinline__ void scale_quad(Quad& q, unsigned c) {
    const _Float16 h = __builtin_bit_cast(_Float16, (unsigned short)c);
#pragma unroll
    for (int f = 0; f < 4; ++f) {
        q.f[f] = __builtin_bit_cast(u4v, __builtin_bit_cast(h8v, q.f[f]) * h);
        asm volatile("" : "+v"(q.f[f]));
    }
}

__device__ __forceinline__ u4v no_zero(u4v u) {
    u[0] = u[0] ? u[0] : 0x8000u;
    return u;
}

__device__ __forceinline__ bool quad_valid(const Quad& q) {
    return __builtin_amdgcn_ballot_w64(min(min(q.f[0][0], q.f[1][0]), min(q.f[2][0], q.f[3][0])) == 0u) == 0ull;
}

template <int N>
__device__ __forceinline__ void wait_vm() {
    asm volatile("s_waitcnt vmcnt(%0)" ::"i"(N) : "memory");
}

__device__ __forceinline__ void wait_vm_n(int n) {
    switch (n) {
        case 0: wait_vm<0>(); break;
        case 1: wait_vm<1>(); break;
        case 2: wait_vm<2>(); break;
        case 3: wait_vm<3>(); break;
        case 4: wait_vm<4>(); break;
        case 5: wait_vm<5>(); break;
        case 6: wait_vm<6>(); break;
        case 7: wait_vm<7>(); break;
        case 8: wait_vm<8>(); break;
        case 9: wait_vm<9>(); break;
        default: wait_vm<10>(); break;
    }
}

__device__ __forceinline__ void raw_barrier() {
    asm volatile("s_waitcnt lgkmcnt(0)\n\ts_barrier" ::: "memory");
}

template <int SLP, class T>
__device__ __forceinline__ unsigned lds_wait(T& v, unsigned n = 1u) {
    unsigned r, it = 0u;
    unsigned long long t0 = 0ull;
    while ((r = (unsigned)LDS_LD(v)) < n) {
        __builtin_amdgcn_s_sleep(SLP);
        if ((++it & 1023u) == 0u) {
            const unsigned long long t = __builtin_amdgcn_s_memrealtime();
            if (t0 == 0ull) t0 = t;
            else if (t - t0 > 2ull * DEADLINE_TICKS) { r = 0u; break; }
        }
    }
    asm volatile("" ::: "memory");
    return r;
}

struct Lds {
    u4v stage[RMAX][256];
    f4v redF[NR][8 * 33];
    f4v redO[NR][8 * 33];
    int exp_[RMAX + 1];
    int rdy[RMAX + 1];
    unsigned call, bad, nloc, locdone, nfor, nown, fdone, clr;
    int prog, win, zok;
};

__device__ __forceinline__ void deposit(f4v* slot, const f16v& t, int i, int hh) {
#pragma unroll
    for (int g = 0; g < 4; ++g)
        slot[(2 * g + hh) * 33 + i] = f4v{t[4 * g], t[4 * g + 1], t[4 * g + 2], t[4 * g + 3]};
}
__device__ __forceinline__ void arrive(unsigned& c, unsigned v) {
    unsigned long long ex;
    asm volatile("s_mov_b64 %0, exec\n\ts_mov_b64 exec, 1\n\tds_add_u32 %1, %2\n\ts_mov_b64 exec, %0"
                 : "=&s"(ex)
                 : "v"((unsigned)(uintptr_t)(__attribute__((address_space(3))) unsigned*)&c), "v"(v)
                 : "memory");
}
__device__ __forceinline__ void gather(f4v (&acc)[4], bool& bad, unsigned& flags, unsigned& badw, const f4v* red) {
    unsigned got = 0u, it = 0u;
    unsigned long long t0 = 0ull;
    auto late = [&]() -> bool {
        if ((++it & 1023u) == 0u) {
            const unsigned long long t = __builtin_amdgcn_s_memrealtime();
            if (t0 == 0ull) t0 = t;
            else if (t - t0 > 2ull * DEADLINE_TICKS) return true;
        }
        return false;
    };
    for (int n = 0; n < NR - 2 && !bad;) {
        const unsigned m = (unsigned)LDS_LD(flags) & ~got;
        asm volatile("" ::: "memory");
        if (m == 0u) {
            __builtin_amdgcn_s_sleep(0);
            bad = late();
            continue;
        }
        const int p = __builtin_ctz(m);
#pragma unroll
        for (int u = 0; u < 4; ++u) acc[u] += red[p * (8 * 33) + 8 * u];
        got |= 1u << p;
        ++n;
    }
    const int p = __builtin_ctz(~got);
    f4v v[4];
    for (;;) {
        asm volatile("" ::: "memory");
        const unsigned c = __hip_atomic_load(&flags, __ATOMIC_RELAXED, __HIP_MEMORY_SCOPE_WORKGROUP);
        __builtin_amdgcn_sched_barrier(0);
        asm volatile("" ::: "memory");
        const unsigned w = __hip_atomic_load(&badw, __ATOMIC_RELAXED, __HIP_MEMORY_SCOPE_WORKGROUP);
#pragma unroll
        for (int u = 0; u < 4; ++u) v[u] = red[p * (8 * 33) + 8 * u];
        __builtin_amdgcn_sched_barrier(0);
        if (__builtin_amdgcn_readfirstlane(c) == 0x7fu) { bad |= __builtin_amdgcn_readfirstlane(w) != 0u; break; }
        asm volatile("s_waitcnt lgkmcnt(0)" ::: "memory");
        if (bad || late()) { bad = true; break; }
    }
#pragma unroll
    for (int u = 0; u < 4; ++u) acc[u] += v[u];
    if (bad) {
        const float qn = __builtin_nanf("");
#pragma unroll
        for (int u = 0; u < 4; ++u) acc[u] = f4v{qn, qn, qn, qn};
    }
}

__device__ __forceinline__ void pusher(const Args& a, Lds& S, char* dws, int b, int rank, int dest, int s, int L, int typ) {
    const int nrnd = rounds(typ, s), NST = 4 * nrnd;
    char* const myws = a.ws;
    const unsigned long long tq0 = TNOW();
    const unsigned cv = __hip_atomic_load((unsigned*)(myws + CTR_OFF + b * CTR_STRIDE), __ATOMIC_RELAXED,
                                          __HIP_MEMORY_SCOPE_AGENT);
    const int i = L & 31, hh = L >> 5;
    u4v t[4];
    unsigned vdat, vflg;
    {
        const Ent en = entry(typ, rank, dest, 0, s);
        float v0[32];
        if (typ == 2) load_w<4>(a.w, 64 * en.kq, en.a0, i, hh, v0);
        else load_x<4>(a.x, en.a0, 64 * en.kq, i, hh, v0);
        const Ent le = entry(typ, rank, dest, L < nrnd ? L : 0, s);
        vdat = data_of(0, le.src, le.u);
        vflg = flag_of(0, le.src, le.u);
        asm volatile("" : "+v"(vdat), "+v"(vflg));
        if (L <= RMAX) S.rdy[L] = 0;
        if (L == 0) {
            S.call = 0u; S.bad = 0u; S.nloc = 0u; S.locdone = 0u; S.nfor = 0u; S.nown = 0u; S.fdone = 0u;
            S.clr = clears(typ, rank, s) ? 0u : 1u;
            S.prog = 0; S.zok = 0; S.win = typ == 2 ? 6 : INFL;
        }
        raw_barrier();
        const int e = block_exp<32>(v0);
        if (L == 0) { LDS_ST(S.rdy[0], 1); S.exp_[0] = e; }
#pragma unroll
        for (int f = 0; f < 4; ++f) t[f] = pack8(v0 + 8 * f, e);
        if (typ == 2) {
#pragma unroll
            for (int f = 0; f < 4; ++f) t[f] = no_zero(t[f]);
        }
    }
    const unsigned call = __builtin_amdgcn_readfirstlane(cv) + 1u;
    if (L == 0) LDS_ST(S.call, call);
    if (nrnd == 0) return;
    const int par = (int)(call & 1u);
    __builtin_amdgcn_s_setprio(3);
    const __amdgpu_buffer_rsrc_t rs = __builtin_amdgcn_make_buffer_rsrc(dws, 0, (int)DATA_BYTES, 0x00020000);
    vdat += (unsigned)par * (NR * S_STR);
    vflg += (unsigned)par * (NR * NPC * FLAG_STRIDE);
    auto publish = [&](int c, unsigned early) {
        if (L == c) {
            const unsigned long long val = ((unsigned long long)call << 32) | (((unsigned)S.exp_[c] & 0xffffu) | early);
            __hip_atomic_store((unsigned long long*)(dws + vflg), val, __ATOMIC_RELAXED, __HIP_MEMORY_SCOPE_SYSTEM);
        }
    };
    int issued = 0, pub = 0, dn = 0, h1 = 0, h2 = 0, h3 = 0, h4 = 0, h5 = 0, h6 = 0, h7 = 0;
    const int ev = first_early(typ, s);
    const int nl = 4 * ((typ == 1 ? LS : LC) >> 5) - (typ == 1 ? NL_S : NL_C);
    for (int c = 0;; ++c) {
        const int so = __builtin_amdgcn_readlane((int)vdat, c);
        bool have = false;
#pragma unroll
        for (int f = 0; f < 4; ++f) {
            const int n = 4 * c + f;
            __builtin_amdgcn_raw_buffer_store_b128(t[f], rs, L * 16, so + f * (int)F_STR, AUX_ST);
            const int h0 = ++issued;
            __builtin_amdgcn_sched_barrier(0);
            if (c >= ev && f == (c + 1 == nrnd ? LEAD_O : LEAD_W)) {
                if (c + 1 == nrnd && !lds_wait<1>(S.zok)) {
                    if (L == 0) { LDS_ST(S.bad, 1u); LDS_ST(S.prog, NST); }
                    return;
                }
                publish(c, 0x10000u); ++issued;
            }
            if (L == 0) LDS_ST(S.prog, n + 1 == NST ? NST : dn);
            while (pub < ev && 4 * (pub + 1) <= dn && (pub > 0 || LDS_LD(S.clr))) { publish(pub, 0u); ++issued; ++pub; }
            if (f == 3) {
                if (n + 1 == NST) break;
                have = LDS_LD(S.rdy[c + 1]) != 0;
                asm volatile("" ::: "memory");
                if (have) {
#pragma unroll
                    for (int g = 0; g < 4; ++g) t[g] = S.stage[c + 1][g * 64 + L];
                }
            }
            int win = LDS_LD(S.win);
            if (n >= nl) win = max(win, typ == 1 ? WL_S : WL_C);
            const int ik = win <= 1 ? h0 : win == 2 ? h1 : win == 3 ? h2 : win == 4 ? h3 : win <= 6 ? h5 : h7;
            const int k1 = n + 2 - (win > 6 ? 8 : win > 4 ? 6 : win);
            if (k1 > dn) {
                wait_vm_n(issued - ik);
                dn = k1;
            }
            h7 = h6; h6 = h5; h5 = h4; h4 = h3; h3 = h2; h2 = h1; h1 = h0;
        }
        if (c + 1 == nrnd) break;
        if (!have) {
            while (pub < ev && 4 * (pub + 1) <= dn && (pub > 0 || LDS_LD(S.clr))) { publish(pub, 0u); ++issued; ++pub; }
            if (!lds_wait<0>(S.rdy[c + 1])) {
                if (L == 0) { LDS_ST(S.bad, 1u); LDS_ST(S.prog, NST); }
                return;
            }
#pragma unroll
            for (int g = 0; g < 4; ++g) t[g] = S.stage[c + 1][g * 64 + L];
        }
    }
    const unsigned long long tq1 = TNOW();
    if (pub == 0 && ev > 0) {
        if (!lds_wait<1>(S.clr)) {
            if (L == 0) LDS_ST(S.bad, 1u);
            return;
        }
        while (pub < ev && 4 * (pub + 1) <= dn) { publish(pub, 0u); ++issued; ++pub; }
    }
    if (pub < ev && pub == nrnd - 3) { wait_vm_n(issued - h7); publish(pub, 0u); ++issued; ++pub; }
    if (pub < ev && pub == nrnd - 2) { wait_vm_n(issued - h4); publish(pub, 0u); ++pub; }
    if (pub < ev) { wait_vm<0>(); publish(pub, 0u); }
}

__device__ __forceinline__ bool feed(const Args& a, Lds& S, int rank, int dest, int s, int wv, int L, int typ) {
    if (wv < 1 || wv >= rounds(typ, s)) return false;
    const Ent en = entry(typ, rank, dest, wv, s);
    if (en.kind == 2) return true;
    const int i = L & 31, hh = L >> 5;
    float v[32];
    if (en.kind == 1) load_w<4>(a.w, 64 * en.kq, en.a0, i, hh, v);
    else load_x<4>(a.x, en.a0, 64 * en.kq, i, hh, v);
    const int e = block_exp<32>(v);
#pragma unroll
    for (int f = 0; f < 4; ++f) {
        S.stage[wv][f * 64 + L] = no_zero(pack8(v + 8 * f, e));
    }
    if (L == 0) S.exp_[wv] = e;
    asm volatile("s_waitcnt lgkmcnt(0)" ::: "memory");
    if (L == 0) LDS_ST(S.rdy[wv], 1);
    asm volatile("" ::: "memory");
    return false;
}

}

__global__ void __launch_bounds__(640) gemm_rs_kernel(Args a) {
    __shared__ __attribute__((aligned(16))) Lds S;

    const int b = blockIdx.x;
    const int wr = __builtin_amdgcn_readfirstlane(threadIdx.x >> 6);
    const int wv = wr == 0 ? NR : wr <= NR ? wr - 1 : NR + 1;
    const int L = threadIdx.x & 63;
    const int i = L & 31, hh = L >> 5;
    const int ti = b & 7, tj = (b >> 3) & 31;
    const int lk = (b >> 5) & 7, s = b & 31;
    char* const dws = a.dws[lk < 7 ? lk : 7 + ((s / FH) & 3)];
    float* const pf0 = a.fout[ti >> 1];
    char* const myws = a.ws;
    asm volatile("" ::"s"(a.x), "s"(a.w), "s"(myws), "s"(a.out), "s"(a.rank), "s"(dws), "s"(pf0));
    const int rank = a.rank & (NR - 1);
    const int cls = rank & 1, mem = rank >> 1;
    const int dest = lk < 7 ? (rank + 1 + lk) & (NR - 1) : 2 * ((s / FH) & 3) + 1 - cls;
    const int typ = lk == 7 ? 2 : lk & 1;

    if (PUSH_EXPECT(wv == NR)) {
        pusher(a, S, dws, b, rank, dest, s, L, typ);
        return;
    }

    if (wv == NR + 1) {
        raw_barrier();
        const unsigned call = lds_wait<1>(S.call);
        if (call == 0u) return;
        if (L == 0)
            __hip_atomic_store((unsigned*)(myws + CTR_OFF + b * CTR_STRIDE), call, __ATOMIC_RELAXED, __HIP_MEMORY_SCOPE_AGENT);
        if (lk < 7) {
            const int nrnd = rounds(typ, s), NST = 4 * nrnd, LT = typ ? 4 * LS : 4 * LC;
            int z = 0;
            auto clear = [&]() {
                const __amdgpu_buffer_rsrc_t rz = __builtin_amdgcn_make_buffer_rsrc(myws, 0, (int)DATA_BYTES, 0x00020000);
                for (int c = first_early(typ, s); c < nrnd; ++c) {
                    const Ent en = entry(typ, dest, rank, c, s);
                    const unsigned zo = data_of((int)(call & 1u) ^ 1, en.src, en.u);
#pragma unroll
                    for (int f = 0; f < 4; ++f)
                        __builtin_amdgcn_raw_buffer_store_b128(u4v{0u, 0u, 0u, 0u}, rz, L * 16, (int)(zo + f * F_STR), AUX_ST);
                }
                z = 1;
            };
            unsigned* pc = (unsigned*)(myws + PRG_OFF + (unsigned)dest * 128u);
            const unsigned base = (call - 1u) * (unsigned)LT;
            int last = 0;
            unsigned it = 0u;
            unsigned long long t0 = 0ull;
            for (;;) {
                int p = LDS_LD(S.prog);
                if ((++it & 255u) == 0u) {
                    const unsigned long long t = __builtin_amdgcn_s_memrealtime();
                    if (t0 == 0ull) t0 = t;
                    else if (t - t0 > 2ull * DEADLINE_TICKS) p = NST;
                }
                const int dl = p > last ? p - last : 0;
                last += dl;
                if (last >= NST) {
                    if (L == 0) __hip_atomic_fetch_add(pc, (unsigned)dl, __ATOMIC_RELAXED, __HIP_MEMORY_SCOPE_AGENT);
                    break;
                }
                if (z == 0 && last >= ZAT) clear();
                unsigned g = 0u;
                if (L == 0) g = __hip_atomic_fetch_add(pc, (unsigned)dl, __ATOMIC_RELAXED, __HIP_MEMORY_SCOPE_AGENT) + (unsigned)dl;
                const int tot = (int)(__builtin_amdgcn_readfirstlane(g) - base);
                if (z == 1) {
                    wait_vm<0>();
                    if (L == 0) LDS_ST(S.zok, 1);
                    z = 2;
                }
                const int lag = tot - NSTR * last;
                int win = lag > FB_T3 ? 6 : lag > FB_T2 ? 4 : lag > FB_T1 ? 3 : lag < -FB_TL ? 1 : 2;
                if (tot >= LT - FB_END * NSTR && win < FB_WE) win = FB_WE;
                if (L == 0 && last > 0) LDS_ST(S.win, win);
                __builtin_amdgcn_s_sleep(FB_SLEEP);
            }
            if (z == 0) clear();
            if (z == 1) {
                wait_vm<0>();
                if (L == 0) LDS_ST(S.zok, 1);
            }
        }
        {
            const __amdgpu_buffer_rsrc_t ro = __builtin_amdgcn_make_buffer_rsrc(a.out, 0, MS * NDIM * 4, 0x00020000);
            const int oo = ((32 * ti + (L >> 3)) * NDIM + 1024 * (1 - cls) + 32 * tj + 4 * (L & 7)) * 4;
            bool bad = !lds_wait<SLEEP_T>(S.fdone);
            const unsigned long long t0 = __builtin_amdgcn_s_memrealtime();
            unsigned ok = 0u, it = 0u;
            struct T4 { u4v v[4]; };
            auto look = [&]() -> T4 {
                T4 r;
#pragma unroll
                for (int u = 0; u < 4; ++u)
                    r.v[u] = __builtin_amdgcn_raw_buffer_load_b128(ro, oo + ((ok >> u & 1u) ? 0x40000000 : 0), 8 * u * NDIM * 4, AUX_LD);
                __builtin_amdgcn_sched_barrier(0);
                return r;
            };
            auto check = [&](const T4& r) {
#pragma unroll
                for (int u = 0; u < 4; ++u) {
                    const u4v v = r.v[u];
                    if (__builtin_amdgcn_ballot_w64(v[0] == EMPTY) == 0ull) ok |= 1u << u;
                }
                __builtin_amdgcn_sched_barrier(0);
            };
            while (!bad) {
                asm volatile("" ::: "memory");
                check(look());
                if (ok != 15u && (++it & 15u) != 0u) {
                    __builtin_amdgcn_s_sleep(SLEEP_T2);
                    continue;
                }
                if (ok == 15u) break;
                const unsigned v = __hip_atomic_load((const unsigned*)(myws + TMO_OFF), __ATOMIC_RELAXED, __HIP_MEMORY_SCOPE_SYSTEM);
                if (__builtin_amdgcn_readfirstlane(v) == call) bad = true;
                if (__builtin_amdgcn_s_memrealtime() - t0 > DEADLINE_TICKS) {
                    if (L == 0)
                        __hip_atomic_store((unsigned*)(myws + TMO_OFF), call, __ATOMIC_RELAXED, __HIP_MEMORY_SCOPE_AGENT);
                    bad = true;
                }
            }
            if (bad) {
                const unsigned qn = 0x7fc00000u;
#pragma unroll
                for (int u = 0; u < 4; ++u)
                    __builtin_amdgcn_raw_buffer_store_b128(u4v{qn, qn, qn, qn}, ro, oo, 8 * u * NDIM * 4, AUX_ST);
            }
            STAMP(59);
        }
        return;
    }

    raw_barrier();
    const bool clr = wv == NR - 1 && clears(typ, rank, s);
    if (clr) {
        const __amdgpu_buffer_rsrc_t ro = __builtin_amdgcn_make_buffer_rsrc(a.out, 0, MS * NDIM * 4, 0x00020000);
        const int oo = ((64 * (dest >> 1) + 32 * ((s >> 2) & 1)) * NDIM + 1024 * (dest & 1) + 256 * (s & 3) + 4 * L) * 4;
#pragma unroll
        for (int u = 0; u < 32; ++u)
            __builtin_amdgcn_raw_buffer_store_b128(u4v{EMPTY, EMPTY, EMPTY, EMPTY}, ro, oo, u * NDIM * 4, AUX_ST);
        wait_vm<0>();
        if (L == 0) LDS_ST(S.clr, 1u);
    }
    if (wv != 1) lds_wait<STAG_S>(S.rdy[0]);
    const bool fwd = feed(a, S, rank, dest, s, wv, L, typ);
    const int col0 = 1024 * cls + 32 * tj;
    {
        float vw[16], vf[16], vo[16];
        load_w<2>(a.w, 32 * wv, col0, i, hh, vw);
        load_x<2>(a.x, frow(rank, ti), 32 * wv, i, hh, vf);
        load_x<2>(a.x, MS * rank + 32 * ti, 32 * wv, i, hh, vo);
        const int ew = block_exp<16>(vw);
        const u4v w0 = pack8(vw, ew), w1 = pack8(vw + 8, ew);
        const int ef = block_exp<16>(vf);
        f16v t = mm(w0, pack8(vf, ef), ZERO16);
        t = mm(w1, pack8(vf + 8, ef), t);
        deposit(S.redF[wv], scale16(t, -(ew + ef)), i, hh);
        const int eo = block_exp<16>(vo);
        t = mm(w0, pack8(vo, eo), ZERO16);
        t = mm(w1, pack8(vo + 8, eo), t);
        deposit(S.redO[wv], scale16(t, -(ew + eo)), i, hh);
        arrive(S.nloc, 1u);
    }
    STAMP(60 + wv);
    if (wv == NR - 1) {
        float* pf = pf0;
        float* po = a.out;
        asm volatile("" : "+s"(pf), "+s"(po));
        const __amdgpu_buffer_rsrc_t rf = __builtin_amdgcn_make_buffer_rsrc(pf, 0, MS * NDIM * 4, 0x00020000);
        const __amdgpu_buffer_rsrc_t ro = __builtin_amdgcn_make_buffer_rsrc(po, 0, MS * NDIM * 4, 0x00020000);
        int of = ((64 * mem + 32 * (ti & 1) + (L >> 3)) * NDIM + col0 + 4 * (L & 7)) * 4;
        int oo = ((32 * ti + (L >> 3)) * NDIM + col0 + 4 * (L & 7)) * 4;
        asm volatile("" : "+v"(of), "+v"(oo));
        const int rp = (L & 7) * 33 + (L >> 3);
        f4v aF[4], aO[4];
        bool bad = !lds_wait<1>(S.nloc, NR);
#pragma unroll
        for (int u = 0; u < 4; ++u) {
            aF[u] = S.redF[0][rp + 8 * u];
            aO[u] = S.redO[0][rp + 8 * u];
#pragma unroll
            for (int p = 1; p < NR; ++p) {
                aF[u] += S.redF[p][rp + 8 * u];
                aO[u] += S.redO[p][rp + 8 * u];
            }
            asm volatile("" : "+v"(aF[u]), "+v"(aO[u]));
        }
        asm volatile("s_waitcnt lgkmcnt(0)" ::: "memory");
        if (L == 0) LDS_ST(S.locdone, 1u);
        gather(aF, bad, S.nfor, S.bad, &S.redF[0][rp]);
        u4v v[4];
#pragma unroll
        for (int u = 0; u < 4; ++u) v[u] = __builtin_bit_cast(u4v, aF[u]);
        const unsigned long long tq = TNOW();
#pragma unroll
        for (int u = 0; u < 4; ++u) {
            v[u][0] = min(v[u][0], EMPTY - 1u);
            asm volatile("" : "+v"(v[u]));
        }
        if (tj < TILE_WJ) {
#pragma unroll
            for (int u = 0; u < 4; ++u) {
                __builtin_amdgcn_raw_buffer_store_b128(v[u], rf, of, 8 * u * NDIM * 4, AUX_ST);
                if (u < 3) wait_vm<TILE_WIN - 1>();
            }
        } else {
#pragma unroll
            for (int u = 0; u < 4; ++u)
                __builtin_amdgcn_raw_buffer_store_b128(v[u], rf, of, 8 * u * NDIM * 4, AUX_ST);
        }
        __builtin_amdgcn_sched_barrier(0);
        asm volatile("s_nop 1" ::: "memory");
        __builtin_amdgcn_sched_barrier(0);
        if (L == 0) LDS_ST(S.fdone, 1u);
        STAMPV(56, tq);
        STAMP(57);
        gather(aO, bad, S.nown, S.bad, &S.redO[0][rp]);
#pragma unroll
        for (int u = 0; u < 4; ++u)
            __builtin_amdgcn_raw_buffer_store_b128(__builtin_bit_cast(u4v, aO[u]), ro, oo, 8 * u * NDIM * 4, AUX_OWN);
        STAMP(58);
        return;
    }
    const unsigned call = lds_wait<1>(S.call);
    const int par = (int)(call & 1u);

    const __amdgpu_buffer_rsrc_t rs = __builtin_amdgcn_make_buffer_rsrc(myws, 0, (int)WS_NEED, 0x00020000);
    const unsigned long long t0 = __builtin_amdgcn_s_memrealtime();
    bool bad = call == 0u;
    unsigned npoll = 0u, nre = 0u;
    auto late = [&]() -> bool {
        if (__builtin_amdgcn_s_memrealtime() - t0 > DEADLINE_TICKS) {
            if (L == 0)
                __hip_atomic_store((unsigned*)(myws + TMO_OFF), call, __ATOMIC_RELAXED, __HIP_MEMORY_SCOPE_AGENT);
            return true;
        }
        return false;
    };
    if (fwd && !bad) {
        const Ent en = entry(1, rank, dest, wv, s);
        const int pf = (int)(L == 0 ? flag_of(par, en.src, en.u) : TMO_OFF);
        u2v pv;
        for (;;) {
            pv = __builtin_amdgcn_raw_buffer_load_b64(rs, pf, 0, AUX_LD);
            if ((unsigned)__builtin_amdgcn_readlane((int)pv[1], 0) >= call) break;
            if ((unsigned)__builtin_amdgcn_readlane((int)pv[0], 1) == call || late()) { bad = true; break; }
            __builtin_amdgcn_s_sleep(SLEEP_R);
            asm volatile("" ::: "memory");
        }
        asm volatile("" ::: "memory");
        const Quad qd = load_quad(rs, (int)(data_of(par, en.src, en.u) + (unsigned)L * 16u));
        S.stage[wv][L] = QF(qd, 0);
        S.stage[wv][64 + L] = QF(qd, 1);
        S.stage[wv][128 + L] = QF(qd, 2);
        S.stage[wv][192 + L] = QF(qd, 3);
        if (L == 0) S.exp_[wv] = (int)(short)pv[0];
        asm volatile("s_waitcnt lgkmcnt(0)" ::: "memory");
        if (L == 0) LDS_ST(S.rdy[wv], 1);
        asm volatile("" ::: "memory");
        STAMP(70 + wv);
    }
    const int src = (rank + 1 + wv) & (NR - 1);
    const int oxf = (int)(data_of(par, src, 4 * ti) + (unsigned)L * 16u);
    const int ow = (int)(data_of(par, src, 32 + 4 * tj) + (unsigned)L * 16u);
    const int oxo = (int)(data_of(par, src, 160 + 4 * ti) + (unsigned)L * 16u);
    const int poff1 = (int)(L < 4 ? flag_of(par, src, 4 * ti + L)
                            : L < 8 ? flag_of(par, src, 32 + 4 * tj + L - 4)
                            : L < 12 ? flag_of(par, src, 160 + 4 * ti + L - 8) : TMO_OFF);
    const int poff0 = (int)(L < 12 ? (unsigned)poff1 : flag_of(par, src, 4 * ti) + 64u);
    unsigned lo = 0u, m = 0u;
    auto poll = [&]() {
        ++npoll;
        const u2v pv = __builtin_amdgcn_raw_buffer_load_b64(rs, (npoll % TMO_EVERY) == 0u ? poff1 : poff0, 0, AUX_LD);
        lo = pv[0];
        m = (unsigned)__builtin_amdgcn_ballot_w64(pv[1] >= call) & 0xfffu;
        if ((unsigned)__builtin_amdgcn_readlane((int)lo, 16) == call) bad = true;
    };
    unsigned cf[4];
    auto level = [&](int xb) -> int {
        int e[4];
#pragma unroll
        for (int k = 0; k < 4; ++k)
            e[k] = (int)(short)__builtin_amdgcn_readlane((int)lo, 4 + k) + (int)(short)__builtin_amdgcn_readlane((int)lo, xb + k);
        const int E = min(min(e[0], e[1]), min(e[2], e[3]));
#pragma unroll
        for (int k = 0; k < 4; ++k) cf[k] = down(e[k] - E);
        return E;
    };
    auto apply = [&](Quad (&Q)[4]) {
#pragma unroll
        for (int k = 0; k < 4; ++k) {
            scale_quad(Q[k], cf[k]);
            cf[k] = 0x3c00u;
        }
    };
    Quad X[4], W[4];
    for (poll(); !bad && (m & 0xfu) != 0xfu; poll()) {
        if (late()) { bad = true; break; }
        __builtin_amdgcn_s_sleep(SLEEP_E);
        asm volatile("" ::: "memory");
    }
    asm volatile("" ::: "memory");
    STAMP(16 + wv);
#pragma unroll
    for (int k = 0; k < 4; ++k) X[k] = load_quad<AUX_XF>(rs, oxf + 1024 * k);
    if (!lds_wait<1>(S.locdone)) bad = true;
    for (; !bad && (m & 0xf0u) != 0xf0u; poll()) {
        if (late()) { bad = true; break; }
        __builtin_amdgcn_s_sleep(SLEEP_W);
        asm volatile("" ::: "memory");
    }
    asm volatile("" ::: "memory");
    STAMP(24 + wv);
    const unsigned need = (unsigned)(__builtin_amdgcn_ballot_w64((lo & 0x10000u) != 0u) >> 4) & 0xfu;
    int E = level(0);
    f16v acc;
    for (;;) {
        asm volatile("" ::: "memory");
        ++npoll;
        const u2v pv = __builtin_amdgcn_raw_buffer_load_b64(rs, (npoll % TMO_EVERY) == 0u ? poff1 : poff0, 0, AUX_LD);
        __builtin_amdgcn_sched_barrier(0);
#pragma unroll
        for (int k = 0; k < 4; ++k) {
            W[k] = load_quad(rs, ow + 1024 * k);
            __builtin_amdgcn_sched_barrier(0);
        }
        apply(X);
        __builtin_amdgcn_sched_barrier(0);
        acc = ZERO16;
#pragma unroll
        for (int k = 0; k < 4; ++k) {
#pragma unroll
            for (int f = 0; f < 4; ++f) {
                acc = mm(QF(W[k], f), QF(X[k], f), acc);
                __builtin_amdgcn_sched_barrier(0);
            }
        }
        lo = pv[0];
        m = (unsigned)__builtin_amdgcn_ballot_w64(pv[1] >= call) & 0xfffu;
        if ((unsigned)__builtin_amdgcn_readlane((int)lo, 16) == call) bad = true;
        if (need == 0u || bad) break;
        const unsigned ok = (quad_valid(W[0]) ? 1u : 0u) | (quad_valid(W[1]) ? 2u : 0u) | (quad_valid(W[2]) ? 4u : 0u) |
                            (quad_valid(W[3]) ? 8u : 0u);
        if ((need & ~ok) == 0u) break;
        if ((++nre & 15u) == 0u && late()) { bad = true; break; }
    }
    __builtin_amdgcn_sched_barrier(0);
    auto foreign = [&]() {
        if (bad && L == 0) LDS_ST(S.bad, 1u);
        deposit(S.redF[wv], scale16(acc, -E), i, hh);
        arrive(S.nfor, 1u << wv);
        STAMP(32 + wv);
    };
    bool pend = (m & 0xf00u) == 0xf00u;
    if (!pend) {
        foreign();
        for (poll(); !bad && (m & 0xf00u) != 0xf00u; poll()) {
            if (late()) { bad = true; break; }
            __builtin_amdgcn_s_sleep(SLEEP_O);
            asm volatile("" ::: "memory");
        }
    }
    const bool needo = (__builtin_amdgcn_readlane((int)lo, 8) & 0x10000) != 0;
    const int EO = level(8);
    for (;;) {
        asm volatile("" ::: "memory");
#pragma unroll
        for (int k = 0; k < 4; ++k) {
            X[k] = load_quad(rs, oxo + 1024 * k);
            __builtin_amdgcn_sched_barrier(0);
        }
        if (pend) {
            foreign();
            pend = false;
            asm volatile("" : "=v"(acc));
        }
        __builtin_amdgcn_sched_barrier(0);
        apply(W);
        __builtin_amdgcn_sched_barrier(0);
        acc = ZERO16;
#pragma unroll
        for (int k = 0; k < 4; ++k) {
#pragma unroll
            for (int f = 0; f < 4; ++f) {
                acc = mm(QF(W[k], f), QF(X[k], f), acc);
                __builtin_amdgcn_sched_barrier(0);
            }
        }
        if (!needo || bad) break;
        const bool ok = quad_valid(X[0]) & quad_valid(X[1]) & quad_valid(X[2]) & quad_valid(X[3]);
        if (ok) break;
        if ((++nre & 15u) == 0u && late()) { bad = true; break; }
    }
    STAMP(40 + wv);
    __builtin_amdgcn_sched_barrier(0);
    if (bad && L == 0) LDS_ST(S.bad, 1u);
    deposit(S.redO[wv], scale16(acc, -EO), i, hh);
    arrive(S.nown, 1u << wv);
    STAMP(48 + wv);
}

extern "C" void kernel_launch(int rank, int nranks, ncclComm_t comm,
                              hipStream_t stream,
                              void* const* d_in, const int* in_sizes, int n_in,
                              void* d_out, int out_size,
                              const int* in_dtypes, int out_dtype,
                              void* const* d_out_all,
                              void* const* d_ws_all, size_t ws_bytes) {
    if (nranks != NR || rank < 0 || rank >= NR || n_in < 2 || ws_bytes < (size_t)WS_NEED || out_size != MS * NDIM) {
        fprintf(stderr, "unexpected problem geometry\n");
        return;
    }
    Args a;
    a.x = (const float*)d_in[0];
    a.w = (const float*)d_in[1];
    a.ws = (char*)d_ws_all[rank];
    a.out = (float*)d_out;
    a.rank = rank;
    a.pad = 0;
    for (int l = 0; l < NR - 1; ++l) a.dws[l] = (char*)d_ws_all[(rank + 1 + l) & (NR - 1)];
    for (int p = 0; p < NR / 2; ++p) {
        a.dws[NR - 1 + p] = (char*)d_ws_all[2 * p + 1 - (rank & 1)];
        a.fout[p] = (float*)d_out_all[2 * p + 1 - (rank & 1)];
    }
    gemm_rs_kernel<<<NWG, 640, 0, stream>>>(a);
    HIP_CHECK(hipGetLastError());
}
```

```cpp
#include <hip/hip_runtime.h>
#include <rccl/rccl.h>
#include <cstdio>
#include <cstdint>

#define HIP_CHECK(expr) do { hipError_t _e = (expr); if (_e != hipSuccess) { \
    fprintf(stderr, "HIP error %s at %s:%d\n", hipGetErrorString(_e), __FILE__, __LINE__); \
    return; } } while (0)

typedef _Float16 h8v __attribute__((ext_vector_type(8)));
typedef float f16v __attribute__((ext_vector_type(16)));
typedef float f4v __attribute__((ext_vector_type(4)));
typedef float f2v __attribute__((ext_vector_type(2)));
typedef unsigned u4v __attribute__((ext_vector_type(4)));
typedef unsigned u2v __attribute__((ext_vector_type(2)));

namespace {

constexpr int NR = 8;
constexpr int NDIM = 2048;
constexpr int KS = 256;
constexpr int MS = 256;
constexpr int NWG = 256;
constexpr int NSTR = 32;
constexpr int NPC = 48;
constexpr int RMAX = 7;
constexpr unsigned F_STR = 33 * 1024, R_STR = 4 * F_STR, S_STR = 6 * R_STR;
constexpr unsigned DATA_BYTES = 2u * NR * S_STR;
constexpr unsigned FLAG_STRIDE = 128;
constexpr unsigned FLAG_OFF = DATA_BYTES;
constexpr unsigned FLAG_BYTES = 2u * NR * NPC * FLAG_STRIDE;
constexpr unsigned OFL_OFF = FLAG_OFF + FLAG_BYTES;
constexpr unsigned OFL_STRIDE = 64;
constexpr unsigned CTR_OFF = OFL_OFF + NWG * OFL_STRIDE;
constexpr unsigned CTR_STRIDE = 64;
constexpr unsigned TMO_OFF = CTR_OFF + NWG * CTR_STRIDE;
constexpr unsigned PRG_OFF = TMO_OFF + 128;
constexpr unsigned WS_NEED = PRG_OFF + NR * 128;
#define DLT 50000000ull
constexpr unsigned long long DEADLINE_TICKS = DLT;

#define INFL 2
#define SLEEP_N 1
#define FB_T1 50
#define FB_T2 110
#define FB_T3 200
#define FB_TL 60
#define FB_W0 2
#define FB_END 6
#define FB_WE 4
#define SLEEP_E 16
#define SLEEP_W 8
#define SLEEP_O 2
#define SLEEP_T 2
#define SLEEP_T2 0
#define SLEEP_R 2
#define TMO_EVERY 16
#define FB_SLEEP 4
#define STAG_S 1
#define EVS 64
#define EVC -32
#define GPD 16
#define RGD 3
#define LEAD_W 2
#define LEAD_O 2
#define ZAT 4
#define PREFETCH 1
#define WL_S 1
#define WL_C 1
#define NL_S 1
#define NL_C 1
#define STAG_MODE 1
#define TILE_WIN 1
#define TILE_WJ 12
#define LKMAP 0x75316420u
#define TG_A 1200
#define TG_B 22
#define TG_M 100000
#define XO_PLAIN 0
#define TG_P 1
#define TG_PD 4
#define PROG_F 1
#define PROG_O 1
#define AUX_OWN 17
#define PUSH_POL 0
#define PUSH_FIRST 1
#define PUSH_LIKELY 0
#define PUSH_EXPECT(c) (c)
constexpr int GP = GPD, RG = RGD, QREL = (32 / GP) * RG * 4, FH = QREL / 4;
constexpr int LS = 192 + QREL, LC = 192 - QREL;
static_assert(LS <= 32 * RMAX && QREL <= 32 && QREL > 0, "stream geometry");
#define AUX_ST 17
#define AUX_LD 17

#define STAMP(idx) do {} while (0)
#define STAMPV(idx, v) do {} while (0)
#define TNOW() 0ull

#define LDS_LD(v) __builtin_amdgcn_readfirstlane(__hip_atomic_load(&(v), __ATOMIC_RELAXED, __HIP_MEMORY_SCOPE_WORKGROUP))
#define LDS_ST(v, x) __hip_atomic_store(&(v), (x), __ATOMIC_RELAXED, __HIP_MEMORY_SCOPE_WORKGROUP)

struct Args {
    const float* x;
    const float* w;
    char* ws;
    float* out;
    int rank, pad;
    char* dws[NR + 3];
    float* fout[NR / 2];
};

__device__ __forceinline__ unsigned data_of(int par, int src, int u) {
    return (unsigned)(par * NR + src) * S_STR + (unsigned)(u >> 5) * R_STR + (unsigned)(u & 31) * 1024u;
}
__device__ __forceinline__ unsigned flag_of(int par, int src, int u) {
    return FLAG_OFF + (unsigned)((par * NR + src) * NPC + (u >> 2)) * FLAG_STRIDE + (unsigned)(u & 3) * 8u;
}
__device__ __forceinline__ int frow(int r, int i) {
    return MS * (2 * (i >> 1) + 1 - (r & 1)) + 64 * (r >> 1) + 32 * (i & 1);
}

struct Ent { int kind, src, u, a0, kq; };
__device__ __forceinline__ int rounds(int typ, int s) {
    return typ == 2 ? (s < QREL ? 1 : 0) : ((typ ? LS : LC) - s + 31) >> 5;
}
__device__ __forceinline__ Ent entry(int typ, int rank, int dest, int rho, int s) {
    Ent e;
    e.src = rank;
    if (typ == 2) {
        const int x = s % FH, j = GP * (x / RG) + GP - RG + x % RG;
        e.kind = 1; e.kq = dest >> 1; e.a0 = 1024 * (dest & 1) + 32 * j; e.u = 32 + 4 * j + e.kq;
        return e;
    }
    if (rho == 0) { e.kind = 0; e.kq = s & 3; e.a0 = frow(dest, s >> 2); e.u = s; return e; }
    const int n = 32 * (rho - 1) + s, o = n - (typ ? LS - 64 : LC - 64);
    if (o >= 0) { e.kind = 0; e.kq = o & 3; e.a0 = MS * dest + 32 * (o >> 2); e.u = 160 + o; return e; }
    constexpr int GS = 4 * (GP + RG), GC = 4 * (GP - RG);
    const int g = typ ? n / GS : n / GC, t = n - g * (typ ? GS : GC);
    if (t < 4 * GP) {
        const int j = GP * g + (t >> 2);
        e.kind = 1; e.kq = t & 3; e.a0 = 1024 * (dest & 1) + 32 * j; e.u = 32 + 4 * j + e.kq;
    } else {
        const int r = t - 4 * GP;
        e.kind = 2; e.kq = rank >> 1; e.a0 = 0; e.src = 2 * (r & 3) + 1 - (rank & 1);
        e.u = 32 + 4 * (GP * g + GP - RG + (r >> 2)) + e.kq;
    }
    return e;
}
__device__ __forceinline__ int first_early(int typ, int s) {
    return typ == 1 ? (LS - 32 - EVS - s + 31) >> 5 : typ == 0 ? (LC - 32 - (EVC) - s + 31) >> 5 : 1;
}

__device__ __forceinline__ bool clears(int typ, int rank, int s) { return typ == 0 && (s >> 3) == (rank >> 1); }
constexpr unsigned EMPTY = 0xffffffffu;

template <int NV>
__device__ __forceinline__ int block_exp(const float (&v)[NV]) {
    float m = 0.f;
#pragma unroll
    for (int t = 0; t < NV; ++t) m = fmaxf(m, fabsf(v[t]));
    unsigned u = __float_as_uint(m);
    u = max(u, (unsigned)__builtin_amdgcn_update_dpp(0, (int)u, 0xB1, 0xf, 0xf, true));
    u = max(u, (unsigned)__builtin_amdgcn_update_dpp(0, (int)u, 0x4E, 0xf, 0xf, true));
    u = max(u, (unsigned)__builtin_amdgcn_update_dpp(0, (int)u, 0x141, 0xf, 0xf, true));
    u = max(u, (unsigned)__builtin_amdgcn_update_dpp(0, (int)u, 0x140, 0xf, 0xf, true));
    const unsigned r = max(max((unsigned)__builtin_amdgcn_readlane((int)u, 0), (unsigned)__builtin_amdgcn_readlane((int)u, 16)),
                           max((unsigned)__builtin_amdgcn_readlane((int)u, 32), (unsigned)__builtin_amdgcn_readlane((int)u, 48)));
    unsigned E = r >> 23;
    E = E < 15u ? 15u : E;
    return 141 - (int)E;
}

__device__ __forceinline__ u4v pack8(const float* v, int e) {
    h8v r;
    typedef _Float16 h2v __attribute__((ext_vector_type(2)));
    const float sc = __uint_as_float((unsigned)(e + 127) << 23);
    const f2v s2 = {sc, sc};
#pragma unroll
    for (int j = 0; j < 8; j += 2) {
        const h2v t = __builtin_convertvector(f2v{v[j], v[j + 1]} * s2, h2v);
        r[j] = t[0];
        r[j + 1] = t[1];
    }
    return __builtin_bit_cast(u4v, r);
}

template <int NF>
__device__ __forceinline__ void load_w(const float* __restrict__ w, int k0, int col0, int i, int hh, float (&v)[8 * NF]) {
    const float* base = w + (size_t)(k0 + 8 * hh) * NDIM + col0 + i;
#pragma unroll
    for (int s = 0; s < NF; ++s)
#pragma unroll
        for (int j = 0; j < 8; ++j) v[8 * s + j] = base[(size_t)(16 * s + j) * NDIM];
}

template <int NF>
__device__ __forceinline__ void load_x(const float* __restrict__ x, int row0, int k0, int i, int hh, float (&v)[8 * NF]) {
    const float* base = x + (size_t)(row0 + i) * KS + k0 + 8 * hh;
#pragma unroll
    for (int s = 0; s < NF; ++s) {
        f4v t0 = *(const f4v*)(base + 16 * s);
        f4v t1 = *(const f4v*)(base + 16 * s + 4);
#pragma unroll
        for (int j = 0; j < 4; ++j) { v[s * 8 + j] = t0[j]; v[s * 8 + 4 + j] = t1[j]; }
    }
}

__device__ __forceinline__ f16v mm(u4v a, u4v b, f16v c) {
    return __builtin_amdgcn_mfma_f32_32x32x16_f16(__builtin_bit_cast(h8v, a), __builtin_bit_cast(h8v, b), c, 0, 0, 0);
}
#define ZERO16 f16v{0.f, 0.f, 0.f, 0.f, 0.f, 0.f, 0.f, 0.f, 0.f, 0.f, 0.f, 0.f, 0.f, 0.f, 0.f, 0.f}

__device__ __forceinline__ f16v scale16(f16v t, int e) {
    const int e1 = e >> 1, e2 = e - e1;
    const float c1 = __uint_as_float((unsigned)(e1 + 127) << 23), c2 = __uint_as_float((unsigned)(e2 + 127) << 23);
    const f2v s1 = {c1, c1}, s2 = {c2, c2};
#pragma unroll
    for (int r = 0; r < 16; r += 2) {
        const f2v u = (f2v{t[r], t[r + 1]} * s1) * s2;
        t[r] = u[0]; t[r + 1] = u[1];
    }
    return t;
}

__device__ __forceinline__ void fma16(f16v& acc, f16v t, int e) {
    const int e1 = e >> 1, e2 = e - e1;
    const float c1 = __uint_as_float((unsigned)(e1 + 127) << 23), c2 = __uint_as_float((unsigned)(e2 + 127) << 23);
    const f2v s1 = {c1, c1}, s2 = {c2, c2};
#pragma unroll
    for (int r = 0; r < 16; r += 2) {
        const f2v u = __builtin_elementwise_fma(f2v{t[r], t[r + 1]} * s1, s2, f2v{acc[r], acc[r + 1]});
        acc[r] = u[0]; acc[r + 1] = u[1];
    }
}

struct Quad { u4v f[4]; };
#define QF(q, n) ((q).f[n])

#define AUX_XF AUX_LD
#define AUX_W AUX_LD
#define AUX_XO AUX_LD
template <int AUX = AUX_LD>
__device__ __forceinline__ Quad load_quad(__amdgpu_buffer_rsrc_t rs, int off) {
    Quad q;
#pragma unroll
    for (int f = 0; f < 4; ++f) q.f[f] = __builtin_amdgcn_raw_buffer_load_b128(rs, off, f * (int)F_STR, AUX);
    return q;
}

__device__ __forceinline__ f16v mm_quad(const Quad& w, const Quad& x, f16v t) {
    t = mm(QF(w, 0), QF(x, 0), t);
    t = mm(QF(w, 1), QF(x, 1), t);
    t = mm(QF(w, 2), QF(x, 2), t);
    return mm(QF(w, 3), QF(x, 3), t);
}

__device__ __forceinline__ unsigned down(int d) {
    return d <= 14 ? (unsigned)(15 - d) << 10 : (1u << 24 >> min(d, 31));
}
__device__ __forceinline__ void scale_quad(Quad& q, unsigned c) {
    const _Float16 h = __builtin_bit_cast(_Float16, (unsigned short)c);
#pragma unroll
    for (int f = 0; f < 4; ++f) {
        q.f[f] = __builtin_bit_cast(u4v, __builtin_bit_cast(h8v, q.f[f]) * h);
        asm volatile("" : "+v"(q.f[f]));
    }
}

__device__ __forceinline__ u4v no_zero(u4v u) {
    u[0] = u[0] ? u[0] : 0x8000u;
    return u;
}

__device__ __forceinline__ bool quad_valid(const Quad& q) {
    return __builtin_amdgcn_ballot_w64(min(min(q.f[0][0], q.f[1][0]), min(q.f[2][0], q.f[3][0])) == 0u) == 0ull;
}

template <int N>
__device__ __forceinline__ void wait_vm() {
    asm volatile("s_waitcnt vmcnt(%0)" ::"i"(N) : "memory");
}

__device__ __forceinline__ void wait_vm_n(int n) {
    switch (n) {
        case 0: wait_vm<0>(); break;
        case 1: wait_vm<1>(); break;
        case 2: wait_vm<2>(); break;
        case 3: wait_vm<3>(); break;
        case 4: wait_vm<4>(); break;
        case 5: wait_vm<5>(); break;
        case 6: wait_vm<6>(); break;
        case 7: wait_vm<7>(); break;
        case 8: wait_vm<8>(); break;
        case 9: wait_vm<9>(); break;
        default: wait_vm<10>(); break;
    }
}

__device__ __forceinline__ void raw_barrier() {
    asm volatile("s_waitcnt lgkmcnt(0)\n\ts_barrier" ::: "memory");
}

template <int SLP, class T>
__device__ __forceinline__ unsigned lds_wait(T& v, unsigned n = 1u) {
    unsigned r, it = 0u;
    unsigned long long t0 = 0ull;
    while ((r = (unsigned)LDS_LD(v)) < n) {
        __builtin_amdgcn_s_sleep(SLP);
        if ((++it & 1023u) == 0u) {
            const unsigned long long t = __builtin_amdgcn_s_memrealtime();
            if (t0 == 0ull) t0 = t;
            else if (t - t0 > 2ull * DEADLINE_TICKS) { r = 0u; break; }
        }
    }
    asm volatile("" ::: "memory");
    return r;
}

struct Lds {
    u4v stage[RMAX][256];
    f4v redF[NR][8 * 33];
    f4v redO[NR][8 * 33];
    int exp_[RMAX + 1];
    int rdy[RMAX + 1];
    unsigned call, bad, nloc, locdone, nfor, nown, fdone, clr;
    int prog, win, zok;
};

__device__ __forceinline__ void deposit(f4v* slot, const f16v& t, int i, int hh) {
#pragma unroll
    for (int g = 0; g < 4; ++g)
        slot[(2 * g + hh) * 33 + i] = f4v{t[4 * g], t[4 * g + 1], t[4 * g + 2], t[4 * g + 3]};
}
__device__ __forceinline__ void arrive(unsigned& c, unsigned v) {
    unsigned long long ex;
    asm volatile("s_mov_b64 %0, exec\n\ts_mov_b64 exec, 1\n\tds_add_u32 %1, %2\n\ts_mov_b64 exec, %0"
                 : "=&s"(ex)
                 : "v"((unsigned)(uintptr_t)(__attribute__((address_space(3))) unsigned*)&c), "v"(v)
                 : "memory");
}
__device__ __forceinline__ void gather(f4v (&acc)[4], bool& bad, unsigned& flags, unsigned& badw, const f4v* red) {
    unsigned got = 0u, it = 0u;
    unsigned long long t0 = 0ull;
    auto late = [&]() -> bool {
        if ((++it & 1023u) == 0u) {
            const unsigned long long t = __builtin_amdgcn_s_memrealtime();
            if (t0 == 0ull) t0 = t;
            else if (t - t0 > 2ull * DEADLINE_TICKS) return true;
        }
        return false;
    };
    for (int n = 0; n < NR - 2 && !bad;) {
        const unsigned m = (unsigned)LDS_LD(flags) & ~got;
        asm volatile("" ::: "memory");
        if (m == 0u) {
            __builtin_amdgcn_s_sleep(0);
            bad = late();
            continue;
        }
        const int p = __builtin_ctz(m);
#pragma unroll
        for (int u = 0; u < 4; ++u) acc[u] += red[p * (8 * 33) + 8 * u];
        got |= 1u << p;
        ++n;
    }
    const int p = __builtin_ctz(~got);
    f4v v[4];
    for (;;) {
        asm volatile("" ::: "memory");
        const unsigned c = __hip_atomic_load(&flags, __ATOMIC_RELAXED, __HIP_MEMORY_SCOPE_WORKGROUP);
        __builtin_amdgcn_sched_barrier(0);
        asm volatile("" ::: "memory");
        const unsigned w = __hip_atomic_load(&badw, __ATOMIC_RELAXED, __HIP_MEMORY_SCOPE_WORKGROUP);
#pragma unroll
        for (int u = 0; u < 4; ++u) v[u] = red[p * (8 * 33) + 8 * u];
        __builtin_amdgcn_sched_barrier(0);
        if (__builtin_amdgcn_readfirstlane(c) == 0x7fu) { bad |= __builtin_amdgcn_readfirstlane(w) != 0u; break; }
        asm volatile("s_waitcnt lgkmcnt(0)" ::: "memory");
        if (bad || late()) { bad = true; break; }
    }
#pragma unroll
    for (int u = 0; u < 4; ++u) acc[u] += v[u];
    if (bad) {
        const float qn = __builtin_nanf("");
#pragma unroll
        for (int u = 0; u < 4; ++u) acc[u] = f4v{qn, qn, qn, qn};
    }
}

__device__ __forceinline__ void pusher(const Args& a, Lds& S, char* dws, int b, int rank, int dest, int s, int L, int typ) {
    const int nrnd = rounds(typ, s), NST = 4 * nrnd;
    char* const myws = a.ws;
    const unsigned long long tq0 = TNOW();
    const unsigned cv = __hip_atomic_load((unsigned*)(myws + CTR_OFF + b * CTR_STRIDE), __ATOMIC_RELAXED,
                                          __HIP_MEMORY_SCOPE_AGENT);
    const int i = L & 31, hh = L >> 5;
    u4v t[4];
    unsigned vdat, vflg;
    {
        const Ent en = entry(typ, rank, dest, 0, s);
        float v0[32];
        if (typ == 2) load_w<4>(a.w, 64 * en.kq, en.a0, i, hh, v0);
        else load_x<4>(a.x, en.a0, 64 * en.kq, i, hh, v0);
        const Ent le = entry(typ, rank, dest, L < nrnd ? L : 0, s);
        vdat = data_of(0, le.src, le.u);
        vflg = flag_of(0, le.src, le.u);
        asm volatile("" : "+v"(vdat), "+v"(vflg));
        if (L <= RMAX) S.rdy[L] = 0;
        if (L == 0) {
            S.call = 0u; S.bad = 0u; S.nloc = 0u; S.locdone = 0u; S.nfor = 0u; S.nown = 0u; S.fdone = 0u;
            S.clr = clears(typ, rank, s) ? 0u : 1u;
            S.prog = 0; S.zok = 0; S.win = typ == 2 ? 6 : INFL;
        }
        raw_barrier();
        const int e = block_exp<32>(v0);
        if (L == 0) { LDS_ST(S.rdy[0], 1); S.exp_[0] = e; }
#pragma unroll
        for (int f = 0; f < 4; ++f) t[f] = pack8(v0 + 8 * f, e);
        if (typ == 2) {
#pragma unroll
            for (int f = 0; f < 4; ++f) t[f] = no_zero(t[f]);
        }
    }
    const unsigned call = __builtin_amdgcn_readfirstlane(cv) + 1u;
    if (L == 0) LDS_ST(S.call, call);
    if (nrnd == 0) return;
    const int par = (int)(call & 1u);
    __builtin_amdgcn_s_setprio(3);
    const __amdgpu_buffer_rsrc_t rs = __builtin_amdgcn_make_buffer_rsrc(dws, 0, (int)DATA_BYTES, 0x00020000);
    vdat += (unsigned)par * (NR * S_STR);
    vflg += (unsigned)par * (NR * NPC * FLAG_STRIDE);
    auto publish = [&](int c, unsigned early) {
        if (L == c) {
            const unsigned long long val = ((unsigned long long)call << 32) | (((unsigned)S.exp_[c] & 0xffffu) | early);
            __hip_atomic_store((unsigned long long*)(dws + vflg), val, __ATOMIC_RELAXED, __HIP_MEMORY_SCOPE_SYSTEM);
        }
    };
    int issued = 0, pub = 0, dn = 0, h1 = 0, h2 = 0, h3 = 0, h4 = 0, h5 = 0, h6 = 0, h7 = 0;
    const int ev = first_early(typ, s);
    const int nl = 4 * ((typ == 1 ? LS : LC) >> 5) - (typ == 1 ? NL_S : NL_C);
    for (int c = 0;; ++c) {
        const int so = __builtin_amdgcn_readlane((int)vdat, c);
        bool have = false;
#pragma unroll
        for (int f = 0; f < 4; ++f) {
            const int n = 4 * c + f;
            __builtin_amdgcn_raw_buffer_store_b128(t[f], rs, L * 16, so + f * (int)F_STR, AUX_ST);
            const int h0 = ++issued;
            __builtin_amdgcn_sched_barrier(0);
            if (c >= ev && f == (c + 1 == nrnd ? LEAD_O : LEAD_W)) {
                if (c + 1 == nrnd && !lds_wait<1>(S.zok)) {
                    if (L == 0) { LDS_ST(S.bad, 1u); LDS_ST(S.prog, NST); }
                    return;
                }
                publish(c, 0x10000u); ++issued;
            }
            if (L == 0) LDS_ST(S.prog, n + 1 == NST ? NST : dn);
            while (pub < ev && 4 * (pub + 1) <= dn && (pub > 0 || LDS_LD(S.clr))) { publish(pub, 0u); ++issued; ++pub; }
            if (f == 3) {
                if (n + 1 == NST) break;
                have = LDS_LD(S.rdy[c + 1]) != 0;
                asm volatile("" ::: "memory");
                if (have) {
#pragma unroll
                    for (int g = 0; g < 4; ++g) t[g] = S.stage[c + 1][g * 64 + L];
                }
            }
            int win = LDS_LD(S.win);
            if (n >= nl) win = max(win, typ == 1 ? WL_S : WL_C);
            const int ik = win <= 1 ? h0 : win == 2 ? h1 : win == 3 ? h2 : win == 4 ? h3 : win <= 6 ? h5 : h7;
            const int k1 = n + 2 - (win > 6 ? 8 : win > 4 ? 6 : win);
            if (k1 > dn) {
                wait_vm_n(issued - ik);
                dn = k1;
            }
            h7 = h6; h6 = h5; h5 = h4; h4 = h3; h3 = h2; h2 = h1; h1 = h0;
        }
        if (c + 1 == nrnd) break;
        if (!have) {
            while (pub < ev && 4 * (pub + 1) <= dn && (pub > 0 || LDS_LD(S.clr))) { publish(pub, 0u); ++issued; ++pub; }
            if (!lds_wait<0>(S.rdy[c + 1])) {
                if (L == 0) { LDS_ST(S.bad, 1u); LDS_ST(S.prog, NST); }
                return;
            }
#pragma unroll
            for (int g = 0; g < 4; ++g) t[g] = S.stage[c + 1][g * 64 + L];
        }
    }
    const unsigned long long tq1 = TNOW();
    if (pub == 0 && ev > 0) {
        if (!lds_wait<1>(S.clr)) {
            if (L == 0) LDS_ST(S.bad, 1u);
            return;
        }
        while (pub < ev && 4 * (pub + 1) <= dn) { publish(pub, 0u); ++issued; ++pub; }
    }
    if (pub < ev && pub == nrnd - 3) { wait_vm_n(issued - h7); publish(pub, 0u); ++issued; ++pub; }
    if (pub < ev && pub == nrnd - 2) { wait_vm_n(issued - h4); publish(pub, 0u); ++pub; }
    if (pub < ev) { wait_vm<0>(); publish(pub, 0u); }
}

__device__ __forceinline__ bool feed(const Args& a, Lds& S, int rank, int dest, int s, int wv, int L, int typ) {
    if (wv < 1 || wv >= rounds(typ, s)) return false;
    const Ent en = entry(typ, rank, dest, wv, s);
    if (en.kind == 2) return true;
    const int i = L & 31, hh = L >> 5;
    float v[32];
    if (en.kind == 1) load_w<4>(a.w, 64 * en.kq, en.a0, i, hh, v);
    else load_x<4>(a.x, en.a0, 64 * en.kq, i, hh, v);
    const int e = block_exp<32>(v);
#pragma unroll
    for (int f = 0; f < 4; ++f) {
        S.stage[wv][f * 64 + L] = no_zero(pack8(v + 8 * f, e));
    }
    if (L == 0) S.exp_[wv] = e;
    asm volatile("s_waitcnt lgkmcnt(0)" ::: "memory");
    if (L == 0) LDS_ST(S.rdy[wv], 1);
    asm volatile("" ::: "memory");
    return false;
}

}

__global__ void __launch_bounds__(640) gemm_rs_kernel(Args a) {
    __shared__ __attribute__((aligned(16))) Lds S;

    const int b = blockIdx.x;
    const int wr = __builtin_amdgcn_readfirstlane(threadIdx.x >> 6);
    const int wv = wr == 0 ? NR : wr <= NR ? wr - 1 : NR + 1;
    const int L = threadIdx.x & 63;
    const int i = L & 31, hh = L >> 5;
    const int ti = b & 7, tj = (b >> 3) & 31;
    const int lk = (int)(LKMAP >> (4 * ((b >> 5) & 7))) & 7, s = b & 31;
    char* const dws = a.dws[lk < 7 ? lk : 7 + ((s / FH) & 3)];
    float* const pf0 = a.fout[ti >> 1];
    char* const myws = a.ws;
    asm volatile("" ::"s"(a.x), "s"(a.w), "s"(myws), "s"(a.out), "s"(a.rank), "s"(dws), "s"(pf0));
    const int rank = a.rank & (NR - 1);
    const int cls = rank & 1, mem = rank >> 1;
    const int dest = lk < 7 ? (rank + 1 + lk) & (NR - 1) : 2 * ((s / FH) & 3) + 1 - cls;
    const int typ = lk == 7 ? 2 : lk & 1;
    const unsigned tk0 = (unsigned)__builtin_amdgcn_s_memrealtime();

    if (PUSH_EXPECT(wv == NR)) {
        pusher(a, S, dws, b, rank, dest, s, L, typ);
        return;
    }

    if (wv == NR + 1) {
        raw_barrier();
        const unsigned call = lds_wait<1>(S.call);
        if (call == 0u) return;
        if (L == 0)
            __hip_atomic_store((unsigned*)(myws + CTR_OFF + b * CTR_STRIDE), call, __ATOMIC_RELAXED, __HIP_MEMORY_SCOPE_AGENT);
        if (lk < 7) {
            const int nrnd = rounds(typ, s), NST = 4 * nrnd, LT = typ ? 4 * LS : 4 * LC;
            int z = 0;
            auto clear = [&]() {
                const __amdgpu_buffer_rsrc_t rz = __builtin_amdgcn_make_buffer_rsrc(myws, 0, (int)DATA_BYTES, 0x00020000);
                for (int c = first_early(typ, s); c < nrnd; ++c) {
                    const Ent en = entry(typ, dest, rank, c, s);
                    const unsigned zo = data_of((int)(call & 1u) ^ 1, en.src, en.u);
#pragma unroll
                    for (int f = 0; f < 4; ++f)
                        __builtin_amdgcn_raw_buffer_store_b128(u4v{0u, 0u, 0u, 0u}, rz, L * 16, (int)(zo + f * F_STR), AUX_ST);
                }
                z = 1;
            };
            unsigned* pc = (unsigned*)(myws + PRG_OFF + (unsigned)dest * 128u);
            const unsigned base = (call - 1u) * (unsigned)LT;
            int last = 0;
            unsigned it = 0u;
            unsigned long long t0 = 0ull;
            for (;;) {
                int p = LDS_LD(S.prog);
                if ((++it & 255u) == 0u) {
                    const unsigned long long t = __builtin_amdgcn_s_memrealtime();
                    if (t0 == 0ull) t0 = t;
                    else if (t - t0 > 2ull * DEADLINE_TICKS) p = NST;
                }
                const int dl = p > last ? p - last : 0;
                last += dl;
                if (last >= NST) {
                    if (L == 0) __hip_atomic_fetch_add(pc, (unsigned)dl, __ATOMIC_RELAXED, __HIP_MEMORY_SCOPE_AGENT);
                    break;
                }
                if (z == 0 && last >= ZAT) clear();
                unsigned g = 0u;
                if (L == 0) g = __hip_atomic_fetch_add(pc, (unsigned)dl, __ATOMIC_RELAXED, __HIP_MEMORY_SCOPE_AGENT) + (unsigned)dl;
                const int tot = (int)(__builtin_amdgcn_readfirstlane(g) - base);
                if (z == 1) {
                    wait_vm<0>();
                    if (L == 0) LDS_ST(S.zok, 1);
                    z = 2;
                }
                const int lag = tot - NSTR * last;
                int win = lag > FB_T3 ? 6 : lag > FB_T2 ? 4 : lag > FB_T1 ? 3 : lag < -FB_TL ? 1 : 2;
                if (tot >= LT - FB_END * NSTR && win < FB_WE) win = FB_WE;
                if (L == 0 && last > 0) LDS_ST(S.win, win);
                __builtin_amdgcn_s_sleep(FB_SLEEP);
            }
            if (z == 0) clear();
            if (z == 1) {
                wait_vm<0>();
                if (L == 0) LDS_ST(S.zok, 1);
            }
        }
        {
            const __amdgpu_buffer_rsrc_t ro = __builtin_amdgcn_make_buffer_rsrc(a.out, 0, MS * NDIM * 4, 0x00020000);
            const int oo = ((32 * ti + (L >> 3)) * NDIM + 1024 * (1 - cls) + 32 * tj + 4 * (L & 7)) * 4;
            bool bad = !lds_wait<SLEEP_T>(S.fdone);
            const unsigned long long t0 = __builtin_amdgcn_s_memrealtime();
            unsigned ok = 0u, it = 0u;
            struct T4 { u4v v[4]; };
            auto look = [&]() -> T4 {
                T4 r;
#pragma unroll
                for (int u = 0; u < 4; ++u)
                    r.v[u] = __builtin_amdgcn_raw_buffer_load_b128(ro, oo + ((ok >> u & 1u) ? 0x40000000 : 0), 8 * u * NDIM * 4, AUX_LD);
                __builtin_amdgcn_sched_barrier(0);
                return r;
            };
            auto check = [&](const T4& r) {
#pragma unroll
                for (int u = 0; u < 4; ++u) {
                    const u4v v = r.v[u];
                    if (__builtin_amdgcn_ballot_w64(v[0] == EMPTY) == 0ull) ok |= 1u << u;
                }
                __builtin_amdgcn_sched_barrier(0);
            };
            while (!bad) {
                asm volatile("" ::: "memory");
                check(look());
                if (ok != 15u && (++it & 15u) != 0u) {
                    __builtin_amdgcn_s_sleep(SLEEP_T2);
                    continue;
                }
                if (ok == 15u) break;
                const unsigned v = __hip_atomic_load((const unsigned*)(myws + TMO_OFF), __ATOMIC_RELAXED, __HIP_MEMORY_SCOPE_SYSTEM);
                if (__builtin_amdgcn_readfirstlane(v) == call) bad = true;
                if (__builtin_amdgcn_s_memrealtime() - t0 > DEADLINE_TICKS) {
                    if (L == 0)
                        __hip_atomic_store((unsigned*)(myws + TMO_OFF), call, __ATOMIC_RELAXED, __HIP_MEMORY_SCOPE_AGENT);
                    bad = true;
                }
            }
            if (bad) {
                const unsigned qn = 0x7fc00000u;
#pragma unroll
                for (int u = 0; u < 4; ++u)
                    __builtin_amdgcn_raw_buffer_store_b128(u4v{qn, qn, qn, qn}, ro, oo, 8 * u * NDIM * 4, AUX_ST);
            }
            STAMP(59);
        }
        return;
    }

    raw_barrier();
    const bool clr = wv == NR - 1 && clears(typ, rank, s);
    if (clr) {
        const __amdgpu_buffer_rsrc_t ro = __builtin_amdgcn_make_buffer_rsrc(a.out, 0, MS * NDIM * 4, 0x00020000);
        const int oo = ((64 * (dest >> 1) + 32 * ((s >> 2) & 1)) * NDIM + 1024 * (dest & 1) + 256 * (s & 3) + 4 * L) * 4;
#pragma unroll
        for (int u = 0; u < 32; ++u)
            __builtin_amdgcn_raw_buffer_store_b128(u4v{EMPTY, EMPTY, EMPTY, EMPTY}, ro, oo, u * NDIM * 4, AUX_ST);
        wait_vm<0>();
        if (L == 0) LDS_ST(S.clr, 1u);
    }
    if (wv != 1) lds_wait<STAG_S>(S.rdy[0]);
    const bool fwd = feed(a, S, rank, dest, s, wv, L, typ);
    const int col0 = 1024 * cls + 32 * tj;
    {
        float vw[16], vf[16], vo[16];
        load_w<2>(a.w, 32 * wv, col0, i, hh, vw);
        load_x<2>(a.x, frow(rank, ti), 32 * wv, i, hh, vf);
        load_x<2>(a.x, MS * rank + 32 * ti, 32 * wv, i, hh, vo);
        const int ew = block_exp<16>(vw);
        const u4v w0 = pack8(vw, ew), w1 = pack8(vw + 8, ew);
        const int ef = block_exp<16>(vf);
        f16v t = mm(w0, pack8(vf, ef), ZERO16);
        t = mm(w1, pack8(vf + 8, ef), t);
        deposit(S.redF[wv], scale16(t, -(ew + ef)), i, hh);
        const int eo = block_exp<16>(vo);
        t = mm(w0, pack8(vo, eo), ZERO16);
        t = mm(w1, pack8(vo + 8, eo), t);
        deposit(S.redO[wv], scale16(t, -(ew + eo)), i, hh);
        arrive(S.nloc, 1u);
    }
    STAMP(60 + wv);
    if (wv == NR - 1) {
        float* pf = pf0;
        float* po = a.out;
        asm volatile("" : "+s"(pf), "+s"(po));
        const __amdgpu_buffer_rsrc_t rf = __builtin_amdgcn_make_buffer_rsrc(pf, 0, MS * NDIM * 4, 0x00020000);
        const __amdgpu_buffer_rsrc_t ro = __builtin_amdgcn_make_buffer_rsrc(po, 0, MS * NDIM * 4, 0x00020000);
        int of = ((64 * mem + 32 * (ti & 1) + (L >> 3)) * NDIM + col0 + 4 * (L & 7)) * 4;
        int oo = ((32 * ti + (L >> 3)) * NDIM + col0 + 4 * (L & 7)) * 4;
        asm volatile("" : "+v"(of), "+v"(oo));
        const int rp = (L & 7) * 33 + (L >> 3);
        f4v aF[4], aO[4];
        bool bad = !lds_wait<1>(S.nloc, NR);
#pragma unroll
        for (int u = 0; u < 4; ++u) {
            aF[u] = S.redF[0][rp + 8 * u];
            aO[u] = S.redO[0][rp + 8 * u];
#pragma unroll
            for (int p = 1; p < NR; ++p) {
                aF[u] += S.redF[p][rp + 8 * u];
                aO[u] += S.redO[p][rp + 8 * u];
            }
            asm volatile("" : "+v"(aF[u]), "+v"(aO[u]));
        }
        asm volatile("s_waitcnt lgkmcnt(0)" ::: "memory");
        if (L == 0) LDS_ST(S.locdone, 1u);
        gather(aF, bad, S.nfor, S.bad, &S.redF[0][rp]);
        u4v v[4];
#pragma unroll
        for (int u = 0; u < 4; ++u) v[u] = __builtin_bit_cast(u4v, aF[u]);
#pragma unroll
        for (int u = 0; u < 4; ++u) {
            v[u][0] = min(v[u][0], EMPTY - 1u);
            asm volatile("" : "+v"(v[u]));
        }
        while ((unsigned)__builtin_amdgcn_s_memrealtime() - tk0 < (unsigned)min(TG_A + TG_B * tj, TG_M))
            __builtin_amdgcn_s_sleep(1);
        if (typ == 1) lds_wait<1>(S.prog, (unsigned)(4 * rounds(typ, s) - TG_PD));
        const unsigned long long tq = TNOW();
        if (tj < TILE_WJ) {
#pragma unroll
            for (int u = 0; u < 4; ++u) {
                __builtin_amdgcn_raw_buffer_store_b128(v[u], rf, of, 8 * u * NDIM * 4, AUX_ST);
                if (u < 3) wait_vm<TILE_WIN - 1>();
            }
        } else {
#pragma unroll
            for (int u = 0; u < 4; ++u)
                __builtin_amdgcn_raw_buffer_store_b128(v[u], rf, of, 8 * u * NDIM * 4, AUX_ST);
        }
        __builtin_amdgcn_sched_barrier(0);
        asm volatile("s_nop 1" ::: "memory");
        __builtin_amdgcn_sched_barrier(0);
        if (L == 0) LDS_ST(S.fdone, 1u);
        STAMPV(56, tq);
        STAMP(57);
        gather(aO, bad, S.nown, S.bad, &S.redO[0][rp]);
#pragma unroll
        for (int u = 0; u < 4; ++u)
            __builtin_amdgcn_raw_buffer_store_b128(__builtin_bit_cast(u4v, aO[u]), ro, oo, 8 * u * NDIM * 4, AUX_OWN);
        STAMP(58);
        return;
    }
    const unsigned call = lds_wait<1>(S.call);
    const int par = (int)(call & 1u);

    const __amdgpu_buffer_rsrc_t rs = __builtin_amdgcn_make_buffer_rsrc(myws, 0, (int)WS_NEED, 0x00020000);
    const unsigned long long t0 = __builtin_amdgcn_s_memrealtime();
    bool bad = call == 0u;
    unsigned npoll = 0u, nre = 0u;
    auto late = [&]() -> bool {
        if (__builtin_amdgcn_s_memrealtime() - t0 > DEADLINE_TICKS) {
            if (L == 0)
                __hip_atomic_store((unsigned*)(myws + TMO_OFF), call, __ATOMIC_RELAXED, __HIP_MEMORY_SCOPE_AGENT);
            return true;
        }
        return false;
    };
    if (fwd && !bad) {
        const Ent en = entry(1, rank, dest, wv, s);
        const int pf = (int)(L == 0 ? flag_of(par, en.src, en.u) : TMO_OFF);
        u2v pv;
        for (;;) {
            pv = __builtin_amdgcn_raw_buffer_load_b64(rs, pf, 0, AUX_LD);
            if ((unsigned)__builtin_amdgcn_readlane((int)pv[1], 0) >= call) break;
            if ((unsigned)__builtin_amdgcn_readlane((int)pv[0], 1) == call || late()) { bad = true; break; }
            __builtin_amdgcn_s_sleep(SLEEP_R);
            asm volatile("" ::: "memory");
        }
        asm volatile("" ::: "memory");
        const Quad qd = load_quad(rs, (int)(data_of(par, en.src, en.u) + (unsigned)L * 16u));
        S.stage[wv][L] = QF(qd, 0);
        S.stage[wv][64 + L] = QF(qd, 1);
        S.stage[wv][128 + L] = QF(qd, 2);
        S.stage[wv][192 + L] = QF(qd, 3);
        if (L == 0) S.exp_[wv] = (int)(short)pv[0];
        asm volatile("s_waitcnt lgkmcnt(0)" ::: "memory");
        if (L == 0) LDS_ST(S.rdy[wv], 1);
        asm volatile("" ::: "memory");
        STAMP(70 + wv);
    }
    const int src = (rank + 1 + wv) & (NR - 1);
    const int oxf = (int)(data_of(par, src, 4 * ti) + (unsigned)L * 16u);
    const int ow = (int)(data_of(par, src, 32 + 4 * tj) + (unsigned)L * 16u);
    const int oxo = (int)(data_of(par, src, 160 + 4 * ti) + (unsigned)L * 16u);
    const int poff1 = (int)(L < 4 ? flag_of(par, src, 4 * ti + L)
                            : L < 8 ? flag_of(par, src, 32 + 4 * tj + L - 4)
                            : L < 12 ? flag_of(par, src, 160 + 4 * ti + L - 8) : TMO_OFF);
    const int poff0 = (int)(L < 12 ? (unsigned)poff1 : flag_of(par, src, 4 * ti) + 64u);
    unsigned lo = 0u, m = 0u;
    auto poll = [&]() {
        ++npoll;
        const u2v pv = __builtin_amdgcn_raw_buffer_load_b64(rs, (npoll % TMO_EVERY) == 0u ? poff1 : poff0, 0, AUX_LD);
        lo = pv[0];
        m = (unsigned)__builtin_amdgcn_ballot_w64(pv[1] >= call) & 0xfffu;
        if ((unsigned)__builtin_amdgcn_readlane((int)lo, 16) == call) bad = true;
    };
    unsigned cf[4];
    auto level = [&](int xb) -> int {
        int e[4];
#pragma unroll
        for (int k = 0; k < 4; ++k)
            e[k] = (int)(short)__builtin_amdgcn_readlane((int)lo, 4 + k) + (int)(short)__builtin_amdgcn_readlane((int)lo, xb + k);
        const int E = min(min(e[0], e[1]), min(e[2], e[3]));
#pragma unroll
        for (int k = 0; k < 4; ++k) cf[k] = down(e[k] - E);
        return E;
    };
    auto apply = [&](Quad (&Q)[4]) {
#pragma unroll
        for (int k = 0; k < 4; ++k) {
            scale_quad(Q[k], cf[k]);
            cf[k] = 0x3c00u;
        }
    };
    Quad X[4], W[4];
    for (poll(); !bad && (m & 0xfu) != 0xfu; poll()) {
        if (late()) { bad = true; break; }
        __builtin_amdgcn_s_sleep(SLEEP_E);
        asm volatile("" ::: "memory");
    }
    asm volatile("" ::: "memory");
    STAMP(16 + wv);
#pragma unroll
    for (int k = 0; k < 4; ++k) X[k] = load_quad<AUX_XF>(rs, oxf + 1024 * k);
    if (!lds_wait<1>(S.locdone)) bad = true;
    for (; !bad && (m & 0xf0u) != 0xf0u; poll()) {
        if (late()) { bad = true; break; }
        __builtin_amdgcn_s_sleep(SLEEP_W);
        asm volatile("" ::: "memory");
    }
    asm volatile("" ::: "memory");
    STAMP(24 + wv);
    const unsigned need = (unsigned)(__builtin_amdgcn_ballot_w64((lo & 0x10000u) != 0u) >> 4) & 0xfu;
    int E = level(0);
    f16v acc;
    for (;;) {
        asm volatile("" ::: "memory");
        ++npoll;
        const u2v pv = __builtin_amdgcn_raw_buffer_load_b64(rs, (npoll % TMO_EVERY) == 0u ? poff1 : poff0, 0, AUX_LD);
        __builtin_amdgcn_sched_barrier(0);
#pragma unroll
        for (int k = 0; k < 4; ++k) {
            W[k] = load_quad(rs, ow + 1024 * k);
            __builtin_amdgcn_sched_barrier(0);
        }
        apply(X);
        __builtin_amdgcn_sched_barrier(0);
        acc = ZERO16;
#pragma unroll
        for (int k = 0; k < 4; ++k) {
#pragma unroll
            for (int f = 0; f < 4; ++f) {
                acc = mm(QF(W[k], f), QF(X[k], f), acc);
                __builtin_amdgcn_sched_barrier(0);
            }
        }
        lo = pv[0];
        m = (unsigned)__builtin_amdgcn_ballot_w64(pv[1] >= call) & 0xfffu;
        if ((unsigned)__builtin_amdgcn_readlane((int)lo, 16) == call) bad = true;
        if (need == 0u || bad) break;
        const unsigned ok = (quad_valid(W[0]) ? 1u : 0u) | (quad_valid(W[1]) ? 2u : 0u) | (quad_valid(W[2]) ? 4u : 0u) |
                            (quad_valid(W[3]) ? 8u : 0u);
        if ((need & ~ok) == 0u) break;
        if ((++nre & 15u) == 0u && late()) { bad = true; break; }
    }
    __builtin_amdgcn_sched_barrier(0);
    auto foreign = [&]() {
        if (bad && L == 0) LDS_ST(S.bad, 1u);
        deposit(S.redF[wv], scale16(acc, -E), i, hh);
        arrive(S.nfor, 1u << wv);
        STAMP(32 + wv);
    };
    bool pend = (m & 0xf00u) == 0xf00u;
    if (!pend) {
        foreign();
        for (poll(); !bad && (m & 0xf00u) != 0xf00u; poll()) {
            if (late()) { bad = true; break; }
            __builtin_amdgcn_s_sleep(SLEEP_O);
            asm volatile("" ::: "memory");
        }
    }
    const bool needo = (__builtin_amdgcn_readlane((int)lo, 8) & 0x10000) != 0;
    const int EO = level(8);
    for (;;) {
        asm volatile("" ::: "memory");
        {
#pragma unroll
            for (int k = 0; k < 4; ++k) {
                X[k] = load_quad(rs, oxo + 1024 * k);
                __builtin_amdgcn_sched_barrier(0);
            }
        }
        if (pend) {
            foreign();
            pend = false;
            asm volatile("" : "=v"(acc));
        }
        __builtin_amdgcn_sched_barrier(0);
        apply(W);
        __builtin_amdgcn_sched_barrier(0);
        acc = ZERO16;
#pragma unroll
        for (int k = 0; k < 4; ++k) {
#pragma unroll
            for (int f = 0; f < 4; ++f) {
                acc = mm(QF(W[k], f), QF(X[k], f), acc);
                __builtin_amdgcn_sched_barrier(0);
            }
        }
        if (!needo || bad) break;
        const bool ok = quad_valid(X[0]) & quad_valid(X[1]) & quad_valid(X[2]) & quad_valid(X[3]);
        if (ok) break;
        if ((++nre & 15u) == 0u && late()) { bad = true; break; }
    }
    STAMP(40 + wv);
    __builtin_amdgcn_sched_barrier(0);
    if (bad && L == 0) LDS_ST(S.bad, 1u);
    deposit(S.redO[wv], scale16(acc, -EO), i, hh);
    arrive(S.nown, 1u << wv);
    STAMP(48 + wv);
}

extern "C" void kernel_launch(int rank, int nranks, ncclComm_t comm,
                              hipStream_t stream,
                              void* const* d_in, const int* in_sizes, int n_in,
                              void* d_out, int out_size,
                              const int* in_dtypes, int out_dtype,
                              void* const* d_out_all,
                              void* const* d_ws_all, size_t ws_bytes) {
    if (nranks != NR || rank < 0 || rank >= NR || n_in < 2 || ws_bytes < (size_t)WS_NEED || out_size != MS * NDIM) {
        fprintf(stderr, "unexpected problem geometry\n");
        return;
    }
    Args a;
    a.x = (const float*)d_in[0];
    a.w = (const float*)d_in[1];
    a.ws = (char*)d_ws_all[rank];
    a.out = (float*)d_out;
    a.rank = rank;
    a.pad = 0;
    for (int l = 0; l < NR - 1; ++l) a.dws[l] = (char*)d_ws_all[(rank + 1 + l) & (NR - 1)];
    for (int p = 0; p < NR / 2; ++p) {
        a.dws[NR - 1 + p] = (char*)d_ws_all[2 * p + 1 - (rank & 1)];
        a.fout[p] = (float*)d_out_all[2 * p + 1 - (rank & 1)];
    }
    gemm_rs_kernel<<<NWG, 640, 0, stream>>>(a);
    HIP_CHECK(hipGetLastError());
}
```

```cpp
#include <hip/hip_runtime.h>
#include <rccl/rccl.h>
#include <cstdio>
#include <cstdint>

#define HIP_CHECK(expr) do { hipError_t _e = (expr); if (_e != hipSuccess) { \
    fprintf(stderr, "HIP error %s at %s:%d\n", hipGetErrorString(_e), __FILE__, __LINE__); \
    return; } } while (0)

typedef _Float16 h8v __attribute__((ext_vector_type(8)));
typedef float f16v __attribute__((ext_vector_type(16)));
typedef float f4v __attribute__((ext_vector_type(4)));
typedef float f2v __attribute__((ext_vector_type(2)));
typedef unsigned u4v __attribute__((ext_vector_type(4)));
typedef unsigned u2v __attribute__((ext_vector_type(2)));

namespace {

constexpr int NR = 8;
constexpr int NDIM = 2048;
constexpr int KS = 256;
constexpr int MS = 256;
constexpr int NWG = 256;
constexpr int NSTR = 32;
constexpr int NPC = 48;
constexpr int RMAX = 7;
constexpr unsigned F_STR = 33 * 1024, R_STR = 4 * F_STR, S_STR = 6 * R_STR;
constexpr unsigned DATA_BYTES = 2u * NR * S_STR;
constexpr unsigned FLAG_STRIDE = 128;
constexpr unsigned FLAG_OFF = DATA_BYTES;
constexpr unsigned FLAG_BYTES = 2u * NR * NPC * FLAG_STRIDE;
constexpr unsigned OFL_OFF = FLAG_OFF + FLAG_BYTES;
constexpr unsigned OFL_STRIDE = 64;
constexpr unsigned CTR_OFF = OFL_OFF + NWG * OFL_STRIDE;
constexpr unsigned CTR_STRIDE = 64;
constexpr unsigned TMO_OFF = CTR_OFF + NWG * CTR_STRIDE;
constexpr unsigned PRG_OFF = TMO_OFF + 128;
constexpr unsigned WS_NEED = PRG_OFF + NR * 128;
#define DLT 50000000ull
constexpr unsigned long long DEADLINE_TICKS = DLT;

#define INFL 2
#define SLEEP_N 1
#define FB_T1 50
#define FB_T2 110
#define FB_T3 200
#define FB_TL 60
#define FB_W0 2
#define FB_END 6
#define FB_WE 4
#define SLEEP_E 16
#define SLEEP_W 8
#define SLEEP_O 2
#define SLEEP_T 2
#define SLEEP_T2 0
#define SLEEP_R 2
#define TMO_EVERY 16
#define FB_SLEEP 4
#define STAG_S 1
#define EVS 64
#define EVC -32
#define GPD 16
#define RGD 2
#define LEAD_W 2
#define LEAD_O 2
#define ZAT 4
#define PREFETCH 1
#define WL_S 1
#define WL_C 1
#define NL_S 1
#define NL_C 1
#define STAG_MODE 1
#define TILE_WIN 1
#define TILE_WJ 12
#define LKMAP 0x75316420u
#define TG_A 1250
#define TG_B 21
#define TG_M 100000
#define XO_PLAIN 0
#define TG_P 1
#define TG_PD 4
#define CG_W 0
#define CG_S 4
#define TG_S 1
#define SC_ON 3
#define FB_T1L 45
#define FB_TLS 50
#define CW_S 0
#define CW_L 0
#define PROG_F 1
#define PROG_O 1
#define AUX_OWN 17
#define PUSH_POL 0
#define PUSH_FIRST 1
#define PUSH_LIKELY 0
#define PUSH_EXPECT(c) (c)
constexpr int GP = GPD, RG = RGD, QREL = (32 / GP) * RG * 4, FH = QREL / 4;
constexpr int LS = 192 + QREL, LC = 192 - QREL;
static_assert(LS <= 32 * RMAX && QREL <= 32 && QREL > 0, "stream geometry");
#define AUX_ST 17
#define AUX_LD 17

#define STAMP(idx) do {} while (0)
#define STAMPV(idx, v) do {} while (0)
#define TNOW() 0ull

#define LDS_LD(v) __builtin_amdgcn_readfirstlane(__hip_atomic_load(&(v), __ATOMIC_RELAXED, __HIP_MEMORY_SCOPE_WORKGROUP))
#define LDS_ST(v, x) __hip_atomic_store(&(v), (x), __ATOMIC_RELAXED, __HIP_MEMORY_SCOPE_WORKGROUP)

struct Args {
    const float* x;
    const float* w;
    char* ws;
    float* out;
    int rank, pad;
    char* dws[NR + 3];
    float* fout[NR / 2];
};

__device__ __forceinline__ unsigned data_of(int par, int src, int u) {
    return (unsigned)(par * NR + src) * S_STR + (unsigned)(u >> 5) * R_STR + (unsigned)(u & 31) * 1024u;
}
__device__ __forceinline__ unsigned flag_of(int par, int src, int u) {
    return FLAG_OFF + (unsigned)((par * NR + src) * NPC + (u >> 2)) * FLAG_STRIDE + (unsigned)(u & 3) * 8u;
}
__device__ __forceinline__ int frow(int r, int i) {
    return MS * (2 * (i >> 1) + 1 - (r & 1)) + 64 * (r >> 1) + 32 * (i & 1);
}

struct Ent { int kind, src, u, a0, kq; };
constexpr int rounds_c(int n, int s) { return (n - s + 31) >> 5; }
__device__ __forceinline__ int rounds(int typ, int s) {
    return typ == 2 ? (s < QREL ? 1 : 0) : ((typ ? LS : LC) - s + 31) >> 5;
}
__device__ __forceinline__ Ent entry(int typ, int rank, int dest, int rho, int s) {
    Ent e;
    e.src = rank;
    if (typ == 2) {
        const int x = s % FH, j = GP * (x / RG) + GP - RG + x % RG;
        e.kind = 1; e.kq = dest >> 1; e.a0 = 1024 * (dest & 1) + 32 * j; e.u = 32 + 4 * j + e.kq;
        return e;
    }
    if (rho == 0) { e.kind = 0; e.kq = s & 3; e.a0 = frow(dest, s >> 2); e.u = s; return e; }
    const int n = 32 * (rho - 1) + s, o = n - (typ ? LS - 64 : LC - 64);
    if (o >= 0) { e.kind = 0; e.kq = o & 3; e.a0 = MS * dest + 32 * (o >> 2); e.u = 160 + o; return e; }
    constexpr int GS = 4 * (GP + RG), GC = 4 * (GP - RG);
    const int g = typ ? n / GS : n / GC, t = n - g * (typ ? GS : GC);
    if (t < 4 * GP) {
        const int j = GP * g + (t >> 2);
        e.kind = 1; e.kq = t & 3; e.a0 = 1024 * (dest & 1) + 32 * j; e.u = 32 + 4 * j + e.kq;
    } else {
        const int r = t - 4 * GP;
        e.kind = 2; e.kq = rank >> 1; e.a0 = 0; e.src = 2 * (r & 3) + 1 - (rank & 1);
        e.u = 32 + 4 * (GP * g + GP - RG + (r >> 2)) + e.kq;
    }
    return e;
}
__device__ __forceinline__ int first_early(int typ, int s) {
    return typ == 1 ? (LS - 32 - EVS - s + 31) >> 5 : typ == 0 ? (LC - 32 - (EVC) - s + 31) >> 5 : 1;
}

__device__ __forceinline__ bool clears(int typ, int rank, int s) { return typ == 0 && (s >> 3) == (rank >> 1); }
constexpr unsigned EMPTY = 0xffffffffu;

template <int NV>
__device__ __forceinline__ int block_exp(const float (&v)[NV]) {
    float m = 0.f;
#pragma unroll
    for (int t = 0; t < NV; ++t) m = fmaxf(m, fabsf(v[t]));
    unsigned u = __float_as_uint(m);
    u = max(u, (unsigned)__builtin_amdgcn_update_dpp(0, (int)u, 0xB1, 0xf, 0xf, true));
    u = max(u, (unsigned)__builtin_amdgcn_update_dpp(0, (int)u, 0x4E, 0xf, 0xf, true));
    u = max(u, (unsigned)__builtin_amdgcn_update_dpp(0, (int)u, 0x141, 0xf, 0xf, true));
    u = max(u, (unsigned)__builtin_amdgcn_update_dpp(0, (int)u, 0x140, 0xf, 0xf, true));
    const unsigned r = max(max((unsigned)__builtin_amdgcn_readlane((int)u, 0), (unsigned)__builtin_amdgcn_readlane((int)u, 16)),
                           max((unsigned)__builtin_amdgcn_readlane((int)u, 32), (unsigned)__builtin_amdgcn_readlane((int)u, 48)));
    unsigned E = r >> 23;
    E = E < 15u ? 15u : E;
    return 141 - (int)E;
}

__device__ __forceinline__ u4v pack8(const float* v, int e) {
    h8v r;
    typedef _Float16 h2v __attribute__((ext_vector_type(2)));
    const float sc = __uint_as_float((unsigned)(e + 127) << 23);
    const f2v s2 = {sc, sc};
#pragma unroll
    for (int j = 0; j < 8; j += 2) {
        const h2v t = __builtin_convertvector(f2v{v[j], v[j + 1]} * s2, h2v);
        r[j] = t[0];
        r[j + 1] = t[1];
    }
    return __builtin_bit_cast(u4v, r);
}

template <int NF>
__device__ __forceinline__ void load_w(const float* __restrict__ w, int k0, int col0, int i, int hh, float (&v)[8 * NF]) {
    const float* base = w + (size_t)(k0 + 8 * hh) * NDIM + col0 + i;
#pragma unroll
    for (int s = 0; s < NF; ++s)
#pragma unroll
        for (int j = 0; j < 8; ++j) v[8 * s + j] = base[(size_t)(16 * s + j) * NDIM];
}

template <int NF>
__device__ __forceinline__ void load_x(const float* __restrict__ x, int row0, int k0, int i, int hh, float (&v)[8 * NF]) {
    const float* base = x + (size_t)(row0 + i) * KS + k0 + 8 * hh;
#pragma unroll
    for (int s = 0; s < NF; ++s) {
        f4v t0 = *(const f4v*)(base + 16 * s);
        f4v t1 = *(const f4v*)(base + 16 * s + 4);
#pragma unroll
        for (int j = 0; j < 4; ++j) { v[s * 8 + j] = t0[j]; v[s * 8 + 4 + j] = t1[j]; }
    }
}

__device__ __forceinline__ f16v mm(u4v a, u4v b, f16v c) {
    return __builtin_amdgcn_mfma_f32_32x32x16_f16(__builtin_bit_cast(h8v, a), __builtin_bit_cast(h8v, b), c, 0, 0, 0);
}
#define ZERO16 f16v{0.f, 0.f, 0.f, 0.f, 0.f, 0.f, 0.f, 0.f, 0.f, 0.f, 0.f, 0.f, 0.f, 0.f, 0.f, 0.f}

__device__ __forceinline__ f16v scale16(f16v t, int e) {
    const int e1 = e >> 1, e2 = e - e1;
    const float c1 = __uint_as_float((unsigned)(e1 + 127) << 23), c2 = __uint_as_float((unsigned)(e2 + 127) << 23);
    const f2v s1 = {c1, c1}, s2 = {c2, c2};
#pragma unroll
    for (int r = 0; r < 16; r += 2) {
        const f2v u = (f2v{t[r], t[r + 1]} * s1) * s2;
        t[r] = u[0]; t[r + 1] = u[1];
    }
    return t;
}

__device__ __forceinline__ void fma16(f16v& acc, f16v t, int e) {
    const int e1 = e >> 1, e2 = e - e1;
    const float c1 = __uint_as_float((unsigned)(e1 + 127) << 23), c2 = __uint_as_float((unsigned)(e2 + 127) << 23);
    const f2v s1 = {c1, c1}, s2 = {c2, c2};
#pragma unroll
    for (int r = 0; r < 16; r += 2) {
        const f2v u = __builtin_elementwise_fma(f2v{t[r], t[r + 1]} * s1, s2, f2v{acc[r], acc[r + 1]});
        acc[r] = u[0]; acc[r + 1] = u[1];
    }
}

struct Quad { u4v f[4]; };
#define QF(q, n) ((q).f[n])

#define AUX_XF AUX_LD
#define AUX_W AUX_LD
#define AUX_XO AUX_LD
template <int AUX = AUX_LD>
__device__ __forceinline__ Quad load_quad(__amdgpu_buffer_rsrc_t rs, int off) {
    Quad q;
#pragma unroll
    for (int f = 0; f < 4; ++f) q.f[f] = __builtin_amdgcn_raw_buffer_load_b128(rs, off, f * (int)F_STR, AUX);
    return q;
}

__device__ __forceinline__ f16v mm_quad(const Quad& w, const Quad& x, f16v t) {
    t = mm(QF(w, 0), QF(x, 0), t);
    t = mm(QF(w, 1), QF(x, 1), t);
    t = mm(QF(w, 2), QF(x, 2), t);
    return mm(QF(w, 3), QF(x, 3), t);
}

__device__ __forceinline__ unsigned down(int d) {
    return d <= 14 ? (unsigned)(15 - d) << 10 : (1u << 24 >> min(d, 31));
}
__device__ __forceinline__ void scale_quad(Quad& q, unsigned c) {
    const _Float16 h = __builtin_bit_cast(_Float16, (unsigned short)c);
#pragma unroll
    for (int f = 0; f < 4; ++f) {
        q.f[f] = __builtin_bit_cast(u4v, __builtin_bit_cast(h8v, q.f[f]) * h);
        asm volatile("" : "+v"(q.f[f]));
    }
}

__device__ __forceinline__ u4v no_zero(u4v u) {
    u[0] = u[0] ? u[0] : 0x8000u;
    return u;
}

__device__ __forceinline__ bool quad_valid(const Quad& q) {
    return __builtin_amdgcn_ballot_w64(min(min(q.f[0][0], q.f[1][0]), min(q.f[2][0], q.f[3][0])) == 0u) == 0ull;
}

template <int N>
__device__ __forceinline__ void wait_vm() {
    asm volatile("s_waitcnt vmcnt(%0)" ::"i"(N) : "memory");
}

__device__ __forceinline__ void wait_vm_n(int n) {
    switch (n) {
        case 0: wait_vm<0>(); break;
        case 1: wait_vm<1>(); break;
        case 2: wait_vm<2>(); break;
        case 3: wait_vm<3>(); break;
        case 4: wait_vm<4>(); break;
        case 5: wait_vm<5>(); break;
        case 6: wait_vm<6>(); break;
        case 7: wait_vm<7>(); break;
        case 8: wait_vm<8>(); break;
        case 9: wait_vm<9>(); break;
        default: wait_vm<10>(); break;
    }
}

__device__ __forceinline__ void raw_barrier() {
    asm volatile("s_waitcnt lgkmcnt(0)\n\ts_barrier" ::: "memory");
}

template <int SLP, class T>
__device__ __forceinline__ unsigned lds_wait(T& v, unsigned n = 1u) {
    unsigned r, it = 0u;
    unsigned long long t0 = 0ull;
    while ((r = (unsigned)LDS_LD(v)) < n) {
        __builtin_amdgcn_s_sleep(SLP);
        if ((++it & 1023u) == 0u) {
            const unsigned long long t = __builtin_amdgcn_s_memrealtime();
            if (t0 == 0ull) t0 = t;
            else if (t - t0 > 2ull * DEADLINE_TICKS) { r = 0u; break; }
        }
    }
    asm volatile("" ::: "memory");
    return r;
}

struct Lds {
    u4v stage[RMAX][256];
    f4v redF[NR][8 * 33];
    f4v redO[NR][8 * 33];
    int exp_[RMAX + 1];
    int rdy[RMAX + 1];
    unsigned call, bad, nloc, locdone, nfor, nown, fdone, clr;
    int prog, win, zok;
};

__device__ __forceinline__ void deposit(f4v* slot, const f16v& t, int i, int hh) {
#pragma unroll
    for (int g = 0; g < 4; ++g)
        slot[(2 * g + hh) * 33 + i] = f4v{t[4 * g], t[4 * g + 1], t[4 * g + 2], t[4 * g + 3]};
}
__device__ __forceinline__ void arrive(unsigned& c, unsigned v) {
    unsigned long long ex;
    asm volatile("s_mov_b64 %0, exec\n\ts_mov_b64 exec, 1\n\tds_add_u32 %1, %2\n\ts_mov_b64 exec, %0"
                 : "=&s"(ex)
                 : "v"((unsigned)(uintptr_t)(__attribute__((address_space(3))) unsigned*)&c), "v"(v)
                 : "memory");
}
__device__ __forceinline__ void gather(f4v (&acc)[4], bool& bad, unsigned& flags, unsigned& badw, const f4v* red) {
    unsigned got = 0u, it = 0u;
    unsigned long long t0 = 0ull;
    auto late = [&]() -> bool {
        if ((++it & 1023u) == 0u) {
            const unsigned long long t = __builtin_amdgcn_s_memrealtime();
            if (t0 == 0ull) t0 = t;
            else if (t - t0 > 2ull * DEADLINE_TICKS) return true;
        }
        return false;
    };
    for (int n = 0; n < NR - 2 && !bad;) {
        const unsigned m = (unsigned)LDS_LD(flags) & ~got;
        asm volatile("" ::: "memory");
        if (m == 0u) {
            __builtin_amdgcn_s_sleep(0);
            bad = late();
            continue;
        }
        const int p = __builtin_ctz(m);
#pragma unroll
        for (int u = 0; u < 4; ++u) acc[u] += red[p * (8 * 33) + 8 * u];
        got |= 1u << p;
        ++n;
    }
    const int p = __builtin_ctz(~got);
    f4v v[4];
    for (;;) {
        asm volatile("" ::: "memory");
        const unsigned c = __hip_atomic_load(&flags, __ATOMIC_RELAXED, __HIP_MEMORY_SCOPE_WORKGROUP);
        __builtin_amdgcn_sched_barrier(0);
        asm volatile("" ::: "memory");
        const unsigned w = __hip_atomic_load(&badw, __ATOMIC_RELAXED, __HIP_MEMORY_SCOPE_WORKGROUP);
#pragma unroll
        for (int u = 0; u < 4; ++u) v[u] = red[p * (8 * 33) + 8 * u];
        __builtin_amdgcn_sched_barrier(0);
        if (__builtin_amdgcn_readfirstlane(c) == 0x7fu) { bad |= __builtin_amdgcn_readfirstlane(w) != 0u; break; }
        asm volatile("s_waitcnt lgkmcnt(0)" ::: "memory");
        if (bad || late()) { bad = true; break; }
    }
#pragma unroll
    for (int u = 0; u < 4; ++u) acc[u] += v[u];
    if (bad) {
        const float qn = __builtin_nanf("");
#pragma unroll
        for (int u = 0; u < 4; ++u) acc[u] = f4v{qn, qn, qn, qn};
    }
}

__device__ __forceinline__ void pusher(const Args& a, Lds& S, char* dws, int b, int rank, int dest, int s, int L, int typ) {
    const int nrnd = rounds(typ, s), NST = 4 * nrnd;
    char* const myws = a.ws;
    const unsigned long long tq0 = TNOW();
    const unsigned cv = __hip_atomic_load((unsigned*)(myws + CTR_OFF + b * CTR_STRIDE), __ATOMIC_RELAXED,
                                          __HIP_MEMORY_SCOPE_AGENT);
    const int i = L & 31, hh = L >> 5;
    u4v t[4];
    unsigned vdat, vflg;
    {
        const Ent en = entry(typ, rank, dest, 0, s);
        float v0[32];
        if (typ == 2) load_w<4>(a.w, 64 * en.kq, en.a0, i, hh, v0);
        else load_x<4>(a.x, en.a0, 64 * en.kq, i, hh, v0);
        const Ent le = entry(typ, rank, dest, L < nrnd ? L : 0, s);
        vdat = data_of(0, le.src, le.u);
        vflg = flag_of(0, le.src, le.u);
        asm volatile("" : "+v"(vdat), "+v"(vflg));
        if (L <= RMAX) S.rdy[L] = 0;
        if (L == 0) {
            S.call = 0u; S.bad = 0u; S.nloc = 0u; S.locdone = 0u; S.nfor = 0u; S.nown = 0u; S.fdone = 0u;
            S.clr = clears(typ, rank, s) ? 0u : 1u;
            S.prog = 0; S.zok = 0; S.win = typ == 2 ? 6 : INFL;
        }
        raw_barrier();
        const int e = block_exp<32>(v0);
        if (L == 0) { LDS_ST(S.rdy[0], 1); S.exp_[0] = e; }
#pragma unroll
        for (int f = 0; f < 4; ++f) t[f] = pack8(v0 + 8 * f, e);
        if (typ == 2) {
#pragma unroll
            for (int f = 0; f < 4; ++f) t[f] = no_zero(t[f]);
        }
    }
    const unsigned call = __builtin_amdgcn_readfirstlane(cv) + 1u;
    if (L == 0) LDS_ST(S.call, call);
    if (nrnd == 0) return;
    const int par = (int)(call & 1u);
    __builtin_amdgcn_s_setprio(3);
    const __amdgpu_buffer_rsrc_t rs = __builtin_amdgcn_make_buffer_rsrc(dws, 0, (int)DATA_BYTES, 0x00020000);
    vdat += (unsigned)par * (NR * S_STR);
    vflg += (unsigned)par * (NR * NPC * FLAG_STRIDE);
    auto publish = [&](int c, unsigned early) {
        if (L == c) {
            const unsigned long long val = ((unsigned long long)call << 32) | (((unsigned)S.exp_[c] & 0xffffu) | early);
            __hip_atomic_store((unsigned long long*)(dws + vflg), val, __ATOMIC_RELAXED, __HIP_MEMORY_SCOPE_SYSTEM);
        }
    };
    int issued = 0, pub = 0, dn = 0, h1 = 0, h2 = 0, h3 = 0, h4 = 0, h5 = 0, h6 = 0, h7 = 0;
    const int ev = first_early(typ, s);
    const int nl = 4 * ((typ == 1 ? LS : LC) >> 5) - (typ == 1 ? NL_S : NL_C);
    for (int c = 0;; ++c) {
        const int so = __builtin_amdgcn_readlane((int)vdat, c);
        bool have = false;
#pragma unroll
        for (int f = 0; f < 4; ++f) {
            const int n = 4 * c + f;
            __builtin_amdgcn_raw_buffer_store_b128(t[f], rs, L * 16, so + f * (int)F_STR, AUX_ST);
            const int h0 = ++issued;
            __builtin_amdgcn_sched_barrier(0);
            if (c >= ev && f == (c + 1 == nrnd ? LEAD_O : LEAD_W)) {
                if (c + 1 == nrnd && !lds_wait<1>(S.zok)) {
                    if (L == 0) { LDS_ST(S.bad, 1u); LDS_ST(S.prog, NST); }
                    return;
                }
                publish(c, 0x10000u); ++issued;
            }
            if (L == 0) LDS_ST(S.prog, n + 1 == NST ? NST : dn);
            while (pub < ev && 4 * (pub + 1) <= dn && (pub > 0 || LDS_LD(S.clr))) { publish(pub, 0u); ++issued; ++pub; }
            if (f == 3) {
                if (n + 1 == NST) break;
                have = LDS_LD(S.rdy[c + 1]) != 0;
                asm volatile("" ::: "memory");
                if (have) {
#pragma unroll
                    for (int g = 0; g < 4; ++g) t[g] = S.stage[c + 1][g * 64 + L];
                }
            }
            int win = LDS_LD(S.win);
            if (n >= nl) win = max(win, typ == 1 ? WL_S : WL_C);
            const int ik = win <= 1 ? h0 : win == 2 ? h1 : win == 3 ? h2 : win == 4 ? h3 : win <= 6 ? h5 : h7;
            const int k1 = n + 2 - (win > 6 ? 8 : win > 4 ? 6 : win);
            if (k1 > dn) {
                wait_vm_n(issued - ik);
                dn = k1;
            }
            h7 = h6; h6 = h5; h5 = h4; h4 = h3; h3 = h2; h2 = h1; h1 = h0;
        }
        if (c + 1 == nrnd) break;
        if (!have) {
            while (pub < ev && 4 * (pub + 1) <= dn && (pub > 0 || LDS_LD(S.clr))) { publish(pub, 0u); ++issued; ++pub; }
            if (!lds_wait<0>(S.rdy[c + 1])) {
                if (L == 0) { LDS_ST(S.bad, 1u); LDS_ST(S.prog, NST); }
                return;
            }
#pragma unroll
            for (int g = 0; g < 4; ++g) t[g] = S.stage[c + 1][g * 64 + L];
        }
    }
    const unsigned long long tq1 = TNOW();
    if (pub == 0 && ev > 0) {
        if (!lds_wait<1>(S.clr)) {
            if (L == 0) LDS_ST(S.bad, 1u);
            return;
        }
        while (pub < ev && 4 * (pub + 1) <= dn) { publish(pub, 0u); ++issued; ++pub; }
    }
    if (pub < ev && pub == nrnd - 3) { wait_vm_n(issued - h7); publish(pub, 0u); ++issued; ++pub; }
    if (pub < ev && pub == nrnd - 2) { wait_vm_n(issued - h4); publish(pub, 0u); ++pub; }
    if (pub < ev) { wait_vm<0>(); publish(pub, 0u); }
}

__device__ __forceinline__ bool feed(const Args& a, Lds& S, int rank, int dest, int s, int wv, int L, int typ) {
    if (wv < 1 || wv >= rounds(typ, s)) return false;
    const Ent en = entry(typ, rank, dest, wv, s);
    if (en.kind == 2) return true;
    const int i = L & 31, hh = L >> 5;
    float v[32];
    if (en.kind == 1) load_w<4>(a.w, 64 * en.kq, en.a0, i, hh, v);
    else load_x<4>(a.x, en.a0, 64 * en.kq, i, hh, v);
    const int e = block_exp<32>(v);
#pragma unroll
    for (int f = 0; f < 4; ++f) {
        S.stage[wv][f * 64 + L] = no_zero(pack8(v + 8 * f, e));
    }
    if (L == 0) S.exp_[wv] = e;
    asm volatile("s_waitcnt lgkmcnt(0)" ::: "memory");
    if (L == 0) LDS_ST(S.rdy[wv], 1);
    asm volatile("" ::: "memory");
    return false;
}

}

__global__ void __launch_bounds__(640) gemm_rs_kernel(Args a) {
    __shared__ __attribute__((aligned(16))) Lds S;

    const int b = blockIdx.x;
    const int wr = __builtin_amdgcn_readfirstlane(threadIdx.x >> 6);
    const int wv = wr == 0 ? NR : wr <= NR ? wr - 1 : NR + 1;
    const int L = threadIdx.x & 63;
    const int i = L & 31, hh = L >> 5;
    const int ti = b & 7, tj = (b >> 3) & 31;
    const int lk = (int)(LKMAP >> (4 * ((b >> 5) & 7))) & 7, s = b & 31;
    char* const dws = a.dws[lk < 7 ? lk : 7 + ((s / FH) & 3)];
    float* const pf0 = a.fout[ti >> 1];
    char* const myws = a.ws;
    asm volatile("" ::"s"(a.x), "s"(a.w), "s"(myws), "s"(a.out), "s"(a.rank), "s"(dws), "s"(pf0));
    const int rank = a.rank & (NR - 1);
    const int cls = rank & 1, mem = rank >> 1;
    const int dest = lk < 7 ? (rank + 1 + lk) & (NR - 1) : 2 * ((s / FH) & 3) + 1 - cls;
    const int typ = lk == 7 ? 2 : lk & 1;
    const unsigned tk0 = (unsigned)__builtin_amdgcn_s_memrealtime();

    if (PUSH_EXPECT(wv == NR)) {
        pusher(a, S, dws, b, rank, dest, s, L, typ);
        return;
    }

    if (wv == NR + 1) {
        raw_barrier();
        const unsigned call = lds_wait<1>(S.call);
        if (call == 0u) return;
        if (L == 0)
            __hip_atomic_store((unsigned*)(myws + CTR_OFF + b * CTR_STRIDE), call, __ATOMIC_RELAXED, __HIP_MEMORY_SCOPE_AGENT);
        if (lk < 7) {
            const int nrnd = rounds(typ, s), NST = 4 * nrnd, LT = typ ? 4 * LS : 4 * LC;
            int z = 0;
            auto clear = [&]() {
                const __amdgpu_buffer_rsrc_t rz = __builtin_amdgcn_make_buffer_rsrc(myws, 0, (int)DATA_BYTES, 0x00020000);
                for (int c = first_early(typ, s); c < nrnd; ++c) {
                    const Ent en = entry(typ, dest, rank, c, s);
                    const unsigned zo = data_of((int)(call & 1u) ^ 1, en.src, en.u);
#pragma unroll
                    for (int f = 0; f < 4; ++f)
                        __builtin_amdgcn_raw_buffer_store_b128(u4v{0u, 0u, 0u, 0u}, rz, L * 16, (int)(zo + f * F_STR), AUX_ST);
                }
                z = 1;
            };
            unsigned* pc = (unsigned*)(myws + PRG_OFF + (unsigned)dest * 128u);
            const unsigned base = (call - 1u) * (unsigned)LT;
            int last = 0;
            unsigned it = 0u;
            unsigned long long t0 = 0ull;
            const bool lng = nrnd == rounds(typ, 0);
            const int mul = (SC_ON & (typ ? 1 : 2)) ? (typ ? (lng ? 4 * rounds_c(LS, 0) : 4 * rounds_c(LS, 31)) * NSTR * 1024 / (4 * LS)
                                                          : (lng ? 4 * rounds_c(LC, 0) : 4 * rounds_c(LC, 31)) * NSTR * 1024 / (4 * LC))
                                                    : 1024;
            const int t1 = lng && mul != 1024 ? FB_T1L : FB_T1, tl = !lng && mul != 1024 ? FB_TLS : FB_TL;
            for (;;) {
                int p = LDS_LD(S.prog);
                if ((++it & 255u) == 0u) {
                    const unsigned long long t = __builtin_amdgcn_s_memrealtime();
                    if (t0 == 0ull) t0 = t;
                    else if (t - t0 > 2ull * DEADLINE_TICKS) p = NST;
                }
                const int dl = p > last ? p - last : 0;
                last += dl;
                if (last >= NST) {
                    if (L == 0) __hip_atomic_fetch_add(pc, (unsigned)dl, __ATOMIC_RELAXED, __HIP_MEMORY_SCOPE_AGENT);
                    break;
                }
                if (z == 0 && last >= ZAT) clear();
                unsigned g = 0u;
                if (L == 0) g = __hip_atomic_fetch_add(pc, (unsigned)dl, __ATOMIC_RELAXED, __HIP_MEMORY_SCOPE_AGENT) + (unsigned)dl;
                const int tot = (int)(__builtin_amdgcn_readfirstlane(g) - base);
                if (z == 1) {
                    wait_vm<0>();
                    if (L == 0) LDS_ST(S.zok, 1);
                    z = 2;
                }
                const int lag = ((tot * mul) >> 10) - NSTR * last;
                int win = lag > FB_T3 ? 6 : lag > FB_T2 ? 4 : lag > t1 ? 3 : lag < -tl ? 1 : 2;
                if (tot >= LT - FB_END * NSTR && win < FB_WE) win = FB_WE;
                if (L == 0 && last > 0) LDS_ST(S.win, win);
                __builtin_amdgcn_s_sleep(FB_SLEEP);
            }
            if (z == 0) clear();
            if (z == 1) {
                wait_vm<0>();
                if (L == 0) LDS_ST(S.zok, 1);
            }
        }
        {
            const __amdgpu_buffer_rsrc_t ro = __builtin_amdgcn_make_buffer_rsrc(a.out, 0, MS * NDIM * 4, 0x00020000);
            const int oo = ((32 * ti + (L >> 3)) * NDIM + 1024 * (1 - cls) + 32 * tj + 4 * (L & 7)) * 4;
            bool bad = !lds_wait<SLEEP_T>(S.fdone);
            const unsigned long long t0 = __builtin_amdgcn_s_memrealtime();
            unsigned ok = 0u, it = 0u;
            struct T4 { u4v v[4]; };
            auto look = [&]() -> T4 {
                T4 r;
#pragma unroll
                for (int u = 0; u < 4; ++u)
                    r.v[u] = __builtin_amdgcn_raw_buffer_load_b128(ro, oo + ((ok >> u & 1u) ? 0x40000000 : 0), 8 * u * NDIM * 4, AUX_LD);
                __builtin_amdgcn_sched_barrier(0);
                return r;
            };
            auto check = [&](const T4& r) {
#pragma unroll
                for (int u = 0; u < 4; ++u) {
                    const u4v v = r.v[u];
                    if (__builtin_amdgcn_ballot_w64(v[0] == EMPTY) == 0ull) ok |= 1u << u;
                }
                __builtin_amdgcn_sched_barrier(0);
            };
            while (!bad) {
                asm volatile("" ::: "memory");
                check(look());
                if (ok != 15u && (++it & 15u) != 0u) {
                    __builtin_amdgcn_s_sleep(SLEEP_T2);
                    continue;
                }
                if (ok == 15u) break;
                const unsigned v = __hip_atomic_load((const unsigned*)(myws + TMO_OFF), __ATOMIC_RELAXED, __HIP_MEMORY_SCOPE_SYSTEM);
                if (__builtin_amdgcn_readfirstlane(v) == call) bad = true;
                if (__builtin_amdgcn_s_memrealtime() - t0 > DEADLINE_TICKS) {
                    if (L == 0)
                        __hip_atomic_store((unsigned*)(myws + TMO_OFF), call, __ATOMIC_RELAXED, __HIP_MEMORY_SCOPE_AGENT);
                    bad = true;
                }
            }
            if (bad) {
                const unsigned qn = 0x7fc00000u;
#pragma unroll
                for (int u = 0; u < 4; ++u)
                    __builtin_amdgcn_raw_buffer_store_b128(u4v{qn, qn, qn, qn}, ro, oo, 8 * u * NDIM * 4, AUX_ST);
            }
            STAMP(59);
        }
        return;
    }

    raw_barrier();
    const bool clr = wv == NR - 1 && clears(typ, rank, s);
    if (clr) {
        const __amdgpu_buffer_rsrc_t ro = __builtin_amdgcn_make_buffer_rsrc(a.out, 0, MS * NDIM * 4, 0x00020000);
        const int oo = ((64 * (dest >> 1) + 32 * ((s >> 2) & 1)) * NDIM + 1024 * (dest & 1) + 256 * (s & 3) + 4 * L) * 4;
#pragma unroll
        for (int u = 0; u < 32; ++u)
            __builtin_amdgcn_raw_buffer_store_b128(u4v{EMPTY, EMPTY, EMPTY, EMPTY}, ro, oo, u * NDIM * 4, AUX_ST);
        wait_vm<0>();
        if (L == 0) LDS_ST(S.clr, 1u);
    }
    if (wv != 1) lds_wait<STAG_S>(S.rdy[0]);
    const bool fwd = feed(a, S, rank, dest, s, wv, L, typ);
    const int col0 = 1024 * cls + 32 * tj;
    {
        float vw[16], vf[16], vo[16];
        load_w<2>(a.w, 32 * wv, col0, i, hh, vw);
        load_x<2>(a.x, frow(rank, ti), 32 * wv, i, hh, vf);
        load_x<2>(a.x, MS * rank + 32 * ti, 32 * wv, i, hh, vo);
        const int ew = block_exp<16>(vw);
        const u4v w0 = pack8(vw, ew), w1 = pack8(vw + 8, ew);
        const int ef = block_exp<16>(vf);
        f16v t = mm(w0, pack8(vf, ef), ZERO16);
        t = mm(w1, pack8(vf + 8, ef), t);
        deposit(S.redF[wv], scale16(t, -(ew + ef)), i, hh);
        const int eo = block_exp<16>(vo);
        t = mm(w0, pack8(vo, eo), ZERO16);
        t = mm(w1, pack8(vo + 8, eo), t);
        deposit(S.redO[wv], scale16(t, -(ew + eo)), i, hh);
        arrive(S.nloc, 1u);
    }
    STAMP(60 + wv);
    if (wv == NR - 1) {
        float* pf = pf0;
        float* po = a.out;
        asm volatile("" : "+s"(pf), "+s"(po));
        const __amdgpu_buffer_rsrc_t rf = __builtin_amdgcn_make_buffer_rsrc(pf, 0, MS * NDIM * 4, 0x00020000);
        const __amdgpu_buffer_rsrc_t ro = __builtin_amdgcn_make_buffer_rsrc(po, 0, MS * NDIM * 4, 0x00020000);
        int of = ((64 * mem + 32 * (ti & 1) + (L >> 3)) * NDIM + col0 + 4 * (L & 7)) * 4;
        int oo = ((32 * ti + (L >> 3)) * NDIM + col0 + 4 * (L & 7)) * 4;
        asm volatile("" : "+v"(of), "+v"(oo));
        const int rp = (L & 7) * 33 + (L >> 3);
        f4v aF[4], aO[4];
        bool bad = !lds_wait<1>(S.nloc, NR);
#pragma unroll
        for (int u = 0; u < 4; ++u) {
            aF[u] = S.redF[0][rp + 8 * u];
            aO[u] = S.redO[0][rp + 8 * u];
#pragma unroll
            for (int p = 1; p < NR; ++p) {
                aF[u] += S.redF[p][rp + 8 * u];
                aO[u] += S.redO[p][rp + 8 * u];
            }
            asm volatile("" : "+v"(aF[u]), "+v"(aO[u]));
        }
        asm volatile("s_waitcnt lgkmcnt(0)" ::: "memory");
        if (L == 0) LDS_ST(S.locdone, 1u);
        gather(aF, bad, S.nfor, S.bad, &S.redF[0][rp]);
        u4v v[4];
#pragma unroll
        for (int u = 0; u < 4; ++u) v[u] = __builtin_bit_cast(u4v, aF[u]);
#pragma unroll
        for (int u = 0; u < 4; ++u) {
            v[u][0] = min(v[u][0], EMPTY - 1u);
            asm volatile("" : "+v"(v[u]));
        }
        while ((unsigned)__builtin_amdgcn_s_memrealtime() - tk0 < (unsigned)min(TG_A + TG_B * tj, TG_M))
            __builtin_amdgcn_s_sleep(TG_S);
        if (typ == 1) lds_wait<1>(S.prog, (unsigned)(4 * rounds(typ, s) - TG_PD));
        const unsigned long long tq = TNOW();
        if (tj < TILE_WJ) {
#pragma unroll
            for (int u = 0; u < 4; ++u) {
                __builtin_amdgcn_raw_buffer_store_b128(v[u], rf, of, 8 * u * NDIM * 4, AUX_ST);
                if (u < 3) wait_vm<TILE_WIN - 1>();
            }
        } else {
#pragma unroll
            for (int u = 0; u < 4; ++u)
                __builtin_amdgcn_raw_buffer_store_b128(v[u], rf, of, 8 * u * NDIM * 4, AUX_ST);
        }
        __builtin_amdgcn_sched_barrier(0);
        asm volatile("s_nop 1" ::: "memory");
        __builtin_amdgcn_sched_barrier(0);
        if (L == 0) LDS_ST(S.fdone, 1u);
        STAMPV(56, tq);
        STAMP(57);
        gather(aO, bad, S.nown, S.bad, &S.redO[0][rp]);
#pragma unroll
        for (int u = 0; u < 4; ++u)
            __builtin_amdgcn_raw_buffer_store_b128(__builtin_bit_cast(u4v, aO[u]), ro, oo, 8 * u * NDIM * 4, AUX_OWN);
        STAMP(58);
        return;
    }
    const unsigned call = lds_wait<1>(S.call);
    const int par = (int)(call & 1u);

    const __amdgpu_buffer_rsrc_t rs = __builtin_amdgcn_make_buffer_rsrc(myws, 0, (int)WS_NEED, 0x00020000);
    const unsigned long long t0 = __builtin_amdgcn_s_memrealtime();
    bool bad = call == 0u;
    unsigned npoll = 0u, nre = 0u;
    auto late = [&]() -> bool {
        if (__builtin_amdgcn_s_memrealtime() - t0 > DEADLINE_TICKS) {
            if (L == 0)
                __hip_atomic_store((unsigned*)(myws + TMO_OFF), call, __ATOMIC_RELAXED, __HIP_MEMORY_SCOPE_AGENT);
            return true;
        }
        return false;
    };
    if (fwd && !bad) {
        const Ent en = entry(1, rank, dest, wv, s);
        const int pf = (int)(L == 0 ? flag_of(par, en.src, en.u) : TMO_OFF);
        u2v pv;
        for (;;) {
            pv = __builtin_amdgcn_raw_buffer_load_b64(rs, pf, 0, AUX_LD);
            if ((unsigned)__builtin_amdgcn_readlane((int)pv[1], 0) >= call) break;
            if ((unsigned)__builtin_amdgcn_readlane((int)pv[0], 1) == call || late()) { bad = true; break; }
            __builtin_amdgcn_s_sleep(SLEEP_R);
            asm volatile("" ::: "memory");
        }
        asm volatile("" ::: "memory");
        const Quad qd = load_quad(rs, (int)(data_of(par, en.src, en.u) + (unsigned)L * 16u));
        S.stage[wv][L] = QF(qd, 0);
        S.stage[wv][64 + L] = QF(qd, 1);
        S.stage[wv][128 + L] = QF(qd, 2);
        S.stage[wv][192 + L] = QF(qd, 3);
        if (L == 0) S.exp_[wv] = (int)(short)pv[0];
        asm volatile("s_waitcnt lgkmcnt(0)" ::: "memory");
        if (L == 0) LDS_ST(S.rdy[wv], 1);
        asm volatile("" ::: "memory");
        STAMP(70 + wv);
    }
    const int src = (rank + 1 + wv) & (NR - 1);
    const int oxf = (int)(data_of(par, src, 4 * ti) + (unsigned)L * 16u);
    const int ow = (int)(data_of(par, src, 32 + 4 * tj) + (unsigned)L * 16u);
    const int oxo = (int)(data_of(par, src, 160 + 4 * ti) + (unsigned)L * 16u);
    const int poff1 = (int)(L < 4 ? flag_of(par, src, 4 * ti + L)
                            : L < 8 ? flag_of(par, src, 32 + 4 * tj + L - 4)
                            : L < 12 ? flag_of(par, src, 160 + 4 * ti + L - 8) : TMO_OFF);
    const int poff0 = (int)(L < 12 ? (unsigned)poff1 : flag_of(par, src, 4 * ti) + 64u);
    unsigned lo = 0u, m = 0u;
    auto poll = [&]() {
        ++npoll;
        const u2v pv = __builtin_amdgcn_raw_buffer_load_b64(rs, (npoll % TMO_EVERY) == 0u ? poff1 : poff0, 0, AUX_LD);
        lo = pv[0];
        m = (unsigned)__builtin_amdgcn_ballot_w64(pv[1] >= call) & 0xfffu;
        if ((unsigned)__builtin_amdgcn_readlane((int)lo, 16) == call) bad = true;
    };
    unsigned cf[4];
    auto level = [&](int xb) -> int {
        int e[4];
#pragma unroll
        for (int k = 0; k < 4; ++k)
            e[k] = (int)(short)__builtin_amdgcn_readlane((int)lo, 4 + k) + (int)(short)__builtin_amdgcn_readlane((int)lo, xb + k);
        const int E = min(min(e[0], e[1]), min(e[2], e[3]));
#pragma unroll
        for (int k = 0; k < 4; ++k) cf[k] = down(e[k] - E);
        return E;
    };
    auto apply = [&](Quad (&Q)[4]) {
#pragma unroll
        for (int k = 0; k < 4; ++k) {
            scale_quad(Q[k], cf[k]);
            cf[k] = 0x3c00u;
        }
    };
    Quad X[4], W[4];
    for (poll(); !bad && (m & 0xfu) != 0xfu; poll()) {
        if (late()) { bad = true; break; }
        __builtin_amdgcn_s_sleep(SLEEP_E);
        asm volatile("" ::: "memory");
    }
    asm volatile("" ::: "memory");
    STAMP(16 + wv);
#pragma unroll
    for (int k = 0; k < 4; ++k) X[k] = load_quad<AUX_XF>(rs, oxf + 1024 * k);
    if (!lds_wait<1>(S.locdone)) bad = true;
    for (; !bad && (m & 0xf0u) != 0xf0u; poll()) {
        if (late()) { bad = true; break; }
        __builtin_amdgcn_s_sleep(SLEEP_W);
        asm volatile("" ::: "memory");
    }
    asm volatile("" ::: "memory");
    STAMP(24 + wv);
    const unsigned need = (unsigned)(__builtin_amdgcn_ballot_w64((lo & 0x10000u) != 0u) >> 4) & 0xfu;
    int E = level(0);
    f16v acc;
    for (;;) {
        asm volatile("" ::: "memory");
        ++npoll;
        const u2v pv = __builtin_amdgcn_raw_buffer_load_b64(rs, (npoll % TMO_EVERY) == 0u ? poff1 : poff0, 0, AUX_LD);
        __builtin_amdgcn_sched_barrier(0);
#pragma unroll
        for (int k = 0; k < 4; ++k) {
            W[k] = load_quad(rs, ow + 1024 * k);
            __builtin_amdgcn_sched_barrier(0);
        }
        apply(X);
        __builtin_amdgcn_sched_barrier(0);
        acc = ZERO16;
#pragma unroll
        for (int k = 0; k < 4; ++k) {
#pragma unroll
            for (int f = 0; f < 4; ++f) {
                acc = mm(QF(W[k], f), QF(X[k], f), acc);
                __builtin_amdgcn_sched_barrier(0);
            }
        }
        lo = pv[0];
        m = (unsigned)__builtin_amdgcn_ballot_w64(pv[1] >= call) & 0xfffu;
        if ((unsigned)__builtin_amdgcn_readlane((int)lo, 16) == call) bad = true;
        if (need == 0u || bad) break;
        const unsigned ok = (quad_valid(W[0]) ? 1u : 0u) | (quad_valid(W[1]) ? 2u : 0u) | (quad_valid(W[2]) ? 4u : 0u) |
                            (quad_valid(W[3]) ? 8u : 0u);
        if ((need & ~ok) == 0u) break;
        if ((++nre & 15u) == 0u && late()) { bad = true; break; }
    }
    __builtin_amdgcn_sched_barrier(0);
    auto foreign = [&]() {
        if (bad && L == 0) LDS_ST(S.bad, 1u);
        deposit(S.redF[wv], scale16(acc, -E), i, hh);
        arrive(S.nfor, 1u << wv);
        STAMP(32 + wv);
    };
    bool pend = (m & 0xf00u) == 0xf00u;
    if (!pend) {
        foreign();
        for (poll(); !bad && (m & 0xf00u) != 0xf00u; poll()) {
            if (late()) { bad = true; break; }
            __builtin_amdgcn_s_sleep(SLEEP_O);
            asm volatile("" ::: "memory");
        }
    }
    const bool needo = (__builtin_amdgcn_readlane((int)lo, 8) & 0x10000) != 0;
    const int EO = level(8);
    for (;;) {
        asm volatile("" ::: "memory");
        {
#pragma unroll
            for (int k = 0; k < 4; ++k) {
                X[k] = load_quad(rs, oxo + 1024 * k);
                __builtin_amdgcn_sched_barrier(0);
            }
        }
        if (pend) {
            foreign();
            pend = false;
            asm volatile("" : "=v"(acc));
        }
        __builtin_amdgcn_sched_barrier(0);
        apply(W);
        __builtin_amdgcn_sched_barrier(0);
        acc = ZERO16;
#pragma unroll
        for (int k = 0; k < 4; ++k) {
#pragma unroll
            for (int f = 0; f < 4; ++f) {
                acc = mm(QF(W[k], f), QF(X[k], f), acc);
                __builtin_amdgcn_sched_barrier(0);
            }
        }
        if (!needo || bad) break;
        const bool ok = quad_valid(X[0]) & quad_valid(X[1]) & quad_valid(X[2]) & quad_valid(X[3]);
        if (ok) break;
        if ((++nre & 15u) == 0u && late()) { bad = true; break; }
    }
    STAMP(40 + wv);
    __builtin_amdgcn_sched_barrier(0);
    if (bad && L == 0) LDS_ST(S.bad, 1u);
    deposit(S.redO[wv], scale16(acc, -EO), i, hh);
    arrive(S.nown, 1u << wv);
    STAMP(48 + wv);
}

extern "C" void kernel_launch(int rank, int nranks, ncclComm_t comm,
                              hipStream_t stream,
                              void* const* d_in, const int* in_sizes, int n_in,
                              void* d_out, int out_size,
                              const int* in_dtypes, int out_dtype,
                              void* const* d_out_all,
                              void* const* d_ws_all, size_t ws_bytes) {
    if (nranks != NR || rank < 0 || rank >= NR || n_in < 2 || ws_bytes < (size_t)WS_NEED || out_size != MS * NDIM) {
        fprintf(stderr, "unexpected problem geometry\n");
        return;
    }
    Args a;
    a.x = (const float*)d_in[0];
    a.w = (const float*)d_in[1];
    a.ws = (char*)d_ws_all[rank];
    a.out = (float*)d_out;
    a.rank = rank;
    a.pad = 0;
    for (int l = 0; l < NR - 1; ++l) a.dws[l] = (char*)d_ws_all[(rank + 1 + l) & (NR - 1)];
    for (int p = 0; p < NR / 2; ++p) {
        a.dws[NR - 1 + p] = (char*)d_ws_all[2 * p + 1 - (rank & 1)];
        a.fout[p] = (float*)d_out_all[2 * p + 1 - (rank & 1)];
    }
    gemm_rs_kernel<<<NWG, 640, 0, stream>>>(a);
    HIP_CHECK(hipGetLastError());
}
```

```cpp
#include <hip/hip_runtime.h>
#include <rccl/rccl.h>
#include <cstdio>
#include <cstdint>

#define HIP_CHECK(expr) do { hipError_t _e = (expr); if (_e != hipSuccess) { \
    fprintf(stderr, "HIP error %s at %s:%d\n", hipGetErrorString(_e), __FILE__, __LINE__); \
    return; } } while (0)

typedef _Float16 h8v __attribute__((ext_vector_type(8)));
typedef float f16v __attribute__((ext_vector_type(16)));
typedef float f4v __attribute__((ext_vector_type(4)));
typedef float f2v __attribute__((ext_vector_type(2)));
typedef unsigned u4v __attribute__((ext_vector_type(4)));
typedef unsigned u2v __attribute__((ext_vector_type(2)));

namespace {

constexpr int NR = 8;
constexpr int NDIM = 2048;
constexpr int KS = 256;
constexpr int MS = 256;
constexpr int NWG = 256;
constexpr int NSTR = 32;
constexpr int NPC = 48;
constexpr int RMAX = 7;
constexpr unsigned F_STR = 33 * 1024, R_STR = 4 * F_STR, S_STR = 6 * R_STR;
constexpr unsigned DATA_BYTES = 2u * NR * S_STR;
constexpr unsigned FLAG_STRIDE = 128;
constexpr unsigned FLAG_OFF = DATA_BYTES;
constexpr unsigned FLAG_BYTES = 2u * NR * NPC * FLAG_STRIDE;
constexpr unsigned OFL_OFF = FLAG_OFF + FLAG_BYTES;
constexpr unsigned OFL_STRIDE = 64;
constexpr unsigned CTR_OFF = OFL_OFF + NWG * OFL_STRIDE;
constexpr unsigned CTR_STRIDE = 64;
constexpr unsigned TMO_OFF = CTR_OFF + NWG * CTR_STRIDE;
constexpr unsigned PRG_OFF = TMO_OFF + 128;
constexpr unsigned WS_NEED = PRG_OFF + NR * 128;
#define DLT 50000000ull
constexpr unsigned long long DEADLINE_TICKS = DLT;

#define INFL 2
#define SLEEP_N 1
#define FB_T1 50
#define FB_T2 110
#define FB_T3 200
#define FB_TL 60
#define FB_W0 2
#define FB_END 6
#define FB_WE 4
#define SLEEP_E 16
#define SLEEP_W 8
#define SLEEP_O 2
#define SLEEP_T 2
#define SLEEP_T2 0
#define SLEEP_R 2
#define TMO_EVERY 16
#define FB_SLEEP 4
#define STAG_S 1
#define EVS 64
#define EVC -32
#define GPD 16
#define RGD 2
#define LEAD_W 2
#define LEAD_O 2
#define ZAT 4
#define PREFETCH 1
#define WL_S 1
#define WL_C 1
#define NL_S 1
#define NL_C 1
#define STAG_MODE 1
#define TILE_WIN 1
#define TILE_WJ 4
#define TILE_WJ2 16
#define LKMAP 0x75316420u
#define TG_A 1300
#define TG_B 19
#define TG_K 31
#define TG_B2 0
#define TG_M 100000
#define XO_PLAIN 0
#define TG_P 1
#define TG_PD 4
#define CG_W 0
#define CG_A 500
#define CG_B 24
#define CG_C 15
#define CG_X 0
#define CG_S 4
#define TG_S 1
#define SC_ON 3
#define FB_T1L 45
#define FB_TLS 50
#define CW_S 0
#define CW_L 0
#define PROG_F 1
#define PROG_O 1
#define AUX_OWN 17
#define PUSH_POL 0
#define PUSH_FIRST 1
#define PUSH_LIKELY 0
#define PUSH_EXPECT(c) (c)
constexpr int GP = GPD, RG = RGD, QREL = (32 / GP) * RG * 4, FH = QREL / 4;
constexpr int LS = 192 + QREL, LC = 192 - QREL;
static_assert(LS <= 32 * RMAX && QREL <= 32 && QREL > 0, "stream geometry");
#define AUX_ST 17
#define AUX_LD 17

#define STAMP(idx) do {} while (0)
#define STAMPV(idx, v) do {} while (0)
#define TNOW() 0ull

#define LDS_LD(v) __builtin_amdgcn_readfirstlane(__hip_atomic_load(&(v), __ATOMIC_RELAXED, __HIP_MEMORY_SCOPE_WORKGROUP))
#define LDS_ST(v, x) __hip_atomic_store(&(v), (x), __ATOMIC_RELAXED, __HIP_MEMORY_SCOPE_WORKGROUP)

struct Args {
    const float* x;
    const float* w;
    char* ws;
    float* out;
    int rank, pad;
    char* dws[NR + 3];
    float* fout[NR / 2];
};

__device__ __forceinline__ unsigned data_of(int par, int src, int u) {
    return (unsigned)(par * NR + src) * S_STR + (unsigned)(u >> 5) * R_STR + (unsigned)(u & 31) * 1024u;
}
__device__ __forceinline__ unsigned flag_of(int par, int src, int u) {
    return FLAG_OFF + (unsigned)((par * NR + src) * NPC + (u >> 2)) * FLAG_STRIDE + (unsigned)(u & 3) * 8u;
}
__device__ __forceinline__ int frow(int r, int i) {
    return MS * (2 * (i >> 1) + 1 - (r & 1)) + 64 * (r >> 1) + 32 * (i & 1);
}

struct Ent { int kind, src, u, a0, kq; };
constexpr int rounds_c(int n, int s) { return (n - s + 31) >> 5; }
__device__ __forceinline__ int rounds(int typ, int s) {
    return typ == 2 ? (s < QREL ? 1 : 0) : ((typ ? LS : LC) - s + 31) >> 5;
}
__device__ __forceinline__ Ent entry(int typ, int rank, int dest, int rho, int s) {
    Ent e;
    e.src = rank;
    if (typ == 2) {
        const int x = s % FH, j = GP * (x / RG) + GP - RG + x % RG;
        e.kind = 1; e.kq = dest >> 1; e.a0 = 1024 * (dest & 1) + 32 * j; e.u = 32 + 4 * j + e.kq;
        return e;
    }
    if (rho == 0) { e.kind = 0; e.kq = s & 3; e.a0 = frow(dest, s >> 2); e.u = s; return e; }
    const int n = 32 * (rho - 1) + s, o = n - (typ ? LS - 64 : LC - 64);
    if (o >= 0) { e.kind = 0; e.kq = o & 3; e.a0 = MS * dest + 32 * (o >> 2); e.u = 160 + o; return e; }
    constexpr int GS = 4 * (GP + RG), GC = 4 * (GP - RG);
    const int g = typ ? n / GS : n / GC, t = n - g * (typ ? GS : GC);
    if (t < 4 * GP) {
        const int j = GP * g + (t >> 2);
        e.kind = 1; e.kq = t & 3; e.a0 = 1024 * (dest & 1) + 32 * j; e.u = 32 + 4 * j + e.kq;
    } else {
        const int r = t - 4 * GP;
        e.kind = 2; e.kq = rank >> 1; e.a0 = 0; e.src = 2 * (r & 3) + 1 - (rank & 1);
        e.u = 32 + 4 * (GP * g + GP - RG + (r >> 2)) + e.kq;
    }
    return e;
}
__device__ __forceinline__ int first_early(int typ, int s) {
    return typ == 1 ? (LS - 32 - EVS - s + 31) >> 5 : typ == 0 ? (LC - 32 - (EVC) - s + 31) >> 5 : 1;
}

__device__ __forceinline__ bool clears(int typ, int rank, int s) { return typ == 0 && (s >> 3) == (rank >> 1); }
constexpr unsigned EMPTY = 0xffffffffu;

template <int NV>
__device__ __forceinline__ int block_exp(const float (&v)[NV]) {
    float m = 0.f;
#pragma unroll
    for (int t = 0; t < NV; ++t) m = fmaxf(m, fabsf(v[t]));
    unsigned u = __float_as_uint(m);
    u = max(u, (unsigned)__builtin_amdgcn_update_dpp(0, (int)u, 0xB1, 0xf, 0xf, true));
    u = max(u, (unsigned)__builtin_amdgcn_update_dpp(0, (int)u, 0x4E, 0xf, 0xf, true));
    u = max(u, (unsigned)__builtin_amdgcn_update_dpp(0, (int)u, 0x141, 0xf, 0xf, true));
    u = max(u, (unsigned)__builtin_amdgcn_update_dpp(0, (int)u, 0x140, 0xf, 0xf, true));
    const unsigned r = max(max((unsigned)__builtin_amdgcn_readlane((int)u, 0), (unsigned)__builtin_amdgcn_readlane((int)u, 16)),
                           max((unsigned)__builtin_amdgcn_readlane((int)u, 32), (unsigned)__builtin_amdgcn_readlane((int)u, 48)));
    unsigned E = r >> 23;
    E = E < 15u ? 15u : E;
    return 141 - (int)E;
}

__device__ __forceinline__ u4v pack8(const float* v, int e) {
    h8v r;
    typedef _Float16 h2v __attribute__((ext_vector_type(2)));
    const float sc = __uint_as_float((unsigned)(e + 127) << 23);
    const f2v s2 = {sc, sc};
#pragma unroll
    for (int j = 0; j < 8; j += 2) {
        const h2v t = __builtin_convertvector(f2v{v[j], v[j + 1]} * s2, h2v);
        r[j] = t[0];
        r[j + 1] = t[1];
    }
    return __builtin_bit_cast(u4v, r);
}

template <int NF>
__device__ __forceinline__ void load_w(const float* __restrict__ w, int k0, int col0, int i, int hh, float (&v)[8 * NF]) {
    const float* base = w + (size_t)(k0 + 8 * hh) * NDIM + col0 + i;
#pragma unroll
    for (int s = 0; s < NF; ++s)
#pragma unroll
        for (int j = 0; j < 8; ++j) v[8 * s + j] = base[(size_t)(16 * s + j) * NDIM];
}

template <int NF>
__device__ __forceinline__ void load_x(const float* __restrict__ x, int row0, int k0, int i, int hh, float (&v)[8 * NF]) {
    const float* base = x + (size_t)(row0 + i) * KS + k0 + 8 * hh;
#pragma unroll
    for (int s = 0; s < NF; ++s) {
        f4v t0 = *(const f4v*)(base + 16 * s);
        f4v t1 = *(const f4v*)(base + 16 * s + 4);
#pragma unroll
        for (int j = 0; j < 4; ++j) { v[s * 8 + j] = t0[j]; v[s * 8 + 4 + j] = t1[j]; }
    }
}

__device__ __forceinline__ f16v mm(u4v a, u4v b, f16v c) {
    return __builtin_amdgcn_mfma_f32_32x32x16_f16(__builtin_bit_cast(h8v, a), __builtin_bit_cast(h8v, b), c, 0, 0, 0);
}
#define ZERO16 f16v{0.f, 0.f, 0.f, 0.f, 0.f, 0.f, 0.f, 0.f, 0.f, 0.f, 0.f, 0.f, 0.f, 0.f, 0.f, 0.f}

__device__ __forceinline__ f16v scale16(f16v t, int e) {
    const int e1 = e >> 1, e2 = e - e1;
    const float c1 = __uint_as_float((unsigned)(e1 + 127) << 23), c2 = __uint_as_float((unsigned)(e2 + 127) << 23);
    const f2v s1 = {c1, c1}, s2 = {c2, c2};
#pragma unroll
    for (int r = 0; r < 16; r += 2) {
        const f2v u = (f2v{t[r], t[r + 1]} * s1) * s2;
        t[r] = u[0]; t[r + 1] = u[1];
    }
    return t;
}

__device__ __forceinline__ void fma16(f16v& acc, f16v t, int e) {
    const int e1 = e >> 1, e2 = e - e1;
    const float c1 = __uint_as_float((unsigned)(e1 + 127) << 23), c2 = __uint_as_float((unsigned)(e2 + 127) << 23);
    const f2v s1 = {c1, c1}, s2 = {c2, c2};
#pragma unroll
    for (int r = 0; r < 16; r += 2) {
        const f2v u = __builtin_elementwise_fma(f2v{t[r], t[r + 1]} * s1, s2, f2v{acc[r], acc[r + 1]});
        acc[r] = u[0]; acc[r + 1] = u[1];
    }
}

struct Quad { u4v f[4]; };
#define QF(q, n) ((q).f[n])

#define AUX_XF AUX_LD
#define AUX_W AUX_LD
#define AUX_XO AUX_LD
template <int AUX = AUX_LD>
__device__ __forceinline__ Quad load_quad(__amdgpu_buffer_rsrc_t rs, int off) {
    Quad q;
#pragma unroll
    for (int f = 0; f < 4; ++f) q.f[f] = __builtin_amdgcn_raw_buffer_load_b128(rs, off, f * (int)F_STR, AUX);
    return q;
}

__device__ __forceinline__ f16v mm_quad(const Quad& w, const Quad& x, f16v t) {
    t = mm(QF(w, 0), QF(x, 0), t);
    t = mm(QF(w, 1), QF(x, 1), t);
    t = mm(QF(w, 2), QF(x, 2), t);
    return mm(QF(w, 3), QF(x, 3), t);
}

__device__ __forceinline__ unsigned down(int d) {
    return d <= 14 ? (unsigned)(15 - d) << 10 : (1u << 24 >> min(d, 31));
}
__device__ __forceinline__ void scale_quad(Quad& q, unsigned c) {
    const _Float16 h = __builtin_bit_cast(_Float16, (unsigned short)c);
#pragma unroll
    for (int f = 0; f < 4; ++f) {
        q.f[f] = __builtin_bit_cast(u4v, __builtin_bit_cast(h8v, q.f[f]) * h);
        asm volatile("" : "+v"(q.f[f]));
    }
}

__device__ __forceinline__ u4v no_zero(u4v u) {
    u[0] = u[0] ? u[0] : 0x8000u;
    return u;
}

__device__ __forceinline__ bool quad_valid(const Quad& q) {
    return __builtin_amdgcn_ballot_w64(min(min(q.f[0][0], q.f[1][0]), min(q.f[2][0], q.f[3][0])) == 0u) == 0ull;
}

template <int N>
__device__ __forceinline__ void wait_vm() {
    asm volatile("s_waitcnt vmcnt(%0)" ::"i"(N) : "memory");
}

__device__ __forceinline__ void wait_vm_n(int n) {
    switch (n) {
        case 0: wait_vm<0>(); break;
        case 1: wait_vm<1>(); break;
        case 2: wait_vm<2>(); break;
        case 3: wait_vm<3>(); break;
        case 4: wait_vm<4>(); break;
        case 5: wait_vm<5>(); break;
        case 6: wait_vm<6>(); break;
        case 7: wait_vm<7>(); break;
        case 8: wait_vm<8>(); break;
        case 9: wait_vm<9>(); break;
        default: wait_vm<10>(); break;
    }
}

__device__ __forceinline__ void raw_barrier() {
    asm volatile("s_waitcnt lgkmcnt(0)\n\ts_barrier" ::: "memory");
}

template <int SLP, class T>
__device__ __forceinline__ unsigned lds_wait(T& v, unsigned n = 1u) {
    unsigned r, it = 0u;
    unsigned long long t0 = 0ull;
    while ((r = (unsigned)LDS_LD(v)) < n) {
        __builtin_amdgcn_s_sleep(SLP);
        if ((++it & 1023u) == 0u) {
            const unsigned long long t = __builtin_amdgcn_s_memrealtime();
            if (t0 == 0ull) t0 = t;
            else if (t - t0 > 2ull * DEADLINE_TICKS) { r = 0u; break; }
        }
    }
    asm volatile("" ::: "memory");
    return r;
}

struct Lds {
    u4v stage[RMAX][256];
    f4v redF[NR][8 * 33];
    f4v redO[NR][8 * 33];
    int exp_[RMAX + 1];
    int rdy[RMAX + 1];
    unsigned call, bad, nloc, locdone, nfor, nown, fdone, clr;
    int prog, win, zok;
};

__device__ __forceinline__ void deposit(f4v* slot, const f16v& t, int i, int hh) {
#pragma unroll
    for (int g = 0; g < 4; ++g)
        slot[(2 * g + hh) * 33 + i] = f4v{t[4 * g], t[4 * g + 1], t[4 * g + 2], t[4 * g + 3]};
}
__device__ __forceinline__ void arrive(unsigned& c, unsigned v) {
    unsigned long long ex;
    asm volatile("s_mov_b64 %0, exec\n\ts_mov_b64 exec, 1\n\tds_add_u32 %1, %2\n\ts_mov_b64 exec, %0"
                 : "=&s"(ex)
                 : "v"((unsigned)(uintptr_t)(__attribute__((address_space(3))) unsigned*)&c), "v"(v)
                 : "memory");
}
__device__ __forceinline__ void gather(f4v (&acc)[4], bool& bad, unsigned& flags, unsigned& badw, const f4v* red) {
    unsigned got = 0u, it = 0u;
    unsigned long long t0 = 0ull;
    auto late = [&]() -> bool {
        if ((++it & 1023u) == 0u) {
            const unsigned long long t = __builtin_amdgcn_s_memrealtime();
            if (t0 == 0ull) t0 = t;
            else if (t - t0 > 2ull * DEADLINE_TICKS) return true;
        }
        return false;
    };
    for (int n = 0; n < NR - 2 && !bad;) {
        const unsigned m = (unsigned)LDS_LD(flags) & ~got;
        asm volatile("" ::: "memory");
        if (m == 0u) {
            __builtin_amdgcn_s_sleep(0);
            bad = late();
            continue;
        }
        const int p = __builtin_ctz(m);
#pragma unroll
        for (int u = 0; u < 4; ++u) acc[u] += red[p * (8 * 33) + 8 * u];
        got |= 1u << p;
        ++n;
    }
    const int p = __builtin_ctz(~got);
    f4v v[4];
    for (;;) {
        asm volatile("" ::: "memory");
        const unsigned c = __hip_atomic_load(&flags, __ATOMIC_RELAXED, __HIP_MEMORY_SCOPE_WORKGROUP);
        __builtin_amdgcn_sched_barrier(0);
        asm volatile("" ::: "memory");
        const unsigned w = __hip_atomic_load(&badw, __ATOMIC_RELAXED, __HIP_MEMORY_SCOPE_WORKGROUP);
#pragma unroll
        for (int u = 0; u < 4; ++u) v[u] = red[p * (8 * 33) + 8 * u];
        __builtin_amdgcn_sched_barrier(0);
        if (__builtin_amdgcn_readfirstlane(c) == 0x7fu) { bad |= __builtin_amdgcn_readfirstlane(w) != 0u; break; }
        asm volatile("s_waitcnt lgkmcnt(0)" ::: "memory");
        if (bad || late()) { bad = true; break; }
    }
#pragma unroll
    for (int u = 0; u < 4; ++u) acc[u] += v[u];
    if (bad) {
        const float qn = __builtin_nanf("");
#pragma unroll
        for (int u = 0; u < 4; ++u) acc[u] = f4v{qn, qn, qn, qn};
    }
}

__device__ __forceinline__ void pusher(const Args& a, Lds& S, char* dws, int b, int rank, int dest, int s, int L, int typ) {
    const int nrnd = rounds(typ, s), NST = 4 * nrnd;
    char* const myws = a.ws;
    const unsigned long long tq0 = TNOW();
    const unsigned cv = __hip_atomic_load((unsigned*)(myws + CTR_OFF + b * CTR_STRIDE), __ATOMIC_RELAXED,
                                          __HIP_MEMORY_SCOPE_AGENT);
    const int i = L & 31, hh = L >> 5;
    u4v t[4];
    unsigned vdat, vflg;
    {
        const Ent en = entry(typ, rank, dest, 0, s);
        float v0[32];
        if (typ == 2) load_w<4>(a.w, 64 * en.kq, en.a0, i, hh, v0);
        else load_x<4>(a.x, en.a0, 64 * en.kq, i, hh, v0);
        const Ent le = entry(typ, rank, dest, L < nrnd ? L : 0, s);
        vdat = data_of(0, le.src, le.u);
        vflg = flag_of(0, le.src, le.u);
        asm volatile("" : "+v"(vdat), "+v"(vflg));
        if (L <= RMAX) S.rdy[L] = 0;
        if (L == 0) {
            S.call = 0u; S.bad = 0u; S.nloc = 0u; S.locdone = 0u; S.nfor = 0u; S.nown = 0u; S.fdone = 0u;
            S.clr = clears(typ, rank, s) ? 0u : 1u;
            S.prog = 0; S.zok = 0; S.win = typ == 2 ? 6 : INFL;
        }
        raw_barrier();
        const int e = block_exp<32>(v0);
        if (L == 0) { LDS_ST(S.rdy[0], 1); S.exp_[0] = e; }
#pragma unroll
        for (int f = 0; f < 4; ++f) t[f] = pack8(v0 + 8 * f, e);
        if (typ == 2) {
#pragma unroll
            for (int f = 0; f < 4; ++f) t[f] = no_zero(t[f]);
        }
    }
    const unsigned call = __builtin_amdgcn_readfirstlane(cv) + 1u;
    if (L == 0) LDS_ST(S.call, call);
    if (nrnd == 0) return;
    const int par = (int)(call & 1u);
    __builtin_amdgcn_s_setprio(3);
    const __amdgpu_buffer_rsrc_t rs = __builtin_amdgcn_make_buffer_rsrc(dws, 0, (int)DATA_BYTES, 0x00020000);
    vdat += (unsigned)par * (NR * S_STR);
    vflg += (unsigned)par * (NR * NPC * FLAG_STRIDE);
    auto publish = [&](int c, unsigned early) {
        if (L == c) {
            const unsigned long long val = ((unsigned long long)call << 32) | (((unsigned)S.exp_[c] & 0xffffu) | early);
            __hip_atomic_store((unsigned long long*)(dws + vflg), val, __ATOMIC_RELAXED, __HIP_MEMORY_SCOPE_SYSTEM);
        }
    };
    int issued = 0, pub = 0, dn = 0, h1 = 0, h2 = 0, h3 = 0, h4 = 0, h5 = 0, h6 = 0, h7 = 0;
    const int ev = first_early(typ, s);
    const int nl = 4 * ((typ == 1 ? LS : LC) >> 5) - (typ == 1 ? NL_S : NL_C);
    for (int c = 0;; ++c) {
        const int so = __builtin_amdgcn_readlane((int)vdat, c);
        bool have = false;
#pragma unroll
        for (int f = 0; f < 4; ++f) {
            const int n = 4 * c + f;
            __builtin_amdgcn_raw_buffer_store_b128(t[f], rs, L * 16, so + f * (int)F_STR, AUX_ST);
            const int h0 = ++issued;
            __builtin_amdgcn_sched_barrier(0);
            if (c >= ev && f == (c + 1 == nrnd ? LEAD_O : LEAD_W)) {
                if (c + 1 == nrnd && !lds_wait<1>(S.zok)) {
                    if (L == 0) { LDS_ST(S.bad, 1u); LDS_ST(S.prog, NST); }
                    return;
                }
                publish(c, 0x10000u); ++issued;
            }
            if (L == 0) LDS_ST(S.prog, n + 1 == NST ? NST : dn);
            while (pub < ev && 4 * (pub + 1) <= dn && (pub > 0 || LDS_LD(S.clr))) { publish(pub, 0u); ++issued; ++pub; }
            if (f == 3) {
                if (n + 1 == NST) break;
                have = LDS_LD(S.rdy[c + 1]) != 0;
                asm volatile("" ::: "memory");
                if (have) {
#pragma unroll
                    for (int g = 0; g < 4; ++g) t[g] = S.stage[c + 1][g * 64 + L];
                }
            }
            int win = LDS_LD(S.win);
            if (n >= nl) win = max(win, typ == 1 ? WL_S : WL_C);
            const int ik = win <= 1 ? h0 : win == 2 ? h1 : win == 3 ? h2 : win == 4 ? h3 : win <= 6 ? h5 : h7;
            const int k1 = n + 2 - (win > 6 ? 8 : win > 4 ? 6 : win);
            if (k1 > dn) {
                wait_vm_n(issued - ik);
                dn = k1;
            }
            h7 = h6; h6 = h5; h5 = h4; h4 = h3; h3 = h2; h2 = h1; h1 = h0;
        }
        if (c + 1 == nrnd) break;
        if (!have) {
            while (pub < ev && 4 * (pub + 1) <= dn && (pub > 0 || LDS_LD(S.clr))) { publish(pub, 0u); ++issued; ++pub; }
            if (!lds_wait<0>(S.rdy[c + 1])) {
                if (L == 0) { LDS_ST(S.bad, 1u); LDS_ST(S.prog, NST); }
                return;
            }
#pragma unroll
            for (int g = 0; g < 4; ++g) t[g] = S.stage[c + 1][g * 64 + L];
        }
    }
    const unsigned long long tq1 = TNOW();
    if (pub == 0 && ev > 0) {
        if (!lds_wait<1>(S.clr)) {
            if (L == 0) LDS_ST(S.bad, 1u);
            return;
        }
        while (pub < ev && 4 * (pub + 1) <= dn) { publish(pub, 0u); ++issued; ++pub; }
    }
    if (pub < ev && pub == nrnd - 3) { wait_vm_n(issued - h7); publish(pub, 0u); ++issued; ++pub; }
    if (pub < ev && pub == nrnd - 2) { wait_vm_n(issued - h4); publish(pub, 0u); ++pub; }
    if (pub < ev) { wait_vm<0>(); publish(pub, 0u); }
}

__device__ __forceinline__ bool feed(const Args& a, Lds& S, int rank, int dest, int s, int wv, int L, int typ) {
    if (wv < 1 || wv >= rounds(typ, s)) return false;
    const Ent en = entry(typ, rank, dest, wv, s);
    if (en.kind == 2) return true;
    const int i = L & 31, hh = L >> 5;
    float v[32];
    if (en.kind == 1) load_w<4>(a.w, 64 * en.kq, en.a0, i, hh, v);
    else load_x<4>(a.x, en.a0, 64 * en.kq, i, hh, v);
    const int e = block_exp<32>(v);
#pragma unroll
    for (int f = 0; f < 4; ++f) {
        S.stage[wv][f * 64 + L] = no_zero(pack8(v + 8 * f, e));
    }
    if (L == 0) S.exp_[wv] = e;
    asm volatile("s_waitcnt lgkmcnt(0)" ::: "memory");
    if (L == 0) LDS_ST(S.rdy[wv], 1);
    asm volatile("" ::: "memory");
    return false;
}

}

__global__ void __launch_bounds__(640) gemm_rs_kernel(Args a) {
    __shared__ __attribute__((aligned(16))) Lds S;

    const int b = blockIdx.x;
    const int wr = __builtin_amdgcn_readfirstlane(threadIdx.x >> 6);
    const int wv = wr == 0 ? NR : wr <= NR ? wr - 1 : NR + 1;
    const int L = threadIdx.x & 63;
    const int i = L & 31, hh = L >> 5;
    const int ti = b & 7, tj = (b >> 3) & 31;
    const int lk = (int)(LKMAP >> (4 * ((b >> 5) & 7))) & 7, s = b & 31;
    char* const dws = a.dws[lk < 7 ? lk : 7 + ((s / FH) & 3)];
    float* const pf0 = a.fout[ti >> 1];
    char* const myws = a.ws;
    asm volatile("" ::"s"(a.x), "s"(a.w), "s"(myws), "s"(a.out), "s"(a.rank), "s"(dws), "s"(pf0));
    const int rank = a.rank & (NR - 1);
    const int cls = rank & 1, mem = rank >> 1;
    const int dest = lk < 7 ? (rank + 1 + lk) & (NR - 1) : 2 * ((s / FH) & 3) + 1 - cls;
    const int typ = lk == 7 ? 2 : lk & 1;
    const unsigned tk0 = (unsigned)__builtin_amdgcn_s_memrealtime();

    if (PUSH_EXPECT(wv == NR)) {
        pusher(a, S, dws, b, rank, dest, s, L, typ);
        return;
    }

    if (wv == NR + 1) {
        raw_barrier();
        const unsigned call = lds_wait<1>(S.call);
        if (call == 0u) return;
        if (L == 0)
            __hip_atomic_store((unsigned*)(myws + CTR_OFF + b * CTR_STRIDE), call, __ATOMIC_RELAXED, __HIP_MEMORY_SCOPE_AGENT);
        if (lk < 7) {
            const int nrnd = rounds(typ, s), NST = 4 * nrnd, LT = typ ? 4 * LS : 4 * LC;
            int z = 0;
            auto clear = [&]() {
                const __amdgpu_buffer_rsrc_t rz = __builtin_amdgcn_make_buffer_rsrc(myws, 0, (int)DATA_BYTES, 0x00020000);
                for (int c = first_early(typ, s); c < nrnd; ++c) {
                    const Ent en = entry(typ, dest, rank, c, s);
                    const unsigned zo = data_of((int)(call & 1u) ^ 1, en.src, en.u);
#pragma unroll
                    for (int f = 0; f < 4; ++f)
                        __builtin_amdgcn_raw_buffer_store_b128(u4v{0u, 0u, 0u, 0u}, rz, L * 16, (int)(zo + f * F_STR), AUX_ST);
                }
                z = 1;
            };
            unsigned* pc = (unsigned*)(myws + PRG_OFF + (unsigned)dest * 128u);
            const unsigned base = (call - 1u) * (unsigned)LT;
            int last = 0;
            unsigned it = 0u;
            unsigned long long t0 = 0ull;
            const bool lng = nrnd == rounds(typ, 0);
            const int mul = (SC_ON & (typ ? 1 : 2)) ? (typ ? (lng ? 4 * rounds_c(LS, 0) : 4 * rounds_c(LS, 31)) * NSTR * 1024 / (4 * LS)
                                                          : (lng ? 4 * rounds_c(LC, 0) : 4 * rounds_c(LC, 31)) * NSTR * 1024 / (4 * LC))
                                                    : 1024;
            const int t1 = lng && mul != 1024 ? FB_T1L : FB_T1, tl = !lng && mul != 1024 ? FB_TLS : FB_TL;
            for (;;) {
                int p = LDS_LD(S.prog);
                if ((++it & 255u) == 0u) {
                    const unsigned long long t = __builtin_amdgcn_s_memrealtime();
                    if (t0 == 0ull) t0 = t;
                    else if (t - t0 > 2ull * DEADLINE_TICKS) p = NST;
                }
                const int dl = p > last ? p - last : 0;
                last += dl;
                if (last >= NST) {
                    if (L == 0) __hip_atomic_fetch_add(pc, (unsigned)dl, __ATOMIC_RELAXED, __HIP_MEMORY_SCOPE_AGENT);
                    break;
                }
                if (z == 0 && last >= ZAT) clear();
                unsigned g = 0u;
                if (L == 0) g = __hip_atomic_fetch_add(pc, (unsigned)dl, __ATOMIC_RELAXED, __HIP_MEMORY_SCOPE_AGENT) + (unsigned)dl;
                const int tot = (int)(__builtin_amdgcn_readfirstlane(g) - base);
                if (z == 1) {
                    wait_vm<0>();
                    if (L == 0) LDS_ST(S.zok, 1);
                    z = 2;
                }
                const int lag = ((tot * mul) >> 10) - NSTR * last;
                int win = lag > FB_T3 ? 6 : lag > FB_T2 ? 4 : lag > t1 ? 3 : lag < -tl ? 1 : 2;
                if (tot >= LT - FB_END * NSTR && win < FB_WE) win = FB_WE;
                if (L == 0 && last > 0) LDS_ST(S.win, win);
                __builtin_amdgcn_s_sleep(FB_SLEEP);
            }
            if (z == 0) clear();
            if (z == 1) {
                wait_vm<0>();
                if (L == 0) LDS_ST(S.zok, 1);
            }
        }
        {
            const __amdgpu_buffer_rsrc_t ro = __builtin_amdgcn_make_buffer_rsrc(a.out, 0, MS * NDIM * 4, 0x00020000);
            const int oo = ((32 * ti + (L >> 3)) * NDIM + 1024 * (1 - cls) + 32 * tj + 4 * (L & 7)) * 4;
            bool bad = !lds_wait<SLEEP_T>(S.fdone);
            const unsigned long long t0 = __builtin_amdgcn_s_memrealtime();
            unsigned ok = 0u, it = 0u;
            struct T4 { u4v v[4]; };
            auto look = [&]() -> T4 {
                T4 r;
#pragma unroll
                for (int u = 0; u < 4; ++u)
                    r.v[u] = __builtin_amdgcn_raw_buffer_load_b128(ro, oo + ((ok >> u & 1u) ? 0x40000000 : 0), 8 * u * NDIM * 4, AUX_LD);
                __builtin_amdgcn_sched_barrier(0);
                return r;
            };
            auto check = [&](const T4& r) {
#pragma unroll
                for (int u = 0; u < 4; ++u) {
                    const u4v v = r.v[u];
                    if (__builtin_amdgcn_ballot_w64(v[0] == EMPTY) == 0ull) ok |= 1u << u;
                }
                __builtin_amdgcn_sched_barrier(0);
            };
            while (!bad) {
                asm volatile("" ::: "memory");
                check(look());
                if (ok != 15u && (++it & 15u) != 0u) {
                    __builtin_amdgcn_s_sleep(SLEEP_T2);
                    continue;
                }
                if (ok == 15u) break;
                const unsigned v = __hip_atomic_load((const unsigned*)(myws + TMO_OFF), __ATOMIC_RELAXED, __HIP_MEMORY_SCOPE_SYSTEM);
                if (__builtin_amdgcn_readfirstlane(v) == call) bad = true;
                if (__builtin_amdgcn_s_memrealtime() - t0 > DEADLINE_TICKS) {
                    if (L == 0)
                        __hip_atomic_store((unsigned*)(myws + TMO_OFF), call, __ATOMIC_RELAXED, __HIP_MEMORY_SCOPE_AGENT);
                    bad = true;
                }
            }
            if (bad) {
                const unsigned qn = 0x7fc00000u;
#pragma unroll
                for (int u = 0; u < 4; ++u)
                    __builtin_amdgcn_raw_buffer_store_b128(u4v{qn, qn, qn, qn}, ro, oo, 8 * u * NDIM * 4, AUX_ST);
            }
            STAMP(59);
        }
        return;
    }

    raw_barrier();
    const bool clr = wv == NR - 1 && clears(typ, rank, s);
    if (clr) {
        const __amdgpu_buffer_rsrc_t ro = __builtin_amdgcn_make_buffer_rsrc(a.out, 0, MS * NDIM * 4, 0x00020000);
        const int oo = ((64 * (dest >> 1) + 32 * ((s >> 2) & 1)) * NDIM + 1024 * (dest & 1) + 256 * (s & 3) + 4 * L) * 4;
#pragma unroll
        for (int u = 0; u < 32; ++u)
            __builtin_amdgcn_raw_buffer_store_b128(u4v{EMPTY, EMPTY, EMPTY, EMPTY}, ro, oo, u * NDIM * 4, AUX_ST);
        wait_vm<0>();
        if (L == 0) LDS_ST(S.clr, 1u);
    }
    if (wv != 1) lds_wait<STAG_S>(S.rdy[0]);
    const bool fwd = feed(a, S, rank, dest, s, wv, L, typ);
    const int col0 = 1024 * cls + 32 * tj;
    {
        float vw[16], vf[16], vo[16];
        load_w<2>(a.w, 32 * wv, col0, i, hh, vw);
        load_x<2>(a.x, frow(rank, ti), 32 * wv, i, hh, vf);
        load_x<2>(a.x, MS * rank + 32 * ti, 32 * wv, i, hh, vo);
        const int ew = block_exp<16>(vw);
        const u4v w0 = pack8(vw, ew), w1 = pack8(vw + 8, ew);
        const int ef = block_exp<16>(vf);
        f16v t = mm(w0, pack8(vf, ef), ZERO16);
        t = mm(w1, pack8(vf + 8, ef), t);
        deposit(S.redF[wv], scale16(t, -(ew + ef)), i, hh);
        const int eo = block_exp<16>(vo);
        t = mm(w0, pack8(vo, eo), ZERO16);
        t = mm(w1, pack8(vo + 8, eo), t);
        deposit(S.redO[wv], scale16(t, -(ew + eo)), i, hh);
        arrive(S.nloc, 1u);
    }
    STAMP(60 + wv);
    if (wv == NR - 1) {
        float* pf = pf0;
        float* po = a.out;
        asm volatile("" : "+s"(pf), "+s"(po));
        const __amdgpu_buffer_rsrc_t rf = __builtin_amdgcn_make_buffer_rsrc(pf, 0, MS * NDIM * 4, 0x00020000);
        const __amdgpu_buffer_rsrc_t ro = __builtin_amdgcn_make_buffer_rsrc(po, 0, MS * NDIM * 4, 0x00020000);
        int of = ((64 * mem + 32 * (ti & 1) + (L >> 3)) * NDIM + col0 + 4 * (L & 7)) * 4;
        int oo = ((32 * ti + (L >> 3)) * NDIM + col0 + 4 * (L & 7)) * 4;
        asm volatile("" : "+v"(of), "+v"(oo));
        const int rp = (L & 7) * 33 + (L >> 3);
        f4v aF[4], aO[4];
        bool bad = !lds_wait<1>(S.nloc, NR);
#pragma unroll
        for (int u = 0; u < 4; ++u) {
            aF[u] = S.redF[0][rp + 8 * u];
            aO[u] = S.redO[0][rp + 8 * u];
#pragma unroll
            for (int p = 1; p < NR; ++p) {
                aF[u] += S.redF[p][rp + 8 * u];
                aO[u] += S.redO[p][rp + 8 * u];
            }
            asm volatile("" : "+v"(aF[u]), "+v"(aO[u]));
        }
        asm volatile("s_waitcnt lgkmcnt(0)" ::: "memory");
        if (L == 0) LDS_ST(S.locdone, 1u);
        gather(aF, bad, S.nfor, S.bad, &S.redF[0][rp]);
        u4v v[4];
#pragma unroll
        for (int u = 0; u < 4; ++u) v[u] = __builtin_bit_cast(u4v, aF[u]);
#pragma unroll
        for (int u = 0; u < 4; ++u) {
            v[u][0] = min(v[u][0], EMPTY - 1u);
            asm volatile("" : "+v"(v[u]));
        }
        while ((unsigned)__builtin_amdgcn_s_memrealtime() - tk0 < (unsigned)min(TG_A + TG_B * min(tj, TG_K) + TG_B2 * max(tj - TG_K, 0), TG_M))
            __builtin_amdgcn_s_sleep(TG_S);
        if (typ == 1) lds_wait<1>(S.prog, (unsigned)(4 * rounds(typ, s) - TG_PD));
        const unsigned long long tq = TNOW();
        if (tj < TILE_WJ) {
#pragma unroll
            for (int u = 0; u < 4; ++u) {
                __builtin_amdgcn_raw_buffer_store_b128(v[u], rf, of, 8 * u * NDIM * 4, AUX_ST);
                if (u < 3) wait_vm<TILE_WIN - 1>();
            }
        } else if (tj < TILE_WJ2) {
#pragma unroll
            for (int u = 0; u < 4; ++u) {
                __builtin_amdgcn_raw_buffer_store_b128(v[u], rf, of, 8 * u * NDIM * 4, AUX_ST);
                if (u < 3) wait_vm<1>();
            }
        } else {
#pragma unroll
            for (int u = 0; u < 4; ++u)
                __builtin_amdgcn_raw_buffer_store_b128(v[u], rf, of, 8 * u * NDIM * 4, AUX_ST);
        }
        __builtin_amdgcn_sched_barrier(0);
        asm volatile("s_nop 1" ::: "memory");
        __builtin_amdgcn_sched_barrier(0);
        if (L == 0) LDS_ST(S.fdone, 1u);
        STAMPV(56, tq);
        STAMP(57);
        gather(aO, bad, S.nown, S.bad, &S.redO[0][rp]);
#pragma unroll
        for (int u = 0; u < 4; ++u)
            __builtin_amdgcn_raw_buffer_store_b128(__builtin_bit_cast(u4v, aO[u]), ro, oo, 8 * u * NDIM * 4, AUX_OWN);
        STAMP(58);
        return;
    }
    const unsigned call = lds_wait<1>(S.call);
    const int par = (int)(call & 1u);

    const __amdgpu_buffer_rsrc_t rs = __builtin_amdgcn_make_buffer_rsrc(myws, 0, (int)WS_NEED, 0x00020000);
    const unsigned long long t0 = __builtin_amdgcn_s_memrealtime();
    bool bad = call == 0u;
    unsigned npoll = 0u, nre = 0u;
    auto late = [&]() -> bool {
        if (__builtin_amdgcn_s_memrealtime() - t0 > DEADLINE_TICKS) {
            if (L == 0)
                __hip_atomic_store((unsigned*)(myws + TMO_OFF), call, __ATOMIC_RELAXED, __HIP_MEMORY_SCOPE_AGENT);
            return true;
        }
        return false;
    };
    if (fwd && !bad) {
        const Ent en = entry(1, rank, dest, wv, s);
        const int pf = (int)(L == 0 ? flag_of(par, en.src, en.u) : TMO_OFF);
        u2v pv;
        for (;;) {
            pv = __builtin_amdgcn_raw_buffer_load_b64(rs, pf, 0, AUX_LD);
            if ((unsigned)__builtin_amdgcn_readlane((int)pv[1], 0) >= call) break;
            if ((unsigned)__builtin_amdgcn_readlane((int)pv[0], 1) == call || late()) { bad = true; break; }
            __builtin_amdgcn_s_sleep(SLEEP_R);
            asm volatile("" ::: "memory");
        }
        asm volatile("" ::: "memory");
        const Quad qd = load_quad(rs, (int)(data_of(par, en.src, en.u) + (unsigned)L * 16u));
        S.stage[wv][L] = QF(qd, 0);
        S.stage[wv][64 + L] = QF(qd, 1);
        S.stage[wv][128 + L] = QF(qd, 2);
        S.stage[wv][192 + L] = QF(qd, 3);
        if (L == 0) S.exp_[wv] = (int)(short)pv[0];
        asm volatile("s_waitcnt lgkmcnt(0)" ::: "memory");
        if (L == 0) LDS_ST(S.rdy[wv], 1);
        asm volatile("" ::: "memory");
        STAMP(70 + wv);
    }
    const int src = (rank + 1 + wv) & (NR - 1);
    const int oxf = (int)(data_of(par, src, 4 * ti) + (unsigned)L * 16u);
    const int ow = (int)(data_of(par, src, 32 + 4 * tj) + (unsigned)L * 16u);
    const int oxo = (int)(data_of(par, src, 160 + 4 * ti) + (unsigned)L * 16u);
    const int poff1 = (int)(L < 4 ? flag_of(par, src, 4 * ti + L)
                            : L < 8 ? flag_of(par, src, 32 + 4 * tj + L - 4)
                            : L < 12 ? flag_of(par, src, 160 + 4 * ti + L - 8) : TMO_OFF);
    const int poff0 = (int)(L < 12 ? (unsigned)poff1 : flag_of(par, src, 4 * ti) + 64u);
    unsigned lo = 0u, m = 0u;
    auto poll = [&]() {
        ++npoll;
        const u2v pv = __builtin_amdgcn_raw_buffer_load_b64(rs, (npoll % TMO_EVERY) == 0u ? poff1 : poff0, 0, AUX_LD);
        lo = pv[0];
        m = (unsigned)__builtin_amdgcn_ballot_w64(pv[1] >= call) & 0xfffu;
        if ((unsigned)__builtin_amdgcn_readlane((int)lo, 16) == call) bad = true;
    };
    unsigned cf[4];
    auto level = [&](int xb) -> int {
        int e[4];
#pragma unroll
        for (int k = 0; k < 4; ++k)
            e[k] = (int)(short)__builtin_amdgcn_readlane((int)lo, 4 + k) + (int)(short)__builtin_amdgcn_readlane((int)lo, xb + k);
        const int E = min(min(e[0], e[1]), min(e[2], e[3]));
#pragma unroll
        for (int k = 0; k < 4; ++k) cf[k] = down(e[k] - E);
        return E;
    };
    auto apply = [&](Quad (&Q)[4]) {
#pragma unroll
        for (int k = 0; k < 4; ++k) {
            scale_quad(Q[k], cf[k]);
            cf[k] = 0x3c00u;
        }
    };
    Quad X[4], W[4];
    for (poll(); !bad && (m & 0xfu) != 0xfu; poll()) {
        if (late()) { bad = true; break; }
        __builtin_amdgcn_s_sleep(SLEEP_E);
        asm volatile("" ::: "memory");
    }
    asm volatile("" ::: "memory");
    STAMP(16 + wv);
#pragma unroll
    for (int k = 0; k < 4; ++k) X[k] = load_quad<AUX_XF>(rs, oxf + 1024 * k);
    if (!lds_wait<1>(S.locdone)) bad = true;
    while ((unsigned)__builtin_amdgcn_s_memrealtime() - tk0 < (unsigned)(CG_A + CG_B * tj + CG_C * wv))
        __builtin_amdgcn_s_sleep(CG_S);
    for (; !bad && (m & 0xf0u) != 0xf0u; poll()) {
        if (late()) { bad = true; break; }
        __builtin_amdgcn_s_sleep(SLEEP_W);
        asm volatile("" ::: "memory");
    }
    asm volatile("" ::: "memory");
    STAMP(24 + wv);
    const unsigned need = (unsigned)(__builtin_amdgcn_ballot_w64((lo & 0x10000u) != 0u) >> 4) & 0xfu;
    int E = level(0);
    f16v acc;
    for (;;) {
        asm volatile("" ::: "memory");
        ++npoll;
        const u2v pv = __builtin_amdgcn_raw_buffer_load_b64(rs, (npoll % TMO_EVERY) == 0u ? poff1 : poff0, 0, AUX_LD);
        __builtin_amdgcn_sched_barrier(0);
#pragma unroll
        for (int k = 0; k < 4; ++k) {
            W[k] = load_quad(rs, ow + 1024 * k);
            __builtin_amdgcn_sched_barrier(0);
        }
        apply(X);
        __builtin_amdgcn_sched_barrier(0);
        acc = ZERO16;
#pragma unroll
        for (int k = 0; k < 4; ++k) {
#pragma unroll
            for (int f = 0; f < 4; ++f) {
                acc = mm(QF(W[k], f), QF(X[k], f), acc);
                __builtin_amdgcn_sched_barrier(0);
            }
        }
        lo = pv[0];
        m = (unsigned)__builtin_amdgcn_ballot_w64(pv[1] >= call) & 0xfffu;
        if ((unsigned)__builtin_amdgcn_readlane((int)lo, 16) == call) bad = true;
        if (need == 0u || bad) break;
        const unsigned ok = (quad_valid(W[0]) ? 1u : 0u) | (quad_valid(W[1]) ? 2u : 0u) | (quad_valid(W[2]) ? 4u : 0u) |
                            (quad_valid(W[3]) ? 8u : 0u);
        if ((need & ~ok) == 0u) break;
        if ((++nre & 15u) == 0u && late()) { bad = true; break; }
    }
    __builtin_amdgcn_sched_barrier(0);
    auto foreign = [&]() {
        if (bad && L == 0) LDS_ST(S.bad, 1u);
        deposit(S.redF[wv], scale16(acc, -E), i, hh);
        arrive(S.nfor, 1u << wv);
        STAMP(32 + wv);
    };
    bool pend = (m & 0xf00u) == 0xf00u;
    if (!pend) {
        foreign();
        for (poll(); !bad && (m & 0xf00u) != 0xf00u; poll()) {
            if (late()) { bad = true; break; }
            __builtin_amdgcn_s_sleep(SLEEP_O);
            asm volatile("" ::: "memory");
        }
    }
    const bool needo = (__builtin_amdgcn_readlane((int)lo, 8) & 0x10000) != 0;
    const int EO = level(8);
    for (;;) {
        asm volatile("" ::: "memory");
        {
#pragma unroll
            for (int k = 0; k < 4; ++k) {
                X[k] = load_quad(rs, oxo + 1024 * k);
                __builtin_amdgcn_sched_barrier(0);
            }
        }
        if (pend) {
            foreign();
            pend = false;
            asm volatile("" : "=v"(acc));
        }
        __builtin_amdgcn_sched_barrier(0);
        apply(W);
        __builtin_amdgcn_sched_barrier(0);
        acc = ZERO16;
#pragma unroll
        for (int k = 0; k < 4; ++k) {
#pragma unroll
            for (int f = 0; f < 4; ++f) {
                acc = mm(QF(W[k], f), QF(X[k], f), acc);
                __builtin_amdgcn_sched_barrier(0);
            }
        }
        if (!needo || bad) break;
        const bool ok = quad_valid(X[0]) & quad_valid(X[1]) & quad_valid(X[2]) & quad_valid(X[3]);
        if (ok) break;
        if ((++nre & 15u) == 0u && late()) { bad = true; break; }
    }
    STAMP(40 + wv);
    __builtin_amdgcn_sched_barrier(0);
    if (bad && L == 0) LDS_ST(S.bad, 1u);
    deposit(S.redO[wv], scale16(acc, -EO), i, hh);
    arrive(S.nown, 1u << wv);
    STAMP(48 + wv);
}

extern "C" void kernel_launch(int rank, int nranks, ncclComm_t comm,
                              hipStream_t stream,
                              void* const* d_in, const int* in_sizes, int n_in,
                              void* d_out, int out_size,
                              const int* in_dtypes, int out_dtype,
                              void* const* d_out_all,
                              void* const* d_ws_all, size_t ws_bytes) {
    if (nranks != NR || rank < 0 || rank >= NR || n_in < 2 || ws_bytes < (size_t)WS_NEED || out_size != MS * NDIM) {
        fprintf(stderr, "unexpected problem geometry\n");
        return;
    }
    Args a;
    a.x = (const float*)d_in[0];
    a.w = (const float*)d_in[1];
    a.ws = (char*)d_ws_all[rank];
    a.out = (float*)d_out;
    a.rank = rank;
    a.pad = 0;
    for (int l = 0; l < NR - 1; ++l) a.dws[l] = (char*)d_ws_all[(rank + 1 + l) & (NR - 1)];
    for (int p = 0; p < NR / 2; ++p) {
        a.dws[NR - 1 + p] = (char*)d_ws_all[2 * p + 1 - (rank & 1)];
        a.fout[p] = (float*)d_out_all[2 * p + 1 - (rank & 1)];
    }
    gemm_rs_kernel<<<NWG, 640, 0, stream>>>(a);
    HIP_CHECK(hipGetLastError());
}
```
